```python
import numpy as np

kernel_name = "gpu_gemm_a2a_m4096_k4096_n4096_e4m3e5m2_dequant"


ABSMAX_RTOL = 0.05

COMPUTE_DTYPE = "float32"
EPILOGUE = "dequant"
IN_QUANT = None
OUT_QUANT = None
IN_FP8 = "e4m3e5m2"
IN_DTYPES = {"x": "e4m3", "w_mat": "e5m2",
             "scale_x": "float32", "scale_w": "float32"}
WORLD_SIZE = 8

def _snap_fp8(x, mant_bits, max_abs, min_normal):
    a = np.abs(x.astype(np.float64))
    _m, _e = np.frexp(a)
    q = np.where(a >= min_normal, np.ldexp(1.0, _e - 1 - mant_bits),
                 min_normal * 2.0 ** -mant_bits)
    snapped = np.minimum(np.round(a / q) * q, max_abs)
    return (np.sign(x) * snapped).astype(np.float32)

def setup_inputs():

    x = _snap_fp8(np.random.randn(4096, 4096).astype(np.float32), 3, 448.0, 2.0 ** -6)
    w_mat = _snap_fp8(np.random.randn(4096, 4096).astype(np.float32), 2, 57344.0, 2.0 ** -14)
    scale_x = (0.02 * (1.0 + np.random.rand(1))).astype(np.float32)
    scale_w = (0.02 * (1.0 + np.random.rand(1))).astype(np.float32)
    return {"x": x, "w_mat": w_mat, "scale_x": scale_x, "scale_w": scale_w}

def reference(x, w_mat, scale_x, scale_w):
    acc = (x @ w_mat).astype(np.float32)
    y = (acc * (scale_x * scale_w)).astype(np.float32)
    return y
```

```cpp
#include <hip/hip_runtime.h>
#include <rccl/rccl.h>
#include <cstdio>
#include <cstdint>

#define HIP_CHECK(expr) do { hipError_t _e = (expr); if (_e != hipSuccess) { \
    fprintf(stderr, "HIP error %s at %s:%d\n", hipGetErrorString(_e), __FILE__, __LINE__); \
    return; } } while (0)

namespace {

typedef int v2i __attribute__((ext_vector_type(2)));
typedef int v4i __attribute__((ext_vector_type(4)));
typedef unsigned v4u __attribute__((ext_vector_type(4)));
typedef int v8i __attribute__((ext_vector_type(8)));
typedef float v4f __attribute__((ext_vector_type(4)));
#define LAS __attribute__((address_space(3)))
#define GAS __attribute__((address_space(1)))
#define CAS __attribute__((address_space(4)))

constexpr int NR = 8;
constexpr int KD = 4096;
constexpr int ND = 4096;
constexpr int MS = 512;
constexpr int NC = 512;
constexpr int RR = 64;
#define HG0 24
constexpr int HG1 = 64 - HG0;
static_assert(HG0 % 2 == 0 && HG0 >= 2 && HG0 <= 62, "heights");
#define WIN_X 2
#define NSW_MAIN 3
#define NSW_LATE 5
#define NSW_G1 8
constexpr int NTHREADS = 512;
constexpr int NWG = 256;
constexpr int KSTEPS = KD / 128;

constexpr unsigned SLOT = 16384 + 8192, NS = 6;
constexpr unsigned AHEAD = NS - 1;
constexpr unsigned STG_OFF = 2 * SLOT;
static_assert((KSTEPS - 1) % NS == 1, "the last two stages use slots 0 and 1");
constexpr unsigned CTL_OFF = NS * SLOT;
constexpr unsigned LDS_BYTES = CTL_OFF + 64;

constexpr size_t WS_CALL = 0;
constexpr size_t WS_TMO = 32768;
constexpr size_t WS_DONE = 65536;
constexpr size_t WS_XFLAG = WS_DONE + 8 * 128;
constexpr size_t WS_STAMP = 131072;
constexpr size_t WS_XST = 1u << 20;
constexpr size_t XST_SLOT = (size_t)RR * KD;
constexpr size_t WS_NEED = WS_XST + 2 * NR * XST_SLOT;
constexpr unsigned long long DEADLINE_TICKS = 40000000ull;

struct Params {
    const uint8_t* x;
    const uint8_t* w;
    const float* sx;
    const float* sw;
    float* out[NR];
    uint8_t* ws[NR];
    int rank;
};

__device__ __forceinline__ unsigned rfl(unsigned v) { return __builtin_amdgcn_readfirstlane(v); }

__device__ __forceinline__ __amdgpu_buffer_rsrc_t make_rsrc(const void* p, unsigned bytes) {
    const uint64_t a = (uint64_t)p;
    const uint64_t u = ((uint64_t)rfl((unsigned)(a >> 32)) << 32) | rfl((unsigned)a);
    return __builtin_amdgcn_make_buffer_rsrc((void*)u, (short)0, (int)rfl(bytes), 0x00020000);
}

__device__ __forceinline__ void dma16(unsigned lds, unsigned voff, __amdgpu_buffer_rsrc_t rs, unsigned soff) {
    asm volatile("s_mov_b32 m0, %0\n\ts_nop 0\n\tbuffer_load_dwordx4 %1, %2, %3 offen lds"
                 :: "s"(lds), "v"(voff), "s"(rs), "s"(soff) : "memory");
}
__device__ __forceinline__ void dma16x4(unsigned lds, unsigned v0, unsigned v1, unsigned v2, unsigned v3,
                                        __amdgpu_buffer_rsrc_t rs, unsigned soff) {
    asm volatile("s_mov_b32 m0, %0\n\ts_nop 0\n\t"
                 "buffer_load_dwordx4 %1, %5, %6 offen lds\n\t"
                 "buffer_load_dwordx4 %2, %5, %6 offen offset:1024 lds\n\t"
                 "buffer_load_dwordx4 %3, %5, %6 offen offset:2048 lds\n\t"
                 "buffer_load_dwordx4 %4, %5, %6 offen offset:3072 lds"
                 :: "s"(lds), "v"(v0), "v"(v1), "v"(v2), "v"(v3), "s"(rs), "s"(soff) : "memory");
}
template <int N> __device__ __forceinline__ void wait_vm() { asm volatile("s_waitcnt vmcnt(%0)" :: "i"(N) : "memory"); }
__device__ __forceinline__ void wg_barrier() { asm volatile("s_barrier" ::: "memory"); }
__device__ __forceinline__ void wait_lds() { asm volatile("s_waitcnt lgkmcnt(0)" ::: "memory"); }
__device__ __forceinline__ void st16_sys(void* p, v4f v) {
    asm volatile("global_store_dwordx4 %0, %1, off sc0 sc1\n\ts_nop 1" :: "v"(p), "v"(v) : "memory");
}

#define STAMP(i) do {} while (0)

__device__ __forceinline__ bool wait_flags(const unsigned* flags, unsigned call, unsigned* tmo, int lane) {
    const unsigned long long t0 = __builtin_amdgcn_s_memrealtime();
    for (;;) {
        const bool ok = __hip_atomic_load(flags + (lane & 31), __ATOMIC_RELAXED, __HIP_MEMORY_SCOPE_SYSTEM) >= call;
        if (__all(ok)) break;
        if (rfl(__hip_atomic_load(tmo, __ATOMIC_RELAXED, __HIP_MEMORY_SCOPE_AGENT)) == call
            || __builtin_amdgcn_s_memrealtime() - t0 > DEADLINE_TICKS) {
            if (lane == 0) __hip_atomic_store(tmo, call, __ATOMIC_RELAXED, __HIP_MEMORY_SCOPE_AGENT);
            return false;
        }
        __builtin_amdgcn_s_sleep(2);
    }
    return rfl(__hip_atomic_load(tmo, __ATOMIC_RELAXED, __HIP_MEMORY_SCOPE_AGENT)) != call;
}

template <int NXW>
__device__ __forceinline__ void load_tile(int w, int lane, __amdgpu_buffer_rsrc_t rw, __amdgpu_buffer_rsrc_t rx) {
    constexpr int CNT = 4 + NXW;
    const unsigned v0 = (lane >> 3) * KD + (((lane & 7) ^ ((lane >> 4) & 3)) << 4);
    unsigned vw[4];
#pragma unroll
    for (int j = 0; j < 4; ++j) vw[j] = (v0 ^ (((j >> 1) & 1) << 6)) + (4 * w + j) * (8u * ND) - j * 1024u;
    const unsigned vx0 = (v0 ^ ((w & 1) << 6)) + w * (8u * KD), vx1 = vx0 + 32u * KD;
    unsigned ioff = w * 1024;
    auto issue = [&](unsigned ks) {
        dma16x4(ioff + w * 3072, vw[0], vw[1], vw[2], vw[3], rw, ks * (128u * ND));
        if (NXW > 0) dma16(ioff + 16384, vx0, rx, ks * 128u);
        if (NXW > 1) dma16(ioff + 20480, vx1, rx, ks * 128u);
        ioff += SLOT;
        if (ioff >= SLOT * NS) ioff -= SLOT * NS;
    };
    asm volatile("s_nop 4" : "+s"(rx), "+s"(rw) :: "memory");
#pragma unroll 1
    for (unsigned i = 0; i < AHEAD; ++i) issue(i);
#pragma unroll 1
    for (unsigned i = AHEAD; i < KSTEPS; ++i) {
        wait_vm<CNT * (AHEAD - 1)>();
        wg_barrier();
        issue(i);
    }
    static_assert(AHEAD == 5, "drain");
    wait_vm<CNT * 4>(); wg_barrier();
    wait_vm<CNT * 3>(); wg_barrier();
    wait_vm<CNT * 2>(); wg_barrier();
    wait_vm<CNT>(); wg_barrier();
    wait_vm<0>(); wg_barrier();
}

template <int NU>
__device__ __forceinline__ void consume_tile(LAS uint8_t* smem, int c, int lane, v4f (&acc)[4][2]) {
    const int g = lane >> 4, q = (lane & 15) >> 1, p = lane & 1, m = lane & 15;
    const unsigned fw = ((q >> 1) & 3) | ((g & 1) << 2);
    const unsigned w0 = (16 * g + q) * 128 + (((2 * c) ^ fw) << 4) + p * 8;
    const unsigned x0 = 16384 + m * 128 + ((g ^ ((m >> 1) & 7)) << 4);
#pragma unroll
    for (int u = 0; u < 4; ++u) { acc[u][0] = v4f{0, 0, 0, 0}; acc[u][1] = v4f{0, 0, 0, 0}; }
    v8i wf[2][2], xf[2][NU];
    auto load = [&](int s, v8i (&wv)[2], v8i (&xv)[NU]) {
        LAS uint8_t* sl = smem + (s % NS) * SLOT;
#pragma unroll
        for (int cb = 0; cb < 2; ++cb) {
            LAS uint8_t* b = sl + (w0 ^ (cb << 4));
            v2i t0 = __builtin_amdgcn_ds_read_tr8_b64_v2i32((LAS v2i*)(b));
            v2i t1 = __builtin_amdgcn_ds_read_tr8_b64_v2i32((LAS v2i*)(b + 1024));
            v2i t2 = __builtin_amdgcn_ds_read_tr8_b64_v2i32((LAS v2i*)(b + 8192));
            v2i t3 = __builtin_amdgcn_ds_read_tr8_b64_v2i32((LAS v2i*)(b + 9216));
            wv[cb] = v8i{t0[0], t0[1], t1[0], t1[1], t2[0], t2[1], t3[0], t3[1]};
        }
#pragma unroll
        for (int u = 0; u < NU; ++u) {
            v4i a = *(LAS v4i*)(sl + x0 + u * 2048);
            v4i b = *(LAS v4i*)(sl + (x0 ^ 64) + u * 2048);
            xv[u] = v8i{a[0], a[1], a[2], a[3], b[0], b[1], b[2], b[3]};
        }
    };
    auto mma = [&](v8i (&wv)[2], v8i (&xv)[NU]) {
#pragma unroll
        for (int u = 0; u < NU; ++u)
#pragma unroll
            for (int cb = 0; cb < 2; ++cb)
                acc[u][cb] = __builtin_amdgcn_mfma_scale_f32_16x16x128_f8f6f4(wv[cb], xv[u], acc[u][cb], 1, 0, 0, 0, 0, 0);
    };
    auto step = [&](int s, v8i (&wn)[2], v8i (&xn)[NU], v8i (&wo)[2], v8i (&xo)[NU]) {
        wg_barrier();
        __builtin_amdgcn_sched_barrier(0);
        load(s, wn, xn);
        mma(wo, xo);
        constexpr int PER = (8 + 2 * NU + 2 * NU - 1) / (2 * NU);
#pragma unroll
        for (int i = 0; i < 2 * NU; ++i) {
            __builtin_amdgcn_sched_group_barrier(0x100, PER, 0);
            __builtin_amdgcn_sched_group_barrier(0x8, 1, 0);
        }
        __builtin_amdgcn_sched_barrier(0);
        wait_lds();
    };
    wg_barrier();
    load(0, wf[0], xf[0]);
    wait_lds();
    step(1, wf[1], xf[1], wf[0], xf[0]);
#pragma unroll 1
    for (int it = 0; it < (KSTEPS - 2) / (int)NS; ++it) {
#pragma unroll
        for (int s = 2; s < 2 + (int)NS; s += 2) {
            step(s, wf[0], xf[0], wf[1], xf[1]);
            step(s + 1, wf[1], xf[1], wf[0], xf[0]);
        }
    }
    static_assert((KSTEPS - 2) % NS == 0 && NS % 2 == 0, "trip count");
    mma(wf[1], xf[1]);
}

__device__ __forceinline__ void run_tile(LAS uint8_t* smem, int wave, int lane, const uint8_t* wcol, unsigned wbytes,
                                         const uint8_t* xrows, int h, v4f (&acc)[4][2]) {
    if (wave >= 4) {
        const int w = wave - 4;
        const __amdgpu_buffer_rsrc_t rw = make_rsrc(wcol, wbytes);
        const __amdgpu_buffer_rsrc_t rx = make_rsrc(xrows, h * KD);
        const int nxw = (8 * w < h) + (8 * w + 32 < h);
        if (nxw == 2) load_tile<2>(w, lane, rw, rx);
        else if (nxw == 1) load_tile<1>(w, lane, rw, rx);
        else load_tile<0>(w, lane, rw, rx);
    } else {
        if (h > 48) consume_tile<4>(smem, wave, lane, acc);
        else if (h > 32) consume_tile<3>(smem, wave, lane, acc);
        else if (h > 16) consume_tile<2>(smem, wave, lane, acc);
        else consume_tile<1>(smem, wave, lane, acc);
    }
}

__global__ void __launch_bounds__(NTHREADS) gemm_a2a_kernel(Params P) {
    extern __shared__ __attribute__((aligned(16))) uint8_t smem_[];
    LAS uint8_t* smem = (LAS uint8_t*)smem_;
    const int lane = threadIdx.x & 63;
    const int wave = rfl(threadIdx.x >> 6);
    const int grp = blockIdx.x & 7, idx = blockIdx.x >> 3, panel = idx & 3, rgp = idx >> 2;
    const unsigned rank = (unsigned)P.rank & 7u;
    const bool local = grp == 7;
    const unsigned dest = (rank + 1 + grp) & 7u;
    uint8_t* myws = P.ws[rank];
    uint8_t* pws = P.ws[dest];
    unsigned* ctr = (unsigned*)(myws + WS_CALL + (size_t)blockIdx.x * 128);
    unsigned* tmo = (unsigned*)(myws + WS_TMO);
    STAMP(wave);
    const unsigned call = *(const CAS unsigned*)ctr + 1;
    float scale = *(const CAS float*)P.sx * *(const CAS float*)P.sw;
    const unsigned coff = dest * NC + panel * 128;
    const int g = lane >> 4, m = lane & 15;
    const unsigned src1 = (rank + 1 + rgp) & 7u;
    const int ntiles = local && rgp < 7 ? 2 : 1;

    if (!local && rgp == 7) {
        constexpr int XI = RR * KD / 1024 / 32;
        const unsigned xo = (panel * 8 + wave) * XI * 1024 + lane * 16;
        v4f xv[XI];
#pragma unroll
        for (int i = 0; i < XI; ++i) xv[i] = *(const GAS v4f*)(P.x + xo + i * 1024);
        uint8_t* xd = pws + WS_XST + (size_t)((call & 1) * NR + rank) * XST_SLOT + xo;
#pragma unroll
        for (int i = 0; i < XI; ++i) asm volatile("" : "+v"(xv[i]));
        STAMP(40);
#pragma unroll
        for (int i = 0; i < XI; ++i) {
            if (i >= WIN_X) wait_vm<WIN_X - 1>();
            st16_sys(xd + i * 1024, xv[i]);
        }
        wait_vm<0>();
        if (lane == 0)
            __hip_atomic_store((unsigned*)(pws + WS_XFLAG + rank * 128) + panel * 8 + wave, call, __ATOMIC_RELAXED, __HIP_MEMORY_SCOPE_SYSTEM);
        STAMP(44);
    }

    v4f acc[4][2];
    const int h0 = local ? 64 : rgp == 0 ? HG0 : rgp == 1 ? HG1 : 64;
    const int row0 = local ? rgp * 64 : rgp == 0 ? RR : rgp == 1 ? RR + HG0 : RR + (rgp - 1) * 64;
#pragma unroll 1
    for (int t = 0; t < ntiles; ++t) {
        const uint8_t* xrows = P.x + (size_t)row0 * KD;
        if (t == 1) {
            if (wave == 4) {
                const bool ok = wait_flags((const unsigned*)(myws + WS_XFLAG + src1 * 128), call, tmo, lane);
                __builtin_amdgcn_fence(__ATOMIC_ACQUIRE, "");
                wait_vm<0>();
                *(volatile LAS unsigned*)(smem + CTL_OFF) = ok ? 0u : 1u;
                wait_lds();
                STAMP(44);
            }
            wg_barrier();
            if (*(volatile LAS unsigned*)(smem + CTL_OFF)) scale = __builtin_nanf("");
            xrows = myws + WS_XST + (size_t)((call & 1) * NR + src1) * XST_SLOT;
        }
        run_tile(smem, wave, lane, P.w + coff, (unsigned)(KD * ND) - coff, xrows, h0, acc);
        STAMP(16 + 16 * t + wave);
        if (local && wave < 4) {
            const size_t orow = t ? (size_t)src1 * MS : (size_t)rank * MS + row0;
            float* o = P.out[rank] + (orow + m) * NC + panel * 128 + wave * 32 + g * 4;
#pragma unroll
            for (int u = 0; u < 4; ++u) {
                st16_sys(o + (size_t)u * 16 * NC, acc[u][0] * scale);
                st16_sys(o + (size_t)u * 16 * NC + 16, acc[u][1] * scale);
            }
        }
    }

    if (local) {
        if (wave == 0) {
            if (lane == 0) *ctr = call;
            if (panel == 0 && rgp < 7) {
                wait_vm<0>();
                const bool ok = wait_flags((const unsigned*)(myws + WS_DONE + src1 * 128), call, tmo, lane);
                if (!ok && lane == 0)
                    __hip_atomic_store(P.out[rank] + ((size_t)rank * MS + row0) * NC, __builtin_nanf(""), __ATOMIC_RELAXED, __HIP_MEMORY_SCOPE_SYSTEM);
                STAMP(52);
            }
        }
        return;
    }

    if (wave < 4) {
#pragma unroll
        for (int u = 0; u < 4; ++u)
            if (u * 16 < h0) {
                const int r = u * 16 + m;
                LAS uint8_t* d = smem + STG_OFF + r * 512 + wave * 128;
                *(LAS v4f*)(d + ((g ^ (r & 7)) << 4)) = acc[u][0] * scale;
                *(LAS v4f*)(d + (((4 + g) ^ (r & 7)) << 4)) = acc[u][1] * scale;
            }
        wait_lds();
    }
    wg_barrier();
    const int nsw = rgp == 0 ? 8 : rgp == 1 ? NSW_G1 : rgp == 7 ? NSW_LATE : NSW_MAIN;
    if (wave >= nsw) return;
    STAMP(48);
    float* o = P.out[dest] + ((size_t)rank * MS + row0 + (lane >> 5)) * NC + panel * 128 + (lane & 31) * 4;
#pragma unroll 1
    for (int it = wave; it < h0 / 2; it += nsw) {
        const int r = it * 2 + (lane >> 5);
        const v4f v = *(LAS v4f*)(smem + STG_OFF + r * 512 + (((lane & 31) ^ (r & 7)) << 4));
        if (rgp != 0) wait_vm<0>();
        st16_sys(o + (size_t)it * 2 * NC, v);
    }
    wait_vm<0>();
    wg_barrier();
    if (wave == 0 && lane == 0) {
        __hip_atomic_store((unsigned*)(pws + WS_DONE + rank * 128) + idx, call, __ATOMIC_RELAXED, __HIP_MEMORY_SCOPE_SYSTEM);
        *ctr = call;
    }
    STAMP(52);
}

}

extern "C" void kernel_launch(int rank, int nranks, ncclComm_t comm,
                              hipStream_t stream,
                              void* const* d_in, const int* in_sizes, int n_in,
                              void* d_out, int out_size,
                              const int* in_dtypes, int out_dtype,
                              void* const* d_out_all,
                              void* const* d_ws_all, size_t ws_bytes) {
    (void)comm; (void)in_dtypes; (void)out_dtype; (void)d_out;
    if (nranks != NR || (unsigned)rank >= (unsigned)NR || n_in != 4 || in_sizes[0] != MS * KD || in_sizes[1] != KD * ND
        || out_size != NR * MS * NC || ws_bytes < WS_NEED) {
        fprintf(stderr, "kernel_launch: unexpected problem shape\n");
        return;
    }
    static bool attr_set = false;
    if (!attr_set) {
        HIP_CHECK(hipFuncSetAttribute((const void*)gemm_a2a_kernel, hipFuncAttributeMaxDynamicSharedMemorySize, LDS_BYTES));
        attr_set = true;
    }
    Params P;
    P.x = (const uint8_t*)d_in[0];
    P.w = (const uint8_t*)d_in[1];
    P.sx = (const float*)d_in[2];
    P.sw = (const float*)d_in[3];
    for (int p = 0; p < NR; ++p) { P.out[p] = (float*)d_out_all[p]; P.ws[p] = (uint8_t*)d_ws_all[p]; }
    P.rank = rank;
    hipLaunchKernelGGL(gemm_a2a_kernel, dim3(NWG), dim3(NTHREADS), LDS_BYTES, stream, P);
    HIP_CHECK(hipGetLastError());
}
```

```cpp
#include <hip/hip_runtime.h>
#include <rccl/rccl.h>
#include <cstdio>
#include <cstdint>

#define HIP_CHECK(expr) do { hipError_t _e = (expr); if (_e != hipSuccess) { \
    fprintf(stderr, "HIP error %s at %s:%d\n", hipGetErrorString(_e), __FILE__, __LINE__); \
    return; } } while (0)

namespace {

typedef int v2i __attribute__((ext_vector_type(2)));
typedef int v4i __attribute__((ext_vector_type(4)));
typedef unsigned v4u __attribute__((ext_vector_type(4)));
typedef int v8i __attribute__((ext_vector_type(8)));
typedef float v4f __attribute__((ext_vector_type(4)));
#define LAS __attribute__((address_space(3)))
#define GAS __attribute__((address_space(1)))
#define CAS __attribute__((address_space(4)))

constexpr int NR = 8;
constexpr int KD = 4096;
constexpr int ND = 4096;
constexpr int MS = 512;
constexpr int NC = 512;
constexpr int RR = 64;
#define HG0 24
constexpr int HG1 = 64 - HG0;
static_assert(HG0 % 2 == 0 && HG0 >= 2 && HG0 <= 62, "heights");
#define WIN_X 2
#define NSW_MAIN 3
#define NSW_LATE 5
#define NSW_G1 8
constexpr int NTHREADS = 512;
constexpr int NWG = 256;
constexpr int KSTEPS = KD / 128;

constexpr unsigned SLOT = 16384 + 8192, NS = 6;
constexpr unsigned AHEAD = NS - 1;
constexpr unsigned STG_OFF = 2 * SLOT;
static_assert((KSTEPS - 1) % NS == 1, "the last two stages use slots 0 and 1");
constexpr unsigned CTL_OFF = NS * SLOT;
constexpr unsigned LDS_BYTES = CTL_OFF + 64;

constexpr size_t WS_CALL = 0;
constexpr size_t WS_TMO = 32768;
constexpr size_t WS_DONE = 65536;
constexpr size_t WS_XFLAG = WS_DONE + 8 * 128;
constexpr size_t WS_STAMP = 131072;
constexpr size_t WS_XST = 1u << 20;
constexpr size_t XST_SLOT = (size_t)RR * KD;
constexpr size_t WS_NEED = WS_XST + 2 * NR * XST_SLOT;
constexpr unsigned long long DEADLINE_TICKS = 40000000ull;

struct Params {
    const uint8_t* x;
    const uint8_t* w;
    const float* sx;
    const float* sw;
    float* out[NR];
    uint8_t* ws[NR];
    int rank;
};

__device__ __forceinline__ unsigned rfl(unsigned v) { return __builtin_amdgcn_readfirstlane(v); }

__device__ __forceinline__ __amdgpu_buffer_rsrc_t make_rsrc(const void* p, unsigned bytes) {
    const uint64_t a = (uint64_t)p;
    const uint64_t u = ((uint64_t)rfl((unsigned)(a >> 32)) << 32) | rfl((unsigned)a);
    return __builtin_amdgcn_make_buffer_rsrc((void*)u, (short)0, (int)rfl(bytes), 0x00020000);
}

__device__ __forceinline__ void dma16(unsigned lds, unsigned voff, __amdgpu_buffer_rsrc_t rs, unsigned soff) {
    asm volatile("s_mov_b32 m0, %0\n\ts_nop 0\n\tbuffer_load_dwordx4 %1, %2, %3 offen lds"
                 :: "s"(lds), "v"(voff), "s"(rs), "s"(soff) : "memory");
}
__device__ __forceinline__ void dma16x4(unsigned lds, unsigned v0, unsigned v1, unsigned v2, unsigned v3,
                                        __amdgpu_buffer_rsrc_t rs, unsigned soff) {
    asm volatile("s_mov_b32 m0, %0\n\ts_nop 0\n\t"
                 "buffer_load_dwordx4 %1, %5, %6 offen lds\n\t"
                 "buffer_load_dwordx4 %2, %5, %6 offen offset:1024 lds\n\t"
                 "buffer_load_dwordx4 %3, %5, %6 offen offset:2048 lds\n\t"
                 "buffer_load_dwordx4 %4, %5, %6 offen offset:3072 lds"
                 :: "s"(lds), "v"(v0), "v"(v1), "v"(v2), "v"(v3), "s"(rs), "s"(soff) : "memory");
}
template <int N> __device__ __forceinline__ void wait_vm() { asm volatile("s_waitcnt vmcnt(%0)" :: "i"(N) : "memory"); }
__device__ __forceinline__ void wg_barrier() { asm volatile("s_barrier" ::: "memory"); }
__device__ __forceinline__ void wait_lds() { asm volatile("s_waitcnt lgkmcnt(0)" ::: "memory"); }
__device__ __forceinline__ void st16_sys(void* p, v4f v) {
    asm volatile("global_store_dwordx4 %0, %1, off sc0 sc1\n\ts_nop 1" :: "v"(p), "v"(v) : "memory");
}

#define STAMP(i) do {} while (0)

__device__ __forceinline__ bool wait_flags(const unsigned* flags, unsigned call, unsigned* tmo, int lane) {
    const unsigned long long t0 = __builtin_amdgcn_s_memrealtime();
    for (;;) {
        const bool ok = __hip_atomic_load(flags + (lane & 31), __ATOMIC_RELAXED, __HIP_MEMORY_SCOPE_SYSTEM) >= call;
        if (__all(ok)) break;
        if (rfl(__hip_atomic_load(tmo, __ATOMIC_RELAXED, __HIP_MEMORY_SCOPE_AGENT)) == call
            || __builtin_amdgcn_s_memrealtime() - t0 > DEADLINE_TICKS) {
            if (lane == 0) __hip_atomic_store(tmo, call, __ATOMIC_RELAXED, __HIP_MEMORY_SCOPE_AGENT);
            return false;
        }
        __builtin_amdgcn_s_sleep(2);
    }
    return rfl(__hip_atomic_load(tmo, __ATOMIC_RELAXED, __HIP_MEMORY_SCOPE_AGENT)) != call;
}

template <int NXW>
__device__ __forceinline__ void load_tile(int w, int lane, __amdgpu_buffer_rsrc_t rw, __amdgpu_buffer_rsrc_t rx, unsigned rot) {
    constexpr int CNT = 4 + NXW;
    const unsigned v0 = (lane >> 3) * KD + (((lane & 7) ^ ((lane >> 4) & 3)) << 4);
    unsigned vw[4];
#pragma unroll
    for (int j = 0; j < 4; ++j) vw[j] = (v0 ^ (((j >> 1) & 1) << 6)) + (4 * w + j) * (8u * ND) - j * 1024u;
    const unsigned vx0 = (v0 ^ ((w & 1) << 6)) + w * (8u * KD), vx1 = vx0 + 32u * KD;
    unsigned ioff = w * 1024;
    auto issue = [&](unsigned i) {
        const unsigned ks = (i + rot) & (KSTEPS - 1);
        dma16x4(ioff + w * 3072, vw[0], vw[1], vw[2], vw[3], rw, ks * (128u * ND));
        if (NXW > 0) dma16(ioff + 16384, vx0, rx, ks * 128u);
        if (NXW > 1) dma16(ioff + 20480, vx1, rx, ks * 128u);
        ioff += SLOT;
        if (ioff >= SLOT * NS) ioff -= SLOT * NS;
    };
    asm volatile("s_nop 4" : "+s"(rx), "+s"(rw) :: "memory");
#pragma unroll 1
    for (unsigned i = 0; i < AHEAD; ++i) issue(i);
#pragma unroll 1
    for (unsigned i = AHEAD; i < KSTEPS; ++i) {
        wait_vm<CNT * (AHEAD - 1)>();
        wg_barrier();
        issue(i);
    }
    static_assert(AHEAD == 5, "drain");
    wait_vm<CNT * 4>(); wg_barrier();
    wait_vm<CNT * 3>(); wg_barrier();
    wait_vm<CNT * 2>(); wg_barrier();
    wait_vm<CNT>(); wg_barrier();
    wait_vm<0>(); wg_barrier();
}

template <int NU>
__device__ __forceinline__ void consume_tile(LAS uint8_t* smem, int c, int lane, v4f (&acc)[4][2]) {
    const int g = lane >> 4, q = (lane & 15) >> 1, p = lane & 1, m = lane & 15;
    const unsigned fw = ((q >> 1) & 3) | ((g & 1) << 2);
    const unsigned w0 = (16 * g + q) * 128 + (((2 * c) ^ fw) << 4) + p * 8;
    const unsigned x0 = 16384 + m * 128 + ((g ^ ((m >> 1) & 7)) << 4);
#pragma unroll
    for (int u = 0; u < 4; ++u) { acc[u][0] = v4f{0, 0, 0, 0}; acc[u][1] = v4f{0, 0, 0, 0}; }
    v8i wf[2][2], xf[2][NU];
    auto load = [&](int s, v8i (&wv)[2], v8i (&xv)[NU]) {
        LAS uint8_t* sl = smem + (s % NS) * SLOT;
#pragma unroll
        for (int cb = 0; cb < 2; ++cb) {
            LAS uint8_t* b = sl + (w0 ^ (cb << 4));
            v2i t0 = __builtin_amdgcn_ds_read_tr8_b64_v2i32((LAS v2i*)(b));
            v2i t1 = __builtin_amdgcn_ds_read_tr8_b64_v2i32((LAS v2i*)(b + 1024));
            v2i t2 = __builtin_amdgcn_ds_read_tr8_b64_v2i32((LAS v2i*)(b + 8192));
            v2i t3 = __builtin_amdgcn_ds_read_tr8_b64_v2i32((LAS v2i*)(b + 9216));
            wv[cb] = v8i{t0[0], t0[1], t1[0], t1[1], t2[0], t2[1], t3[0], t3[1]};
        }
#pragma unroll
        for (int u = 0; u < NU; ++u) {
            v4i a = *(LAS v4i*)(sl + x0 + u * 2048);
            v4i b = *(LAS v4i*)(sl + (x0 ^ 64) + u * 2048);
            xv[u] = v8i{a[0], a[1], a[2], a[3], b[0], b[1], b[2], b[3]};
        }
    };
    auto mma = [&](v8i (&wv)[2], v8i (&xv)[NU]) {
#pragma unroll
        for (int u = 0; u < NU; ++u)
#pragma unroll
            for (int cb = 0; cb < 2; ++cb)
                acc[u][cb] = __builtin_amdgcn_mfma_scale_f32_16x16x128_f8f6f4(wv[cb], xv[u], acc[u][cb], 1, 0, 0, 0, 0, 0);
    };
    auto step = [&](int s, v8i (&wn)[2], v8i (&xn)[NU], v8i (&wo)[2], v8i (&xo)[NU]) {
        wg_barrier();
        __builtin_amdgcn_sched_barrier(0);
        load(s, wn, xn);
        mma(wo, xo);
        constexpr int PER = (8 + 2 * NU + 2 * NU - 1) / (2 * NU);
#pragma unroll
        for (int i = 0; i < 2 * NU; ++i) {
            __builtin_amdgcn_sched_group_barrier(0x100, PER, 0);
            __builtin_amdgcn_sched_group_barrier(0x8, 1, 0);
        }
        __builtin_amdgcn_sched_barrier(0);
        wait_lds();
    };
    wg_barrier();
    load(0, wf[0], xf[0]);
    wait_lds();
    step(1, wf[1], xf[1], wf[0], xf[0]);
#pragma unroll 1
    for (int it = 0; it < (KSTEPS - 2) / (int)NS; ++it) {
#pragma unroll
        for (int s = 2; s < 2 + (int)NS; s += 2) {
            step(s, wf[0], xf[0], wf[1], xf[1]);
            step(s + 1, wf[1], xf[1], wf[0], xf[0]);
        }
    }
    static_assert((KSTEPS - 2) % NS == 0 && NS % 2 == 0, "trip count");
    mma(wf[1], xf[1]);
}

__device__ __forceinline__ void run_tile(LAS uint8_t* smem, int wave, int lane, const uint8_t* wcol, unsigned wbytes,
                                         const uint8_t* xrows, int h, unsigned rot, v4f (&acc)[4][2]) {
    if (wave >= 4) {
        const int w = wave - 4;
        const __amdgpu_buffer_rsrc_t rw = make_rsrc(wcol, wbytes);
        const __amdgpu_buffer_rsrc_t rx = make_rsrc(xrows, h * KD);
        const int nxw = (8 * w < h) + (8 * w + 32 < h);
        if (nxw == 2) load_tile<2>(w, lane, rw, rx, rot);
        else if (nxw == 1) load_tile<1>(w, lane, rw, rx, rot);
        else load_tile<0>(w, lane, rw, rx, rot);
    } else {
        if (h > 48) consume_tile<4>(smem, wave, lane, acc);
        else if (h > 32) consume_tile<3>(smem, wave, lane, acc);
        else if (h > 16) consume_tile<2>(smem, wave, lane, acc);
        else consume_tile<1>(smem, wave, lane, acc);
    }
}

__global__ void __launch_bounds__(NTHREADS) gemm_a2a_kernel(Params P) {
    extern __shared__ __attribute__((aligned(16))) uint8_t smem_[];
    LAS uint8_t* smem = (LAS uint8_t*)smem_;
    const int lane = threadIdx.x & 63;
    const int wave = rfl(threadIdx.x >> 6);
    const int grp = blockIdx.x & 7, idx = blockIdx.x >> 3, panel = idx & 3, rgp = idx >> 2;
    const unsigned rank = (unsigned)P.rank & 7u;
    const bool local = grp == 7;
    const unsigned dest = (rank + 1 + grp) & 7u;
    uint8_t* myws = P.ws[rank];
    uint8_t* pws = P.ws[dest];
    unsigned* ctr = (unsigned*)(myws + WS_CALL + (size_t)blockIdx.x * 128);
    unsigned* tmo = (unsigned*)(myws + WS_TMO);
    STAMP(wave);
    const unsigned call = *(const CAS unsigned*)ctr + 1;
    float scale = *(const CAS float*)P.sx * *(const CAS float*)P.sw;
    const unsigned coff = dest * NC + panel * 128;
    const int g = lane >> 4, m = lane & 15;
    const unsigned src1 = (rank + 1 + rgp) & 7u;
    const int ntiles = local && rgp < 7 ? 2 : 1;

    if (!local && rgp == 7) {
        constexpr int XI = RR * KD / 1024 / 32;
        const unsigned xo = (panel * 8 + wave) * XI * 1024 + lane * 16;
        v4f xv[XI];
#pragma unroll
        for (int i = 0; i < XI; ++i) xv[i] = *(const GAS v4f*)(P.x + xo + i * 1024);
        uint8_t* xd = pws + WS_XST + (size_t)((call & 1) * NR + rank) * XST_SLOT + xo;
#pragma unroll
        for (int i = 0; i < XI; ++i) asm volatile("" : "+v"(xv[i]));
        STAMP(40);
#pragma unroll
        for (int i = 0; i < XI; ++i) {
            if (i >= WIN_X) wait_vm<WIN_X - 1>();
            st16_sys(xd + i * 1024, xv[i]);
        }
        wait_vm<0>();
        if (lane == 0)
            __hip_atomic_store((unsigned*)(pws + WS_XFLAG + rank * 128) + panel * 8 + wave, call, __ATOMIC_RELAXED, __HIP_MEMORY_SCOPE_SYSTEM);
        STAMP(44);
    }

#define ROT 1
    const unsigned rot = ROT == 0 ? 0 : ROT == 1 ? rgp * 4 + panel * 8 + grp : ROT == 2 ? rgp * 4 : ROT == 3 ? (local ? rgp * 4 : (rgp * 32 + 3) / 7) + panel * 8 + grp : rgp * 4 + panel * 8;
    v4f acc[4][2];
    const int h0 = local ? 64 : rgp == 0 ? HG0 : rgp == 1 ? HG1 : 64;
    const int row0 = local ? rgp * 64 : rgp == 0 ? RR : rgp == 1 ? RR + HG0 : RR + (rgp - 1) * 64;
#pragma unroll 1
    for (int t = 0; t < ntiles; ++t) {
        const uint8_t* xrows = P.x + (size_t)row0 * KD;
        if (t == 1) {
            if (wave == 4) {
                const bool ok = wait_flags((const unsigned*)(myws + WS_XFLAG + src1 * 128), call, tmo, lane);
                __builtin_amdgcn_fence(__ATOMIC_ACQUIRE, "");
                wait_vm<0>();
                *(volatile LAS unsigned*)(smem + CTL_OFF) = ok ? 0u : 1u;
                wait_lds();
                STAMP(44);
            }
            wg_barrier();
            if (*(volatile LAS unsigned*)(smem + CTL_OFF)) scale = __builtin_nanf("");
            xrows = myws + WS_XST + (size_t)((call & 1) * NR + src1) * XST_SLOT;
        }
        run_tile(smem, wave, lane, P.w + coff, (unsigned)(KD * ND) - coff, xrows, h0, rot, acc);
        STAMP(16 + 16 * t + wave);
        if (local && wave < 4) {
            const size_t orow = t ? (size_t)src1 * MS : (size_t)rank * MS + row0;
            float* o = P.out[rank] + (orow + m) * NC + panel * 128 + wave * 32 + g * 4;
#pragma unroll
            for (int u = 0; u < 4; ++u) {
                st16_sys(o + (size_t)u * 16 * NC, acc[u][0] * scale);
                st16_sys(o + (size_t)u * 16 * NC + 16, acc[u][1] * scale);
            }
        }
    }

    if (local) {
        if (wave == 0) {
            if (lane == 0) *ctr = call;
            if (panel == 0 && rgp < 7) {
                wait_vm<0>();
                const bool ok = wait_flags((const unsigned*)(myws + WS_DONE + src1 * 128), call, tmo, lane);
                if (!ok && lane == 0)
                    __hip_atomic_store(P.out[rank] + ((size_t)rank * MS + row0) * NC, __builtin_nanf(""), __ATOMIC_RELAXED, __HIP_MEMORY_SCOPE_SYSTEM);
                STAMP(52);
            }
        }
        return;
    }

    if (wave < 4) {
#pragma unroll
        for (int u = 0; u < 4; ++u)
            if (u * 16 < h0) {
                const int r = u * 16 + m;
                LAS uint8_t* d = smem + STG_OFF + r * 512 + wave * 128;
                *(LAS v4f*)(d + ((g ^ (r & 7)) << 4)) = acc[u][0] * scale;
                *(LAS v4f*)(d + (((4 + g) ^ (r & 7)) << 4)) = acc[u][1] * scale;
            }
        wait_lds();
    }
    wg_barrier();
    const int nsw = rgp == 0 ? 8 : rgp == 1 ? NSW_G1 : rgp == 7 ? NSW_LATE : NSW_MAIN;
    if (wave >= nsw) return;
    STAMP(48);
    float* o = P.out[dest] + ((size_t)rank * MS + row0 + (lane >> 5)) * NC + panel * 128 + (lane & 31) * 4;
#pragma unroll 1
    for (int it = wave; it < h0 / 2; it += nsw) {
        const int r = it * 2 + (lane >> 5);
        const v4f v = *(LAS v4f*)(smem + STG_OFF + r * 512 + (((lane & 31) ^ (r & 7)) << 4));
        if (rgp != 0) wait_vm<0>();
        st16_sys(o + (size_t)it * 2 * NC, v);
    }
    wait_vm<0>();
    wg_barrier();
    if (wave == 0 && lane == 0) {
        __hip_atomic_store((unsigned*)(pws + WS_DONE + rank * 128) + idx, call, __ATOMIC_RELAXED, __HIP_MEMORY_SCOPE_SYSTEM);
        *ctr = call;
    }
    STAMP(52);
}

}

extern "C" void kernel_launch(int rank, int nranks, ncclComm_t comm,
                              hipStream_t stream,
                              void* const* d_in, const int* in_sizes, int n_in,
                              void* d_out, int out_size,
                              const int* in_dtypes, int out_dtype,
                              void* const* d_out_all,
                              void* const* d_ws_all, size_t ws_bytes) {
    (void)comm; (void)in_dtypes; (void)out_dtype; (void)d_out;
    if (nranks != NR || (unsigned)rank >= (unsigned)NR || n_in != 4 || in_sizes[0] != MS * KD || in_sizes[1] != KD * ND
        || out_size != NR * MS * NC || ws_bytes < WS_NEED) {
        fprintf(stderr, "kernel_launch: unexpected problem shape\n");
        return;
    }
    static bool attr_set = false;
    if (!attr_set) {
        HIP_CHECK(hipFuncSetAttribute((const void*)gemm_a2a_kernel, hipFuncAttributeMaxDynamicSharedMemorySize, LDS_BYTES));
        attr_set = true;
    }
    Params P;
    P.x = (const uint8_t*)d_in[0];
    P.w = (const uint8_t*)d_in[1];
    P.sx = (const float*)d_in[2];
    P.sw = (const float*)d_in[3];
    for (int p = 0; p < NR; ++p) { P.out[p] = (float*)d_out_all[p]; P.ws[p] = (uint8_t*)d_ws_all[p]; }
    P.rank = rank;
    hipLaunchKernelGGL(gemm_a2a_kernel, dim3(NWG), dim3(NTHREADS), LDS_BYTES, stream, P);
    HIP_CHECK(hipGetLastError());
}
```

```cpp
#include <hip/hip_runtime.h>
#include <rccl/rccl.h>
#include <cstdio>
#include <cstdint>

#define HIP_CHECK(expr) do { hipError_t _e = (expr); if (_e != hipSuccess) { \
    fprintf(stderr, "HIP error %s at %s:%d\n", hipGetErrorString(_e), __FILE__, __LINE__); \
    return; } } while (0)

namespace {

typedef int v2i __attribute__((ext_vector_type(2)));
typedef int v4i __attribute__((ext_vector_type(4)));
typedef unsigned v4u __attribute__((ext_vector_type(4)));
typedef int v8i __attribute__((ext_vector_type(8)));
typedef float v4f __attribute__((ext_vector_type(4)));
#define LAS __attribute__((address_space(3)))
#define GAS __attribute__((address_space(1)))
#define CAS __attribute__((address_space(4)))

constexpr int NR = 8;
constexpr int KD = 4096;
constexpr int ND = 4096;
constexpr int MS = 512;
constexpr int NC = 512;
#define RR 64
constexpr int RA = RR > 64 ? RR / 2 : RR;
constexpr int RB = RR - RA;
#define HG0 24
#define HG1 40
constexpr int HG2 = 128 - HG0 - HG1;
#define HF 64
constexpr int HP = MS - RR - 128 - 4 * HF;
static_assert(RR % 16 == 0 && RR >= 48 && RR <= 128 && HG0 % 2 == 0 && HG1 % 2 == 0 && HF % 2 == 0, "heights");
static_assert(HG0 >= 2 && HG0 <= 64 && HG1 >= 2 && HG1 <= 64 && HG2 >= 2 && HG2 <= 64 && HF <= 64 && HP >= 2 && HP <= 64, "heights");
#define NTE 2
#define NSW_F 4
#define NSW_LATE 8
#define WIN_E 2
constexpr int NTHREADS = 512;
constexpr int NWG = 256;
constexpr int KSTEPS = KD / 128;

constexpr unsigned SLOT = 16384 + 8192, NS = 6;
constexpr unsigned AHEAD = NS - 1;
constexpr unsigned STG_OFF = 2 * SLOT;
static_assert((KSTEPS - 1) % NS == 1, "the last two stages use slots 0 and 1");
constexpr unsigned CTL_OFF = NS * SLOT;
constexpr unsigned JUNK_OFF = CTL_OFF + 64;
constexpr unsigned LDS_BYTES = JUNK_OFF + 1536;

constexpr size_t WS_CALL = 0;
constexpr size_t WS_TMO = 32768;
constexpr size_t WS_DONE = 65536;
constexpr size_t WS_XFLAG = WS_DONE + 8 * 128;
constexpr size_t WS_STAMP = 131072;
constexpr size_t WS_XST = 1u << 20;
constexpr size_t XST_SLOT = (size_t)RR * KD;
constexpr size_t WS_NEED = WS_XST + 2 * NR * XST_SLOT;
constexpr unsigned long long DEADLINE_TICKS = 40000000ull;

struct Params {
    const uint8_t* x;
    const uint8_t* w;
    unsigned rank, pad;
    const float* sx;
    const float* sw;
    float* outd[NR];
    uint8_t* wsd[NR];
};

__device__ __forceinline__ unsigned rfl(unsigned v) { return __builtin_amdgcn_readfirstlane(v); }

__device__ __forceinline__ __amdgpu_buffer_rsrc_t make_rsrc(const void* p, unsigned bytes) {
    const uint64_t a = (uint64_t)p;
    const uint64_t u = ((uint64_t)rfl((unsigned)(a >> 32)) << 32) | rfl((unsigned)a);
    return __builtin_amdgcn_make_buffer_rsrc((void*)u, (short)0, (int)rfl(bytes), 0x00020000);
}

__device__ __forceinline__ void dma16(unsigned lds, unsigned voff, __amdgpu_buffer_rsrc_t rs, unsigned soff) {
    asm volatile("s_mov_b32 m0, %0\n\ts_nop 0\n\tbuffer_load_dwordx4 %1, %2, %3 offen lds"
                 :: "s"(lds), "v"(voff), "s"(rs), "s"(soff) : "memory");
}
__device__ __forceinline__ void dma16x4(unsigned lds, unsigned v0, unsigned v1, unsigned v2, unsigned v3,
                                        __amdgpu_buffer_rsrc_t rs, unsigned soff) {
    asm volatile("s_mov_b32 m0, %0\n\ts_nop 0\n\t"
                 "buffer_load_dwordx4 %1, %5, %6 offen lds\n\t"
                 "buffer_load_dwordx4 %2, %5, %6 offen offset:1024 lds\n\t"
                 "buffer_load_dwordx4 %3, %5, %6 offen offset:2048 lds\n\t"
                 "buffer_load_dwordx4 %4, %5, %6 offen offset:3072 lds"
                 :: "s"(lds), "v"(v0), "v"(v1), "v"(v2), "v"(v3), "s"(rs), "s"(soff) : "memory");
}
__device__ __forceinline__ void dma16x4_nt(unsigned lds, unsigned v0, unsigned v1, unsigned v2, unsigned v3,
                                           __amdgpu_buffer_rsrc_t rs, unsigned soff) {
    asm volatile("s_mov_b32 m0, %0\n\ts_nop 0\n\t"
                 "buffer_load_dwordx4 %1, %5, %6 offen nt lds\n\t"
                 "buffer_load_dwordx4 %2, %5, %6 offen offset:1024 nt lds\n\t"
                 "buffer_load_dwordx4 %3, %5, %6 offen offset:2048 nt lds\n\t"
                 "buffer_load_dwordx4 %4, %5, %6 offen offset:3072 nt lds"
                 :: "s"(lds), "v"(v0), "v"(v1), "v"(v2), "v"(v3), "s"(rs), "s"(soff) : "memory");
}
__device__ __forceinline__ void dma16_nt(unsigned lds, unsigned voff, __amdgpu_buffer_rsrc_t rs, unsigned soff) {
    asm volatile("s_mov_b32 m0, %0\n\ts_nop 0\n\tbuffer_load_dwordx4 %1, %2, %3 offen nt lds"
                 :: "s"(lds), "v"(voff), "s"(rs), "s"(soff) : "memory");
}
__device__ __forceinline__ void pf4_nt(unsigned lds, unsigned v0, __amdgpu_buffer_rsrc_t rs, unsigned soff) {
    asm volatile("s_mov_b32 m0, %0\n\ts_nop 0\n\t"
                 "buffer_load_dwordx4 %1, %2, %3 offen nt lds\n\t"
                 "buffer_load_dwordx4 %1, %2, %3 offen offset:128 nt lds\n\t"
                 "buffer_load_dwordx4 %1, %2, %3 offen offset:256 nt lds\n\t"
                 "buffer_load_dwordx4 %1, %2, %3 offen offset:384 nt lds"
                 :: "s"(lds), "v"(v0), "s"(rs), "s"(soff) : "memory");
}
template <int N> __device__ __forceinline__ void wait_vm() { asm volatile("s_waitcnt vmcnt(%0)" :: "i"(N) : "memory"); }
__device__ __forceinline__ void wg_barrier() { asm volatile("s_barrier" ::: "memory"); }
__device__ __forceinline__ void wait_lds() { asm volatile("s_waitcnt lgkmcnt(0)" ::: "memory"); }
__device__ __forceinline__ void st16_sys(void* p, v4f v) {
    asm volatile("global_store_dwordx4 %0, %1, off sc0 sc1\n\ts_nop 1" :: "v"(p), "v"(v) : "memory");
}

#define STAMP(i) do {} while (0)

__device__ __forceinline__ bool wait_flags(const GAS unsigned* flags, unsigned call, GAS unsigned* tmo, int lane) {
    const unsigned long long t0 = __builtin_amdgcn_s_memrealtime();
    for (;;) {
        const bool ok = __hip_atomic_load(flags + (lane & 31), __ATOMIC_RELAXED, __HIP_MEMORY_SCOPE_SYSTEM) >= call;
        if (__all(ok)) break;
        if (rfl(__hip_atomic_load(tmo, __ATOMIC_RELAXED, __HIP_MEMORY_SCOPE_AGENT)) == call
            || __builtin_amdgcn_s_memrealtime() - t0 > DEADLINE_TICKS) {
            if (lane == 0) __hip_atomic_store(tmo, call, __ATOMIC_RELAXED, __HIP_MEMORY_SCOPE_AGENT);
            return false;
        }
        __builtin_amdgcn_s_sleep(2);
    }
    return rfl(__hip_atomic_load(tmo, __ATOMIC_RELAXED, __HIP_MEMORY_SCOPE_AGENT)) != call;
}

template <int NXW>
__device__ __forceinline__ void load_tile(int w, int lane, __amdgpu_buffer_rsrc_t rw, __amdgpu_buffer_rsrc_t rx,
                                          int fr, unsigned nnt, __amdgpu_buffer_rsrc_t rp, unsigned vp, int npf,
                                          unsigned long long* ts = nullptr) {
    constexpr int CNT = 4 + NXW;
    const unsigned v0 = (lane >> 3) * KD + (((lane & 7) ^ ((lane >> 4) & 3)) << 4);
    unsigned vw[4];
#pragma unroll
    for (int j = 0; j < 4; ++j) vw[j] = (v0 ^ (((j >> 1) & 1) << 6)) + (4 * w + j) * (8u * ND) - j * 1024u;
    const unsigned vx0 = (v0 ^ ((w & 1) << 6)) + w * (8u * KD), vx1 = vx0 + 32u * KD;
    unsigned ioff = w * 1024;
    unsigned ks = fr < 0 ? 0 : fr;
    auto issue = [&](unsigned i) {
        if (i < nnt) {
            if (npf > 0) dma16(JUNK_OFF, vp, rp, (4 * i + w) * 128u);
            if (npf > 1) dma16(JUNK_OFF, vp, rp, (4 * i + w) * 128u + 128u * KD);
            dma16x4_nt(ioff + w * 3072, vw[0], vw[1], vw[2], vw[3], rw, ks * (128u * ND));
        } else {
            dma16x4(ioff + w * 3072, vw[0], vw[1], vw[2], vw[3], rw, ks * (128u * ND));
        }
        if (NXW > 0) dma16(ioff + 16384, vx0, rx, ks * 128u);
        if (NXW > 1) dma16(ioff + 20480, vx1, rx, ks * 128u);
        ioff += SLOT;
        if (ioff >= SLOT * NS) ioff -= SLOT * NS;
        if (fr >= 0 && i < 8) ks = i < 7 ? ks + 4 : (fr == 0 ? 1 : 0);
        else { ++ks; if ((int)(ks & 3) == fr) ++ks; }
    };
    asm volatile("s_nop 4" : "+s"(rx), "+s"(rw), "+s"(rp) :: "memory");
#pragma unroll 1
    for (unsigned i = 0; i < AHEAD; ++i) issue(i);
    unsigned i = AHEAD;
#pragma unroll 1
    for (; i < nnt; ++i) {
        wait_vm<CNT * (AHEAD - 1)>();
        wg_barrier();
        issue(i);
    }
#pragma unroll 1
    for (; i < KSTEPS; ++i) {
        const unsigned mw = ioff + w * 3072, mx = ioff + 16384, mx1 = ioff + 20480;
        const unsigned sow = ks * (128u * ND), sox = ks * 128u;
#define STEP_HEAD "s_mov_b32 m0, %0\n\ts_waitcnt vmcnt(%13)\n\ts_barrier\n\t" \
                  "buffer_load_dwordx4 %3, %9, %11 offen lds\n\t" \
                  "buffer_load_dwordx4 %4, %9, %11 offen offset:1024 lds\n\t" \
                  "buffer_load_dwordx4 %5, %9, %11 offen offset:2048 lds\n\t" \
                  "buffer_load_dwordx4 %6, %9, %11 offen offset:3072 lds"
#define STEP_X0 "\n\ts_mov_b32 m0, %1\n\ts_nop 0\n\tbuffer_load_dwordx4 %7, %10, %12 offen lds"
#define STEP_X1 "\n\ts_mov_b32 m0, %2\n\ts_nop 0\n\tbuffer_load_dwordx4 %8, %10, %12 offen lds"
#define STEP_OPS :: "s"(mw), "s"(mx), "s"(mx1), "v"(vw[0]), "v"(vw[1]), "v"(vw[2]), "v"(vw[3]), "v"(vx0), "v"(vx1), \
                    "s"(rw), "s"(rx), "s"(sow), "s"(sox), "i"(CNT * (AHEAD - 1)) : "memory"
        if (NXW == 0) asm volatile(STEP_HEAD STEP_OPS);
        if (NXW == 1) asm volatile(STEP_HEAD STEP_X0 STEP_OPS);
        if (NXW == 2) asm volatile(STEP_HEAD STEP_X0 STEP_X1 STEP_OPS);
        ioff += SLOT;
        if (ioff >= SLOT * NS) ioff -= SLOT * NS;
        ++ks;
        if ((int)(ks & 3) == fr) ++ks;
    }
    static_assert(AHEAD == 5, "drain");
    wait_vm<CNT * 4>(); wg_barrier();
    wait_vm<CNT * 3>(); wg_barrier();
    wait_vm<CNT * 2>(); wg_barrier();
    wait_vm<CNT>(); wg_barrier();
    wait_vm<0>(); wg_barrier();
}

template <int NU>
__device__ __forceinline__ void consume_tile(LAS uint8_t* smem, int c, int lane, v4f (&acc)[4][2]) {
    const int g = lane >> 4, q = (lane & 15) >> 1, p = lane & 1, m = lane & 15;
    const unsigned fw = ((q >> 1) & 3) | ((g & 1) << 2);
    const unsigned w0 = (16 * g + q) * 128 + (((2 * c) ^ fw) << 4) + p * 8;
    const unsigned x0 = 16384 + m * 128 + ((g ^ ((m >> 1) & 7)) << 4);
#pragma unroll
    for (int u = 0; u < 4; ++u) { acc[u][0] = v4f{0, 0, 0, 0}; acc[u][1] = v4f{0, 0, 0, 0}; }
    v8i wf[2][2], xf[2][NU];
    auto load = [&](int s, v8i (&wv)[2], v8i (&xv)[NU]) {
        LAS uint8_t* sl = smem + (s % NS) * SLOT;
#pragma unroll
        for (int cb = 0; cb < 2; ++cb) {
            LAS uint8_t* b = sl + (w0 ^ (cb << 4));
            v2i t0 = __builtin_amdgcn_ds_read_tr8_b64_v2i32((LAS v2i*)(b));
            v2i t1 = __builtin_amdgcn_ds_read_tr8_b64_v2i32((LAS v2i*)(b + 1024));
            v2i t2 = __builtin_amdgcn_ds_read_tr8_b64_v2i32((LAS v2i*)(b + 8192));
            v2i t3 = __builtin_amdgcn_ds_read_tr8_b64_v2i32((LAS v2i*)(b + 9216));
            wv[cb] = v8i{t0[0], t0[1], t1[0], t1[1], t2[0], t2[1], t3[0], t3[1]};
        }
#pragma unroll
        for (int u = 0; u < NU; ++u) {
            v4i a = *(LAS v4i*)(sl + x0 + u * 2048);
            v4i b = *(LAS v4i*)(sl + (x0 ^ 64) + u * 2048);
            xv[u] = v8i{a[0], a[1], a[2], a[3], b[0], b[1], b[2], b[3]};
        }
    };
    auto mma = [&](v8i (&wv)[2], v8i (&xv)[NU]) {
#pragma unroll
        for (int u = 0; u < NU; ++u)
#pragma unroll
            for (int cb = 0; cb < 2; ++cb)
                acc[u][cb] = __builtin_amdgcn_mfma_scale_f32_16x16x128_f8f6f4(wv[cb], xv[u], acc[u][cb], 1, 0, 0, 0, 0, 0);
    };
    auto step = [&](int s, v8i (&wn)[2], v8i (&xn)[NU], v8i (&wo)[2], v8i (&xo)[NU]) {
        wg_barrier();
        __builtin_amdgcn_sched_barrier(0);
        load(s, wn, xn);
        mma(wo, xo);
        constexpr int PER = (8 + 2 * NU + 2 * NU - 1) / (2 * NU);
#pragma unroll
        for (int i = 0; i < 2 * NU; ++i) {
            __builtin_amdgcn_sched_group_barrier(0x100, PER, 0);
            __builtin_amdgcn_sched_group_barrier(0x8, 1, 0);
        }
        __builtin_amdgcn_sched_barrier(0);
        wait_lds();
    };
    wg_barrier();
    load(0, wf[0], xf[0]);
    wait_lds();
    step(1, wf[1], xf[1], wf[0], xf[0]);
#pragma unroll 1
    for (int it = 0; it < (KSTEPS - 2) / (int)NS; ++it) {
#pragma unroll
        for (int s = 2; s < 2 + (int)NS; s += 2) {
            step(s, wf[0], xf[0], wf[1], xf[1]);
            step(s + 1, wf[1], xf[1], wf[0], xf[0]);
        }
    }
    static_assert((KSTEPS - 2) % NS == 0 && NS % 2 == 0, "trip count");
    mma(wf[1], xf[1]);
}

__global__ void __launch_bounds__(NTHREADS) gemm_a2a_kernel(Params P) {
    extern __shared__ __attribute__((aligned(16))) uint8_t smem_[];
    LAS uint8_t* smem = (LAS uint8_t*)smem_;
    const int lane = threadIdx.x & 63;
    const int wave = rfl(threadIdx.x >> 6);
    const int grp = blockIdx.x & 7, idx = blockIdx.x >> 3, panel = idx & 3, rgp = idx >> 2;
    const uint8_t* px = P.x;
    const uint8_t* pw = P.w;
    unsigned rank = P.rank;
    uint8_t* myws = P.wsd[7];
    uint8_t* pws = P.wsd[grp];
    float* pout = P.outd[grp];
    asm volatile("" : "+s"(px), "+s"(pw), "+s"(rank), "+s"(myws), "+s"(pws), "+s"(pout));
    rank &= 7u;
    const bool local = grp == 7;
    const int h0 = local ? 64 : rgp == 0 ? HG0 : rgp == 1 ? HG1 : rgp == 2 ? HG2 : rgp == 7 ? HP : HF;
    const int row0 = local ? rgp * 64 : rgp == 0 ? RR : rgp == 1 ? RR + HG0 : rgp == 2 ? RR + HG0 + HG1 : RR + 128 + (rgp - 3) * HF;
#define CTR ((GAS unsigned*)(myws + WS_CALL + (size_t)blockIdx.x * 128))
#define TMO ((GAS unsigned*)(myws + WS_TMO))
#define LOAD_CALL() do { call = *(const CAS unsigned*)(uintptr_t)CTR + 1; asm volatile("" : "+s"(call)); } while (0)
    unsigned call = 0;

    if (__builtin_expect(!local && rgp == 7, 0)) {
        constexpr int XI = RR / 8;
        const unsigned xo = (panel * 8 + wave) * XI * 1024;
        const unsigned lb = wave * XI * 1024;
        __amdgpu_buffer_rsrc_t rp = make_rsrc(px, RR * KD);
        asm volatile("s_nop 4" : "+s"(rp) :: "memory");
        dma16(lb, lane * 16, rp, xo);
        dma16(lb + 1024, lane * 16, rp, xo + 1024);
        wg_barrier();
        dma16(lb + 2048, lane * 16, rp, xo + 2048);
        dma16(lb + 3072, lane * 16, rp, xo + 3072);
        LOAD_CALL();
        uint8_t* xd = pws + WS_XST + (size_t)(rank * 2 + (call & 1)) * XST_SLOT + xo + lane * 16;
        LAS uint8_t* ls = smem + lb + lane * 16;
        wait_vm<3>();
        v4f v = *(LAS v4f*)ls;
#pragma unroll
        for (int i = 0; i < XI; ++i) {
            if (i >= 2) {
                if (i + 3 < XI) wait_vm<3>(); else if (i + 2 < XI) wait_vm<2>(); else wait_vm<1>();
            }
            st16_sys(xd + i * 1024, v);
            if (i + 4 < XI) dma16(lb + (i + 4) * 1024, lane * 16, rp, xo + (i + 4) * 1024);
            if (i == 0) wait_vm<4>();
            if (i == 1) wait_vm<5>();
            if (i + 1 < XI) v = *(LAS v4f*)(ls + (i + 1) * 1024);
        }
        static_assert(XI >= 6 && XI <= 16, "wait counts");
        wait_vm<0>();
        if (lane == 0)
            __hip_atomic_store((GAS unsigned*)(pws + WS_XFLAG + rank * 128) + panel * 8 + wave, call, __ATOMIC_RELAXED, __HIP_MEMORY_SCOPE_SYSTEM);
        STAMP(44);
        wg_barrier();
    }

    float scale = 0.f;
    const int g = lane >> 4, m = lane & 15;
    if (wave >= 4) {
        const int w = wave - 4;
        const unsigned coff = ((rank + 1 + grp) & 7u) * NC + panel * 128;
        const __amdgpu_buffer_rsrc_t rw = make_rsrc(pw + coff, (unsigned)(KD * ND) - coff);
        const __amdgpu_buffer_rsrc_t rx = make_rsrc(px + (size_t)row0 * KD, h0 * KD);
        const int f = rgp - (local ? 4 : 3);
        const bool fetch = (unsigned)f < 4u;
        const int fr = fetch ? (f + panel) & 3 : -1;
        const unsigned nnt = fetch ? 8u : f < 0 ? (unsigned)NTE : 0u;
        const __amdgpu_buffer_rsrc_t rp = make_rsrc(px, (unsigned)(MS * KD));
        const unsigned vp = ((local ? 0 : RR) + (f * 4 + panel) * 8 + (lane >> 3)) * KD + (lane & 7) * 16;
        const int npf = fetch ? (local ? 2 : 1) : 0;
        const int nxw = (8 * w < h0) + (8 * w + 32 < h0);
        unsigned long long ts[3];
        if (nxw == 2) load_tile<2>(w, lane, rw, rx, fr, nnt, rp, vp, npf, ts);
        else if (nxw == 1) load_tile<1>(w, lane, rw, rx, fr, nnt, rp, vp, npf, ts);
        else load_tile<0>(w, lane, rw, rx, fr, nnt, rp, vp, npf, ts);
        LOAD_CALL();
    } else {
        v4f acc[4][2];
        float sa = *(const CAS float*)P.sx, sb = *(const CAS float*)P.sw;
        LOAD_CALL();
        asm volatile("" : "+s"(sa), "+s"(sb));
        scale = sa * sb;
        STAMP(60 + (wave & 1));
        if (h0 > 48) consume_tile<4>(smem, wave, lane, acc);
        else if (h0 > 32) consume_tile<3>(smem, wave, lane, acc);
        else if (h0 > 16) consume_tile<2>(smem, wave, lane, acc);
        else consume_tile<1>(smem, wave, lane, acc);
        STAMP(16 + wave);
        if (!local) {
#pragma unroll
            for (int u = 0; u < 4; ++u)
                if (u * 16 < h0) {
                    const int r = u * 16 + m;
                    LAS uint8_t* d = smem + STG_OFF + r * 512 + wave * 128;
                    *(LAS v4f*)(d + ((g ^ (r & 7)) << 4)) = acc[u][0] * scale;
                    *(LAS v4f*)(d + (((4 + g) ^ (r & 7)) << 4)) = acc[u][1] * scale;
                }
            wait_lds();
        } else {
            float* o = pout + ((size_t)rank * MS + row0 + m) * NC + panel * 128 + wave * 32 + g * 4;
#pragma unroll
            for (int u = 0; u < 4; ++u) {
                st16_sys(o + (size_t)u * 16 * NC, acc[u][0] * scale);
                st16_sys(o + (size_t)u * 16 * NC + 16, acc[u][1] * scale);
            }
        }
    }

    if (!local) {
        float* o = pout + ((size_t)rank * MS + row0 + (lane >> 5)) * NC + panel * 128 + (lane & 31) * 4;
        wg_barrier();
        const int nsw = rgp < 3 ? 8 : rgp == 7 ? NSW_LATE : NSW_F;
        if (wave >= nsw) return;
        STAMP(48);
#pragma unroll 1
        for (int it = wave; it < h0 / 2; it += nsw) {
            const int r = it * 2 + (lane >> 5);
            const v4f v = *(LAS v4f*)(smem + STG_OFF + r * 512 + (((lane & 31) ^ (r & 7)) << 4));
            if (rgp < 3) wait_vm<WIN_E - 1>(); else wait_vm<0>();
            st16_sys(o + (size_t)it * 2 * NC, v);
        }
        wait_vm<0>();
        wg_barrier();
        if (wave == 0 && lane == 0) {
            __hip_atomic_store((GAS unsigned*)(pws + WS_DONE + rank * 128) + idx, call, __ATOMIC_RELAXED, __HIP_MEMORY_SCOPE_SYSTEM);
            *CTR = call;
        }
        STAMP(52);
        if (RB == 0 || rgp != 0) return;
    } else if (rgp == 7) {
        wait_vm<0>();
        wg_barrier();
        if (wave == 7 && lane == 0) *CTR = call;
        if (panel == 0 && wave < 7) {
            const unsigned src = (rank + 1 + wave) & 7u;
            const bool ok = wait_flags((const GAS unsigned*)(myws + WS_DONE + src * 128), call, TMO, lane);
            if (!ok && lane == 0)
                __hip_atomic_store((GAS float*)pout + ((size_t)rank * MS + row0) * NC, __builtin_nanf(""), __ATOMIC_RELAXED, __HIP_MEMORY_SCOPE_SYSTEM);
            STAMP(45 + wave);
        }
        return;
    }

    const unsigned src = (rank + 1 + (local ? rgp : grp)) & 7u;
    const int r2 = local ? 0 : RA;
    if (wave == 4) {
        const bool ok = wait_flags((const GAS unsigned*)(myws + WS_XFLAG + src * 128), call, TMO, lane);
        __builtin_amdgcn_fence(__ATOMIC_ACQUIRE, "");
        wait_vm<0>();
        *(volatile LAS unsigned*)(smem + CTL_OFF) = ok ? 0u : 1u;
        wait_lds();
        STAMP(44);
    }
    wg_barrier();
    if (wave >= 4) {
        const int w = wave - 4;
        const unsigned coff = rank * NC + panel * 128;
        const __amdgpu_buffer_rsrc_t rw = make_rsrc(pw + coff, (unsigned)(KD * ND) - coff);
        const __amdgpu_buffer_rsrc_t rx = make_rsrc(myws + WS_XST + (size_t)(src * 2 + (call & 1)) * XST_SLOT + (size_t)r2 * KD, RA * KD);
        if (8 * w + 32 < RA) load_tile<2>(w, lane, rw, rx, -1, 0, rx, 0, 0);
        else load_tile<1>(w, lane, rw, rx, -1, 0, rx, 0, 0);
    } else {
        if (local && wave == 0 && lane == 0) *CTR = call;
        v4f acc[4][2];
        consume_tile<(RA + 15) / 16>(smem, wave, lane, acc);
        STAMP(32 + wave);
        if (*(volatile LAS unsigned*)(smem + CTL_OFF)) scale = __builtin_nanf("");
        float* o = P.outd[7] + ((size_t)src * MS + r2 + m) * NC + panel * 128 + wave * 32 + g * 4;
#pragma unroll
        for (int u = 0; u < (RA + 15) / 16; ++u)
            if (u * 16 + m < RA) {
                st16_sys(o + (size_t)u * 16 * NC, acc[u][0] * scale);
                st16_sys(o + (size_t)u * 16 * NC + 16, acc[u][1] * scale);
            }
    }
}

}

extern "C" void kernel_launch(int rank, int nranks, ncclComm_t comm,
                              hipStream_t stream,
                              void* const* d_in, const int* in_sizes, int n_in,
                              void* d_out, int out_size,
                              const int* in_dtypes, int out_dtype,
                              void* const* d_out_all,
                              void* const* d_ws_all, size_t ws_bytes) {
    (void)comm; (void)in_dtypes; (void)out_dtype; (void)d_out;
    if (nranks != NR || (unsigned)rank >= (unsigned)NR || n_in != 4 || in_sizes[0] != MS * KD || in_sizes[1] != KD * ND
        || out_size != NR * MS * NC || ws_bytes < WS_NEED) {
        fprintf(stderr, "kernel_launch: unexpected problem shape\n");
        return;
    }
    static bool attr_set = false;
    if (!attr_set) {
        HIP_CHECK(hipFuncSetAttribute((const void*)gemm_a2a_kernel, hipFuncAttributeMaxDynamicSharedMemorySize, LDS_BYTES));
        attr_set = true;
    }
    Params P;
    P.x = (const uint8_t*)d_in[0];
    P.w = (const uint8_t*)d_in[1];
    P.sx = (const float*)d_in[2];
    P.sw = (const float*)d_in[3];
    for (int g = 0; g < NR; ++g) {
        const int p = (rank + 1 + g) % NR;
        P.outd[g] = (float*)d_out_all[p];
        P.wsd[g] = (uint8_t*)d_ws_all[p];
    }
    P.rank = (unsigned)rank;
    P.pad = 0;
    hipLaunchKernelGGL(gemm_a2a_kernel, dim3(NWG), dim3(NTHREADS), LDS_BYTES, stream, P);
    HIP_CHECK(hipGetLastError());
}
```

```cpp
#include <hip/hip_runtime.h>
#include <rccl/rccl.h>
#include <cstdio>
#include <cstdint>

#define HIP_CHECK(expr) do { hipError_t _e = (expr); if (_e != hipSuccess) { \
    fprintf(stderr, "HIP error %s at %s:%d\n", hipGetErrorString(_e), __FILE__, __LINE__); \
    return; } } while (0)

namespace {

typedef int v2i __attribute__((ext_vector_type(2)));
typedef int v4i __attribute__((ext_vector_type(4)));
typedef int v8i __attribute__((ext_vector_type(8)));
typedef float v4f __attribute__((ext_vector_type(4)));
typedef unsigned long long u64;
#define LAS __attribute__((address_space(3)))
#define GAS __attribute__((address_space(1)))
#define CAS __attribute__((address_space(4)))

constexpr int NR = 8;
constexpr int KD = 4096;
constexpr int ND = 4096;
constexpr int MS = 512;
constexpr int NC = 512;
#define RR 112
constexpr int RA = RR / 2;
#define NF 4
constexpr int NE = 7 - NF;
#define HG0 24
#define HG1 56
#define HG2 64
constexpr int ESUM = (NE > 0 ? HG0 : 0) + (NE > 1 ? HG1 : 0) + (NE > 2 ? (NE - 2) * HG2 : 0);
constexpr int FSUM = MS - RR - ESUM;
constexpr int HF = ((FSUM + NF - 1) / NF + 1) & ~1;
constexpr int HFL = FSUM - (NF - 1) * HF;
static_assert(RR % 8 == 0 && RA % 2 == 0 && RA > 32 && RA <= 64 && NF >= 2 && NF <= 7 && ESUM <= 384, "shape");
static_assert(HG0 % 2 == 0 && HG1 % 2 == 0 && HG2 % 2 == 0 && HG0 <= 64 && HG1 <= 64 && HG2 <= 64 && HF <= 64 && HFL >= 2 && ESUM % 8 == 0, "heights");
#define NFL 4
#define SLOWNF 0
#define NOXPF 0
#define TAU 112
#define LEAD 35
#define XLEAD 70
#define FENCE 0
constexpr int NTHREADS = 512;
constexpr int NWG = 256;
constexpr int KSTEPS = KD / 128;

constexpr unsigned SLOT = 16384 + 8192, NS = 6;
constexpr unsigned AHEAD = NS - 1;
constexpr unsigned STG_OFF = 2 * SLOT;
static_assert((KSTEPS - 1) % NS == 1, "the last two stages use slots 0 and 1");
constexpr unsigned CTL_OFF = NS * SLOT;
constexpr unsigned JUNK_OFF = CTL_OFF + 64;
constexpr unsigned LDS_BYTES = JUNK_OFF + 1024;

constexpr size_t WS_CALL = 0;
constexpr size_t WS_TMO = 32768;
constexpr size_t WS_PACE = 49152;
constexpr size_t WS_DONE = 65536;
constexpr size_t WS_XFLAG = WS_DONE + 8 * 128;
constexpr size_t WS_STAMP = 131072;
constexpr size_t WS_XST = 1u << 20;
constexpr size_t XST_SLOT = (size_t)RR * KD;
constexpr size_t WS_NEED = WS_XST + 2 * NR * XST_SLOT;
constexpr u64 DEADLINE_TICKS = 40000000ull;

struct Params {
    const uint8_t* x;
    const uint8_t* w;
    unsigned rank, pad;
    const float* sx;
    const float* sw;
    float* outd[NR];
    uint8_t* wsd[NR];
};

constexpr unsigned comb(int n) { unsigned m = 0; for (int i = 0; i < 32; i += n) m |= 1u << i; return m; }

__device__ __forceinline__ unsigned rfl(unsigned v) { return __builtin_amdgcn_readfirstlane(v); }
__device__ __forceinline__ u64 rtime() { return __builtin_amdgcn_s_memrealtime(); }

__device__ __forceinline__ __amdgpu_buffer_rsrc_t make_rsrc(const void* p, unsigned bytes) {
    const uint64_t a = (uint64_t)p;
    const uint64_t u = ((uint64_t)rfl((unsigned)(a >> 32)) << 32) | rfl((unsigned)a);
    return __builtin_amdgcn_make_buffer_rsrc((void*)u, (short)0, (int)rfl(bytes), 0x00020000);
}

template <bool SYS = false>
__device__ __forceinline__ void dma16(unsigned lds, unsigned voff, __amdgpu_buffer_rsrc_t rs, unsigned soff) {
    if (SYS) asm volatile("s_mov_b32 m0, %0\n\ts_nop 0\n\tbuffer_load_dwordx4 %1, %2, %3 offen sc0 sc1 lds"
                          :: "s"(lds), "v"(voff), "s"(rs), "s"(soff) : "memory");
    else     asm volatile("s_mov_b32 m0, %0\n\ts_nop 0\n\tbuffer_load_dwordx4 %1, %2, %3 offen lds"
                          :: "s"(lds), "v"(voff), "s"(rs), "s"(soff) : "memory");
}
__device__ __forceinline__ void dma16x4(unsigned lds, unsigned v0, unsigned v1, unsigned v2, unsigned v3,
                                        __amdgpu_buffer_rsrc_t rs, unsigned soff) {
    asm volatile("s_mov_b32 m0, %0\n\ts_nop 0\n\t"
                 "buffer_load_dwordx4 %1, %5, %6 offen lds\n\t"
                 "buffer_load_dwordx4 %2, %5, %6 offen offset:1024 lds\n\t"
                 "buffer_load_dwordx4 %3, %5, %6 offen offset:2048 lds\n\t"
                 "buffer_load_dwordx4 %4, %5, %6 offen offset:3072 lds"
                 :: "s"(lds), "v"(v0), "v"(v1), "v"(v2), "v"(v3), "s"(rs), "s"(soff) : "memory");
}
template <int N> __device__ __forceinline__ void wait_vm() { asm volatile("s_waitcnt vmcnt(%0)" :: "i"(N) : "memory"); }
__device__ __forceinline__ void wg_barrier() { asm volatile("s_barrier" ::: "memory"); }
__device__ __forceinline__ void wait_lds() { asm volatile("s_waitcnt lgkmcnt(0)" ::: "memory"); }
__device__ __forceinline__ void st16_sys(void* p, v4f v) {
    asm volatile("global_store_dwordx4 %0, %1, off sc0 sc1\n\ts_nop 1" :: "v"(p), "v"(v) : "memory");
}

#define STAMP(i) do {} while (0)
#define LOGT(k) do {} while (0)
#define LOGOUT(half) do {} while (0)

__device__ __forceinline__ bool wait_flags(const GAS unsigned* flags, unsigned ignore, unsigned call, GAS unsigned* tmo, int lane) {
    const GAS unsigned* p = lane < 32 ? flags + lane : tmo;
    const u64 t0 = rtime();
    for (;;) {
        const u64 now = rtime();
        const unsigned v = __hip_atomic_load(p, __ATOMIC_RELAXED, __HIP_MEMORY_SCOPE_SYSTEM);
        const u64 ge = __ballot(v >= call), eq = __ballot(v == call);
        if ((eq >> 32) || now - t0 > DEADLINE_TICKS) break;
        if (((unsigned)ge | ignore) == 0xffffffffu) return true;
    }
    if (lane == 0) __hip_atomic_store(tmo, call, __ATOMIC_RELAXED, __HIP_MEMORY_SCOPE_AGENT);
    return false;
}

template <int NXW, bool SYS>
__device__ __forceinline__ void load_tile(int w, int lane, __amdgpu_buffer_rsrc_t rw, __amdgpu_buffer_rsrc_t rx,
                                          unsigned first, __amdgpu_buffer_rsrc_t rp, unsigned vp, int npf, unsigned pstep,
                                          u64* ts = nullptr) {
    constexpr int CNT = 4 + NXW;
    const unsigned v0 = (lane >> 3) * KD + (((lane & 7) ^ ((lane >> 4) & 3)) << 4);
    unsigned vw[4];
#pragma unroll
    for (int j = 0; j < 4; ++j) vw[j] = (v0 ^ (((j >> 1) & 1) << 6)) + (4 * w + j) * (8u * ND) - j * 1024u;
    const unsigned vx0 = (v0 ^ ((w & 1) << 6)) + w * (8u * KD), vx1 = vx0 + 32u * KD;
    unsigned ioff = w * 1024;
    u64 todo = ((u64)~first << 32) | first;
    unsigned poff = 0;
    auto issue = [&]() {
        const unsigned ks = __builtin_ctzll(todo) & 31;
        todo &= todo - 1;
        if (poff < (unsigned)KD) {
            if (npf > 0) dma16(JUNK_OFF, vp, rp, poff);
            if (npf > 1) dma16(JUNK_OFF, vp, rp, poff + 128u * KD);
            if (npf > 2) dma16(JUNK_OFF, vp, rp, poff + 256u * KD);
            poff += pstep;
        }
        dma16x4(ioff + w * 3072, vw[0], vw[1], vw[2], vw[3], rw, ks * (128u * ND));
        if (NXW > 0) dma16<SYS>(ioff + 16384, vx0, rx, ks * 128u);
        if (NXW > 1) dma16<SYS>(ioff + 20480, vx1, rx, ks * 128u);
        ioff += SLOT;
        if (ioff >= SLOT * NS) ioff -= SLOT * NS;
    };
    asm volatile("s_nop 4" : "+s"(rx), "+s"(rw), "+s"(rp) :: "memory");
#pragma unroll 1
    for (unsigned i = 0; i < AHEAD; ++i) issue();
#pragma unroll 1
    while (npf > 0 && poff < (unsigned)KD) {
        wait_vm<CNT * (AHEAD - 1)>();
        wg_barrier();
        issue();
    }
#pragma unroll 1
    while (todo) {
        const unsigned ks = __builtin_ctzll(todo) & 31;
        todo &= todo - 1;
        const unsigned mw = ioff + w * 3072, mx = ioff + 16384, mx1 = ioff + 20480;
        const unsigned sow = ks * (128u * ND), sox = ks * 128u;
#define STEP_HEAD "s_mov_b32 m0, %0\n\ts_waitcnt vmcnt(%13)\n\ts_barrier\n\t" \
                  "buffer_load_dwordx4 %3, %9, %11 offen lds\n\t" \
                  "buffer_load_dwordx4 %4, %9, %11 offen offset:1024 lds\n\t" \
                  "buffer_load_dwordx4 %5, %9, %11 offen offset:2048 lds\n\t" \
                  "buffer_load_dwordx4 %6, %9, %11 offen offset:3072 lds"
#define STEP_X0(pol) "\n\ts_mov_b32 m0, %1\n\ts_nop 0\n\tbuffer_load_dwordx4 %7, %10, %12 offen" pol " lds"
#define STEP_X1(pol) "\n\ts_mov_b32 m0, %2\n\ts_nop 0\n\tbuffer_load_dwordx4 %8, %10, %12 offen" pol " lds"
#define STEP_OPS :: "s"(mw), "s"(mx), "s"(mx1), "v"(vw[0]), "v"(vw[1]), "v"(vw[2]), "v"(vw[3]), "v"(vx0), "v"(vx1), \
                    "s"(rw), "s"(rx), "s"(sow), "s"(sox), "i"(CNT * (AHEAD - 1)) : "memory"
        if (NXW == 0) asm volatile(STEP_HEAD STEP_OPS);
        if (NXW == 1 && !SYS) asm volatile(STEP_HEAD STEP_X0("") STEP_OPS);
        if (NXW == 2 && !SYS) asm volatile(STEP_HEAD STEP_X0("") STEP_X1("") STEP_OPS);
        if (NXW == 1 && SYS) asm volatile(STEP_HEAD STEP_X0(" sc0 sc1") STEP_OPS);
        if (NXW == 2 && SYS) asm volatile(STEP_HEAD STEP_X0(" sc0 sc1") STEP_X1(" sc0 sc1") STEP_OPS);
        ioff += SLOT;
        if (ioff >= SLOT * NS) ioff -= SLOT * NS;
    }
    static_assert(AHEAD == 5, "drain");
    wait_vm<CNT * 4>(); wg_barrier();
    wait_vm<CNT * 3>(); wg_barrier();
    wait_vm<CNT * 2>(); wg_barrier();
    wait_vm<CNT>(); wg_barrier();
    wait_vm<0>(); wg_barrier();
}

template <int NU>
__device__ __forceinline__ void consume_tile(LAS uint8_t* smem, int c, int lane, v4f (&acc)[4][2]) {
    const int g = lane >> 4, q = (lane & 15) >> 1, p = lane & 1, m = lane & 15;
    const unsigned fw = ((q >> 1) & 3) | ((g & 1) << 2);
    const unsigned w0 = (16 * g + q) * 128 + (((2 * c) ^ fw) << 4) + p * 8;
    const unsigned x0 = 16384 + m * 128 + ((g ^ ((m >> 1) & 7)) << 4);
#pragma unroll
    for (int u = 0; u < 4; ++u) { acc[u][0] = v4f{0, 0, 0, 0}; acc[u][1] = v4f{0, 0, 0, 0}; }
    v8i wf[2][2], xf[2][NU];
    auto load = [&](int s, v8i (&wv)[2], v8i (&xv)[NU]) {
        LAS uint8_t* sl = smem + (s % NS) * SLOT;
#pragma unroll
        for (int cb = 0; cb < 2; ++cb) {
            LAS uint8_t* b = sl + (w0 ^ (cb << 4));
            v2i t0 = __builtin_amdgcn_ds_read_tr8_b64_v2i32((LAS v2i*)(b));
            v2i t1 = __builtin_amdgcn_ds_read_tr8_b64_v2i32((LAS v2i*)(b + 1024));
            v2i t2 = __builtin_amdgcn_ds_read_tr8_b64_v2i32((LAS v2i*)(b + 8192));
            v2i t3 = __builtin_amdgcn_ds_read_tr8_b64_v2i32((LAS v2i*)(b + 9216));
            wv[cb] = v8i{t0[0], t0[1], t1[0], t1[1], t2[0], t2[1], t3[0], t3[1]};
        }
#pragma unroll
        for (int u = 0; u < NU; ++u) {
            v4i a = *(LAS v4i*)(sl + x0 + u * 2048);
            v4i b = *(LAS v4i*)(sl + (x0 ^ 64) + u * 2048);
            xv[u] = v8i{a[0], a[1], a[2], a[3], b[0], b[1], b[2], b[3]};
        }
    };
    auto mma = [&](v8i (&wv)[2], v8i (&xv)[NU]) {
#pragma unroll
        for (int u = 0; u < NU; ++u)
#pragma unroll
            for (int cb = 0; cb < 2; ++cb)
                acc[u][cb] = __builtin_amdgcn_mfma_scale_f32_16x16x128_f8f6f4(wv[cb], xv[u], acc[u][cb], 1, 0, 0, 0, 0, 0);
    };
    auto step = [&](int s, v8i (&wn)[2], v8i (&xn)[NU], v8i (&wo)[2], v8i (&xo)[NU]) {
        wg_barrier();
        __builtin_amdgcn_sched_barrier(0);
        load(s, wn, xn);
        mma(wo, xo);
        constexpr int PER = (8 + 2 * NU + 2 * NU - 1) / (2 * NU);
#pragma unroll
        for (int i = 0; i < 2 * NU; ++i) {
            __builtin_amdgcn_sched_group_barrier(0x100, PER, 0);
            __builtin_amdgcn_sched_group_barrier(0x8, 1, 0);
        }
        __builtin_amdgcn_sched_barrier(0);
        wait_lds();
    };
    wg_barrier();
    load(0, wf[0], xf[0]);
    wait_lds();
    step(1, wf[1], xf[1], wf[0], xf[0]);
#pragma unroll 1
    for (int it = 0; it < (KSTEPS - 2) / (int)NS; ++it) {
#pragma unroll
        for (int s = 2; s < 2 + (int)NS; s += 2) {
            step(s, wf[0], xf[0], wf[1], xf[1]);
            step(s + 1, wf[1], xf[1], wf[0], xf[0]);
        }
    }
    static_assert((KSTEPS - 2) % NS == 0 && NS % 2 == 0, "trip count");
    mma(wf[1], xf[1]);
}

__global__ void __launch_bounds__(NTHREADS) gemm_a2a_kernel(Params P) {
    extern __shared__ __attribute__((aligned(16))) uint8_t smem_[];
    LAS uint8_t* smem = (LAS uint8_t*)smem_;
    const int lane = threadIdx.x & 63;
    const int wave = rfl(threadIdx.x >> 6);
    const int grp = blockIdx.x & 7, idx = blockIdx.x >> 3, panel = idx & 3, rgp = idx >> 2;
    const uint8_t* px = P.x;
    const uint8_t* pw = P.w;
    unsigned rank = P.rank;
    uint8_t* myws = P.wsd[7];
    uint8_t* pws = P.wsd[grp];
    float* pout = P.outd[grp];
    asm volatile("" : "+s"(px), "+s"(pw), "+s"(rank), "+s"(myws), "+s"(pws), "+s"(pout));
    rank &= 7u;
    const bool local = grp == 7;
    const bool pusher = !local && rgp == 7;
#define CTR ((GAS unsigned*)(myws + WS_CALL + (size_t)blockIdx.x * 128))
#define TMO ((GAS unsigned*)(myws + WS_TMO))
#define PACE ((GAS u64*)(myws + WS_PACE + grp * 128))
#define LOAD_CALL() do { call = *(const CAS unsigned*)(uintptr_t)CTR + 1; asm volatile("" : "+s"(call)); } while (0)
    unsigned call = 0;
    float scale = 0.f;
    const int g = lane >> 4, m = lane & 15;

    if (__builtin_expect(pusher, 0)) {
        constexpr int XI = RR / 8;
        const unsigned xo = (panel * 8 + wave) * XI * 1024;
        const unsigned lb = wave * XI * 1024;
        __amdgpu_buffer_rsrc_t rp = make_rsrc(px, RR * KD);
        asm volatile("s_nop 4" : "+s"(rp) :: "memory");
        if (wave == 7 && lane == 0) {
            __hip_atomic_fetch_max(PACE, (rtime() + XLEAD) << 6, __ATOMIC_RELAXED, __HIP_MEMORY_SCOPE_WORKGROUP);
            __hip_atomic_fetch_add(PACE, (u64)(RR * TAU), __ATOMIC_RELAXED, __HIP_MEMORY_SCOPE_WORKGROUP);
        }
        dma16(lb, lane * 16, rp, xo);
        dma16(lb + 1024, lane * 16, rp, xo + 1024);
        wg_barrier();
        dma16(lb + 2048, lane * 16, rp, xo + 2048);
        dma16(lb + 3072, lane * 16, rp, xo + 3072);
        LOAD_CALL();
        uint8_t* xd = pws + WS_XST + (size_t)(rank * 2 + (call & 1)) * XST_SLOT + xo + lane * 16;
        LAS uint8_t* ls = smem + lb + lane * 16;
        wait_vm<3>();
        v4f v = *(LAS v4f*)ls;
#pragma unroll
        for (int i = 0; i < XI; ++i) {
            if (i >= 2) {
                if (i + 3 < XI) wait_vm<3>(); else if (i + 2 < XI) wait_vm<2>(); else wait_vm<1>();
            }
            st16_sys(xd + i * 1024, v);
            LOGT(i);
            if (i + 4 < XI) dma16(lb + (i + 4) * 1024, lane * 16, rp, xo + (i + 4) * 1024);
            if (i == 0) wait_vm<4>();
            if (i == 1) wait_vm<5>();
            if (i + 1 < XI) v = *(LAS v4f*)(ls + (i + 1) * 1024);
        }
        static_assert(XI >= 6 && XI <= 16, "wait counts");
        scale = *(const CAS float*)P.sx * *(const CAS float*)P.sw;
        wait_vm<0>();
        if (lane == 0)
            __hip_atomic_store((GAS unsigned*)(pws + WS_XFLAG + rank * 128) + panel * 8 + wave, call, __ATOMIC_RELAXED, __HIP_MEMORY_SCOPE_SYSTEM);
        STAMP(40);
        LOGOUT(0);
    } else {
        const int h0 = local ? 64 : rgp >= NE ? (rgp == 6 ? HFL : HF) : rgp == 0 ? HG0 : rgp == 1 ? HG1 : HG2;
        const int row0 = local ? rgp * 64 : rgp >= NE ? RR + ESUM + (rgp - NE) * HF : rgp == 0 ? RR : rgp == 1 ? RR + HG0 : RR + HG0 + HG1 + (rgp - 2) * HG2;
        if (wave >= 4) {
            const int w = wave - 4;
            const unsigned coff = ((rank + 1 + grp) & 7u) * NC + panel * 128;
            const __amdgpu_buffer_rsrc_t rw = make_rsrc(pw + coff, (unsigned)(KD * ND) - coff);
            const __amdgpu_buffer_rsrc_t rx = make_rsrc(px + (size_t)row0 * KD, h0 * KD);
            const int nf = local ? NFL : NF;
            const int f = rgp - (local ? 8 - NFL : NE);
            unsigned first = 0;
            if (f >= 0) first = local ? comb(NFL) << ((f + panel) % NFL) : comb(NF) << ((f + panel) % NF);
            if (SLOWNF && ((((unsigned)(uintptr_t)pw + coff) >> 7) & 7) == 3) first = (local ? comb(8) : comb(7)) << rgp;
            const int nb = local ? 16 : (ESUM > 128 ? 16 : ESUM / 8), erows = local ? (8 - NFL) * 64 : ESUM;
            const int q = (f * 4 + panel) * 4 + w;
            const int qs = local ? q >> 4 : q / (ESUM > 128 ? 16 : (ESUM > 0 ? ESUM / 8 : 1)), qb = q - qs * nb;
            const __amdgpu_buffer_rsrc_t rp = make_rsrc(px + (local ? 0 : RR * KD), (unsigned)(erows * KD));
            const unsigned vp = (qb * 8 + (lane >> 3)) * KD + qs * 128 + (lane & 7) * 16;
            const int npf = !NOXPF && f >= 0 && qs < nf && erows > 0 ? (256 + qb * 8 < erows ? 3 : 128 + qb * 8 < erows ? 2 : 1) : 0;
            const int nxw = (8 * w < h0) + (8 * w + 32 < h0);
            u64 ts[3];
            if (nxw == 2) load_tile<2, false>(w, lane, rw, rx, first, rp, vp, npf, nf * 128u, ts);
            else if (nxw == 1) load_tile<1, false>(w, lane, rw, rx, first, rp, vp, npf, nf * 128u, ts);
            else load_tile<0, false>(w, lane, rw, rx, first, rp, vp, npf, nf * 128u, ts);
            if (!local && w == 0) {
                if (lane == 0) {
                    const u64 now = (rtime() + LEAD) << 6;
                    __hip_atomic_fetch_max(PACE, now, __ATOMIC_RELAXED, __HIP_MEMORY_SCOPE_WORKGROUP);
                    const u64 t = __hip_atomic_fetch_add(PACE, (u64)(h0 / 2) * TAU, __ATOMIC_RELAXED, __HIP_MEMORY_SCOPE_WORKGROUP);
                    *(volatile LAS u64*)(smem + CTL_OFF + 8) = t - now < (4000ull << 6) ? t : now;
                }
                wait_lds();
            }
            LOAD_CALL();
        } else {
            v4f acc[4][2];
            float sa = *(const CAS float*)P.sx, sb = *(const CAS float*)P.sw;
            LOAD_CALL();
            asm volatile("" : "+s"(sa), "+s"(sb));
            scale = sa * sb;
            if (h0 > 48) consume_tile<4>(smem, wave, lane, acc);
            else if (h0 > 32) consume_tile<3>(smem, wave, lane, acc);
            else if (h0 > 16) consume_tile<2>(smem, wave, lane, acc);
            else consume_tile<1>(smem, wave, lane, acc);
            STAMP(16 + wave);
            if (!local) {
#pragma unroll
                for (int u = 0; u < 4; ++u)
                    if (u * 16 < h0) {
                        const int r = u * 16 + m;
                        LAS uint8_t* d = smem + STG_OFF + r * 512 + wave * 128;
                        *(LAS v4f*)(d + ((g ^ (r & 7)) << 4)) = acc[u][0] * scale;
                        *(LAS v4f*)(d + (((4 + g) ^ (r & 7)) << 4)) = acc[u][1] * scale;
                    }
                wait_lds();
            } else {
                float* o = pout + ((size_t)rank * MS + row0 + m) * NC + panel * 128 + wave * 32 + g * 4;
#pragma unroll
                for (int u = 0; u < 4; ++u) {
                    st16_sys(o + (size_t)u * 16 * NC, acc[u][0] * scale);
                    st16_sys(o + (size_t)u * 16 * NC + 16, acc[u][1] * scale);
                }
            }
        }

        if (!local) {
            float* o = pout + ((size_t)rank * MS + row0 + (lane >> 5)) * NC + panel * 128 + (lane & 31) * 4;
            wg_barrier();
            STAMP(48);
            const u64 st = *(volatile LAS u64*)(smem + CTL_OFF + 8);
            u64 slot = (((u64)rfl((unsigned)(st >> 32)) << 32) | rfl((unsigned)st)) + wave * TAU;
#pragma unroll 1
            for (int it = wave; it < h0 / 2; it += 8) {
                const int r = it * 2 + (lane >> 5);
                const v4f v = *(LAS v4f*)(smem + STG_OFF + r * 512 + (((lane & 31) ^ (r & 7)) << 4));
                for (;;) {
                    const long long d = (long long)(slot >> 6) - (long long)rtime();
                    if (d <= 0) break;
                    if (d > 30) __builtin_amdgcn_s_sleep(8);
                }
                st16_sys(o + (size_t)it * 2 * NC, v);
                LOGT(it / 8);
                slot += 8 * TAU;
            }
            wait_vm<0>();
            LOGOUT(1);
            wg_barrier();
            STAMP(52);
            if (wave == 0 && lane == 0) {
                __hip_atomic_store((GAS unsigned*)(pws + WS_DONE + rank * 128) + idx, call, __ATOMIC_RELAXED, __HIP_MEMORY_SCOPE_SYSTEM);
                *CTR = call;
            }
            return;
        }
        if (rgp == 7) {
            wait_vm<0>();
            wg_barrier();
            if (wave == 7 && lane == 0) *CTR = call;
            if (panel == 0 && wave < 7) {
                const unsigned src = (rank + 1 + wave) & 7u;
                const bool ok = wait_flags((const GAS unsigned*)(myws + WS_DONE + src * 128), 0xf0000000u, call, TMO, lane);
                if (!ok && lane == 0)
                    __hip_atomic_store((GAS float*)pout + ((size_t)rank * MS + row0) * NC, __builtin_nanf(""), __ATOMIC_RELAXED, __HIP_MEMORY_SCOPE_SYSTEM);
                STAMP(45 + wave);
            }
            return;
        }
    }

    const unsigned src = (rank + 1 + (local ? rgp : grp)) & 7u;
    const int r2 = local ? 0 : RA;
    if (wave == 4) {
        const bool ok = wait_flags((const GAS unsigned*)(myws + WS_XFLAG + src * 128), 0u, call, TMO, lane);
        *(volatile LAS unsigned*)(smem + CTL_OFF) = ok ? 0u : 1u;
        wait_lds();
        STAMP(44);
    }
    wg_barrier();
    if (wave >= 4) {
        const int w = wave - 4;
        const unsigned coff = rank * NC + panel * 128;
        const __amdgpu_buffer_rsrc_t rw = make_rsrc(pw + coff, (unsigned)(KD * ND) - coff);
        const __amdgpu_buffer_rsrc_t rx = make_rsrc(myws + WS_XST + (size_t)(src * 2 + (call & 1)) * XST_SLOT + (size_t)r2 * KD, RA * KD);
        if (8 * w + 32 < RA) load_tile<2, true>(w, lane, rw, rx, 0, rx, 0, 0, KD);
        else load_tile<1, true>(w, lane, rw, rx, 0, rx, 0, 0, KD);
    } else {
        if (wave == 0 && lane == 0) *CTR = call;
        v4f acc[4][2];
        consume_tile<(RA + 15) / 16>(smem, wave, lane, acc);
        STAMP(32 + wave);
        if (*(volatile LAS unsigned*)(smem + CTL_OFF)) scale = __builtin_nanf("");
        float* o = P.outd[7] + ((size_t)src * MS + r2 + m) * NC + panel * 128 + wave * 32 + g * 4;
#pragma unroll
        for (int u = 0; u < (RA + 15) / 16; ++u)
            if (u * 16 + m < RA) {
                st16_sys(o + (size_t)u * 16 * NC, acc[u][0] * scale);
                st16_sys(o + (size_t)u * 16 * NC + 16, acc[u][1] * scale);
            }
    }
}

}

extern "C" void kernel_launch(int rank, int nranks, ncclComm_t comm,
                              hipStream_t stream,
                              void* const* d_in, const int* in_sizes, int n_in,
                              void* d_out, int out_size,
                              const int* in_dtypes, int out_dtype,
                              void* const* d_out_all,
                              void* const* d_ws_all, size_t ws_bytes) {
    (void)comm; (void)in_dtypes; (void)out_dtype; (void)d_out;
    if (nranks != NR || (unsigned)rank >= (unsigned)NR || n_in != 4 || in_sizes[0] != MS * KD || in_sizes[1] != KD * ND
        || out_size != NR * MS * NC || ws_bytes < WS_NEED) {
        fprintf(stderr, "kernel_launch: unexpected problem shape\n");
        return;
    }
    static bool attr_set = false;
    if (!attr_set) {
        HIP_CHECK(hipFuncSetAttribute((const void*)gemm_a2a_kernel, hipFuncAttributeMaxDynamicSharedMemorySize, LDS_BYTES));
        attr_set = true;
    }
    Params P;
    P.x = (const uint8_t*)d_in[0];
    P.w = (const uint8_t*)d_in[1];
    P.sx = (const float*)d_in[2];
    P.sw = (const float*)d_in[3];
    for (int g = 0; g < NR; ++g) {
        const int p = (rank + 1 + g) % NR;
        P.outd[g] = (float*)d_out_all[p];
        P.wsd[g] = (uint8_t*)d_ws_all[p];
    }
    P.rank = (unsigned)rank;
    P.pad = 0;
    hipLaunchKernelGGL(gemm_a2a_kernel, dim3(NWG), dim3(NTHREADS), LDS_BYTES, stream, P);
    HIP_CHECK(hipGetLastError());
}
```

```cpp
#include <hip/hip_runtime.h>
#include <rccl/rccl.h>
#include <cstdio>
#include <cstdint>

#define HIP_CHECK(expr) do { hipError_t _e = (expr); if (_e != hipSuccess) { \
    fprintf(stderr, "HIP error %s at %s:%d\n", hipGetErrorString(_e), __FILE__, __LINE__); \
    return; } } while (0)

namespace {

typedef int v2i __attribute__((ext_vector_type(2)));
typedef int v4i __attribute__((ext_vector_type(4)));
typedef int v8i __attribute__((ext_vector_type(8)));
typedef float v4f __attribute__((ext_vector_type(4)));
typedef unsigned long long u64;
#define LAS __attribute__((address_space(3)))
#define GAS __attribute__((address_space(1)))
#define CAS __attribute__((address_space(4)))

constexpr int NR = 8;
constexpr int KD = 4096;
constexpr int ND = 4096;
constexpr int MS = 512;
constexpr int NC = 512;
#define RR 88
constexpr int RA = RR / 2;
#define NF 4
constexpr int NE = 7 - NF;
constexpr int HMAX = 80;
#define HG0 24
#define HG1 48
#define HG2 64
constexpr int ESUM = (NE > 0 ? HG0 : 0) + (NE > 1 ? HG1 : 0) + (NE > 2 ? (NE - 2) * HG2 : 0);
constexpr int FSUM = MS - RR - ESUM;
constexpr int HF = ((FSUM + NF - 1) / NF + 1) & ~1;
constexpr int HFL = FSUM - (NF - 1) * HF;
static_assert(RR % 8 == 0 && RA % 2 == 0 && RA > 32 && RA <= 64 && NF >= 2 && NF <= 7 && ESUM <= 384, "shape");
static_assert(HG0 % 2 == 0 && HG1 % 2 == 0 && HG2 % 2 == 0 && HG0 <= HMAX && HG1 <= HMAX && HG2 <= HMAX && HF <= HMAX && HFL >= 2 && ESUM % 8 == 0, "heights");
#define NFL 4
#define SLOWNF 0
#define NOXPF 0
#define RGP(i) (((i) + 7) & 7)
#define AH1 3
#define AH2 5
#define PEXP 1
#define TAU 112
#define LEAD 25
#define XLEAD 60
#define FENCE 1
constexpr int NTHREADS = 512;
constexpr int NWG = 256;
constexpr int KSTEPS = KD / 128;

constexpr unsigned SLOT = 16384 + HMAX * 128, NS = 6;
constexpr unsigned STG_OFF = 2 * SLOT;
static_assert((KSTEPS - 1) % NS == 1, "the last two stages use slots 0 and 1");
constexpr unsigned CTL_OFF = NS * SLOT;
constexpr unsigned JUNK_OFF = CTL_OFF + 64;
constexpr unsigned LDS_BYTES = JUNK_OFF + 1024;

constexpr size_t WS_CALL = 0;
constexpr size_t WS_TMO = 32768;
constexpr size_t WS_PACE = 49152;
constexpr size_t WS_DONE = 65536;
constexpr size_t WS_XFLAG = WS_DONE + 8 * 128;
constexpr size_t WS_STAMP = 131072;
constexpr size_t WS_XST = 1u << 20;
constexpr size_t XST_SLOT = (size_t)RR * KD;
constexpr size_t WS_NEED = WS_XST + 2 * NR * XST_SLOT;
constexpr u64 DEADLINE_TICKS = 40000000ull;

struct Params {
    const uint8_t* x;
    const uint8_t* w;
    unsigned rank, pad;
    const float* sx;
    const float* sw;
    float* outd[NR];
    uint8_t* wsd[NR];
};

constexpr unsigned comb(int n) { unsigned m = 0; for (int i = 0; i < 32; i += n) m |= 1u << i; return m; }

__device__ __forceinline__ unsigned rfl(unsigned v) { return __builtin_amdgcn_readfirstlane(v); }
__device__ __forceinline__ u64 rtime() { return __builtin_amdgcn_s_memrealtime(); }

__device__ __forceinline__ __amdgpu_buffer_rsrc_t make_rsrc(const void* p, unsigned bytes) {
    const uint64_t a = (uint64_t)p;
    const uint64_t u = ((uint64_t)rfl((unsigned)(a >> 32)) << 32) | rfl((unsigned)a);
    return __builtin_amdgcn_make_buffer_rsrc((void*)u, (short)0, (int)rfl(bytes), 0x00020000);
}

template <bool SYS = false>
__device__ __forceinline__ void dma16(unsigned lds, unsigned voff, __amdgpu_buffer_rsrc_t rs, unsigned soff) {
    if (SYS) asm volatile("s_mov_b32 m0, %0\n\ts_nop 0\n\tbuffer_load_dwordx4 %1, %2, %3 offen sc0 sc1 lds"
                          :: "s"(lds), "v"(voff), "s"(rs), "s"(soff) : "memory");
    else     asm volatile("s_mov_b32 m0, %0\n\ts_nop 0\n\tbuffer_load_dwordx4 %1, %2, %3 offen lds"
                          :: "s"(lds), "v"(voff), "s"(rs), "s"(soff) : "memory");
}
__device__ __forceinline__ void dma16x4(unsigned lds, unsigned v0, unsigned v1, unsigned v2, unsigned v3,
                                        __amdgpu_buffer_rsrc_t rs, unsigned soff) {
    asm volatile("s_mov_b32 m0, %0\n\ts_nop 0\n\t"
                 "buffer_load_dwordx4 %1, %5, %6 offen lds\n\t"
                 "buffer_load_dwordx4 %2, %5, %6 offen offset:1024 lds\n\t"
                 "buffer_load_dwordx4 %3, %5, %6 offen offset:2048 lds\n\t"
                 "buffer_load_dwordx4 %4, %5, %6 offen offset:3072 lds"
                 :: "s"(lds), "v"(v0), "v"(v1), "v"(v2), "v"(v3), "s"(rs), "s"(soff) : "memory");
}
template <int N> __device__ __forceinline__ void wait_vm() { asm volatile("s_waitcnt vmcnt(%0)" :: "i"(N) : "memory"); }
__device__ __forceinline__ void wg_barrier() { asm volatile("s_barrier" ::: "memory"); }
__device__ __forceinline__ void wait_lds() { asm volatile("s_waitcnt lgkmcnt(0)" ::: "memory"); }
__device__ __forceinline__ void st16_sys(void* p, v4f v) {
    asm volatile("global_store_dwordx4 %0, %1, off sc0 sc1\n\ts_nop 1" :: "v"(p), "v"(v) : "memory");
}

#define STAMP(i) do {} while (0)
#define LOGT(k) do {} while (0)
#define LOGOUT(half) do {} while (0)

__device__ __forceinline__ bool wait_flags(const GAS unsigned* flags, unsigned ignore, unsigned call, GAS unsigned* tmo, int lane) {
    const GAS unsigned* p = lane < 32 ? flags + lane : tmo;
    const u64 t0 = rtime();
    for (;;) {
        const u64 now = rtime();
        const unsigned v = __hip_atomic_load(p, __ATOMIC_RELAXED, __HIP_MEMORY_SCOPE_SYSTEM);
        const u64 ge = __ballot(v >= call), eq = __ballot(v == call);
        if ((eq >> 32) || now - t0 > DEADLINE_TICKS) break;
        if (((unsigned)ge | ignore) == 0xffffffffu) return true;
    }
    if (lane == 0) __hip_atomic_store(tmo, call, __ATOMIC_RELAXED, __HIP_MEMORY_SCOPE_AGENT);
    return false;
}

template <int NXW, bool SYS, int AHEAD>
__device__ __forceinline__ void load_tile(int w, int lane, __amdgpu_buffer_rsrc_t rw, __amdgpu_buffer_rsrc_t rx,
                                          unsigned first, __amdgpu_buffer_rsrc_t rp, unsigned vp, int npf, unsigned pstep,
                                          u64* ts = nullptr) {
    constexpr int CNT = 4 + NXW;
    const unsigned v0 = (lane >> 3) * KD + (((lane & 7) ^ ((lane >> 4) & 3)) << 4);
    unsigned vw[4];
#pragma unroll
    for (int j = 0; j < 4; ++j) vw[j] = (v0 ^ (((j >> 1) & 1) << 6)) + (4 * w + j) * (8u * ND) - j * 1024u;
    const unsigned vx0 = (v0 ^ ((w & 1) << 6)) + w * (8u * KD), vx1 = vx0 + 32u * KD, vx2 = vx0 + 64u * KD;
    unsigned ioff = w * 1024;
    u64 todo = ((u64)~first << 32) | first;
    unsigned poff = 0;
    auto issue = [&]() {
        const unsigned ks = __builtin_ctzll(todo) & 31;
        todo &= todo - 1;
        if (poff < (unsigned)KD) {
            if (npf > 0) dma16(JUNK_OFF, vp, rp, poff);
            if (npf > 1) dma16(JUNK_OFF, vp, rp, poff + 128u * KD);
            if (npf > 2) dma16(JUNK_OFF, vp, rp, poff + 256u * KD);
            poff += pstep;
        }
        dma16x4(ioff + w * 3072, vw[0], vw[1], vw[2], vw[3], rw, ks * (128u * ND));
        if (NXW > 0) dma16<SYS>(ioff + 16384, vx0, rx, ks * 128u);
        if (NXW > 1) dma16<SYS>(ioff + 20480, vx1, rx, ks * 128u);
        if (NXW > 2) dma16<SYS>(ioff + 24576, vx2, rx, ks * 128u);
        ioff += SLOT;
        if (ioff >= SLOT * NS) ioff -= SLOT * NS;
    };
    asm volatile("s_nop 4" : "+s"(rx), "+s"(rw), "+s"(rp) :: "memory");
#pragma unroll 1
    for (int i = 0; i < AHEAD; ++i) issue();
#pragma unroll 1
    while (npf > 0 && poff < (unsigned)KD) {
        wait_vm<CNT * (AHEAD - 1)>();
        wg_barrier();
        issue();
    }
#pragma unroll 1
    while (todo) {
        const unsigned ks = __builtin_ctzll(todo) & 31;
        todo &= todo - 1;
        const unsigned mw = ioff + w * 3072, mx = ioff + 16384, mx1 = ioff + 20480;
        const unsigned sow = ks * (128u * ND), sox = ks * 128u;
#define STEP_HEAD "s_mov_b32 m0, %0\n\ts_waitcnt vmcnt(%13)\n\ts_barrier\n\t" \
                  "buffer_load_dwordx4 %3, %9, %11 offen lds\n\t" \
                  "buffer_load_dwordx4 %4, %9, %11 offen offset:1024 lds\n\t" \
                  "buffer_load_dwordx4 %5, %9, %11 offen offset:2048 lds\n\t" \
                  "buffer_load_dwordx4 %6, %9, %11 offen offset:3072 lds"
#define STEP_X0(pol) "\n\ts_mov_b32 m0, %1\n\ts_nop 0\n\tbuffer_load_dwordx4 %7, %10, %12 offen" pol " lds"
#define STEP_X1(pol) "\n\ts_mov_b32 m0, %2\n\ts_nop 0\n\tbuffer_load_dwordx4 %8, %10, %12 offen" pol " lds"
#define STEP_OPS :: "s"(mw), "s"(mx), "s"(mx1), "v"(vw[0]), "v"(vw[1]), "v"(vw[2]), "v"(vw[3]), "v"(vx0), "v"(vx1), \
                    "s"(rw), "s"(rx), "s"(sow), "s"(sox), "i"(CNT * (AHEAD - 1)) : "memory"
        if (NXW == 0) asm volatile(STEP_HEAD STEP_OPS);
        if (NXW == 1 && !SYS) asm volatile(STEP_HEAD STEP_X0("") STEP_OPS);
        if (NXW >= 2 && !SYS) asm volatile(STEP_HEAD STEP_X0("") STEP_X1("") STEP_OPS);
        if (NXW == 1 && SYS) asm volatile(STEP_HEAD STEP_X0(" sc0 sc1") STEP_OPS);
        if (NXW >= 2 && SYS) asm volatile(STEP_HEAD STEP_X0(" sc0 sc1") STEP_X1(" sc0 sc1") STEP_OPS);
        if (NXW > 2) dma16<SYS>(ioff + 24576, vx2, rx, sox);
        ioff += SLOT;
        if (ioff >= SLOT * NS) ioff -= SLOT * NS;
    }
    static_assert(AHEAD >= 2 && AHEAD <= 5 && AHEAD < (int)NS, "drain");
    if (AHEAD > 4) { wait_vm<CNT * 4>(); wg_barrier(); }
    if (AHEAD > 3) { wait_vm<CNT * 3>(); wg_barrier(); }
    if (AHEAD > 2) { wait_vm<CNT * 2>(); wg_barrier(); }
    wait_vm<CNT>(); wg_barrier();
    wait_vm<0>(); wg_barrier();
}

template <int NU>
__device__ __forceinline__ void consume_tile(LAS uint8_t* smem, int c, int lane, v4f (&acc)[5][2]) {
    const int g = lane >> 4, q = (lane & 15) >> 1, p = lane & 1, m = lane & 15;
    const unsigned fw = ((q >> 1) & 3) | ((g & 1) << 2);
    const unsigned w0 = (16 * g + q) * 128 + (((2 * c) ^ fw) << 4) + p * 8;
    const unsigned x0 = 16384 + m * 128 + ((g ^ ((m >> 1) & 7)) << 4);
#pragma unroll
    for (int u = 0; u < 5; ++u) { acc[u][0] = v4f{0, 0, 0, 0}; acc[u][1] = v4f{0, 0, 0, 0}; }
    v8i wf[2][2], xf[2][NU];
    auto load = [&](int s, v8i (&wv)[2], v8i (&xv)[NU]) {
        LAS uint8_t* sl = smem + (s % NS) * SLOT;
#pragma unroll
        for (int cb = 0; cb < 2; ++cb) {
            LAS uint8_t* b = sl + (w0 ^ (cb << 4));
            v2i t0 = __builtin_amdgcn_ds_read_tr8_b64_v2i32((LAS v2i*)(b));
            v2i t1 = __builtin_amdgcn_ds_read_tr8_b64_v2i32((LAS v2i*)(b + 1024));
            v2i t2 = __builtin_amdgcn_ds_read_tr8_b64_v2i32((LAS v2i*)(b + 8192));
            v2i t3 = __builtin_amdgcn_ds_read_tr8_b64_v2i32((LAS v2i*)(b + 9216));
            wv[cb] = v8i{t0[0], t0[1], t1[0], t1[1], t2[0], t2[1], t3[0], t3[1]};
        }
#pragma unroll
        for (int u = 0; u < NU; ++u) {
            v4i a = *(LAS v4i*)(sl + x0 + u * 2048);
            v4i b = *(LAS v4i*)(sl + (x0 ^ 64) + u * 2048);
            xv[u] = v8i{a[0], a[1], a[2], a[3], b[0], b[1], b[2], b[3]};
        }
    };
    auto mma = [&](v8i (&wv)[2], v8i (&xv)[NU]) {
#pragma unroll
        for (int u = 0; u < NU; ++u)
#pragma unroll
            for (int cb = 0; cb < 2; ++cb)
                acc[u][cb] = __builtin_amdgcn_mfma_scale_f32_16x16x128_f8f6f4(wv[cb], xv[u], acc[u][cb], 1, 0, 0, 0, 0, 0);
    };
    auto step = [&](int s, v8i (&wn)[2], v8i (&xn)[NU], v8i (&wo)[2], v8i (&xo)[NU]) {
        wg_barrier();
        __builtin_amdgcn_sched_barrier(0);
        load(s, wn, xn);
        mma(wo, xo);
        constexpr int PER = (8 + 2 * NU + 2 * NU - 1) / (2 * NU);
#pragma unroll
        for (int i = 0; i < 2 * NU; ++i) {
            __builtin_amdgcn_sched_group_barrier(0x100, PER, 0);
            __builtin_amdgcn_sched_group_barrier(0x8, 1, 0);
        }
        __builtin_amdgcn_sched_barrier(0);
        wait_lds();
    };
    wg_barrier();
    load(0, wf[0], xf[0]);
    wait_lds();
    step(1, wf[1], xf[1], wf[0], xf[0]);
#pragma unroll 1
    for (int it = 0; it < (KSTEPS - 2) / (int)NS; ++it) {
#pragma unroll
        for (int s = 2; s < 2 + (int)NS; s += 2) {
            step(s, wf[0], xf[0], wf[1], xf[1]);
            step(s + 1, wf[1], xf[1], wf[0], xf[0]);
        }
    }
    static_assert((KSTEPS - 2) % NS == 0 && NS % 2 == 0, "trip count");
    mma(wf[1], xf[1]);
}

__global__ void __launch_bounds__(NTHREADS) gemm_a2a_kernel(Params P) {
    extern __shared__ __attribute__((aligned(16))) uint8_t smem_[];
    LAS uint8_t* smem = (LAS uint8_t*)smem_;
    const int lane = threadIdx.x & 63;
    const int wave = rfl(threadIdx.x >> 6);
    const int grp = blockIdx.x & 7, idx = blockIdx.x >> 3, panel = idx & 3, rgp = RGP(idx >> 2);
    const uint8_t* px = P.x;
    const uint8_t* pw = P.w;
    unsigned rank = P.rank;
    uint8_t* myws = P.wsd[7];
    uint8_t* pws = P.wsd[grp];
    float* pout = P.outd[grp];
    asm volatile("" : "+s"(px), "+s"(pw), "+s"(rank), "+s"(myws), "+s"(pws), "+s"(pout));
    rank &= 7u;
    const bool local = grp == 7;
    const bool pusher = !local && rgp == 7;
#define CTR ((GAS unsigned*)(myws + WS_CALL + (size_t)blockIdx.x * 128))
#define TMO ((GAS unsigned*)(myws + WS_TMO))
#define PACE ((GAS u64*)(myws + WS_PACE + grp * 128))
#define LOAD_CALL() do { call = *(const CAS unsigned*)(uintptr_t)CTR + 1; asm volatile("" : "+s"(call)); } while (0)
    unsigned call = 0;
    float scale = 0.f;
    const int g = lane >> 4, m = lane & 15;

    if (__builtin_expect(pusher, PEXP)) {
        constexpr int XI = RR / 8;
        const unsigned xo = (panel * 8 + wave) * XI * 1024;
        const unsigned lb = wave * XI * 1024;
        __amdgpu_buffer_rsrc_t rp = make_rsrc(px, RR * KD);
        asm volatile("s_nop 4" : "+s"(rp) :: "memory");
        if (wave == 7 && lane == 0) {
            __hip_atomic_fetch_max(PACE, (rtime() + XLEAD) << 6, __ATOMIC_RELAXED, __HIP_MEMORY_SCOPE_WORKGROUP);
            __hip_atomic_fetch_add(PACE, (u64)(RR * TAU), __ATOMIC_RELAXED, __HIP_MEMORY_SCOPE_WORKGROUP);
        }
        dma16(lb, lane * 16, rp, xo);
        dma16(lb + 1024, lane * 16, rp, xo + 1024);
        wg_barrier();
        dma16(lb + 2048, lane * 16, rp, xo + 2048);
        dma16(lb + 3072, lane * 16, rp, xo + 3072);
        LOAD_CALL();
        uint8_t* xd = pws + WS_XST + (size_t)(rank * 2 + (call & 1)) * XST_SLOT + xo + lane * 16;
        LAS uint8_t* ls = smem + lb + lane * 16;
        wait_vm<3>();
        v4f v = *(LAS v4f*)ls;
#pragma unroll
        for (int i = 0; i < XI; ++i) {
            if (i >= 2) {
                if (i + 3 < XI) wait_vm<3>(); else if (i + 2 < XI) wait_vm<2>(); else wait_vm<1>();
            }
            st16_sys(xd + i * 1024, v);
            LOGT(i);
            if (i + 4 < XI) dma16(lb + (i + 4) * 1024, lane * 16, rp, xo + (i + 4) * 1024);
            if (i == 0) wait_vm<4>();
            if (i == 1) wait_vm<5>();
            if (i + 1 < XI) v = *(LAS v4f*)(ls + (i + 1) * 1024);
        }
        static_assert(XI >= 6 && XI <= 16, "wait counts");
        scale = *(const CAS float*)P.sx * *(const CAS float*)P.sw;
        wait_vm<0>();
        if (lane == 0)
            __hip_atomic_store((GAS unsigned*)(pws + WS_XFLAG + rank * 128) + panel * 8 + wave, call, __ATOMIC_RELAXED, __HIP_MEMORY_SCOPE_SYSTEM);
        STAMP(40);
        LOGOUT(0);
    } else {
        const int h0 = local ? 64 : rgp >= NE ? (rgp == 6 ? HFL : HF) : rgp == 0 ? HG0 : rgp == 1 ? HG1 : HG2;
        const int row0 = local ? rgp * 64 : rgp >= NE ? RR + ESUM + (rgp - NE) * HF : rgp == 0 ? RR : rgp == 1 ? RR + HG0 : RR + HG0 + HG1 + (rgp - 2) * HG2;
        if (wave >= 4) {
            const int w = wave - 4;
            const unsigned coff = ((rank + 1 + grp) & 7u) * NC + panel * 128;
            const __amdgpu_buffer_rsrc_t rw = make_rsrc(pw + coff, (unsigned)(KD * ND) - coff);
            const __amdgpu_buffer_rsrc_t rx = make_rsrc(px + (size_t)row0 * KD, h0 * KD);
            const int nf = local ? NFL : NF;
            const int f = rgp - (local ? 8 - NFL : NE);
            unsigned first = 0;
            if (f >= 0) first = local ? comb(NFL) << ((f + panel) % NFL) : comb(NF) << ((f + panel) % NF);
            if (SLOWNF && ((((unsigned)(uintptr_t)pw + coff) >> 7) & 7) == 3) first = (local ? comb(8) : comb(7)) << rgp;
            const int nb = local ? 16 : (ESUM > 128 ? 16 : ESUM / 8), erows = local ? (8 - NFL) * 64 : ESUM;
            const int q = (f * 4 + panel) * 4 + w;
            const int qs = local ? q >> 4 : q / (ESUM > 128 ? 16 : (ESUM > 0 ? ESUM / 8 : 1)), qb = q - qs * nb;
            const __amdgpu_buffer_rsrc_t rp = make_rsrc(px + (local ? 0 : RR * KD), (unsigned)(erows * KD));
            const unsigned vp = (qb * 8 + (lane >> 3)) * KD + qs * 128 + (lane & 7) * 16;
            const int npf = !NOXPF && f >= 0 && qs < nf && erows > 0 ? (256 + qb * 8 < erows ? 3 : 128 + qb * 8 < erows ? 2 : 1) : 0;
            const int nxw = (8 * w < h0) + (8 * w + 32 < h0) + (8 * w + 64 < h0);
            u64 ts[3];
            if (nxw == 3) load_tile<3, false, AH1>(w, lane, rw, rx, first, rp, vp, npf, nf * 128u, ts);
            else if (nxw == 2) load_tile<2, false, AH1>(w, lane, rw, rx, first, rp, vp, npf, nf * 128u, ts);
            else if (nxw == 1) load_tile<1, false, AH1>(w, lane, rw, rx, first, rp, vp, npf, nf * 128u, ts);
            else load_tile<0, false, AH1>(w, lane, rw, rx, first, rp, vp, npf, nf * 128u, ts);
            if (!local && w == 0) {
                if (lane == 0) {
                    const u64 now = (rtime() + LEAD) << 6;
                    __hip_atomic_fetch_max(PACE, now, __ATOMIC_RELAXED, __HIP_MEMORY_SCOPE_WORKGROUP);
                    const u64 t = __hip_atomic_fetch_add(PACE, (u64)(h0 / 2) * TAU, __ATOMIC_RELAXED, __HIP_MEMORY_SCOPE_WORKGROUP);
                    *(volatile LAS u64*)(smem + CTL_OFF + 8) = t - now < (4000ull << 6) ? t : now;
                }
                wait_lds();
            }
            LOAD_CALL();
        } else {
            v4f acc[5][2];
            float sa = *(const CAS float*)P.sx, sb = *(const CAS float*)P.sw;
            LOAD_CALL();
            asm volatile("" : "+s"(sa), "+s"(sb));
            scale = sa * sb;
            if (h0 > 64) consume_tile<5>(smem, wave, lane, acc);
            else if (h0 > 48) consume_tile<4>(smem, wave, lane, acc);
            else if (h0 > 32) consume_tile<3>(smem, wave, lane, acc);
            else if (h0 > 16) consume_tile<2>(smem, wave, lane, acc);
            else consume_tile<1>(smem, wave, lane, acc);
            STAMP(16 + wave);
            if (!local) {
#pragma unroll
                for (int u = 0; u < 5; ++u)
                    if (u * 16 < h0) {
                        const int r = u * 16 + m;
                        LAS uint8_t* d = smem + STG_OFF + r * 512 + wave * 128;
                        *(LAS v4f*)(d + ((g ^ (r & 7)) << 4)) = acc[u][0] * scale;
                        *(LAS v4f*)(d + (((4 + g) ^ (r & 7)) << 4)) = acc[u][1] * scale;
                    }
                wait_lds();
            } else {
                float* o = pout + ((size_t)rank * MS + row0 + m) * NC + panel * 128 + wave * 32 + g * 4;
#pragma unroll
                for (int u = 0; u < 4; ++u) {
                    st16_sys(o + (size_t)u * 16 * NC, acc[u][0] * scale);
                    st16_sys(o + (size_t)u * 16 * NC + 16, acc[u][1] * scale);
                }
            }
        }

        if (!local) {
            float* o = pout + ((size_t)rank * MS + row0 + (lane >> 5)) * NC + panel * 128 + (lane & 31) * 4;
            wg_barrier();
            STAMP(48);
            const u64 st = *(volatile LAS u64*)(smem + CTL_OFF + 8);
            u64 slot = (((u64)rfl((unsigned)(st >> 32)) << 32) | rfl((unsigned)st)) + wave * TAU;
#pragma unroll 1
            for (int it = wave; it < h0 / 2; it += 8) {
                const int r = it * 2 + (lane >> 5);
                const v4f v = *(LAS v4f*)(smem + STG_OFF + r * 512 + (((lane & 31) ^ (r & 7)) << 4));
                for (;;) {
                    const long long d = (long long)(slot >> 6) - (long long)rtime();
                    if (d <= 0) break;
                    if (d > 30) __builtin_amdgcn_s_sleep(8);
                }
                st16_sys(o + (size_t)it * 2 * NC, v);
                LOGT(it / 8);
                slot += 8 * TAU;
            }
            wait_vm<0>();
            LOGOUT(1);
            wg_barrier();
            STAMP(52);
            if (wave == 0 && lane == 0) {
                __hip_atomic_store((GAS unsigned*)(pws + WS_DONE + rank * 128) + rgp * 4 + panel, call, __ATOMIC_RELAXED, __HIP_MEMORY_SCOPE_SYSTEM);
                *CTR = call;
            }
            return;
        }
        if (rgp == 7) {
            wait_vm<0>();
            wg_barrier();
            if (wave == 7 && lane == 0) *CTR = call;
            if (panel == 0 && wave < 7) {
                const unsigned src = (rank + 1 + wave) & 7u;
                const bool ok = wait_flags((const GAS unsigned*)(myws + WS_DONE + src * 128), 0xf0000000u, call, TMO, lane);
                if (!ok && lane == 0)
                    __hip_atomic_store((GAS float*)pout + ((size_t)rank * MS + row0) * NC, __builtin_nanf(""), __ATOMIC_RELAXED, __HIP_MEMORY_SCOPE_SYSTEM);
                STAMP(45 + wave);
            }
            return;
        }
    }

    const unsigned src = (rank + 1 + (local ? rgp : grp)) & 7u;
    const int r2 = local ? 0 : RA;
    if (wave == 4) {
        const bool ok = wait_flags((const GAS unsigned*)(myws + WS_XFLAG + src * 128), 0u, call, TMO, lane);
        __builtin_amdgcn_fence(__ATOMIC_ACQUIRE, "");
        wait_vm<0>();
        *(volatile LAS unsigned*)(smem + CTL_OFF) = ok ? 0u : 1u;
        wait_lds();
        STAMP(44);
    }
    wg_barrier();
    if (wave >= 4) {
        const int w = wave - 4;
        const unsigned coff = rank * NC + panel * 128;
        const __amdgpu_buffer_rsrc_t rw = make_rsrc(pw + coff, (unsigned)(KD * ND) - coff);
        const __amdgpu_buffer_rsrc_t rx = make_rsrc(myws + WS_XST + (size_t)(src * 2 + (call & 1)) * XST_SLOT + (size_t)r2 * KD, RA * KD);
        if (8 * w + 32 < RA) load_tile<2, true, AH2>(w, lane, rw, rx, 0, rx, 0, 0, KD);
        else load_tile<1, true, AH2>(w, lane, rw, rx, 0, rx, 0, 0, KD);
    } else {
        if (wave == 0 && lane == 0) *CTR = call;
        v4f acc[5][2];
        consume_tile<(RA + 15) / 16>(smem, wave, lane, acc);
        STAMP(32 + wave);
        if (*(volatile LAS unsigned*)(smem + CTL_OFF)) scale = __builtin_nanf("");
        float* o = P.outd[7] + ((size_t)src * MS + r2 + m) * NC + panel * 128 + wave * 32 + g * 4;
#pragma unroll
        for (int u = 0; u < (RA + 15) / 16; ++u)
            if (u * 16 + m < RA) {
                st16_sys(o + (size_t)u * 16 * NC, acc[u][0] * scale);
                st16_sys(o + (size_t)u * 16 * NC + 16, acc[u][1] * scale);
            }
    }
}

}

extern "C" void kernel_launch(int rank, int nranks, ncclComm_t comm,
                              hipStream_t stream,
                              void* const* d_in, const int* in_sizes, int n_in,
                              void* d_out, int out_size,
                              const int* in_dtypes, int out_dtype,
                              void* const* d_out_all,
                              void* const* d_ws_all, size_t ws_bytes) {
    (void)comm; (void)in_dtypes; (void)out_dtype; (void)d_out;
    if (nranks != NR || (unsigned)rank >= (unsigned)NR || n_in != 4 || in_sizes[0] != MS * KD || in_sizes[1] != KD * ND
        || out_size != NR * MS * NC || ws_bytes < WS_NEED) {
        fprintf(stderr, "kernel_launch: unexpected problem shape\n");
        return;
    }
    static bool attr_set = false;
    if (!attr_set) {
        HIP_CHECK(hipFuncSetAttribute((const void*)gemm_a2a_kernel, hipFuncAttributeMaxDynamicSharedMemorySize, LDS_BYTES));
        attr_set = true;
    }
    Params P;
    P.x = (const uint8_t*)d_in[0];
    P.w = (const uint8_t*)d_in[1];
    P.sx = (const float*)d_in[2];
    P.sw = (const float*)d_in[3];
    for (int g = 0; g < NR; ++g) {
        const int p = (rank + 1 + g) % NR;
        P.outd[g] = (float*)d_out_all[p];
        P.wsd[g] = (uint8_t*)d_ws_all[p];
    }
    P.rank = (unsigned)rank;
    P.pad = 0;
    hipLaunchKernelGGL(gemm_a2a_kernel, dim3(NWG), dim3(NTHREADS), LDS_BYTES, stream, P);
    HIP_CHECK(hipGetLastError());
}
```

```cpp
#include <hip/hip_runtime.h>
#include <rccl/rccl.h>
#include <cstdio>
#include <cstdint>

#define HIP_CHECK(expr) do { hipError_t _e = (expr); if (_e != hipSuccess) { \
    fprintf(stderr, "HIP error %s at %s:%d\n", hipGetErrorString(_e), __FILE__, __LINE__); \
    return; } } while (0)

namespace {

typedef int v2i __attribute__((ext_vector_type(2)));
typedef int v4i __attribute__((ext_vector_type(4)));
typedef int v8i __attribute__((ext_vector_type(8)));
typedef float v4f __attribute__((ext_vector_type(4)));
typedef unsigned long long u64;
#define LAS __attribute__((address_space(3)))
#define GAS __attribute__((address_space(1)))
#define CAS __attribute__((address_space(4)))

constexpr int NR = 8;
constexpr int KD = 4096;
constexpr int ND = 4096;
constexpr int MS = 512;
constexpr int NC = 512;
#define RR 104
#define RA 40
constexpr int RB = RR - RA;
constexpr int RM = RA > RB ? RA : RB;
#define NF 4
constexpr int NE = 7 - NF;
constexpr int HMAX = 80;
#define HG0 24
#define HG1 48
#define HG2 64
constexpr int ESUM = (NE > 0 ? HG0 : 0) + (NE > 1 ? HG1 : 0) + (NE > 2 ? (NE - 2) * HG2 : 0);
constexpr int FSUM = MS - RR - ESUM;
constexpr int HF = ((FSUM + NF - 1) / NF + 1) & ~1;
constexpr int HFL = FSUM - (NF - 1) * HF;
static_assert(RR % 8 == 0 && RA >= 8 && RB >= 8 && RM <= HMAX && NF >= 2 && NF <= 7 && ESUM <= 384, "shape");
static_assert(HG0 % 2 == 0 && HG1 % 2 == 0 && HG2 % 2 == 0 && HG0 <= HMAX && HG1 <= HMAX && HG2 <= HMAX && HF <= HMAX && HFL >= 2 && ESUM % 8 == 0, "heights");
#define NFL 4
#define SLOWNF 2
#define NOXPF 0
#define RGP(i) (((i) + 7) & 7)
#define AH1 3
#define AH2 3
#define PEXP 1
#define HELP0 2
#define ROT2 0
#define TAU 112
#define LEAD 25
#define XLEAD 60
#define FENCE 0
constexpr int NTHREADS = 512;
constexpr int NWG = 256;
constexpr int KSTEPS = KD / 128;

constexpr unsigned SLOT = 16384 + HMAX * 128, NS = 6;
constexpr unsigned STG_OFF = 2 * SLOT;
static_assert((KSTEPS - 1) % NS == 1, "the last two stages use slots 0 and 1");
constexpr unsigned CTL_OFF = NS * SLOT;
constexpr unsigned JUNK_OFF = CTL_OFF + 64;
constexpr unsigned LDS_BYTES = JUNK_OFF + 1024;

constexpr size_t WS_CALL = 0;
constexpr size_t WS_TMO = 32768;
constexpr size_t WS_PACE = 49152;
constexpr size_t WS_DONE = 65536;
constexpr size_t WS_XFLAG = WS_DONE + 8 * 128;
constexpr size_t WS_STAMP = 131072;
constexpr size_t WS_XST = 1u << 20;
constexpr size_t XST_SLOT = (size_t)RR * KD;
constexpr size_t WS_NEED = WS_XST + 2 * NR * XST_SLOT;
constexpr u64 DEADLINE_TICKS = 40000000ull;

struct Params {
    const uint8_t* x;
    const uint8_t* w;
    unsigned rank, pad;
    const float* sx;
    const float* sw;
    float* outd[NR];
    uint8_t* wsd[NR];
};

constexpr unsigned comb(int n) { unsigned m = 0; for (int i = 0; i < 32; i += n) m |= 1u << i; return m; }

__device__ __forceinline__ unsigned rfl(unsigned v) { return __builtin_amdgcn_readfirstlane(v); }
__device__ __forceinline__ u64 rtime() { return __builtin_amdgcn_s_memrealtime(); }

__device__ __forceinline__ __amdgpu_buffer_rsrc_t make_rsrc(const void* p, unsigned bytes) {
    const uint64_t a = (uint64_t)p;
    const uint64_t u = ((uint64_t)rfl((unsigned)(a >> 32)) << 32) | rfl((unsigned)a);
    return __builtin_amdgcn_make_buffer_rsrc((void*)u, (short)0, (int)rfl(bytes), 0x00020000);
}

template <bool SYS = false>
__device__ __forceinline__ void dma16(unsigned lds, unsigned voff, __amdgpu_buffer_rsrc_t rs, unsigned soff) {
    if (SYS) asm volatile("s_mov_b32 m0, %0\n\ts_nop 0\n\tbuffer_load_dwordx4 %1, %2, %3 offen sc0 sc1 lds"
                          :: "s"(lds), "v"(voff), "s"(rs), "s"(soff) : "memory");
    else     asm volatile("s_mov_b32 m0, %0\n\ts_nop 0\n\tbuffer_load_dwordx4 %1, %2, %3 offen lds"
                          :: "s"(lds), "v"(voff), "s"(rs), "s"(soff) : "memory");
}
__device__ __forceinline__ void dma16x4(unsigned lds, unsigned v0, unsigned v1, unsigned v2, unsigned v3,
                                        __amdgpu_buffer_rsrc_t rs, unsigned soff) {
    asm volatile("s_mov_b32 m0, %0\n\ts_nop 0\n\t"
                 "buffer_load_dwordx4 %1, %5, %6 offen lds\n\t"
                 "buffer_load_dwordx4 %2, %5, %6 offen offset:1024 lds\n\t"
                 "buffer_load_dwordx4 %3, %5, %6 offen offset:2048 lds\n\t"
                 "buffer_load_dwordx4 %4, %5, %6 offen offset:3072 lds"
                 :: "s"(lds), "v"(v0), "v"(v1), "v"(v2), "v"(v3), "s"(rs), "s"(soff) : "memory");
}
template <int N> __device__ __forceinline__ void wait_vm() { asm volatile("s_waitcnt vmcnt(%0)" :: "i"(N) : "memory"); }
__device__ __forceinline__ void wg_barrier() { asm volatile("s_barrier" ::: "memory"); }
__device__ __forceinline__ void wait_lds() { asm volatile("s_waitcnt lgkmcnt(0)" ::: "memory"); }
__device__ __forceinline__ void st16_sys(void* p, v4f v) {
    asm volatile("global_store_dwordx4 %0, %1, off sc0 sc1\n\ts_nop 1" :: "v"(p), "v"(v) : "memory");
}

#define STAMP(i) do {} while (0)
#define LOGT(k) do {} while (0)
#define LOGOUT(half) do {} while (0)

__device__ __forceinline__ bool wait_flags(const GAS unsigned* flags, unsigned ignore, unsigned call, GAS unsigned* tmo, int lane) {
    const GAS unsigned* p = lane < 32 ? flags + lane : tmo;
    const u64 t0 = rtime();
    for (;;) {
        const u64 now = rtime();
        const unsigned v = __hip_atomic_load(p, __ATOMIC_RELAXED, __HIP_MEMORY_SCOPE_SYSTEM);
        const u64 ge = __ballot(v >= call), eq = __ballot(v == call);
        if ((eq >> 32) || now - t0 > DEADLINE_TICKS) break;
        if (((unsigned)ge | ignore) == 0xffffffffu) return true;
    }
    if (lane == 0) __hip_atomic_store(tmo, call, __ATOMIC_RELAXED, __HIP_MEMORY_SCOPE_AGENT);
    return false;
}

template <int NXW, bool SYS, int AHEAD>
__device__ __forceinline__ void load_tile(int w, int lane, __amdgpu_buffer_rsrc_t rw, __amdgpu_buffer_rsrc_t rx,
                                          unsigned first, __amdgpu_buffer_rsrc_t rp, unsigned vp, int npf, unsigned pstep,
                                          u64* ts = nullptr) {
    constexpr int CNT = 4 + NXW;
    const unsigned v0 = (lane >> 3) * KD + (((lane & 7) ^ ((lane >> 4) & 3)) << 4);
    unsigned vw[4];
#pragma unroll
    for (int j = 0; j < 4; ++j) vw[j] = (v0 ^ (((j >> 1) & 1) << 6)) + (4 * w + j) * (8u * ND) - j * 1024u;
    const unsigned vx0 = (v0 ^ ((w & 1) << 6)) + w * (8u * KD), vx1 = vx0 + 32u * KD, vx2 = vx0 + 64u * KD;
    unsigned ioff = w * 1024;
    u64 todo = ((u64)~first << 32) | first;
    unsigned poff = 0;
    auto issue = [&]() {
        const unsigned ks = __builtin_ctzll(todo) & 31;
        todo &= todo - 1;
        if (poff < (unsigned)KD) {
            if (npf > 0) dma16(JUNK_OFF, vp, rp, poff);
            if (npf > 1) dma16(JUNK_OFF, vp, rp, poff + 128u * KD);
            if (npf > 2) dma16(JUNK_OFF, vp, rp, poff + 256u * KD);
            poff += pstep;
        }
        dma16x4(ioff + w * 3072, vw[0], vw[1], vw[2], vw[3], rw, ks * (128u * ND));
        if (NXW > 0) dma16<SYS>(ioff + 16384, vx0, rx, ks * 128u);
        if (NXW > 1) dma16<SYS>(ioff + 20480, vx1, rx, ks * 128u);
        if (NXW > 2) dma16<SYS>(ioff + 24576, vx2, rx, ks * 128u);
        ioff += SLOT;
        if (ioff >= SLOT * NS) ioff -= SLOT * NS;
    };
    asm volatile("s_nop 4" : "+s"(rx), "+s"(rw), "+s"(rp) :: "memory");
#pragma unroll 1
    for (int i = 0; i < AHEAD; ++i) issue();
#pragma unroll 1
    while (npf > 0 && poff < (unsigned)KD) {
        wait_vm<CNT * (AHEAD - 1)>();
        wg_barrier();
        issue();
    }
#pragma unroll 1
    while (todo) {
        const unsigned ks = __builtin_ctzll(todo) & 31;
        todo &= todo - 1;
        const unsigned mw = ioff + w * 3072, mx = ioff + 16384, mx1 = ioff + 20480;
        const unsigned sow = ks * (128u * ND), sox = ks * 128u;
#define STEP_HEAD "s_mov_b32 m0, %0\n\ts_waitcnt vmcnt(%13)\n\ts_barrier\n\t" \
                  "buffer_load_dwordx4 %3, %9, %11 offen lds\n\t" \
                  "buffer_load_dwordx4 %4, %9, %11 offen offset:1024 lds\n\t" \
                  "buffer_load_dwordx4 %5, %9, %11 offen offset:2048 lds\n\t" \
                  "buffer_load_dwordx4 %6, %9, %11 offen offset:3072 lds"
#define STEP_X0(pol) "\n\ts_mov_b32 m0, %1\n\ts_nop 0\n\tbuffer_load_dwordx4 %7, %10, %12 offen" pol " lds"
#define STEP_X1(pol) "\n\ts_mov_b32 m0, %2\n\ts_nop 0\n\tbuffer_load_dwordx4 %8, %10, %12 offen" pol " lds"
#define STEP_OPS :: "s"(mw), "s"(mx), "s"(mx1), "v"(vw[0]), "v"(vw[1]), "v"(vw[2]), "v"(vw[3]), "v"(vx0), "v"(vx1), \
                    "s"(rw), "s"(rx), "s"(sow), "s"(sox), "i"(CNT * (AHEAD - 1)) : "memory"
        if (NXW == 0) asm volatile(STEP_HEAD STEP_OPS);
        if (NXW == 1 && !SYS) asm volatile(STEP_HEAD STEP_X0("") STEP_OPS);
        if (NXW >= 2 && !SYS) asm volatile(STEP_HEAD STEP_X0("") STEP_X1("") STEP_OPS);
        if (NXW == 1 && SYS) asm volatile(STEP_HEAD STEP_X0(" sc0 sc1") STEP_OPS);
        if (NXW >= 2 && SYS) asm volatile(STEP_HEAD STEP_X0(" sc0 sc1") STEP_X1(" sc0 sc1") STEP_OPS);
        if (NXW > 2) dma16<SYS>(ioff + 24576, vx2, rx, sox);
        ioff += SLOT;
        if (ioff >= SLOT * NS) ioff -= SLOT * NS;
    }
    static_assert(AHEAD >= 2 && AHEAD <= 5 && AHEAD < (int)NS, "drain");
    if (AHEAD > 4) { wait_vm<CNT * 4>(); wg_barrier(); }
    if (AHEAD > 3) { wait_vm<CNT * 3>(); wg_barrier(); }
    if (AHEAD > 2) { wait_vm<CNT * 2>(); wg_barrier(); }
    wait_vm<CNT>(); wg_barrier();
    wait_vm<0>(); wg_barrier();
}

template <int NU>
__device__ __forceinline__ void consume_tile(LAS uint8_t* smem, int c, int lane, v4f (&acc)[5][2]) {
    const int g = lane >> 4, q = (lane & 15) >> 1, p = lane & 1, m = lane & 15;
    const unsigned fw = ((q >> 1) & 3) | ((g & 1) << 2);
    const unsigned w0 = (16 * g + q) * 128 + (((2 * c) ^ fw) << 4) + p * 8;
    const unsigned x0 = 16384 + m * 128 + ((g ^ ((m >> 1) & 7)) << 4);
#pragma unroll
    for (int u = 0; u < 5; ++u) { acc[u][0] = v4f{0, 0, 0, 0}; acc[u][1] = v4f{0, 0, 0, 0}; }
    v8i wf[2][2], xf[2][NU];
    auto load = [&](int s, v8i (&wv)[2], v8i (&xv)[NU]) {
        LAS uint8_t* sl = smem + (s % NS) * SLOT;
#pragma unroll
        for (int cb = 0; cb < 2; ++cb) {
            LAS uint8_t* b = sl + (w0 ^ (cb << 4));
            v2i t0 = __builtin_amdgcn_ds_read_tr8_b64_v2i32((LAS v2i*)(b));
            v2i t1 = __builtin_amdgcn_ds_read_tr8_b64_v2i32((LAS v2i*)(b + 1024));
            v2i t2 = __builtin_amdgcn_ds_read_tr8_b64_v2i32((LAS v2i*)(b + 8192));
            v2i t3 = __builtin_amdgcn_ds_read_tr8_b64_v2i32((LAS v2i*)(b + 9216));
            wv[cb] = v8i{t0[0], t0[1], t1[0], t1[1], t2[0], t2[1], t3[0], t3[1]};
        }
#pragma unroll
        for (int u = 0; u < NU; ++u) {
            v4i a = *(LAS v4i*)(sl + x0 + u * 2048);
            v4i b = *(LAS v4i*)(sl + (x0 ^ 64) + u * 2048);
            xv[u] = v8i{a[0], a[1], a[2], a[3], b[0], b[1], b[2], b[3]};
        }
    };
    auto mma = [&](v8i (&wv)[2], v8i (&xv)[NU]) {
#pragma unroll
        for (int u = 0; u < NU; ++u)
#pragma unroll
            for (int cb = 0; cb < 2; ++cb)
                acc[u][cb] = __builtin_amdgcn_mfma_scale_f32_16x16x128_f8f6f4(wv[cb], xv[u], acc[u][cb], 1, 0, 0, 0, 0, 0);
    };
    auto step = [&](int s, v8i (&wn)[2], v8i (&xn)[NU], v8i (&wo)[2], v8i (&xo)[NU]) {
        wg_barrier();
        __builtin_amdgcn_sched_barrier(0);
        load(s, wn, xn);
        mma(wo, xo);
        constexpr int PER = (8 + 2 * NU + 2 * NU - 1) / (2 * NU);
#pragma unroll
        for (int i = 0; i < 2 * NU; ++i) {
            __builtin_amdgcn_sched_group_barrier(0x100, PER, 0);
            __builtin_amdgcn_sched_group_barrier(0x8, 1, 0);
        }
        __builtin_amdgcn_sched_barrier(0);
        wait_lds();
    };
    wg_barrier();
    load(0, wf[0], xf[0]);
    wait_lds();
    step(1, wf[1], xf[1], wf[0], xf[0]);
#pragma unroll 1
    for (int it = 0; it < (KSTEPS - 2) / (int)NS; ++it) {
#pragma unroll
        for (int s = 2; s < 2 + (int)NS; s += 2) {
            step(s, wf[0], xf[0], wf[1], xf[1]);
            step(s + 1, wf[1], xf[1], wf[0], xf[0]);
        }
    }
    static_assert((KSTEPS - 2) % NS == 0 && NS % 2 == 0, "trip count");
    mma(wf[1], xf[1]);
}

__global__ void __launch_bounds__(NTHREADS) gemm_a2a_kernel(Params P) {
    extern __shared__ __attribute__((aligned(16))) uint8_t smem_[];
    LAS uint8_t* smem = (LAS uint8_t*)smem_;
    const int lane = threadIdx.x & 63;
    const int wave = rfl(threadIdx.x >> 6);
    const int grp = blockIdx.x & 7, idx = blockIdx.x >> 3, panel = idx & 3, rgp = RGP(idx >> 2);
    const uint8_t* px = P.x;
    const uint8_t* pw = P.w;
    unsigned rank = P.rank;
    uint8_t* myws = P.wsd[7];
    uint8_t* pws = P.wsd[grp];
    float* pout = P.outd[grp];
    asm volatile("" : "+s"(px), "+s"(pw), "+s"(rank), "+s"(myws), "+s"(pws), "+s"(pout));
    rank &= 7u;
    const bool local = grp == 7;
    const bool pusher = !local && rgp == 7;
#define CTR ((GAS unsigned*)(myws + WS_CALL + (size_t)blockIdx.x * 128))
#define TMO ((GAS unsigned*)(myws + WS_TMO))
#define PACE ((GAS u64*)(myws + WS_PACE + grp * 128))
#define LOAD_CALL() do { call = *(const CAS unsigned*)(uintptr_t)CTR + 1; asm volatile("" : "+s"(call)); } while (0)
    unsigned call = 0;
    float scale = 0.f;
    const int g = lane >> 4, m = lane & 15;

    if (__builtin_expect(pusher, PEXP)) {
        constexpr int XI = RR / 8;
        const unsigned xo = (panel * 8 + wave) * XI * 1024;
        const unsigned lb = wave * XI * 1024;
        __amdgpu_buffer_rsrc_t rp = make_rsrc(px, RR * KD);
        asm volatile("s_nop 4" : "+s"(rp) :: "memory");
        if (wave == 7 && lane == 0) {
            __hip_atomic_fetch_max(PACE, (rtime() + XLEAD) << 6, __ATOMIC_RELAXED, __HIP_MEMORY_SCOPE_WORKGROUP);
            __hip_atomic_fetch_add(PACE, (u64)(RR * TAU), __ATOMIC_RELAXED, __HIP_MEMORY_SCOPE_WORKGROUP);
        }
        dma16(lb, lane * 16, rp, xo);
        dma16(lb + 1024, lane * 16, rp, xo + 1024);
        wg_barrier();
        dma16(lb + 2048, lane * 16, rp, xo + 2048);
        dma16(lb + 3072, lane * 16, rp, xo + 3072);
        LOAD_CALL();
        uint8_t* xd = pws + WS_XST + (size_t)(rank * 2 + (call & 1)) * XST_SLOT + xo + lane * 16;
        LAS uint8_t* ls = smem + lb + lane * 16;
        wait_vm<3>();
        v4f v = *(LAS v4f*)ls;
#pragma unroll
        for (int i = 0; i < XI; ++i) {
            if (i >= 2) {
                if (i + 3 < XI) wait_vm<3>(); else if (i + 2 < XI) wait_vm<2>(); else wait_vm<1>();
            }
            st16_sys(xd + i * 1024, v);
            LOGT(i);
            if (i + 4 < XI) dma16(lb + (i + 4) * 1024, lane * 16, rp, xo + (i + 4) * 1024);
            if (i == 0) wait_vm<4>();
            if (i == 1) wait_vm<5>();
            if (i + 1 < XI) v = *(LAS v4f*)(ls + (i + 1) * 1024);
        }
        static_assert(XI >= 6 && XI <= 16, "wait counts");
        scale = *(const CAS float*)P.sx * *(const CAS float*)P.sw;
        wait_vm<0>();
        if (lane == 0)
            __hip_atomic_store((GAS unsigned*)(pws + WS_XFLAG + rank * 128) + panel * 8 + wave, call, __ATOMIC_RELAXED, __HIP_MEMORY_SCOPE_SYSTEM);
        STAMP(40);
        LOGOUT(0);
    } else {
        const int h0 = local ? 64 : rgp >= NE ? (rgp == 6 ? HFL : HF) : rgp == 0 ? HG0 : rgp == 1 ? HG1 : HG2;
        const int row0 = local ? rgp * 64 : rgp >= NE ? RR + ESUM + (rgp - NE) * HF : rgp == 0 ? RR : rgp == 1 ? RR + HG0 : RR + HG0 + HG1 + (rgp - 2) * HG2;
        if (wave >= 4) {
            const int w = wave - 4;
            const unsigned coff = ((rank + 1 + grp) & 7u) * NC + panel * 128;
            const __amdgpu_buffer_rsrc_t rw = make_rsrc(pw + coff, (unsigned)(KD * ND) - coff);
            const __amdgpu_buffer_rsrc_t rx = make_rsrc(px + (size_t)row0 * KD, h0 * KD);
            const int nf = local ? NFL : NF;
            const int f = rgp - (local ? 8 - NFL : NE);
            unsigned first = 0;
            if (f >= 0) first = local ? comb(NFL) << ((f + panel) % NFL) : comb(NF) << ((f + panel) % NF);
            if ((SLOWNF == 1 || (SLOWNF == 2 && local)) && ((((unsigned)(uintptr_t)pw + coff) >> 7) & 7) == 3) first = (local ? comb(8) : comb(7)) << rgp;
            const int nb = local ? 16 : (ESUM > 128 ? 16 : ESUM / 8), erows = local ? (8 - NFL) * 64 : ESUM;
            const int q = (f * 4 + panel) * 4 + w;
            const int qs = local ? q >> 4 : q / (ESUM > 128 ? 16 : (ESUM > 0 ? ESUM / 8 : 1)), qb = q - qs * nb;
            const __amdgpu_buffer_rsrc_t rp = make_rsrc(px + (local ? 0 : RR * KD), (unsigned)(erows * KD));
            const unsigned vp = (qb * 8 + (lane >> 3)) * KD + qs * 128 + (lane & 7) * 16;
            const int npf = !NOXPF && f >= 0 && qs < nf && erows > 0 ? (256 + qb * 8 < erows ? 3 : 128 + qb * 8 < erows ? 2 : 1) : 0;
            const int nxw = (8 * w < h0) + (8 * w + 32 < h0) + (8 * w + 64 < h0);
            u64 ts[3];
            if (nxw == 3) load_tile<3, false, AH1>(w, lane, rw, rx, first, rp, vp, npf, nf * 128u, ts);
            else if (nxw == 2) load_tile<2, false, AH1>(w, lane, rw, rx, first, rp, vp, npf, nf * 128u, ts);
            else if (nxw == 1) load_tile<1, false, AH1>(w, lane, rw, rx, first, rp, vp, npf, nf * 128u, ts);
            else load_tile<0, false, AH1>(w, lane, rw, rx, first, rp, vp, npf, nf * 128u, ts);
            if (!local && w == 0) {
                if (lane == 0) {
                    const u64 now = (rtime() + LEAD) << 6;
                    __hip_atomic_fetch_max(PACE, now, __ATOMIC_RELAXED, __HIP_MEMORY_SCOPE_WORKGROUP);
                    const u64 t = __hip_atomic_fetch_add(PACE, (u64)(h0 / 2) * TAU, __ATOMIC_RELAXED, __HIP_MEMORY_SCOPE_WORKGROUP);
                    *(volatile LAS u64*)(smem + CTL_OFF + 8) = t - now < (4000ull << 6) ? t : now;
                }
                wait_lds();
            }
            LOAD_CALL();
        } else {
            v4f acc[5][2];
            float sa = *(const CAS float*)P.sx, sb = *(const CAS float*)P.sw;
            LOAD_CALL();
            asm volatile("" : "+s"(sa), "+s"(sb));
            scale = sa * sb;
            if (h0 > 64) consume_tile<5>(smem, wave, lane, acc);
            else if (h0 > 48) consume_tile<4>(smem, wave, lane, acc);
            else if (h0 > 32) consume_tile<3>(smem, wave, lane, acc);
            else if (h0 > 16) consume_tile<2>(smem, wave, lane, acc);
            else consume_tile<1>(smem, wave, lane, acc);
            STAMP(16 + wave);
            if (!local) {
#pragma unroll
                for (int u = 0; u < 5; ++u)
                    if (u * 16 < h0) {
                        const int r = u * 16 + m;
                        LAS uint8_t* d = smem + STG_OFF + r * 512 + wave * 128;
                        *(LAS v4f*)(d + ((g ^ (r & 7)) << 4)) = acc[u][0] * scale;
                        *(LAS v4f*)(d + (((4 + g) ^ (r & 7)) << 4)) = acc[u][1] * scale;
                    }
                wait_lds();
            } else {
                float* o = pout + ((size_t)rank * MS + row0 + m) * NC + panel * 128 + wave * 32 + g * 4;
#pragma unroll
                for (int u = 0; u < 4; ++u) {
                    st16_sys(o + (size_t)u * 16 * NC, acc[u][0] * scale);
                    st16_sys(o + (size_t)u * 16 * NC + 16, acc[u][1] * scale);
                }
            }
        }

        if (!local) {
            float* o = pout + ((size_t)rank * MS + row0 + (lane >> 5)) * NC + panel * 128 + (lane & 31) * 4;
            wg_barrier();
            STAMP(48);
            const u64 st = *(volatile LAS u64*)(smem + CTL_OFF + 8);
            u64 slot = (((u64)rfl((unsigned)(st >> 32)) << 32) | rfl((unsigned)st)) + wave * TAU;
#pragma unroll 1
            for (int it = wave; it < h0 / 2; it += 8) {
                const int r = it * 2 + (lane >> 5);
                const v4f v = *(LAS v4f*)(smem + STG_OFF + r * 512 + (((lane & 31) ^ (r & 7)) << 4));
                for (;;) {
                    const long long d = (long long)(slot >> 6) - (long long)rtime();
                    if (d <= 0) break;
                    if (d > 30) __builtin_amdgcn_s_sleep(8);
                }
                st16_sys(o + (size_t)it * 2 * NC, v);
                LOGT(it / 8);
                slot += 8 * TAU;
            }
            wait_vm<0>();
            LOGOUT(1);
            wg_barrier();
            STAMP(52);
            if (wave == 0 && lane == 0) {
                __hip_atomic_store((GAS unsigned*)(pws + WS_DONE + rank * 128) + rgp * 4 + panel, call, __ATOMIC_RELAXED, __HIP_MEMORY_SCOPE_SYSTEM);
                *CTR = call;
            }
            if (rgp == 0) {
                const unsigned c3 = rank * NC + 3 * 128;
                if (((((unsigned)(uintptr_t)pw + c3) >> 7) & 7) == 3) {
                    __amdgpu_buffer_rsrc_t rw = make_rsrc(pw + c3, (unsigned)(KD * ND) - c3);
                    asm volatile("s_nop 4" : "+s"(rw) :: "memory");
                    const unsigned vl = ((panel * 8 + wave) * 8 + (lane >> 3)) * ND + (lane & 7) * 16;
#pragma unroll 1
                    for (int i = HELP0; i < 16; ++i) dma16(JUNK_OFF, vl, rw, i * (256u * ND));
                }
            }
            return;
        }
        if (rgp == 7) {
            wait_vm<0>();
            wg_barrier();
            if (wave == 7 && lane == 0) *CTR = call;
            if (panel == 0 && wave < 7) {
                const unsigned src = (rank + 1 + wave) & 7u;
                const bool ok = wait_flags((const GAS unsigned*)(myws + WS_DONE + src * 128), 0xf0000000u, call, TMO, lane);
                if (!ok && lane == 0)
                    __hip_atomic_store((GAS float*)pout + ((size_t)rank * MS + row0) * NC, __builtin_nanf(""), __ATOMIC_RELAXED, __HIP_MEMORY_SCOPE_SYSTEM);
                STAMP(45 + wave);
            }
            return;
        }
    }

    const unsigned src = (rank + 1 + (local ? rgp : grp)) & 7u;
    const int r2 = local ? 0 : RA, h2 = local ? RA : RB;
    if (wave == 4) {
        const bool ok = wait_flags((const GAS unsigned*)(myws + WS_XFLAG + src * 128), 0u, call, TMO, lane);
        *(volatile LAS unsigned*)(smem + CTL_OFF) = ok ? 0u : 1u;
        wait_lds();
        STAMP(44);
    }
    wg_barrier();
    if (wave >= 4) {
        const int w = wave - 4;
        const unsigned coff = rank * NC + panel * 128;
        const __amdgpu_buffer_rsrc_t rw = make_rsrc(pw + coff, (unsigned)(KD * ND) - coff);
        const __amdgpu_buffer_rsrc_t rx = make_rsrc(myws + WS_XST + (size_t)(src * 2 + (call & 1)) * XST_SLOT + (size_t)r2 * KD, h2 * KD);
        const int nxw = (8 * w < h2) + (8 * w + 32 < h2) + (8 * w + 64 < h2);
        const unsigned first2 = ROT2 == 1 && local ? ~0u << (8 * panel) : ROT2 == 2 && local ? comb(4) << panel : 0u;
        if (RM > 64 && nxw == 3) load_tile<3, true, AH2>(w, lane, rw, rx, first2, rx, 0, 0, KD);
        else if (nxw == 2) load_tile<2, true, AH2>(w, lane, rw, rx, first2, rx, 0, 0, KD);
        else if (nxw == 1) load_tile<1, true, AH2>(w, lane, rw, rx, first2, rx, 0, 0, KD);
        else load_tile<0, true, AH2>(w, lane, rw, rx, first2, rx, 0, 0, KD);
    } else {
        if (wave == 0 && lane == 0) *CTR = call;
        v4f acc[5][2];
        if (local) consume_tile<(RA + 15) / 16>(smem, wave, lane, acc);
        else consume_tile<(RB + 15) / 16>(smem, wave, lane, acc);
        STAMP(32 + wave);
        if (*(volatile LAS unsigned*)(smem + CTL_OFF)) scale = __builtin_nanf("");
        float* o = P.outd[7] + ((size_t)src * MS + r2 + m) * NC + panel * 128 + wave * 32 + g * 4;
#pragma unroll
        for (int u = 0; u < (RM + 15) / 16; ++u)
            if (u * 16 + m < h2) {
                st16_sys(o + (size_t)u * 16 * NC, acc[u][0] * scale);
                st16_sys(o + (size_t)u * 16 * NC + 16, acc[u][1] * scale);
            }
    }
}

}

extern "C" void kernel_launch(int rank, int nranks, ncclComm_t comm,
                              hipStream_t stream,
                              void* const* d_in, const int* in_sizes, int n_in,
                              void* d_out, int out_size,
                              const int* in_dtypes, int out_dtype,
                              void* const* d_out_all,
                              void* const* d_ws_all, size_t ws_bytes) {
    (void)comm; (void)in_dtypes; (void)out_dtype; (void)d_out;
    if (nranks != NR || (unsigned)rank >= (unsigned)NR || n_in != 4 || in_sizes[0] != MS * KD || in_sizes[1] != KD * ND
        || out_size != NR * MS * NC || ws_bytes < WS_NEED) {
        fprintf(stderr, "kernel_launch: unexpected problem shape\n");
        return;
    }
    static bool attr_set = false;
    if (!attr_set) {
        HIP_CHECK(hipFuncSetAttribute((const void*)gemm_a2a_kernel, hipFuncAttributeMaxDynamicSharedMemorySize, LDS_BYTES));
        attr_set = true;
    }
    Params P;
    P.x = (const uint8_t*)d_in[0];
    P.w = (const uint8_t*)d_in[1];
    P.sx = (const float*)d_in[2];
    P.sw = (const float*)d_in[3];
    for (int g = 0; g < NR; ++g) {
        const int p = (rank + 1 + g) % NR;
        P.outd[g] = (float*)d_out_all[p];
        P.wsd[g] = (uint8_t*)d_ws_all[p];
    }
    P.rank = (unsigned)rank;
    P.pad = 0;
    hipLaunchKernelGGL(gemm_a2a_kernel, dim3(NWG), dim3(NTHREADS), LDS_BYTES, stream, P);
    HIP_CHECK(hipGetLastError());
}
```

```cpp
#include <hip/hip_runtime.h>
#include <rccl/rccl.h>
#include <cstdio>
#include <cstdint>

#define HIP_CHECK(expr) do { hipError_t _e = (expr); if (_e != hipSuccess) { \
    fprintf(stderr, "HIP error %s at %s:%d\n", hipGetErrorString(_e), __FILE__, __LINE__); \
    return; } } while (0)

namespace {

typedef int v2i __attribute__((ext_vector_type(2)));
typedef int v4i __attribute__((ext_vector_type(4)));
typedef int v8i __attribute__((ext_vector_type(8)));
typedef float v4f __attribute__((ext_vector_type(4)));
typedef unsigned long long u64;
#define LAS __attribute__((address_space(3)))
#define GAS __attribute__((address_space(1)))
#define CAS __attribute__((address_space(4)))

constexpr int NR = 8;
constexpr int KD = 4096;
constexpr int ND = 4096;
constexpr int MS = 512;
constexpr int NC = 512;
#define RR 88
#define RA 40
constexpr int RB = RR - RA;
constexpr int RM = RA > RB ? RA : RB;
#define NF 0
constexpr int NE = 7 - NF;
#define HMAX 80
#define HLIST 32, 48, 64, 72, 72, 72, 64
constexpr int HL[7] = {HLIST};
constexpr int hsum(int n) { int t = 0; for (int i = 0; i < n; ++i) t += HL[i]; return t; }
constexpr u64 hpack() { u64 t = 0; for (int i = 0; i < 7; ++i) t |= (u64)HL[i] << (8 * i); return t; }
constexpr u64 rpack() { u64 t = 0; for (int i = 0; i < 7; ++i) t |= (u64)(hsum(i) / 2) << (8 * i); return t; }
constexpr bool hok() { for (int i = 0; i < 7; ++i) if (HL[i] < 2 || HL[i] > HMAX || HL[i] % 2) return false; return true; }
constexpr int ESUM = (hsum(NE) + 7) & ~7;
static_assert(RR % 8 == 0 && RA >= 8 && RB >= 8 && RM <= HMAX && NF >= 0 && NF <= 7, "shape");
static_assert(hok() && hsum(7) == MS - RR, "heights");
#define EP 2
#define NFL 4
#define SLOWNF 2
#define NOXPF 0
#define RGP(i) (((i) + 7) & 7)
#define AH1 3
#define AH2 3
#define PEXP 1
#define HELP0 2
#define ROT2 0
#define XPF_ON local
#define PWIN 6
#define SEMI 1
#define TAU 112
#define LEAD 25
#define XLEAD 60
#define FENCE 0
constexpr int NTHREADS = 512;
constexpr int NWG = 256;
constexpr int KSTEPS = KD / 128;

constexpr unsigned SLOT = 16384 + HMAX * 128, NS = 6;
constexpr unsigned STG_OFF = 2 * SLOT;
static_assert((KSTEPS - 1) % NS == 1, "the last two stages use slots 0 and 1");
constexpr unsigned CTL_OFF = NS * SLOT;
constexpr unsigned JUNK_OFF = CTL_OFF + 64;
constexpr unsigned LDS_BYTES = JUNK_OFF + 1024;

constexpr size_t WS_CALL = 0;
constexpr size_t WS_TMO = 32768;
constexpr size_t WS_PACE = 49152;
constexpr size_t WS_DONE = 65536;
constexpr size_t WS_XFLAG = WS_DONE + 8 * 128;
constexpr size_t WS_STAMP = 131072;
constexpr size_t WS_XST = 1u << 20;
constexpr size_t XST_SLOT = (size_t)RR * KD;
constexpr size_t WS_NEED = WS_XST + 2 * NR * XST_SLOT;
constexpr u64 DEADLINE_TICKS = 40000000ull;

struct Params {
    const uint8_t* x;
    const uint8_t* w;
    unsigned rank, pad;
    const float* sx;
    const float* sw;
    float* outd[NR];
    uint8_t* wsd[NR];
};

constexpr unsigned comb(int n) { unsigned m = 0; for (int i = 0; i < 32; i += n) m |= 1u << i; return m; }

__device__ __forceinline__ unsigned rfl(unsigned v) { return __builtin_amdgcn_readfirstlane(v); }
__device__ __forceinline__ u64 rtime() { return __builtin_amdgcn_s_memrealtime(); }

__device__ __forceinline__ __amdgpu_buffer_rsrc_t make_rsrc(const void* p, unsigned bytes) {
    const uint64_t a = (uint64_t)p;
    const uint64_t u = ((uint64_t)rfl((unsigned)(a >> 32)) << 32) | rfl((unsigned)a);
    return __builtin_amdgcn_make_buffer_rsrc((void*)u, (short)0, (int)rfl(bytes), 0x00020000);
}

template <bool SYS = false>
__device__ __forceinline__ void dma16(unsigned lds, unsigned voff, __amdgpu_buffer_rsrc_t rs, unsigned soff) {
    if (SYS) asm volatile("s_mov_b32 m0, %0\n\ts_nop 0\n\tbuffer_load_dwordx4 %1, %2, %3 offen sc0 sc1 lds"
                          :: "s"(lds), "v"(voff), "s"(rs), "s"(soff) : "memory");
    else     asm volatile("s_mov_b32 m0, %0\n\ts_nop 0\n\tbuffer_load_dwordx4 %1, %2, %3 offen lds"
                          :: "s"(lds), "v"(voff), "s"(rs), "s"(soff) : "memory");
}
__device__ __forceinline__ void dma16x4(unsigned lds, unsigned v0, unsigned v1, unsigned v2, unsigned v3,
                                        __amdgpu_buffer_rsrc_t rs, unsigned soff) {
    asm volatile("s_mov_b32 m0, %0\n\ts_nop 0\n\t"
                 "buffer_load_dwordx4 %1, %5, %6 offen lds\n\t"
                 "buffer_load_dwordx4 %2, %5, %6 offen offset:1024 lds\n\t"
                 "buffer_load_dwordx4 %3, %5, %6 offen offset:2048 lds\n\t"
                 "buffer_load_dwordx4 %4, %5, %6 offen offset:3072 lds"
                 :: "s"(lds), "v"(v0), "v"(v1), "v"(v2), "v"(v3), "s"(rs), "s"(soff) : "memory");
}
template <int N> __device__ __forceinline__ void wait_vm() { asm volatile("s_waitcnt vmcnt(%0)" :: "i"(N) : "memory"); }
__device__ __forceinline__ void wg_barrier() { asm volatile("s_barrier" ::: "memory"); }
__device__ __forceinline__ void wait_lds() { asm volatile("s_waitcnt lgkmcnt(0)" ::: "memory"); }
__device__ __forceinline__ void st16_sys(void* p, v4f v) {
    asm volatile("global_store_dwordx4 %0, %1, off sc0 sc1\n\ts_nop 1" :: "v"(p), "v"(v) : "memory");
}

#define STAMP(i) do {} while (0)
#define LOGT(k) do {} while (0)
#define LOGOUT(half) do {} while (0)

__device__ __forceinline__ bool wait_flags(const GAS unsigned* flags, unsigned ignore, unsigned call, GAS unsigned* tmo, int lane) {
    const GAS unsigned* p = lane < 32 ? flags + lane : tmo;
    const u64 t0 = rtime();
    for (;;) {
        const u64 now = rtime();
        const unsigned v = __hip_atomic_load(p, __ATOMIC_RELAXED, __HIP_MEMORY_SCOPE_SYSTEM);
        const u64 ge = __ballot(v >= call), eq = __ballot(v == call);
        if ((eq >> 32) || now - t0 > DEADLINE_TICKS) break;
        if (((unsigned)ge | ignore) == 0xffffffffu) return true;
    }
    if (lane == 0) __hip_atomic_store(tmo, call, __ATOMIC_RELAXED, __HIP_MEMORY_SCOPE_AGENT);
    return false;
}

template <int NXW, bool SYS, int AHEAD>
__device__ __forceinline__ void load_tile(int w, int lane, __amdgpu_buffer_rsrc_t rw, __amdgpu_buffer_rsrc_t rx,
                                          unsigned first, __amdgpu_buffer_rsrc_t rp, unsigned vp, int npf, unsigned pstep,
                                          u64* ts = nullptr) {
    constexpr int CNT = 4 + NXW;
    const unsigned v0 = (lane >> 3) * KD + (((lane & 7) ^ ((lane >> 4) & 3)) << 4);
    unsigned vw[4];
#pragma unroll
    for (int j = 0; j < 4; ++j) vw[j] = (v0 ^ (((j >> 1) & 1) << 6)) + (4 * w + j) * (8u * ND) - j * 1024u;
    const unsigned vx0 = (v0 ^ ((w & 1) << 6)) + w * (8u * KD), vx1 = vx0 + 32u * KD, vx2 = vx0 + 64u * KD;
    unsigned ioff = w * 1024;
    u64 todo = ((u64)~first << 32) | first;
    unsigned poff = 0;
    auto issue = [&]() {
        const unsigned ks = __builtin_ctzll(todo) & 31;
        todo &= todo - 1;
        if (poff < (unsigned)KD) {
            if (npf > 0) dma16(JUNK_OFF, vp, rp, poff);
            if (npf > 1) dma16(JUNK_OFF, vp, rp, poff + 128u * KD);
            if (npf > 2) dma16(JUNK_OFF, vp, rp, poff + 256u * KD);
            poff += pstep;
        }
        dma16x4(ioff + w * 3072, vw[0], vw[1], vw[2], vw[3], rw, ks * (128u * ND));
        if (NXW > 0) dma16<SYS>(ioff + 16384, vx0, rx, ks * 128u);
        if (NXW > 1) dma16<SYS>(ioff + 20480, vx1, rx, ks * 128u);
        if (NXW > 2) dma16<SYS>(ioff + 24576, vx2, rx, ks * 128u);
        ioff += SLOT;
        if (ioff >= SLOT * NS) ioff -= SLOT * NS;
    };
    asm volatile("s_nop 4" : "+s"(rx), "+s"(rw), "+s"(rp) :: "memory");
#pragma unroll 1
    for (int i = 0; i < AHEAD; ++i) issue();
#pragma unroll 1
    while (npf > 0 && poff < (unsigned)KD) {
        wait_vm<CNT * (AHEAD - 1)>();
        wg_barrier();
        issue();
    }
#pragma unroll 1
    while (todo) {
        const unsigned ks = __builtin_ctzll(todo) & 31;
        todo &= todo - 1;
        const unsigned mw = ioff + w * 3072, mx = ioff + 16384, mx1 = ioff + 20480;
        const unsigned sow = ks * (128u * ND), sox = ks * 128u;
#define STEP_HEAD "s_mov_b32 m0, %0\n\ts_waitcnt vmcnt(%13)\n\ts_barrier\n\t" \
                  "buffer_load_dwordx4 %3, %9, %11 offen lds\n\t" \
                  "buffer_load_dwordx4 %4, %9, %11 offen offset:1024 lds\n\t" \
                  "buffer_load_dwordx4 %5, %9, %11 offen offset:2048 lds\n\t" \
                  "buffer_load_dwordx4 %6, %9, %11 offen offset:3072 lds"
#define STEP_X0(pol) "\n\ts_mov_b32 m0, %1\n\ts_nop 0\n\tbuffer_load_dwordx4 %7, %10, %12 offen" pol " lds"
#define STEP_X1(pol) "\n\ts_mov_b32 m0, %2\n\ts_nop 0\n\tbuffer_load_dwordx4 %8, %10, %12 offen" pol " lds"
#define STEP_OPS :: "s"(mw), "s"(mx), "s"(mx1), "v"(vw[0]), "v"(vw[1]), "v"(vw[2]), "v"(vw[3]), "v"(vx0), "v"(vx1), \
                    "s"(rw), "s"(rx), "s"(sow), "s"(sox), "i"(CNT * (AHEAD - 1)) : "memory"
        if (NXW == 0) asm volatile(STEP_HEAD STEP_OPS);
        if (NXW == 1 && !SYS) asm volatile(STEP_HEAD STEP_X0("") STEP_OPS);
        if (NXW >= 2 && !SYS) asm volatile(STEP_HEAD STEP_X0("") STEP_X1("") STEP_OPS);
        if (NXW == 1 && SYS) asm volatile(STEP_HEAD STEP_X0(" sc0 sc1") STEP_OPS);
        if (NXW >= 2 && SYS) asm volatile(STEP_HEAD STEP_X0(" sc0 sc1") STEP_X1(" sc0 sc1") STEP_OPS);
        if (NXW > 2) dma16<SYS>(ioff + 24576, vx2, rx, sox);
        ioff += SLOT;
        if (ioff >= SLOT * NS) ioff -= SLOT * NS;
    }
    static_assert(AHEAD >= 2 && AHEAD <= 5 && AHEAD < (int)NS, "drain");
    if (AHEAD > 4) { wait_vm<CNT * 4>(); wg_barrier(); }
    if (AHEAD > 3) { wait_vm<CNT * 3>(); wg_barrier(); }
    if (AHEAD > 2) { wait_vm<CNT * 2>(); wg_barrier(); }
    wait_vm<CNT>(); wg_barrier();
    wait_vm<0>(); wg_barrier();
}

template <int NU>
__device__ __forceinline__ void consume_tile(LAS uint8_t* smem, int c, int lane, v4f (&acc)[5][2]) {
    const int g = lane >> 4, q = (lane & 15) >> 1, p = lane & 1, m = lane & 15;
    const unsigned fw = ((q >> 1) & 3) | ((g & 1) << 2);
    const unsigned w0 = (16 * g + q) * 128 + (((2 * c) ^ fw) << 4) + p * 8;
    const unsigned x0 = 16384 + m * 128 + ((g ^ ((m >> 1) & 7)) << 4);
#pragma unroll
    for (int u = 0; u < 5; ++u) { acc[u][0] = v4f{0, 0, 0, 0}; acc[u][1] = v4f{0, 0, 0, 0}; }
    v8i wf[2][2], xf[2][NU];
    auto load = [&](int s, v8i (&wv)[2], v8i (&xv)[NU]) {
        LAS uint8_t* sl = smem + (s % NS) * SLOT;
#pragma unroll
        for (int cb = 0; cb < 2; ++cb) {
            LAS uint8_t* b = sl + (w0 ^ (cb << 4));
            v2i t0 = __builtin_amdgcn_ds_read_tr8_b64_v2i32((LAS v2i*)(b));
            v2i t1 = __builtin_amdgcn_ds_read_tr8_b64_v2i32((LAS v2i*)(b + 1024));
            v2i t2 = __builtin_amdgcn_ds_read_tr8_b64_v2i32((LAS v2i*)(b + 8192));
            v2i t3 = __builtin_amdgcn_ds_read_tr8_b64_v2i32((LAS v2i*)(b + 9216));
            wv[cb] = v8i{t0[0], t0[1], t1[0], t1[1], t2[0], t2[1], t3[0], t3[1]};
        }
#pragma unroll
        for (int u = 0; u < NU; ++u) {
            v4i a = *(LAS v4i*)(sl + x0 + u * 2048);
            v4i b = *(LAS v4i*)(sl + (x0 ^ 64) + u * 2048);
            xv[u] = v8i{a[0], a[1], a[2], a[3], b[0], b[1], b[2], b[3]};
        }
    };
    auto mma = [&](v8i (&wv)[2], v8i (&xv)[NU]) {
#pragma unroll
        for (int u = 0; u < NU; ++u)
#pragma unroll
            for (int cb = 0; cb < 2; ++cb)
                acc[u][cb] = __builtin_amdgcn_mfma_scale_f32_16x16x128_f8f6f4(wv[cb], xv[u], acc[u][cb], 1, 0, 0, 0, 0, 0);
    };
    auto step = [&](int s, v8i (&wn)[2], v8i (&xn)[NU], v8i (&wo)[2], v8i (&xo)[NU]) {
        wg_barrier();
        __builtin_amdgcn_sched_barrier(0);
        load(s, wn, xn);
        mma(wo, xo);
        constexpr int PER = (8 + 2 * NU + 2 * NU - 1) / (2 * NU);
#pragma unroll
        for (int i = 0; i < 2 * NU; ++i) {
            __builtin_amdgcn_sched_group_barrier(0x100, PER, 0);
            __builtin_amdgcn_sched_group_barrier(0x8, 1, 0);
        }
        __builtin_amdgcn_sched_barrier(0);
        wait_lds();
    };
    wg_barrier();
    load(0, wf[0], xf[0]);
    wait_lds();
    step(1, wf[1], xf[1], wf[0], xf[0]);
#pragma unroll 1
    for (int it = 0; it < (KSTEPS - 2) / (int)NS; ++it) {
#pragma unroll
        for (int s = 2; s < 2 + (int)NS; s += 2) {
            step(s, wf[0], xf[0], wf[1], xf[1]);
            step(s + 1, wf[1], xf[1], wf[0], xf[0]);
        }
    }
    static_assert((KSTEPS - 2) % NS == 0 && NS % 2 == 0, "trip count");
    mma(wf[1], xf[1]);
}

__global__ void __launch_bounds__(NTHREADS) gemm_a2a_kernel(Params P) {
    extern __shared__ __attribute__((aligned(16))) uint8_t smem_[];
    LAS uint8_t* smem = (LAS uint8_t*)smem_;
    const int lane = threadIdx.x & 63;
    const int wave = rfl(threadIdx.x >> 6);
    const int grp = blockIdx.x & 7, idx = blockIdx.x >> 3, panel = idx & 3, rgp = RGP(idx >> 2);
    const uint8_t* px = P.x;
    const uint8_t* pw = P.w;
    unsigned rank = P.rank;
    uint8_t* myws = P.wsd[7];
    uint8_t* pws = P.wsd[grp];
    float* pout = P.outd[grp];
    asm volatile("" : "+s"(px), "+s"(pw), "+s"(rank), "+s"(myws), "+s"(pws), "+s"(pout));
    rank &= 7u;
    const bool local = grp == 7;
    const bool pusher = !local && rgp == 7;
#define CTR ((GAS unsigned*)(myws + WS_CALL + (size_t)blockIdx.x * 128))
#define TMO ((GAS unsigned*)(myws + WS_TMO))
#define PACE ((GAS u64*)(myws + WS_PACE + grp * 128))
#define LOAD_CALL() do { call = *(const CAS unsigned*)(uintptr_t)CTR + 1; asm volatile("" : "+s"(call)); } while (0)
    unsigned call = 0;
    float scale = 0.f;
    const int g = lane >> 4, m = lane & 15;

    if (__builtin_expect(pusher, PEXP)) {
        constexpr int XI = RR / 8;
        const unsigned xo = (panel * 8 + wave) * XI * 1024;
        const unsigned lb = wave * XI * 1024;
        __amdgpu_buffer_rsrc_t rp = make_rsrc(px, RR * KD);
        asm volatile("s_nop 4" : "+s"(rp) :: "memory");
        if (wave == 7 && lane == 0) {
            __hip_atomic_fetch_max(PACE, (rtime() + XLEAD) << 6, __ATOMIC_RELAXED, __HIP_MEMORY_SCOPE_WORKGROUP);
            __hip_atomic_fetch_add(PACE, (u64)(RR * TAU), __ATOMIC_RELAXED, __HIP_MEMORY_SCOPE_WORKGROUP);
        }
        dma16(lb, lane * 16, rp, xo);
        dma16(lb + 1024, lane * 16, rp, xo + 1024);
        wg_barrier();
        dma16(lb + 2048, lane * 16, rp, xo + 2048);
        dma16(lb + 3072, lane * 16, rp, xo + 3072);
        LOAD_CALL();
        uint8_t* xd = pws + WS_XST + (size_t)(rank * 2 + (call & 1)) * XST_SLOT + xo + lane * 16;
        LAS uint8_t* ls = smem + lb + lane * 16;
        wait_vm<3>();
        v4f v = *(LAS v4f*)ls;
#pragma unroll
        for (int i = 0; i < XI; ++i) {
            if (i >= 2) {
                if (i + 3 < XI) wait_vm<3>(); else if (i + 2 < XI) wait_vm<2>(); else wait_vm<1>();
            }
            st16_sys(xd + i * 1024, v);
            LOGT(i);
            if (i + 4 < XI) dma16(lb + (i + 4) * 1024, lane * 16, rp, xo + (i + 4) * 1024);
            if (i == 0) wait_vm<4>();
            if (i == 1) wait_vm<5>();
            if (i + 1 < XI) v = *(LAS v4f*)(ls + (i + 1) * 1024);
        }
        static_assert(XI >= 6 && XI <= 16, "wait counts");
        scale = *(const CAS float*)P.sx * *(const CAS float*)P.sw;
        wait_vm<0>();
        if (lane == 0)
            __hip_atomic_store((GAS unsigned*)(pws + WS_XFLAG + rank * 128) + panel * 8 + wave, call, __ATOMIC_RELAXED, __HIP_MEMORY_SCOPE_SYSTEM);
        STAMP(40);
        LOGOUT(0);
    } else {
        const int h0 = local ? 64 : (int)(hpack() >> (8 * rgp)) & 255;
        const int row0 = local ? rgp * 64 : RR + 2 * ((int)(rpack() >> (8 * rgp)) & 255);
        if (!local && (panel >= EP || (!SEMI && rgp >= NE))) {
            constexpr int NH = SEMI ? (4 - EP) * 7 : 28 - EP * NE, NP = 16 * EP + ESUM / 8, PH = NH * 8 / NP;
            static_assert(PH >= 1, "helpers");
            const int hi = panel >= EP ? (panel - EP) * 7 + rgp : (4 - EP) * 7 + panel * NF + rgp - NE;
            const int id = wave * NH + hi;
            if (id < NP * PH) {
                const int ph = id / NP, i = id - ph * NP;
                const bool isw = i < 16 * EP;
                const unsigned cb = ((rank + 1 + grp) & 7u) * NC;
                __amdgpu_buffer_rsrc_t rs = make_rsrc(isw ? pw + cb : px + RR * KD, isw ? (unsigned)(KD * ND) - cb : (unsigned)(ESUM * KD));
                const unsigned vo = ((isw ? i & 15 : i - 16 * EP) * 8 + (lane >> 3)) * KD + (isw ? (i >> 4) * 128 : 0) + (lane & 7) * 16;
                const unsigned st = isw ? 128u * ND : 128u;
                asm volatile("s_nop 4" : "+s"(rs) :: "memory");
#pragma unroll 1
                for (unsigned sg = ph; sg < KSTEPS; sg += PH) { dma16(JUNK_OFF, vo, rs, sg * st); wait_vm<PWIN>(); }
            }
            wg_barrier();
        }
        if (wave >= 4) {
            const int w = wave - 4;
            const unsigned coff = ((rank + 1 + grp) & 7u) * NC + panel * 128;
            const __amdgpu_buffer_rsrc_t rw = make_rsrc(pw + coff, (unsigned)(KD * ND) - coff);
            const __amdgpu_buffer_rsrc_t rx = make_rsrc(px + (size_t)row0 * KD, h0 * KD);
            const int nf = local ? NFL : NF;
            const int f = rgp - (local ? 8 - NFL : NE);
            unsigned first = 0;
            if (f >= 0) first = local ? comb(NFL) << ((f + panel) % NFL) : comb(NF ? NF : 1) << ((f + panel) % (NF ? NF : 1));
            if ((SLOWNF == 1 || (SLOWNF == 2 && local)) && ((((unsigned)(uintptr_t)pw + coff) >> 7) & 7) == 3) first = (local ? comb(8) : comb(7)) << rgp;
            if (!local) first = panel < EP && (SEMI || rgp < NE) ? 0u : comb(7) << ((rgp + 2 * panel) % 7);
            const int nb = local ? 16 : (ESUM > 128 ? 16 : ESUM / 8), erows = local ? (8 - NFL) * 64 : ESUM;
            const int q = (f * 4 + panel) * 4 + w;
            const int qs = local ? q >> 4 : q / (ESUM > 128 ? 16 : (ESUM > 0 ? ESUM / 8 : 1)), qb = q - qs * nb;
            const __amdgpu_buffer_rsrc_t rp = make_rsrc(px + (local ? 0 : RR * KD), (unsigned)(erows * KD));
            const unsigned vp = (qb * 8 + (lane >> 3)) * KD + qs * 128 + (lane & 7) * 16;
            const int npf = !NOXPF && XPF_ON && f >= 0 && qs < nf && erows > 0 ? (256 + qb * 8 < erows ? 3 : 128 + qb * 8 < erows ? 2 : 1) : 0;
            const int nxw = (8 * w < h0) + (8 * w + 32 < h0) + (8 * w + 64 < h0);
            u64 ts[3];
            if (HMAX > 64 && nxw == 3) load_tile<3, false, AH1>(w, lane, rw, rx, first, rp, vp, npf, nf * 128u, ts);
            else if (nxw == 2) load_tile<2, false, AH1>(w, lane, rw, rx, first, rp, vp, npf, nf * 128u, ts);
            else if (nxw == 1) load_tile<1, false, AH1>(w, lane, rw, rx, first, rp, vp, npf, nf * 128u, ts);
            else load_tile<0, false, AH1>(w, lane, rw, rx, first, rp, vp, npf, nf * 128u, ts);
            if (!local && w == 0) {
                if (lane == 0) {
                    const u64 now = (rtime() + LEAD) << 6;
                    __hip_atomic_fetch_max(PACE, now, __ATOMIC_RELAXED, __HIP_MEMORY_SCOPE_WORKGROUP);
                    const u64 t = __hip_atomic_fetch_add(PACE, (u64)(h0 / 2) * TAU, __ATOMIC_RELAXED, __HIP_MEMORY_SCOPE_WORKGROUP);
                    *(volatile LAS u64*)(smem + CTL_OFF + 8) = t - now < (4000ull << 6) ? t : now;
                }
                wait_lds();
            }
            LOAD_CALL();
        } else {
            v4f acc[5][2];
            float sa = *(const CAS float*)P.sx, sb = *(const CAS float*)P.sw;
            LOAD_CALL();
            asm volatile("" : "+s"(sa), "+s"(sb));
            scale = sa * sb;
            if (HMAX > 64 && h0 > 64) consume_tile<5>(smem, wave, lane, acc);
            else if (h0 > 48) consume_tile<4>(smem, wave, lane, acc);
            else if (h0 > 32) consume_tile<3>(smem, wave, lane, acc);
            else if (h0 > 16) consume_tile<2>(smem, wave, lane, acc);
            else consume_tile<1>(smem, wave, lane, acc);
            STAMP(16 + wave);
            if (!local) {
#pragma unroll
                for (int u = 0; u < 5; ++u)
                    if (u * 16 < h0) {
                        const int r = u * 16 + m;
                        LAS uint8_t* d = smem + STG_OFF + r * 512 + wave * 128;
                        *(LAS v4f*)(d + ((g ^ (r & 7)) << 4)) = acc[u][0] * scale;
                        *(LAS v4f*)(d + (((4 + g) ^ (r & 7)) << 4)) = acc[u][1] * scale;
                    }
                wait_lds();
            } else {
                float* o = pout + ((size_t)rank * MS + row0 + m) * NC + panel * 128 + wave * 32 + g * 4;
#pragma unroll
                for (int u = 0; u < 4; ++u) {
                    st16_sys(o + (size_t)u * 16 * NC, acc[u][0] * scale);
                    st16_sys(o + (size_t)u * 16 * NC + 16, acc[u][1] * scale);
                }
            }
        }

        if (!local) {
            float* o = pout + ((size_t)rank * MS + row0 + (lane >> 5)) * NC + panel * 128 + (lane & 31) * 4;
            wg_barrier();
            STAMP(48);
            const u64 st = *(volatile LAS u64*)(smem + CTL_OFF + 8);
            u64 slot = (((u64)rfl((unsigned)(st >> 32)) << 32) | rfl((unsigned)st)) + wave * TAU;
#pragma unroll 1
            for (int it = wave; it < h0 / 2; it += 8) {
                const int r = it * 2 + (lane >> 5);
                const v4f v = *(LAS v4f*)(smem + STG_OFF + r * 512 + (((lane & 31) ^ (r & 7)) << 4));
                for (;;) {
                    const long long d = (long long)(slot >> 6) - (long long)rtime();
                    if (d <= 0) break;
                    if (d > 30) __builtin_amdgcn_s_sleep(8);
                }
                st16_sys(o + (size_t)it * 2 * NC, v);
                LOGT(it / 8);
                slot += 8 * TAU;
            }
            wait_vm<0>();
            LOGOUT(1);
            wg_barrier();
            STAMP(52);
            if (wave == 0 && lane == 0) {
                __hip_atomic_store((GAS unsigned*)(pws + WS_DONE + rank * 128) + rgp * 4 + panel, call, __ATOMIC_RELAXED, __HIP_MEMORY_SCOPE_SYSTEM);
                *CTR = call;
            }
            if (rgp == 0) {
                const unsigned c3 = rank * NC + 3 * 128;
                if (((((unsigned)(uintptr_t)pw + c3) >> 7) & 7) == 3) {
                    __amdgpu_buffer_rsrc_t rw = make_rsrc(pw + c3, (unsigned)(KD * ND) - c3);
                    asm volatile("s_nop 4" : "+s"(rw) :: "memory");
                    const unsigned vl = ((panel * 8 + wave) * 8 + (lane >> 3)) * ND + (lane & 7) * 16;
#pragma unroll 1
                    for (int i = HELP0; i < 16; ++i) dma16(JUNK_OFF, vl, rw, i * (256u * ND));
                }
            }
            return;
        }
        if (rgp == 7) {
            wait_vm<0>();
            wg_barrier();
            if (wave == 7 && lane == 0) *CTR = call;
            if (panel == 0 && wave < 7) {
                const unsigned src = (rank + 1 + wave) & 7u;
                const bool ok = wait_flags((const GAS unsigned*)(myws + WS_DONE + src * 128), 0xf0000000u, call, TMO, lane);
                if (!ok && lane == 0)
                    __hip_atomic_store((GAS float*)pout + ((size_t)rank * MS + row0) * NC, __builtin_nanf(""), __ATOMIC_RELAXED, __HIP_MEMORY_SCOPE_SYSTEM);
                STAMP(45 + wave);
            }
            return;
        }
    }

    const unsigned src = (rank + 1 + (local ? rgp : grp)) & 7u;
    const int r2 = local ? 0 : RA, h2 = local ? RA : RB;
    if (wave == 4) {
        const bool ok = wait_flags((const GAS unsigned*)(myws + WS_XFLAG + src * 128), 0u, call, TMO, lane);
        *(volatile LAS unsigned*)(smem + CTL_OFF) = ok ? 0u : 1u;
        wait_lds();
        STAMP(44);
    }
    wg_barrier();
    if (wave >= 4) {
        const int w = wave - 4;
        const unsigned coff = rank * NC + panel * 128;
        const __amdgpu_buffer_rsrc_t rw = make_rsrc(pw + coff, (unsigned)(KD * ND) - coff);
        const __amdgpu_buffer_rsrc_t rx = make_rsrc(myws + WS_XST + (size_t)(src * 2 + (call & 1)) * XST_SLOT + (size_t)r2 * KD, h2 * KD);
        const int nxw = (8 * w < h2) + (8 * w + 32 < h2) + (8 * w + 64 < h2);
        const unsigned first2 = ROT2 == 1 && local ? ~0u << (8 * panel) : ROT2 == 2 && local ? comb(4) << panel : 0u;
        if (RM > 64 && nxw == 3) load_tile<3, true, AH2>(w, lane, rw, rx, first2, rx, 0, 0, KD);
        else if (nxw == 2) load_tile<2, true, AH2>(w, lane, rw, rx, first2, rx, 0, 0, KD);
        else if (nxw == 1) load_tile<1, true, AH2>(w, lane, rw, rx, first2, rx, 0, 0, KD);
        else load_tile<0, true, AH2>(w, lane, rw, rx, first2, rx, 0, 0, KD);
    } else {
        if (wave == 0 && lane == 0) *CTR = call;
        v4f acc[5][2];
        if (local) consume_tile<(RA + 15) / 16>(smem, wave, lane, acc);
        else consume_tile<(RB + 15) / 16>(smem, wave, lane, acc);
        STAMP(32 + wave);
        if (*(volatile LAS unsigned*)(smem + CTL_OFF)) scale = __builtin_nanf("");
        float* o = P.outd[7] + ((size_t)src * MS + r2 + m) * NC + panel * 128 + wave * 32 + g * 4;
#pragma unroll
        for (int u = 0; u < (RM + 15) / 16; ++u)
            if (u * 16 + m < h2) {
                st16_sys(o + (size_t)u * 16 * NC, acc[u][0] * scale);
                st16_sys(o + (size_t)u * 16 * NC + 16, acc[u][1] * scale);
            }
    }
}

}

extern "C" void kernel_launch(int rank, int nranks, ncclComm_t comm,
                              hipStream_t stream,
                              void* const* d_in, const int* in_sizes, int n_in,
                              void* d_out, int out_size,
                              const int* in_dtypes, int out_dtype,
                              void* const* d_out_all,
                              void* const* d_ws_all, size_t ws_bytes) {
    (void)comm; (void)in_dtypes; (void)out_dtype; (void)d_out;
    if (nranks != NR || (unsigned)rank >= (unsigned)NR || n_in != 4 || in_sizes[0] != MS * KD || in_sizes[1] != KD * ND
        || out_size != NR * MS * NC || ws_bytes < WS_NEED) {
        fprintf(stderr, "kernel_launch: unexpected problem shape\n");
        return;
    }
    static bool attr_set = false;
    if (!attr_set) {
        HIP_CHECK(hipFuncSetAttribute((const void*)gemm_a2a_kernel, hipFuncAttributeMaxDynamicSharedMemorySize, LDS_BYTES));
        attr_set = true;
    }
    Params P;
    P.x = (const uint8_t*)d_in[0];
    P.w = (const uint8_t*)d_in[1];
    P.sx = (const float*)d_in[2];
    P.sw = (const float*)d_in[3];
    for (int g = 0; g < NR; ++g) {
        const int p = (rank + 1 + g) % NR;
        P.outd[g] = (float*)d_out_all[p];
        P.wsd[g] = (uint8_t*)d_ws_all[p];
    }
    P.rank = (unsigned)rank;
    P.pad = 0;
    hipLaunchKernelGGL(gemm_a2a_kernel, dim3(NWG), dim3(NTHREADS), LDS_BYTES, stream, P);
    HIP_CHECK(hipGetLastError());
}
```

```cpp
#include <hip/hip_runtime.h>
#include <rccl/rccl.h>
#include <cstdio>
#include <cstdint>
#include <cstddef>

#define HIP_CHECK(expr) do { hipError_t _e = (expr); if (_e != hipSuccess) { \
    fprintf(stderr, "HIP error %s at %s:%d\n", hipGetErrorString(_e), __FILE__, __LINE__); \
    return; } } while (0)

namespace {

typedef int v2i __attribute__((ext_vector_type(2)));
typedef int v4i __attribute__((ext_vector_type(4)));
typedef int v8i __attribute__((ext_vector_type(8)));
typedef float v4f __attribute__((ext_vector_type(4)));
typedef unsigned long long u64;
#define LAS __attribute__((address_space(3)))
#define GAS __attribute__((address_space(1)))
#define CAS __attribute__((address_space(4)))

constexpr int NR = 8;
constexpr int KD = 4096;
constexpr int ND = 4096;
constexpr int MS = 512;
constexpr int NC = 512;
#define RR 96
#define RA 40
constexpr int RB = RR - RA;
constexpr int RM = RA > RB ? RA : RB;
constexpr int HMAX = 80;
#define HLIST 32, 48, 64, 64, 64, 64, 80
#define DLIST 0, 0, 0, 10, 30, 45, 50
#define LLIST 0, 0, 0, 0, 0, 0, 0
constexpr int HL[7] = {HLIST}, DL[7] = {DLIST}, LL[7] = {LLIST};
constexpr int hsum(int n) { int t = 0; for (int i = 0; i < n; ++i) t += HL[i]; return t; }
constexpr u64 pack(const int (&a)[7]) { u64 t = 0; for (int i = 0; i < 7; ++i) t |= (u64)a[i] << (8 * i); return t; }
constexpr u64 rpack() { u64 t = 0; for (int i = 0; i < 7; ++i) t |= (u64)(hsum(i) / 2) << (8 * i); return t; }
constexpr bool hok() { for (int i = 0; i < 7; ++i) if (HL[i] < 2 || HL[i] > HMAX || HL[i] % 2) return false; return true; }
static_assert(RR % 8 == 0 && RA >= 8 && RB >= 8 && RM <= HMAX && hok() && hsum(7) == MS - RR, "shape");
#define EP 2
#define PWIN 6
#define AH1 3
#define TAU 112
#define XTAU 104
#define LEAD 25
constexpr int NTHREADS = 512;
constexpr int NWG = 256;
constexpr int KSTEPS = KD / 128;

constexpr unsigned SLOT = 16384 + HMAX * 128, NS = 6;
constexpr unsigned STG_OFF = 2 * SLOT;
static_assert((KSTEPS - 1) % NS == 1, "the last two stages use slots 0 and 1");
constexpr unsigned CTL_OFF = NS * SLOT;
constexpr unsigned JUNK_OFF = CTL_OFF + 64;
constexpr unsigned LDS_BYTES = JUNK_OFF + 1024;

constexpr size_t WS_STAMP = 131072;
constexpr size_t WS_SRC = 2u << 20, SRC_BYTES = 512u << 10;
constexpr size_t A_DONE = 0;
constexpr size_t A_XFLAG = 256;
constexpr size_t A_ROWS = 4096;
constexpr size_t C_TMO = 0;
constexpr size_t C_PACE = 1024;
constexpr size_t C_CALL = 4096;
static_assert(A_ROWS + RR * KD <= SRC_BYTES, "area");
constexpr size_t WS_NEED = WS_SRC + NR * SRC_BYTES;
constexpr u64 DEADLINE_TICKS = 40000000ull;
constexpr unsigned WRANGE = KD * ND - (ND - NC);

struct Group {
    const uint8_t* wcol;
    uint8_t* parea;
    float* prows;
    uint8_t* marea;
};
struct Params {
    const uint8_t* x;
    uint8_t* ctl;
    const float* sx;
    const float* sw;
    Group g[NR];
    float* mrows[NR];
    const uint8_t* w;
    uint8_t* ws;
    unsigned rank;
};

constexpr unsigned comb(int n) { unsigned m = 0; for (int i = 0; i < 32; i += n) m |= 1u << i; return m; }

__device__ __forceinline__ unsigned rfl(unsigned v) { return __builtin_amdgcn_readfirstlane(v); }
__device__ __forceinline__ u64 rtime() { return __builtin_amdgcn_s_memrealtime(); }
__device__ __forceinline__ void delay(u64 t0, unsigned ticks) { while (rtime() - t0 < ticks) __builtin_amdgcn_s_sleep(4); }

__device__ __forceinline__ __amdgpu_buffer_rsrc_t make_rsrc(u64 p, unsigned bytes) {
    return __builtin_amdgcn_make_buffer_rsrc((void*)p, (short)0, (int)bytes, 0x00020000);
}

template <bool SYS = false>
__device__ __forceinline__ void dma16(unsigned lds, unsigned voff, __amdgpu_buffer_rsrc_t rs, unsigned soff) {
    if (SYS) asm volatile("s_mov_b32 m0, %0\n\ts_nop 0\n\tbuffer_load_dwordx4 %1, %2, %3 offen sc0 sc1 lds"
                          :: "s"(lds), "v"(voff), "s"(rs), "s"(soff) : "memory");
    else     asm volatile("s_mov_b32 m0, %0\n\ts_nop 0\n\tbuffer_load_dwordx4 %1, %2, %3 offen lds"
                          :: "s"(lds), "v"(voff), "s"(rs), "s"(soff) : "memory");
}
template <int N> __device__ __forceinline__ void wait_vm() { asm volatile("s_waitcnt vmcnt(%0)" :: "i"(N) : "memory"); }
__device__ __forceinline__ void wg_barrier() { asm volatile("s_barrier" ::: "memory"); }
__device__ __forceinline__ void wait_lds() { asm volatile("s_waitcnt lgkmcnt(0)" ::: "memory"); }
__device__ __forceinline__ void st16_sys(u64 p, v4f v) {
    asm volatile("global_store_dwordx4 %0, %1, off sc0 sc1\n\ts_nop 1" :: "v"(p), "v"(v) : "memory");
}
__device__ __forceinline__ void flag_sys(u64 p, unsigned v) {
    __hip_atomic_store((GAS unsigned*)p, v, __ATOMIC_RELAXED, __HIP_MEMORY_SCOPE_SYSTEM);
}

#define STAMP(i) do {} while (0)
#define LOGT(k) do {} while (0)
#define LOGOUT(half) do {} while (0)

__device__ __forceinline__ bool wait_flags(u64 flags, int n, unsigned call, u64 tmo, int lane) {
    const GAS unsigned* p = (const GAS unsigned*)(lane < n ? flags + 4 * lane : tmo);
    const u64 t0 = rtime();
    for (;;) {
        const u64 now = rtime();
        const unsigned v = __hip_atomic_load(p, __ATOMIC_RELAXED, __HIP_MEMORY_SCOPE_SYSTEM);
        const u64 ge = __ballot(v >= call), eq = __ballot(v == call);
        if ((eq >> 63) || now - t0 > DEADLINE_TICKS) break;
        if ((ge | (~0ull << n)) == ~0ull) return true;
    }
    if (lane == 0) __hip_atomic_store((GAS unsigned*)tmo, call, __ATOMIC_RELAXED, __HIP_MEMORY_SCOPE_AGENT);
    return false;
}

template <int NXW, bool SYS>
__device__ __forceinline__ void load_tile(int w, int lane, unsigned woff, unsigned xoff, unsigned first,
                                          u64 pw, u64 px, unsigned xbytes, u64* ts = nullptr) {
    constexpr int CNT = 4 + NXW, AHEAD = AH1;
    const unsigned v0 = (lane >> 3) * KD + (((lane & 7) ^ ((lane >> 4) & 3)) << 4);
    unsigned vw[4];
#pragma unroll
    for (int j = 0; j < 4; ++j) vw[j] = (v0 ^ (((j >> 1) & 1) << 6)) + (4 * w + j) * (8u * ND) - j * 1024u + woff;
    unsigned vx0 = (v0 ^ ((w & 1) << 6)) + w * (8u * KD) + xoff, vx1 = vx0 + 32u * KD, vx2 = vx0 + 64u * KD;
    unsigned ioff = w * 1024;
    u64 todo = ((u64)~first << 32) | first;
    asm volatile("" : "+v"(vw[0]), "+v"(vw[1]), "+v"(vw[2]), "+v"(vw[3]), "+v"(vx0), "+v"(vx1), "+v"(vx2), "+s"(ioff), "+s"(todo));
    __builtin_amdgcn_sched_barrier(0);
    const __amdgpu_buffer_rsrc_t rw = make_rsrc(pw, WRANGE), rx = make_rsrc(px, xbytes);
#define STEP_W "buffer_load_dwordx4 %4, %11, %13 offen lds\n\t" \
               "buffer_load_dwordx4 %5, %11, %13 offen offset:1024 lds\n\t" \
               "buffer_load_dwordx4 %6, %11, %13 offen offset:2048 lds\n\t" \
               "buffer_load_dwordx4 %7, %11, %13 offen offset:3072 lds"
#define STEP_X(m, v, pol) "\n\ts_mov_b32 m0, %" m "\n\ts_nop 0\n\tbuffer_load_dwordx4 %" v ", %12, %14 offen" pol " lds"
#define STEP_OPS :: "s"(ioff + w * 3072), "s"(ioff + 16384), "s"(ioff + 20480), "s"(ioff + 24576), \
                    "v"(vw[0]), "v"(vw[1]), "v"(vw[2]), "v"(vw[3]), "v"(vx0), "v"(vx1), "v"(vx2), \
                    "s"(rw), "s"(rx), "s"(ks * (128u * ND)), "s"(ks * 128u), "i"(CNT * (AHEAD - 1)) : "memory"
#define STEP_P(head, pol) do { \
        if (NXW == 0) asm volatile(head STEP_W STEP_OPS); \
        if (NXW == 1) asm volatile(head STEP_W STEP_X("1", "8", pol) STEP_OPS); \
        if (NXW == 2) asm volatile(head STEP_W STEP_X("1", "8", pol) STEP_X("2", "9", pol) STEP_OPS); \
        if (NXW == 3) asm volatile(head STEP_W STEP_X("1", "8", pol) STEP_X("2", "9", pol) STEP_X("3", "10", pol) STEP_OPS); \
    } while (0)
#define STEP(head) do { \
        const unsigned ks = __builtin_ctzll(todo) & 31; \
        todo &= todo - 1; \
        if (SYS) STEP_P(head, " sc0 sc1"); else STEP_P(head, ""); \
        ioff += SLOT; \
        if (ioff >= SLOT * NS) ioff -= SLOT * NS; \
    } while (0)
#pragma unroll 1
    for (int i = 0; i < AHEAD; ++i) STEP("s_mov_b32 m0, %0\n\ts_nop 0\n\t");
#pragma unroll 1
    while (todo) STEP("s_mov_b32 m0, %0\n\ts_waitcnt vmcnt(%15)\n\ts_barrier\n\t");
    static_assert(AHEAD >= 2 && AHEAD <= 5 && AHEAD < (int)NS, "drain");
    if (AHEAD > 4) { wait_vm<CNT * 4>(); wg_barrier(); }
    if (AHEAD > 3) { wait_vm<CNT * 3>(); wg_barrier(); }
    if (AHEAD > 2) { wait_vm<CNT * 2>(); wg_barrier(); }
    wait_vm<CNT>(); wg_barrier();
    wait_vm<0>(); wg_barrier();
}
template <bool SYS>
__device__ __forceinline__ void load_tile_h(int h, int w, int lane, unsigned woff, unsigned xoff, unsigned first,
                                            u64 pw, u64 px, unsigned xbytes, u64* ts = nullptr) {
    const int nxw = (8 * w < h) + (8 * w + 32 < h) + (8 * w + 64 < h);
    if (nxw == 2) load_tile<2, SYS>(w, lane, woff, xoff, first, pw, px, xbytes, ts);
    else if (nxw == 1) load_tile<1, SYS>(w, lane, woff, xoff, first, pw, px, xbytes, ts);
    else if (nxw == 3) load_tile<3, SYS>(w, lane, woff, xoff, first, pw, px, xbytes, ts);
    else load_tile<0, SYS>(w, lane, woff, xoff, first, pw, px, xbytes, ts);
}

template <int NU>
__device__ __forceinline__ void consume_tile(LAS uint8_t* smem, int c, int lane, v4f (&acc)[5][2]) {
    const int g = lane >> 4, q = (lane & 15) >> 1, p = lane & 1, m = lane & 15;
    const unsigned fw = ((q >> 1) & 3) | ((g & 1) << 2);
    const unsigned w0 = (16 * g + q) * 128 + (((2 * c) ^ fw) << 4) + p * 8;
    const unsigned x0 = 16384 + m * 128 + ((g ^ ((m >> 1) & 7)) << 4);
#pragma unroll
    for (int u = 0; u < 5; ++u) { acc[u][0] = v4f{0, 0, 0, 0}; acc[u][1] = v4f{0, 0, 0, 0}; }
    v8i wf[2][2], xf[2][NU];
    auto load = [&](int s, v8i (&wv)[2], v8i (&xv)[NU]) {
        LAS uint8_t* sl = smem + (s % NS) * SLOT;
#pragma unroll
        for (int cb = 0; cb < 2; ++cb) {
            LAS uint8_t* b = sl + (w0 ^ (cb << 4));
            v2i t0 = __builtin_amdgcn_ds_read_tr8_b64_v2i32((LAS v2i*)(b));
            v2i t1 = __builtin_amdgcn_ds_read_tr8_b64_v2i32((LAS v2i*)(b + 1024));
            v2i t2 = __builtin_amdgcn_ds_read_tr8_b64_v2i32((LAS v2i*)(b + 8192));
            v2i t3 = __builtin_amdgcn_ds_read_tr8_b64_v2i32((LAS v2i*)(b + 9216));
            wv[cb] = v8i{t0[0], t0[1], t1[0], t1[1], t2[0], t2[1], t3[0], t3[1]};
        }
#pragma unroll
        for (int u = 0; u < NU; ++u) {
            v4i a = *(LAS v4i*)(sl + x0 + u * 2048);
            v4i b = *(LAS v4i*)(sl + (x0 ^ 64) + u * 2048);
            xv[u] = v8i{a[0], a[1], a[2], a[3], b[0], b[1], b[2], b[3]};
        }
    };
    auto mma = [&](v8i (&wv)[2], v8i (&xv)[NU]) {
#pragma unroll
        for (int u = 0; u < NU; ++u)
#pragma unroll
            for (int cb = 0; cb < 2; ++cb)
                acc[u][cb] = __builtin_amdgcn_mfma_scale_f32_16x16x128_f8f6f4(wv[cb], xv[u], acc[u][cb], 1, 0, 0, 0, 0, 0);
    };
    auto step = [&](int s, v8i (&wn)[2], v8i (&xn)[NU], v8i (&wo)[2], v8i (&xo)[NU]) {
        wg_barrier();
        __builtin_amdgcn_sched_barrier(0);
        load(s, wn, xn);
        mma(wo, xo);
        constexpr int PER = (8 + 2 * NU + 2 * NU - 1) / (2 * NU);
#pragma unroll
        for (int i = 0; i < 2 * NU; ++i) {
            __builtin_amdgcn_sched_group_barrier(0x100, PER, 0);
            __builtin_amdgcn_sched_group_barrier(0x8, 1, 0);
        }
        __builtin_amdgcn_sched_barrier(0);
        wait_lds();
    };
    wg_barrier();
    load(0, wf[0], xf[0]);
    wait_lds();
    step(1, wf[1], xf[1], wf[0], xf[0]);
#pragma unroll 1
    for (int it = 0; it < (KSTEPS - 2) / (int)NS; ++it) {
#pragma unroll
        for (int s = 2; s < 2 + (int)NS; s += 2) {
            step(s, wf[0], xf[0], wf[1], xf[1]);
            step(s + 1, wf[1], xf[1], wf[0], xf[0]);
        }
    }
    static_assert((KSTEPS - 2) % NS == 0 && NS % 2 == 0, "trip count");
    mma(wf[1], xf[1]);
}

__global__ void __launch_bounds__(NTHREADS) gemm_a2a_kernel(Params P) {
    extern __shared__ __attribute__((aligned(16))) uint8_t smem_[];
    LAS uint8_t* smem = (LAS uint8_t*)smem_;
    const int grp = blockIdx.x & 7, idx = blockIdx.x >> 3, panel = idx & 3, rgp = ((idx >> 2) + 7) & 7;
    const CAS char* ka = (const CAS char*)__builtin_amdgcn_kernarg_segment_ptr();
#define KARG(off) (*(const volatile CAS u64*)(ka + (off)))
    const u64 px = KARG(offsetof(Params, x)), ctl = KARG(offsetof(Params, ctl));
    const u64 psx = KARG(offsetof(Params, sx)), psw = KARG(offsetof(Params, sw));
    const u64 wcol = KARG(offsetof(Params, g) + grp * sizeof(Group)), parea = KARG(offsetof(Params, g) + grp * sizeof(Group) + 8);
    const u64 prows = KARG(offsetof(Params, g) + grp * sizeof(Group) + 16);
    const int lane = threadIdx.x & 63;
    const int wave = rfl(threadIdx.x >> 6);
    const bool local = grp == 7;
    const bool pusher = !local && rgp == 7;
#define CTR (ctl + C_CALL + blockIdx.x * 128)
#define PACE ((GAS u64*)(ctl + C_PACE + grp * 128))
#define SLOAD(a) (*(const volatile CAS unsigned*)(a))
#define TOUCH(a) do { unsigned t_ = SLOAD(a); asm volatile("" :: "s"(t_)); } while (0)
    unsigned call = 0;
    float scale = 0.f;
    const int g = lane >> 4, m = lane & 15;

    if (__builtin_expect(pusher, 1)) {
        constexpr int XI = RR / 8;
        const unsigned xo = (panel * 8 + wave) * XI * 1024;
        const unsigned lb = wave * XI * 1024;
        const __amdgpu_buffer_rsrc_t rp = make_rsrc(px, RR * KD);
        dma16(lb, lane * 16, rp, xo);
        dma16(lb + 1024, lane * 16, rp, xo + 1024);
        wg_barrier();
        dma16(lb + 2048, lane * 16, rp, xo + 2048);
        dma16(lb + 3072, lane * 16, rp, xo + 3072);
        const u64 xd = parea + A_ROWS + xo + lane * 16;
        LAS uint8_t* ls = smem + lb + lane * 16;
        wait_vm<3>();
        v4f v = *(LAS v4f*)ls;
#pragma unroll
        for (int i = 0; i < XI; ++i) {
            if (i >= 2) {
                if (i + 3 < XI) wait_vm<3>(); else if (i + 2 < XI) wait_vm<2>(); else wait_vm<1>();
            }
            st16_sys(xd + i * 1024, v);
            LOGT(i);
            if (i + 4 < XI) dma16(lb + (i + 4) * 1024, lane * 16, rp, xo + (i + 4) * 1024);
            if (i == 1 && wave == 7 && lane == 0) {
                __hip_atomic_fetch_max(PACE, rtime() << 6, __ATOMIC_RELAXED, __HIP_MEMORY_SCOPE_WORKGROUP);
                __hip_atomic_fetch_add(PACE, (u64)(RR * XTAU), __ATOMIC_RELAXED, __HIP_MEMORY_SCOPE_WORKGROUP);
            }
            if (i == 0) wait_vm<4>();
            if (i == 1) wait_vm<5>();
            if (i + 1 < XI) v = *(LAS v4f*)(ls + (i + 1) * 1024);
        }
        static_assert(XI >= 6 && XI <= 16, "wait counts");
        scale = __uint_as_float(SLOAD(psx)) * __uint_as_float(SLOAD(psw));
        call = SLOAD(CTR) + 1;
        wait_vm<0>();
        if (lane == 0) flag_sys(parea + A_XFLAG + (panel * 8 + wave) * 4, call);
        STAMP(40);
        LOGOUT(0);
    } else {
        const int h0 = local ? 64 : (int)(pack(HL) >> (8 * rgp)) & 255;
        const int row0 = local ? rgp * 64 : RR + 2 * ((int)(rpack() >> (8 * rgp)) & 255);
        if (!local && panel >= EP) {
            constexpr int NH = (4 - EP) * 7, NP = 16 * EP + (MS - RR) / 8;
            static_assert(NH * 8 - NP >= 20, "spare waves");
            const int id = wave * NH + (panel - EP) * 7 + rgp;
            const u64 t0 = pack(LL) ? rtime() : 0;
            if (id < NP) {
                const bool isw = id < 16 * EP;
                unsigned vo = ((isw ? id & 15 : id - 16 * EP + RR / 8) * 8 + (lane >> 3)) * KD + (isw ? (id >> 4) * 128 : 0) + (lane & 7) * 16;
                unsigned st = isw ? 128u * ND : 128u, sz = isw ? WRANGE : (unsigned)(MS * KD);
                asm volatile("" : "+v"(vo), "+s"(st), "+s"(sz));
                __builtin_amdgcn_sched_barrier(0);
                const __amdgpu_buffer_rsrc_t rs = make_rsrc(isw ? wcol : px, sz);
#pragma unroll 1
                for (unsigned sg = 0; sg < KSTEPS; ++sg) { dma16(JUNK_OFF, vo, rs, sg * st); wait_vm<PWIN>(); }
            } else if (id - NP < 15) {
                const int j = id - NP;
                const u64 a = j < 1 ? parea + 4
                            : j < 9 ? (u64)P.w + ((u64)j << 21) - 4
                            : j < 10 ? px + MS * KD - 4
                            : j < 12 ? prows + (j - 10) * (MS * NC * 4 - 4)
                            : j < 13 ? (u64)P.g[grp].marea
                            : (u64)P.mrows[grp] + (j & 1) * (RR * NC * 4);
                if (j != 0 && j != 9) delay(rtime(), 60 + j * 10);
                TOUCH(a);
            } else {
                u64 pc; unsigned len;
                asm volatile("s_getpc_b64 %0\n.Lcodepf_pc:\n\ts_mov_b32 %1, codepf_end-.Lcodepf_pc" : "=s"(pc), "=s"(len));
                if (len > 65536u) len = 0;
                const __amdgpu_buffer_rsrc_t rc = make_rsrc(pc & ~15ull, len & ~15u);
#pragma unroll 1
                for (unsigned o = (id - NP - 15) * 1024u; o < len; o += (NH * 8 - NP - 15) * 1024u) dma16(JUNK_OFF, lane * 16, rc, o);
            }
            if (pack(LL)) delay(t0, 8 * ((unsigned)(pack(LL) >> (8 * rgp)) & 255));
            wg_barrier();
        } else if (!local && ((pack(DL) >> (8 * rgp)) & 255)) {
            delay(rtime(), 4 * ((unsigned)(pack(DL) >> (8 * rgp)) & 255));
        }
        if (wave >= 4) {
            const int w = wave - 4;
            const int r7 = rgp + 3 * (panel & 1);
            const unsigned first = local ? comb(8) << ((rgp + 2 * panel) & 7) : panel < EP ? 0u : comb(7) << (r7 >= 7 ? r7 - 7 : r7);
            u64 ts[3];
            load_tile_h<false>(h0, w, lane, panel * 128, row0 * KD, first, wcol, px, (row0 + h0) * KD, ts);
            if (!local && w == 0) {
                if (lane == 0) {
                    const u64 now = (rtime() + LEAD) << 6;
                    __hip_atomic_fetch_max(PACE, now, __ATOMIC_RELAXED, __HIP_MEMORY_SCOPE_WORKGROUP);
                    const u64 t = __hip_atomic_fetch_add(PACE, (u64)(h0 / 2) * TAU, __ATOMIC_RELAXED, __HIP_MEMORY_SCOPE_WORKGROUP);
                    *(volatile LAS u64*)(smem + CTL_OFF + 8) = t - now < (4000ull << 6) ? t : now;
                }
                wait_lds();
            }
            call = SLOAD(CTR) + 1;
            asm volatile("" : "+s"(call));
        } else {
            v4f acc[5][2];
            if (local && rgp >= 4) {
                const int j = (rgp - 4) * 16 + panel * 4 + wave;
                const u64 a = j < 8 ? (u64)P.w + ((u64)(j + 1) << 21) - 4 : j < 16 ? (u64)P.mrows[j - 8] : (u64)P.g[(j - 16) & 7].marea;
                if (j < 23) TOUCH(a);
            }
            float sa = __uint_as_float(SLOAD(psx)), sb = __uint_as_float(SLOAD(psw));
            call = SLOAD(CTR) + 1;
            asm volatile("" : "+s"(sa), "+s"(sb), "+s"(call));
            scale = sa * sb;
            if (h0 > 64) consume_tile<5>(smem, wave, lane, acc);
            else if (h0 > 48) consume_tile<4>(smem, wave, lane, acc);
            else if (h0 > 32) consume_tile<3>(smem, wave, lane, acc);
            else if (h0 > 16) consume_tile<2>(smem, wave, lane, acc);
            else consume_tile<1>(smem, wave, lane, acc);
            if (!local) {
#pragma unroll
                for (int u = 0; u < 5; ++u)
                    if (u * 16 < h0) {
                        const int r = u * 16 + m;
                        LAS uint8_t* d = smem + STG_OFF + r * 512 + wave * 128;
                        *(LAS v4f*)(d + ((g ^ (r & 7)) << 4)) = acc[u][0] * scale;
                        *(LAS v4f*)(d + (((4 + g) ^ (r & 7)) << 4)) = acc[u][1] * scale;
                    }
                wait_lds();
            } else {
                const u64 o = prows + ((row0 + m) * NC + panel * 128 + wave * 32 + g * 4) * 4;
#pragma unroll
                for (int u = 0; u < 4; ++u) {
                    st16_sys(o + u * 16 * NC * 4, acc[u][0] * scale);
                    st16_sys(o + u * 16 * NC * 4 + 64, acc[u][1] * scale);
                }
            }
        }

        if (!local) {
            const u64 o = prows + ((row0 + (lane >> 5)) * NC + panel * 128 + (lane & 31) * 4) * 4;
            wg_barrier();
            const u64 st = *(volatile LAS u64*)(smem + CTL_OFF + 8);
            u64 slot = (((u64)rfl((unsigned)(st >> 32)) << 32) | rfl((unsigned)st)) + wave * TAU;
#pragma unroll 1
            for (int it = wave; it < h0 / 2; it += 8) {
                const int r = it * 2 + (lane >> 5);
                const v4f v = *(LAS v4f*)(smem + STG_OFF + r * 512 + (((lane & 31) ^ (r & 7)) << 4));
                for (;;) {
                    const long long d = (long long)(slot >> 6) - (long long)rtime();
                    if (d <= 0) break;
                    if (d > 30) __builtin_amdgcn_s_sleep(8);
                }
                st16_sys(o + it * (2 * NC * 4), v);
                LOGT(it / 8);
                slot += 8 * TAU;
            }
            asm volatile("codepf_end:");
            wait_vm<0>();
            wg_barrier();
            if (wave == 0 && lane == 0) {
                flag_sys(parea + A_DONE + (rgp * 4 + panel) * 4, call);
                *(GAS unsigned*)CTR = call;
            }
            STAMP(52);
            LOGOUT(1);
            if (rgp == 0 && panel < EP) {
                const u64 c3 = (u64)P.g[7].wcol + 3 * 128;
                if (((c3 >> 7) & 7) == 3) {
                    const __amdgpu_buffer_rsrc_t rw = make_rsrc(c3, WRANGE - 3 * 128);
                    const unsigned vl = ((panel * 8 + wave) * 8 + (lane >> 3)) * ND + (lane & 7) * 16;
#pragma unroll 1
                    for (int i = 4; i < KSTEPS; ++i) dma16(JUNK_OFF, vl, rw, i * (128u * ND));
                }
            }
            return;
        }
        if (rgp == 7) {
            wait_vm<0>();
            wg_barrier();
            if (wave == 7 && lane == 0) *(GAS unsigned*)CTR = call;
            if (panel == 0 && wave < 7) {
                const bool ok = wait_flags((u64)P.g[wave].marea + A_DONE, 36, call, ctl + C_TMO, lane);
                if (!ok && lane == 0)
                    flag_sys(prows + row0 * NC * 4, 0x7fc00000u);
                STAMP(45 + wave);
            }
            return;
        }
    }

    const int sg = local ? rgp : grp;
    const int r2 = local ? 0 : RA, h2 = local ? RA : RB;
    const u64 marea = (u64)P.g[sg].marea;
    if (wave == 4) {
        const bool ok = wait_flags(marea + A_XFLAG, 32, call, ctl + C_TMO, lane);
        *(volatile LAS unsigned*)(smem + CTL_OFF) = ok ? 0u : 1u;
        wait_lds();
        STAMP(44);
    }
    wg_barrier();
    if (wave >= 4) {
        const int w = wave - 4;
        load_tile_h<true>(h2, w, lane, panel * 128, r2 * KD, 0u, (u64)P.g[7].wcol, marea + A_ROWS, (r2 + h2) * KD);
        if (w == 0 && lane == 0) flag_sys((u64)P.g[sg].parea + A_DONE + ((local ? 32 : 28) + panel) * 4, call);
    } else {
        if (wave == 0 && lane == 0) *(GAS unsigned*)CTR = call;
        v4f acc[5][2];
        if (local) consume_tile<(RA + 15) / 16>(smem, wave, lane, acc);
        else consume_tile<(RB + 15) / 16>(smem, wave, lane, acc);
        STAMP(32 + wave);
        if (*(volatile LAS unsigned*)(smem + CTL_OFF)) scale = __builtin_nanf("");
        const u64 o = (u64)P.mrows[sg] + ((r2 + m) * NC + panel * 128 + wave * 32 + g * 4) * 4;
#pragma unroll
        for (int u = 0; u < (RM + 15) / 16; ++u)
            if (u * 16 + m < h2) {
                st16_sys(o + u * 16 * NC * 4, acc[u][0] * scale);
                st16_sys(o + u * 16 * NC * 4 + 64, acc[u][1] * scale);
            }
    }
}

}

extern "C" void kernel_launch(int rank, int nranks, ncclComm_t comm,
                              hipStream_t stream,
                              void* const* d_in, const int* in_sizes, int n_in,
                              void* d_out, int out_size,
                              const int* in_dtypes, int out_dtype,
                              void* const* d_out_all,
                              void* const* d_ws_all, size_t ws_bytes) {
    (void)comm; (void)in_dtypes; (void)out_dtype;
    if (nranks != NR || (unsigned)rank >= (unsigned)NR || n_in != 4 || in_sizes[0] != MS * KD || in_sizes[1] != KD * ND
        || out_size != NR * MS * NC || ws_bytes < WS_NEED) {
        fprintf(stderr, "kernel_launch: unexpected problem shape\n");
        return;
    }
    static bool attr_set = false;
    if (!attr_set) {
        HIP_CHECK(hipFuncSetAttribute((const void*)gemm_a2a_kernel, hipFuncAttributeMaxDynamicSharedMemorySize, LDS_BYTES));
        attr_set = true;
    }
    Params P;
    P.x = (const uint8_t*)d_in[0];
    P.w = (const uint8_t*)d_in[1];
    P.sx = (const float*)d_in[2];
    P.sw = (const float*)d_in[3];
    P.ws = (uint8_t*)d_ws_all[rank];
    P.ctl = P.ws + WS_SRC + (size_t)rank * SRC_BYTES;
    P.rank = (unsigned)rank;
    for (int g = 0; g < NR; ++g) {
        const int p = (rank + 1 + g) % NR;
        P.g[g].wcol = P.w + (size_t)p * NC;
        P.g[g].parea = (uint8_t*)d_ws_all[p] + WS_SRC + (size_t)rank * SRC_BYTES;
        P.g[g].prows = (float*)d_out_all[p] + (size_t)rank * MS * NC;
        P.g[g].marea = P.ws + WS_SRC + (size_t)p * SRC_BYTES;
        P.mrows[g] = (float*)d_out + (size_t)p * MS * NC;
    }
    hipLaunchKernelGGL(gemm_a2a_kernel, dim3(NWG), dim3(NTHREADS), LDS_BYTES, stream, P);
    HIP_CHECK(hipGetLastError());
}
```

```cpp
#include <hip/hip_runtime.h>
#include <rccl/rccl.h>
#include <cstdio>
#include <cstdint>
#include <cstddef>

#define HIP_CHECK(expr) do { hipError_t _e = (expr); if (_e != hipSuccess) { \
    fprintf(stderr, "HIP error %s at %s:%d\n", hipGetErrorString(_e), __FILE__, __LINE__); \
    return; } } while (0)

namespace {

typedef int v2i __attribute__((ext_vector_type(2)));
typedef int v4i __attribute__((ext_vector_type(4)));
typedef int v8i __attribute__((ext_vector_type(8)));
typedef float v4f __attribute__((ext_vector_type(4)));
typedef unsigned long long u64;
#define LAS __attribute__((address_space(3)))
#define GAS __attribute__((address_space(1)))
#define CAS __attribute__((address_space(4)))

constexpr int NR = 8;
constexpr int KD = 4096;
constexpr int ND = 4096;
constexpr int MS = 512;
constexpr int NC = 512;
#define RR 96
#define RA 40
constexpr int RB = RR - RA;
constexpr int RM = RA > RB ? RA : RB;
constexpr int HMAX = 80;
#define HLIST 32, 48, 64, 64, 64, 64, 80
#define DLIST 0, 0, 0, 10, 30, 45, 50
#define LLIST 0, 0, 0, 0, 0, 0, 0
constexpr int HL[7] = {HLIST}, DL[7] = {DLIST}, LL[7] = {LLIST};
constexpr int hsum(int n) { int t = 0; for (int i = 0; i < n; ++i) t += HL[i]; return t; }
constexpr u64 pack(const int (&a)[7]) { u64 t = 0; for (int i = 0; i < 7; ++i) t |= (u64)a[i] << (8 * i); return t; }
constexpr u64 rpack() { u64 t = 0; for (int i = 0; i < 7; ++i) t |= (u64)(hsum(i) / 2) << (8 * i); return t; }
constexpr bool hok() { for (int i = 0; i < 7; ++i) if (HL[i] < 2 || HL[i] > HMAX || HL[i] % 2) return false; return true; }
static_assert(RR % 8 == 0 && RA >= 8 && RB >= 8 && RM <= HMAX && hok() && hsum(7) == MS - RR, "shape");
#define EP 2
#define PWIN 6
#define AH1 3
#define TAU 110
#define XTAU 104
#define STAU 135
#define LEAD 25
constexpr int NTHREADS = 512;
constexpr int NWG = 256;
constexpr int KSTEPS = KD / 128;

constexpr unsigned SLOT = 16384 + HMAX * 128, NS = 6;
constexpr unsigned STG_OFF = 2 * SLOT;
static_assert((KSTEPS - 1) % NS == 1, "the last two stages use slots 0 and 1");
constexpr unsigned CTL_OFF = NS * SLOT;
constexpr unsigned JUNK_OFF = CTL_OFF + 64;
constexpr unsigned LDS_BYTES = JUNK_OFF + 1024;

constexpr size_t WS_STAMP = 131072;
constexpr size_t WS_SRC = 2u << 20, SRC_BYTES = 512u << 10;
constexpr size_t A_DONE = 0;
constexpr size_t A_XFLAG = 256;
constexpr size_t A_ROWS = 4096;
constexpr size_t C_TMO = 0;
constexpr size_t C_PACE = 1024;
constexpr size_t C_CALL = 4096;
static_assert(A_ROWS + RR * KD <= SRC_BYTES, "area");
constexpr size_t WS_NEED = WS_SRC + NR * SRC_BYTES;
constexpr u64 DEADLINE_TICKS = 40000000ull;
constexpr unsigned WRANGE = KD * ND - (ND - NC);

struct Group {
    const uint8_t* wcol;
    uint8_t* parea;
    float* prows;
    uint8_t* marea;
};
struct Params {
    const uint8_t* x;
    uint8_t* ctl;
    const float* sx;
    const float* sw;
    Group g[NR];
    float* mrows[NR];
    const uint8_t* w;
    uint8_t* ws;
    unsigned rank;
};

constexpr unsigned comb(int n) { unsigned m = 0; for (int i = 0; i < 32; i += n) m |= 1u << i; return m; }

__device__ __forceinline__ unsigned rfl(unsigned v) { return __builtin_amdgcn_readfirstlane(v); }
__device__ __forceinline__ u64 rtime() { return __builtin_amdgcn_s_memrealtime(); }
__device__ __forceinline__ void delay(u64 t0, unsigned ticks) { while (rtime() - t0 < ticks) __builtin_amdgcn_s_sleep(4); }

__device__ __forceinline__ __amdgpu_buffer_rsrc_t make_rsrc(u64 p, unsigned bytes) {
    return __builtin_amdgcn_make_buffer_rsrc((void*)p, (short)0, (int)bytes, 0x00020000);
}

template <bool SYS = false>
__device__ __forceinline__ void dma16(unsigned lds, unsigned voff, __amdgpu_buffer_rsrc_t rs, unsigned soff) {
    if (SYS) asm volatile("s_mov_b32 m0, %0\n\ts_nop 0\n\tbuffer_load_dwordx4 %1, %2, %3 offen sc0 sc1 lds"
                          :: "s"(lds), "v"(voff), "s"(rs), "s"(soff) : "memory");
    else     asm volatile("s_mov_b32 m0, %0\n\ts_nop 0\n\tbuffer_load_dwordx4 %1, %2, %3 offen lds"
                          :: "s"(lds), "v"(voff), "s"(rs), "s"(soff) : "memory");
}
template <int N> __device__ __forceinline__ void wait_vm() { asm volatile("s_waitcnt vmcnt(%0)" :: "i"(N) : "memory"); }
__device__ __forceinline__ void wg_barrier() { asm volatile("s_barrier" ::: "memory"); }
__device__ __forceinline__ void wait_lds() { asm volatile("s_waitcnt lgkmcnt(0)" ::: "memory"); }
__device__ __forceinline__ void st16_sys(u64 p, v4f v) {
    asm volatile("global_store_dwordx4 %0, %1, off sc0 sc1\n\ts_nop 1" :: "v"(p), "v"(v) : "memory");
}
__device__ __forceinline__ void flag_sys(u64 p, unsigned v) {
    __hip_atomic_store((GAS unsigned*)p, v, __ATOMIC_RELAXED, __HIP_MEMORY_SCOPE_SYSTEM);
}

#define STAMP(i) do {} while (0)
#define LOGT(k) do {} while (0)
#define LOGOUT(half) do {} while (0)

__device__ __forceinline__ bool wait_flags(u64 flags, int n, unsigned call, u64 tmo, int lane) {
    const GAS unsigned* p = (const GAS unsigned*)(lane < n ? flags + 4 * lane : tmo);
    const u64 t0 = rtime();
    for (;;) {
        const u64 now = rtime();
        const unsigned v = __hip_atomic_load(p, __ATOMIC_RELAXED, __HIP_MEMORY_SCOPE_SYSTEM);
        const u64 ge = __ballot(v >= call), eq = __ballot(v == call);
        if ((eq >> 63) || now - t0 > DEADLINE_TICKS) break;
        if ((ge | (~0ull << n)) == ~0ull) return true;
    }
    if (lane == 0) __hip_atomic_store((GAS unsigned*)tmo, call, __ATOMIC_RELAXED, __HIP_MEMORY_SCOPE_AGENT);
    return false;
}

template <int NXW, bool SYS>
__device__ __forceinline__ void load_tile(int w, int lane, unsigned woff, unsigned xoff, unsigned first,
                                          u64 pw, u64 px, unsigned xbytes, u64* ts = nullptr) {
    constexpr int CNT = 4 + NXW, AHEAD = AH1;
    const unsigned v0 = (lane >> 3) * KD + (((lane & 7) ^ ((lane >> 4) & 3)) << 4);
    unsigned vw[4];
#pragma unroll
    for (int j = 0; j < 4; ++j) vw[j] = (v0 ^ (((j >> 1) & 1) << 6)) + (4 * w + j) * (8u * ND) - j * 1024u + woff;
    unsigned vx0 = (v0 ^ ((w & 1) << 6)) + w * (8u * KD) + xoff, vx1 = vx0 + 32u * KD, vx2 = vx0 + 64u * KD;
    unsigned ioff = w * 1024;
    u64 todo = ((u64)~first << 32) | first;
    asm volatile("" : "+v"(vw[0]), "+v"(vw[1]), "+v"(vw[2]), "+v"(vw[3]), "+v"(vx0), "+v"(vx1), "+v"(vx2), "+s"(ioff), "+s"(todo));
    __builtin_amdgcn_sched_barrier(0);
    const __amdgpu_buffer_rsrc_t rw = make_rsrc(pw, WRANGE), rx = make_rsrc(px, xbytes);
#define STEP_W "buffer_load_dwordx4 %4, %11, %13 offen lds\n\t" \
               "buffer_load_dwordx4 %5, %11, %13 offen offset:1024 lds\n\t" \
               "buffer_load_dwordx4 %6, %11, %13 offen offset:2048 lds\n\t" \
               "buffer_load_dwordx4 %7, %11, %13 offen offset:3072 lds"
#define STEP_X(m, v, pol) "\n\ts_mov_b32 m0, %" m "\n\ts_nop 0\n\tbuffer_load_dwordx4 %" v ", %12, %14 offen" pol " lds"
#define STEP_OPS :: "s"(ioff + w * 3072), "s"(ioff + 16384), "s"(ioff + 20480), "s"(ioff + 24576), \
                    "v"(vw[0]), "v"(vw[1]), "v"(vw[2]), "v"(vw[3]), "v"(vx0), "v"(vx1), "v"(vx2), \
                    "s"(rw), "s"(rx), "s"(ks * (128u * ND)), "s"(ks * 128u), "i"(CNT * (AHEAD - 1)) : "memory"
#define STEP_P(head, pol) do { \
        if (NXW == 0) asm volatile(head STEP_W STEP_OPS); \
        if (NXW == 1) asm volatile(head STEP_W STEP_X("1", "8", pol) STEP_OPS); \
        if (NXW == 2) asm volatile(head STEP_W STEP_X("1", "8", pol) STEP_X("2", "9", pol) STEP_OPS); \
        if (NXW == 3) asm volatile(head STEP_W STEP_X("1", "8", pol) STEP_X("2", "9", pol) STEP_X("3", "10", pol) STEP_OPS); \
    } while (0)
#define STEP(head) do { \
        const unsigned ks = __builtin_ctzll(todo) & 31; \
        todo &= todo - 1; \
        if (SYS) STEP_P(head, " sc0 sc1"); else STEP_P(head, ""); \
        ioff += SLOT; \
        if (ioff >= SLOT * NS) ioff -= SLOT * NS; \
    } while (0)
#pragma unroll 1
    for (int i = 0; i < AHEAD; ++i) STEP("s_mov_b32 m0, %0\n\ts_nop 0\n\t");
#pragma unroll 1
    while (todo) STEP("s_mov_b32 m0, %0\n\ts_waitcnt vmcnt(%15)\n\ts_barrier\n\t");
    static_assert(AHEAD >= 2 && AHEAD <= 5 && AHEAD < (int)NS, "drain");
    if (AHEAD > 4) { wait_vm<CNT * 4>(); wg_barrier(); }
    if (AHEAD > 3) { wait_vm<CNT * 3>(); wg_barrier(); }
    if (AHEAD > 2) { wait_vm<CNT * 2>(); wg_barrier(); }
    wait_vm<CNT>(); wg_barrier();
    wait_vm<0>(); wg_barrier();
}
template <bool SYS>
__device__ __forceinline__ void load_tile_h(int h, int w, int lane, unsigned woff, unsigned xoff, unsigned first,
                                            u64 pw, u64 px, unsigned xbytes, u64* ts = nullptr) {
    const int nxw = (8 * w < h) + (8 * w + 32 < h) + (8 * w + 64 < h);
    if (nxw == 2) load_tile<2, SYS>(w, lane, woff, xoff, first, pw, px, xbytes, ts);
    else if (nxw == 1) load_tile<1, SYS>(w, lane, woff, xoff, first, pw, px, xbytes, ts);
    else if (nxw == 3) load_tile<3, SYS>(w, lane, woff, xoff, first, pw, px, xbytes, ts);
    else load_tile<0, SYS>(w, lane, woff, xoff, first, pw, px, xbytes, ts);
}

template <int NU>
__device__ __forceinline__ void consume_tile(LAS uint8_t* smem, int c, int lane, v4f (&acc)[5][2]) {
    const int g = lane >> 4, q = (lane & 15) >> 1, p = lane & 1, m = lane & 15;
    const unsigned fw = ((q >> 1) & 3) | ((g & 1) << 2);
    const unsigned w0 = (16 * g + q) * 128 + (((2 * c) ^ fw) << 4) + p * 8;
    const unsigned x0 = 16384 + m * 128 + ((g ^ ((m >> 1) & 7)) << 4);
#pragma unroll
    for (int u = 0; u < 5; ++u) { acc[u][0] = v4f{0, 0, 0, 0}; acc[u][1] = v4f{0, 0, 0, 0}; }
    v8i wf[2][2], xf[2][NU];
    auto load = [&](int s, v8i (&wv)[2], v8i (&xv)[NU]) {
        LAS uint8_t* sl = smem + (s % NS) * SLOT;
#pragma unroll
        for (int cb = 0; cb < 2; ++cb) {
            LAS uint8_t* b = sl + (w0 ^ (cb << 4));
            v2i t0 = __builtin_amdgcn_ds_read_tr8_b64_v2i32((LAS v2i*)(b));
            v2i t1 = __builtin_amdgcn_ds_read_tr8_b64_v2i32((LAS v2i*)(b + 1024));
            v2i t2 = __builtin_amdgcn_ds_read_tr8_b64_v2i32((LAS v2i*)(b + 8192));
            v2i t3 = __builtin_amdgcn_ds_read_tr8_b64_v2i32((LAS v2i*)(b + 9216));
            wv[cb] = v8i{t0[0], t0[1], t1[0], t1[1], t2[0], t2[1], t3[0], t3[1]};
        }
#pragma unroll
        for (int u = 0; u < NU; ++u) {
            v4i a = *(LAS v4i*)(sl + x0 + u * 2048);
            v4i b = *(LAS v4i*)(sl + (x0 ^ 64) + u * 2048);
            xv[u] = v8i{a[0], a[1], a[2], a[3], b[0], b[1], b[2], b[3]};
        }
    };
    auto mma = [&](v8i (&wv)[2], v8i (&xv)[NU]) {
#pragma unroll
        for (int u = 0; u < NU; ++u)
#pragma unroll
            for (int cb = 0; cb < 2; ++cb)
                acc[u][cb] = __builtin_amdgcn_mfma_scale_f32_16x16x128_f8f6f4(wv[cb], xv[u], acc[u][cb], 1, 0, 0, 0, 0, 0);
    };
    auto step = [&](int s, v8i (&wn)[2], v8i (&xn)[NU], v8i (&wo)[2], v8i (&xo)[NU]) {
        wg_barrier();
        __builtin_amdgcn_sched_barrier(0);
        load(s, wn, xn);
        mma(wo, xo);
        constexpr int PER = (8 + 2 * NU + 2 * NU - 1) / (2 * NU);
#pragma unroll
        for (int i = 0; i < 2 * NU; ++i) {
            __builtin_amdgcn_sched_group_barrier(0x100, PER, 0);
            __builtin_amdgcn_sched_group_barrier(0x8, 1, 0);
        }
        __builtin_amdgcn_sched_barrier(0);
        wait_lds();
    };
    wg_barrier();
    load(0, wf[0], xf[0]);
    wait_lds();
    step(1, wf[1], xf[1], wf[0], xf[0]);
#pragma unroll 1
    for (int it = 0; it < (KSTEPS - 2) / (int)NS; ++it) {
#pragma unroll
        for (int s = 2; s < 2 + (int)NS; s += 2) {
            step(s, wf[0], xf[0], wf[1], xf[1]);
            step(s + 1, wf[1], xf[1], wf[0], xf[0]);
        }
    }
    static_assert((KSTEPS - 2) % NS == 0 && NS % 2 == 0, "trip count");
    mma(wf[1], xf[1]);
}

__global__ void __launch_bounds__(NTHREADS) gemm_a2a_kernel(Params P) {
    extern __shared__ __attribute__((aligned(16))) uint8_t smem_[];
    LAS uint8_t* smem = (LAS uint8_t*)smem_;
    const int grp = blockIdx.x & 7, idx = blockIdx.x >> 3, panel = idx & 3, rgp = ((idx >> 2) + 7) & 7;
    const CAS char* ka = (const CAS char*)__builtin_amdgcn_kernarg_segment_ptr();
#define KARG(off) (*(const volatile CAS u64*)(ka + (off)))
    const u64 px = KARG(offsetof(Params, x)), ctl = KARG(offsetof(Params, ctl));
    const u64 psx = KARG(offsetof(Params, sx)), psw = KARG(offsetof(Params, sw));
    const u64 wcol = KARG(offsetof(Params, g) + grp * sizeof(Group)), parea = KARG(offsetof(Params, g) + grp * sizeof(Group) + 8);
    const u64 prows = KARG(offsetof(Params, g) + grp * sizeof(Group) + 16);
    const int lane = threadIdx.x & 63;
    const int wave = rfl(threadIdx.x >> 6);
    const bool local = grp == 7;
    const bool pusher = !local && rgp == 7;
#define CTR (ctl + C_CALL + blockIdx.x * 128)
#define PACE ((GAS u64*)(ctl + C_PACE + grp * 128))
#define SLOAD(a) (*(const volatile CAS unsigned*)(a))
#define TOUCH(a) do { unsigned t_ = SLOAD(a); asm volatile("" :: "s"(t_)); } while (0)
    unsigned call = 0;
    float scale = 0.f;
    const int g = lane >> 4, m = lane & 15;

    if (__builtin_expect(pusher, 1)) {
        constexpr int XI = RR / 8;
        const unsigned xo = (panel * 8 + wave) * XI * 1024;
        const unsigned lb = wave * XI * 1024;
        const __amdgpu_buffer_rsrc_t rp = make_rsrc(px, RR * KD);
        dma16(lb, lane * 16, rp, xo);
        dma16(lb + 1024, lane * 16, rp, xo + 1024);
        wg_barrier();
        dma16(lb + 2048, lane * 16, rp, xo + 2048);
        dma16(lb + 3072, lane * 16, rp, xo + 3072);
        const u64 xd = parea + A_ROWS + xo + lane * 16;
        LAS uint8_t* ls = smem + lb + lane * 16;
        wait_vm<3>();
        v4f v = *(LAS v4f*)ls;
#pragma unroll
        for (int i = 0; i < XI; ++i) {
            if (i >= 2) {
                if (i + 3 < XI) wait_vm<3>(); else if (i + 2 < XI) wait_vm<2>(); else wait_vm<1>();
            }
            st16_sys(xd + i * 1024, v);
            LOGT(i);
            if (i + 4 < XI) dma16(lb + (i + 4) * 1024, lane * 16, rp, xo + (i + 4) * 1024);
            if (i == 1 && wave == 7 && lane == 0) {
                __hip_atomic_fetch_max(PACE, rtime() << 6, __ATOMIC_RELAXED, __HIP_MEMORY_SCOPE_WORKGROUP);
                __hip_atomic_fetch_add(PACE, (u64)(RR * XTAU), __ATOMIC_RELAXED, __HIP_MEMORY_SCOPE_WORKGROUP);
            }
            if (i == 0) wait_vm<4>();
            if (i == 1) wait_vm<5>();
            if (i + 1 < XI) v = *(LAS v4f*)(ls + (i + 1) * 1024);
        }
        static_assert(XI >= 6 && XI <= 16, "wait counts");
        scale = __uint_as_float(SLOAD(psx)) * __uint_as_float(SLOAD(psw));
        call = SLOAD(CTR) + 1;
        wait_vm<0>();
        if (lane == 0) flag_sys(parea + A_XFLAG + (panel * 8 + wave) * 4, call);
        STAMP(40);
        LOGOUT(0);
    } else {
        const int h0 = local ? 64 : (int)(pack(HL) >> (8 * rgp)) & 255;
        const int row0 = local ? rgp * 64 : RR + 2 * ((int)(rpack() >> (8 * rgp)) & 255);
        if (!local && panel >= EP) {
            constexpr int NH = (4 - EP) * 7, NP = 16 * EP + (MS - RR) / 8;
            static_assert(NH * 8 - NP >= 20, "spare waves");
            const int id = wave * NH + (panel - EP) * 7 + rgp;
            const u64 t0 = pack(LL) ? rtime() : 0;
            if (id < NP) {
                const bool isw = id < 16 * EP;
                unsigned vo = ((isw ? id & 15 : id - 16 * EP + RR / 8) * 8 + (lane >> 3)) * KD + (isw ? (id >> 4) * 128 : 0) + (lane & 7) * 16;
                unsigned st = isw ? 128u * ND : 128u, sz = isw ? WRANGE : (unsigned)(MS * KD);
                asm volatile("" : "+v"(vo), "+s"(st), "+s"(sz));
                __builtin_amdgcn_sched_barrier(0);
                const __amdgpu_buffer_rsrc_t rs = make_rsrc(isw ? wcol : px, sz);
#pragma unroll 1
                for (unsigned sg = 0; sg < KSTEPS; ++sg) { dma16(JUNK_OFF, vo, rs, sg * st); wait_vm<PWIN>(); }
            } else if (id - NP < 15) {
                const int j = id - NP;
                const u64 a = j < 1 ? parea + 4
                            : j < 9 ? (u64)P.w + ((u64)j << 21) - 4
                            : j < 10 ? px + MS * KD - 4
                            : j < 12 ? prows + (j - 10) * (MS * NC * 4 - 4)
                            : j < 13 ? (u64)P.g[grp].marea
                            : (u64)P.mrows[grp] + (j & 1) * (RR * NC * 4);
                if (j != 0 && j != 9) delay(rtime(), 60 + j * 10);
                TOUCH(a);
            } else {
                u64 pc; unsigned len;
                asm volatile("s_getpc_b64 %0\n.Lcodepf_pc:\n\ts_mov_b32 %1, .Lcodepf_end-.Lcodepf_pc" : "=s"(pc), "=s"(len));
                if (len > 65536u) len = 0;
                const __amdgpu_buffer_rsrc_t rc = make_rsrc(pc & ~15ull, len & ~15u);
#pragma unroll 1
                for (unsigned o = (id - NP - 15) * 1024u; o < len; o += (NH * 8 - NP - 15) * 1024u) dma16(JUNK_OFF, lane * 16, rc, o);
            }
            if (pack(LL)) delay(t0, 8 * ((unsigned)(pack(LL) >> (8 * rgp)) & 255));
            wg_barrier();
        } else if (!local && ((pack(DL) >> (8 * rgp)) & 255)) {
            delay(rtime(), 4 * ((unsigned)(pack(DL) >> (8 * rgp)) & 255));
        }
        if (wave >= 4) {
            const int w = wave - 4;
            const int r7 = rgp + 3 * (panel & 1);
            const unsigned first = local ? comb(8) << ((rgp + 2 * panel) & 7) : panel < EP ? 0u : comb(7) << (r7 >= 7 ? r7 - 7 : r7);
            u64 ts[3];
            load_tile_h<false>(h0, w, lane, panel * 128, row0 * KD, first, wcol, px, (row0 + h0) * KD, ts);
            if (!local && w == 0) {
                if (lane == 0) {
                    const u64 now = (rtime() + LEAD) << 6;
                    __hip_atomic_fetch_max(PACE, now, __ATOMIC_RELAXED, __HIP_MEMORY_SCOPE_WORKGROUP);
                    const u64 t = __hip_atomic_fetch_add(PACE, (u64)(h0 / 2) * TAU, __ATOMIC_RELAXED, __HIP_MEMORY_SCOPE_WORKGROUP);
                    *(volatile LAS u64*)(smem + CTL_OFF + 8) = t - now < (4000ull << 6) ? t : now;
                }
                wait_lds();
            }
            call = SLOAD(CTR) + 1;
            asm volatile("" : "+s"(call));
        } else {
            v4f acc[5][2];
            if (local && rgp >= 4) {
                const int j = (rgp - 4) * 16 + panel * 4 + wave;
                const u64 a = j < 8 ? (u64)P.w + ((u64)(j + 1) << 21) - 4 : j < 16 ? (u64)P.mrows[j - 8] : (u64)P.g[(j - 16) & 7].marea;
                if (j < 23) TOUCH(a);
            }
            float sa = __uint_as_float(SLOAD(psx)), sb = __uint_as_float(SLOAD(psw));
            call = SLOAD(CTR) + 1;
            asm volatile("" : "+s"(sa), "+s"(sb), "+s"(call));
            scale = sa * sb;
            if (h0 > 64) consume_tile<5>(smem, wave, lane, acc);
            else if (h0 > 48) consume_tile<4>(smem, wave, lane, acc);
            else if (h0 > 32) consume_tile<3>(smem, wave, lane, acc);
            else if (h0 > 16) consume_tile<2>(smem, wave, lane, acc);
            else consume_tile<1>(smem, wave, lane, acc);
            if (!local) {
#pragma unroll
                for (int u = 0; u < 5; ++u)
                    if (u * 16 < h0) {
                        const int r = u * 16 + m;
                        LAS uint8_t* d = smem + STG_OFF + r * 512 + wave * 128;
                        *(LAS v4f*)(d + ((g ^ (r & 7)) << 4)) = acc[u][0] * scale;
                        *(LAS v4f*)(d + (((4 + g) ^ (r & 7)) << 4)) = acc[u][1] * scale;
                    }
                wait_lds();
            } else {
                const u64 o = prows + ((row0 + m) * NC + panel * 128 + wave * 32 + g * 4) * 4;
#pragma unroll
                for (int u = 0; u < 4; ++u) {
                    st16_sys(o + u * 16 * NC * 4, acc[u][0] * scale);
                    st16_sys(o + u * 16 * NC * 4 + 64, acc[u][1] * scale);
                }
            }
        }

        if (!local) {
            const u64 o = prows + ((row0 + (lane >> 5)) * NC + panel * 128 + (lane & 31) * 4) * 4;
            wg_barrier();
            const u64 st = *(volatile LAS u64*)(smem + CTL_OFF + 8);
            u64 slot = (((u64)rfl((unsigned)(st >> 32)) << 32) | rfl((unsigned)st)) + wave * STAU;
#pragma unroll 1
            for (int it = wave; it < h0 / 2; it += 8) {
                const int r = it * 2 + (lane >> 5);
                const v4f v = *(LAS v4f*)(smem + STG_OFF + r * 512 + (((lane & 31) ^ (r & 7)) << 4));
                for (;;) {
                    const long long d = (long long)(slot >> 6) - (long long)rtime();
                    if (d <= 0) break;
                    if (d > 30) __builtin_amdgcn_s_sleep(8);
                }
                st16_sys(o + it * (2 * NC * 4), v);
                LOGT(it / 8);
                slot += 8 * STAU;
            }
            asm volatile(".Lcodepf_end:");
            wait_vm<0>();
            wg_barrier();
            if (wave == 0 && lane == 0) {
                flag_sys(parea + A_DONE + (rgp * 4 + panel) * 4, call);
                *(GAS unsigned*)CTR = call;
            }
            STAMP(52);
            LOGOUT(1);
            if (rgp == 0 && panel < EP) {
                const u64 c3 = (u64)P.g[7].wcol + 3 * 128;
                if (((c3 >> 7) & 7) == 3) {
                    const __amdgpu_buffer_rsrc_t rw = make_rsrc(c3, WRANGE - 3 * 128);
                    const unsigned vl = ((panel * 8 + wave) * 8 + (lane >> 3)) * ND + (lane & 7) * 16;
#pragma unroll 1
                    for (int i = 4; i < KSTEPS; ++i) dma16(JUNK_OFF, vl, rw, i * (128u * ND));
                }
            }
            return;
        }
        if (rgp == 7) {
            wait_vm<0>();
            wg_barrier();
            if (wave == 7 && lane == 0) *(GAS unsigned*)CTR = call;
            if (panel == 0 && wave < 7) {
                const bool ok = wait_flags((u64)P.g[wave].marea + A_DONE, 36, call, ctl + C_TMO, lane);
                if (!ok && lane == 0)
                    flag_sys(prows + row0 * NC * 4, 0x7fc00000u);
                STAMP(45 + wave);
            }
            return;
        }
    }

    const int sg = local ? rgp : grp;
    const int r2 = local ? 0 : RA, h2 = local ? RA : RB;
    const u64 marea = (u64)P.g[sg].marea;
    if (wave == 4) {
        const bool ok = wait_flags(marea + A_XFLAG, 32, call, ctl + C_TMO, lane);
        *(volatile LAS unsigned*)(smem + CTL_OFF) = ok ? 0u : 1u;
        wait_lds();
        STAMP(44);
    }
    wg_barrier();
    if (wave >= 4) {
        const int w = wave - 4;
        load_tile_h<true>(h2, w, lane, panel * 128, r2 * KD, 0u, (u64)P.g[7].wcol, marea + A_ROWS, (r2 + h2) * KD);
        if (w == 0 && lane == 0) flag_sys((u64)P.g[sg].parea + A_DONE + ((local ? 32 : 28) + panel) * 4, call);
    } else {
        if (wave == 0 && lane == 0) *(GAS unsigned*)CTR = call;
        v4f acc[5][2];
        if (local) consume_tile<(RA + 15) / 16>(smem, wave, lane, acc);
        else consume_tile<(RB + 15) / 16>(smem, wave, lane, acc);
        STAMP(32 + wave);
        if (*(volatile LAS unsigned*)(smem + CTL_OFF)) scale = __builtin_nanf("");
        const u64 o = (u64)P.mrows[sg] + ((r2 + m) * NC + panel * 128 + wave * 32 + g * 4) * 4;
#pragma unroll
        for (int u = 0; u < (RM + 15) / 16; ++u)
            if (u * 16 + m < h2) {
                st16_sys(o + u * 16 * NC * 4, acc[u][0] * scale);
                st16_sys(o + u * 16 * NC * 4 + 64, acc[u][1] * scale);
            }
    }
}

}

extern "C" void kernel_launch(int rank, int nranks, ncclComm_t comm,
                              hipStream_t stream,
                              void* const* d_in, const int* in_sizes, int n_in,
                              void* d_out, int out_size,
                              const int* in_dtypes, int out_dtype,
                              void* const* d_out_all,
                              void* const* d_ws_all, size_t ws_bytes) {
    (void)comm; (void)in_dtypes; (void)out_dtype;
    if (nranks != NR || (unsigned)rank >= (unsigned)NR || n_in != 4 || in_sizes[0] != MS * KD || in_sizes[1] != KD * ND
        || out_size != NR * MS * NC || ws_bytes < WS_NEED) {
        fprintf(stderr, "kernel_launch: unexpected problem shape\n");
        return;
    }
    static bool attr_set = false;
    if (!attr_set) {
        HIP_CHECK(hipFuncSetAttribute((const void*)gemm_a2a_kernel, hipFuncAttributeMaxDynamicSharedMemorySize, LDS_BYTES));
        attr_set = true;
    }
    Params P;
    P.x = (const uint8_t*)d_in[0];
    P.w = (const uint8_t*)d_in[1];
    P.sx = (const float*)d_in[2];
    P.sw = (const float*)d_in[3];
    P.ws = (uint8_t*)d_ws_all[rank];
    P.ctl = P.ws + WS_SRC + (size_t)rank * SRC_BYTES;
    P.rank = (unsigned)rank;
    for (int g = 0; g < NR; ++g) {
        const int p = (rank + 1 + g) % NR;
        P.g[g].wcol = P.w + (size_t)p * NC;
        P.g[g].parea = (uint8_t*)d_ws_all[p] + WS_SRC + (size_t)rank * SRC_BYTES;
        P.g[g].prows = (float*)d_out_all[p] + (size_t)rank * MS * NC;
        P.g[g].marea = P.ws + WS_SRC + (size_t)p * SRC_BYTES;
        P.mrows[g] = (float*)d_out + (size_t)p * MS * NC;
    }
    hipLaunchKernelGGL(gemm_a2a_kernel, dim3(NWG), dim3(NTHREADS), LDS_BYTES, stream, P);
    HIP_CHECK(hipGetLastError());
}
```

```cpp
#include <hip/hip_runtime.h>
#include <rccl/rccl.h>
#include <cstdio>
#include <cstdint>
#include <cstddef>

#define HIP_CHECK(expr) do { hipError_t _e = (expr); if (_e != hipSuccess) { \
    fprintf(stderr, "HIP error %s at %s:%d\n", hipGetErrorString(_e), __FILE__, __LINE__); \
    return; } } while (0)

namespace {

typedef int v2i __attribute__((ext_vector_type(2)));
typedef int v4i __attribute__((ext_vector_type(4)));
typedef int v8i __attribute__((ext_vector_type(8)));
typedef float v4f __attribute__((ext_vector_type(4)));
typedef unsigned long long u64;
#define LAS __attribute__((address_space(3)))
#define GAS __attribute__((address_space(1)))
#define CAS __attribute__((address_space(4)))

constexpr int NR = 8;
constexpr int KD = 4096;
constexpr int ND = 4096;
constexpr int MS = 512;
constexpr int NC = 512;
#define RR 96
#define RA 40
constexpr int RB = RR - RA;
constexpr int RM = RA > RB ? RA : RB;
constexpr int HMAX = 80;
#define HLIST 32, 48, 64, 64, 64, 64, 80
#define DLIST 0, 0, 0, 10, 30, 45, 50
#define LLIST 0, 0, 0, 0, 0, 0, 0
constexpr int HL[7] = {HLIST}, DL[7] = {DLIST}, LL[7] = {LLIST};
constexpr int hsum(int n) { int t = 0; for (int i = 0; i < n; ++i) t += HL[i]; return t; }
constexpr u64 pack(const int (&a)[7]) { u64 t = 0; for (int i = 0; i < 7; ++i) t |= (u64)a[i] << (8 * i); return t; }
constexpr u64 rpack() { u64 t = 0; for (int i = 0; i < 7; ++i) t |= (u64)(hsum(i) / 2) << (8 * i); return t; }
constexpr bool hok() { for (int i = 0; i < 7; ++i) if (HL[i] < 2 || HL[i] > HMAX || HL[i] % 2) return false; return true; }
static_assert(RR % 8 == 0 && RA >= 8 && RB >= 8 && RM <= HMAX && hok() && hsum(7) == MS - RR, "shape");
#define EP 2
#define PWIN 6
#define AH1 3
#define TAU 110
#define XTAU 104
#define STAU 135
#define PRG 5
#define GOLEAD 70
#define PSP 3
#define OWNLEAD 50
#define LEAD 25
constexpr int NTHREADS = 512;
constexpr int NWG = 256;
constexpr int KSTEPS = KD / 128;

constexpr unsigned SLOT = 16384 + HMAX * 128, NS = 6;
constexpr unsigned STG_OFF = 2 * SLOT;
static_assert((KSTEPS - 1) % NS == 1, "the last two stages use slots 0 and 1");
constexpr unsigned CTL_OFF = NS * SLOT;
constexpr unsigned JUNK_OFF = CTL_OFF + 64;
constexpr unsigned LDS_BYTES = JUNK_OFF + 1024;

constexpr size_t WS_STAMP = 131072;
constexpr size_t WS_SRC = 2u << 20, SRC_BYTES = 1u << 20;
constexpr size_t A_DONE = 0;
constexpr size_t A_XFLAG = 256;
constexpr size_t A_GO = 384;
constexpr size_t A_PULL = 512u << 10;
constexpr size_t A_ROWS = 4096;
constexpr size_t C_TMO = 0;
constexpr size_t C_PACE = 1024;
constexpr size_t C_CALL = 4096;
static_assert(A_ROWS + RR * KD <= A_PULL && A_PULL + 4 * HMAX * 512 <= SRC_BYTES, "area");
constexpr size_t WS_NEED = WS_SRC + NR * SRC_BYTES;
constexpr u64 DEADLINE_TICKS = 40000000ull;
constexpr unsigned WRANGE = KD * ND - (ND - NC);

struct Group {
    const uint8_t* wcol;
    uint8_t* parea;
    float* prows;
    uint8_t* marea;
};
struct Params {
    const uint8_t* x;
    uint8_t* ctl;
    const float* sx;
    const float* sw;
    Group g[NR];
    float* mrows[NR];
    const uint8_t* w;
    uint8_t* ws;
    unsigned rank;
};

constexpr unsigned comb(int n) { unsigned m = 0; for (int i = 0; i < 32; i += n) m |= 1u << i; return m; }

__device__ __forceinline__ unsigned rfl(unsigned v) { return __builtin_amdgcn_readfirstlane(v); }
__device__ __forceinline__ u64 rtime() { return __builtin_amdgcn_s_memrealtime(); }
__device__ __forceinline__ void delay(u64 t0, unsigned ticks) { while (rtime() - t0 < ticks) __builtin_amdgcn_s_sleep(4); }

__device__ __forceinline__ __amdgpu_buffer_rsrc_t make_rsrc(u64 p, unsigned bytes) {
    return __builtin_amdgcn_make_buffer_rsrc((void*)p, (short)0, (int)bytes, 0x00020000);
}

template <bool SYS = false>
__device__ __forceinline__ void dma16(unsigned lds, unsigned voff, __amdgpu_buffer_rsrc_t rs, unsigned soff) {
    if (SYS) asm volatile("s_mov_b32 m0, %0\n\ts_nop 0\n\tbuffer_load_dwordx4 %1, %2, %3 offen sc0 sc1 lds"
                          :: "s"(lds), "v"(voff), "s"(rs), "s"(soff) : "memory");
    else     asm volatile("s_mov_b32 m0, %0\n\ts_nop 0\n\tbuffer_load_dwordx4 %1, %2, %3 offen lds"
                          :: "s"(lds), "v"(voff), "s"(rs), "s"(soff) : "memory");
}
template <int N> __device__ __forceinline__ void wait_vm() { asm volatile("s_waitcnt vmcnt(%0)" :: "i"(N) : "memory"); }
__device__ __forceinline__ void wg_barrier() { asm volatile("s_barrier" ::: "memory"); }
__device__ __forceinline__ void wait_lds() { asm volatile("s_waitcnt lgkmcnt(0)" ::: "memory"); }
__device__ __forceinline__ void st16_sys(u64 p, v4f v) {
    asm volatile("global_store_dwordx4 %0, %1, off sc0 sc1\n\ts_nop 1" :: "v"(p), "v"(v) : "memory");
}
__device__ __forceinline__ void flag_sys(u64 p, unsigned v) {
    __hip_atomic_store((GAS unsigned*)p, v, __ATOMIC_RELAXED, __HIP_MEMORY_SCOPE_SYSTEM);
}

#define STAMP(i) do {} while (0)
#define LOGT(k) do {} while (0)
#define LOGOUT(half) do {} while (0)

template <int SLEEP = 0>
__device__ __forceinline__ bool wait_flags(u64 flags, int n, unsigned call, u64 tmo, int lane, u64 ignore = 0) {
    const GAS unsigned* p = (const GAS unsigned*)(lane < n ? flags + 4 * lane : tmo);
    const u64 t0 = rtime();
    for (;;) {
        const u64 now = rtime();
        const unsigned v = __hip_atomic_load(p, __ATOMIC_RELAXED, __HIP_MEMORY_SCOPE_SYSTEM);
        const u64 ge = __ballot(v >= call), eq = __ballot(v == call);
        if ((eq >> 63) || now - t0 > DEADLINE_TICKS) break;
        if ((ge | (~0ull << n) | ignore) == ~0ull) return true;
        if (SLEEP) __builtin_amdgcn_s_sleep(SLEEP);
    }
    if (lane == 0) __hip_atomic_store((GAS unsigned*)tmo, call, __ATOMIC_RELAXED, __HIP_MEMORY_SCOPE_AGENT);
    return false;
}

template <int NXW, bool SYS>
__device__ __forceinline__ void load_tile(int w, int lane, unsigned woff, unsigned xoff, unsigned first,
                                          u64 pw, u64 px, unsigned xbytes, u64* ts = nullptr) {
    constexpr int CNT = 4 + NXW, AHEAD = AH1;
    const unsigned v0 = (lane >> 3) * KD + (((lane & 7) ^ ((lane >> 4) & 3)) << 4);
    unsigned vw[4];
#pragma unroll
    for (int j = 0; j < 4; ++j) vw[j] = (v0 ^ (((j >> 1) & 1) << 6)) + (4 * w + j) * (8u * ND) - j * 1024u + woff;
    unsigned vx0 = (v0 ^ ((w & 1) << 6)) + w * (8u * KD) + xoff, vx1 = vx0 + 32u * KD, vx2 = vx0 + 64u * KD;
    unsigned ioff = w * 1024;
    u64 todo = ((u64)~first << 32) | first;
    asm volatile("" : "+v"(vw[0]), "+v"(vw[1]), "+v"(vw[2]), "+v"(vw[3]), "+v"(vx0), "+v"(vx1), "+v"(vx2), "+s"(ioff), "+s"(todo));
    __builtin_amdgcn_sched_barrier(0);
    asm volatile("" : "+s"(pw), "+s"(px));
    const __amdgpu_buffer_rsrc_t rw = make_rsrc(pw, WRANGE), rx = make_rsrc(px, xbytes);
#define STEP_W "buffer_load_dwordx4 %4, %11, %13 offen lds\n\t" \
               "buffer_load_dwordx4 %5, %11, %13 offen offset:1024 lds\n\t" \
               "buffer_load_dwordx4 %6, %11, %13 offen offset:2048 lds\n\t" \
               "buffer_load_dwordx4 %7, %11, %13 offen offset:3072 lds"
#define STEP_X(m, v, pol) "\n\ts_mov_b32 m0, %" m "\n\ts_nop 0\n\tbuffer_load_dwordx4 %" v ", %12, %14 offen" pol " lds"
#define STEP_OPS :: "s"(ioff + w * 3072), "s"(ioff + 16384), "s"(ioff + 20480), "s"(ioff + 24576), \
                    "v"(vw[0]), "v"(vw[1]), "v"(vw[2]), "v"(vw[3]), "v"(vx0), "v"(vx1), "v"(vx2), \
                    "s"(rw), "s"(rx), "s"(ks * (128u * ND)), "s"(ks * 128u), "i"(CNT * (AHEAD - 1)) : "memory"
#define STEP_P(head, pol) do { \
        if (NXW == 0) asm volatile(head STEP_W STEP_OPS); \
        if (NXW == 1) asm volatile(head STEP_W STEP_X("1", "8", pol) STEP_OPS); \
        if (NXW == 2) asm volatile(head STEP_W STEP_X("1", "8", pol) STEP_X("2", "9", pol) STEP_OPS); \
        if (NXW == 3) asm volatile(head STEP_W STEP_X("1", "8", pol) STEP_X("2", "9", pol) STEP_X("3", "10", pol) STEP_OPS); \
    } while (0)
#define STEP(head) do { \
        const unsigned ks = __builtin_ctzll(todo) & 31; \
        todo &= todo - 1; \
        if (SYS) STEP_P(head, " sc0 sc1"); else STEP_P(head, ""); \
        ioff += SLOT; \
        if (ioff >= SLOT * NS) ioff -= SLOT * NS; \
    } while (0)
#pragma unroll 1
    for (int i = 0; i < AHEAD; ++i) STEP("s_mov_b32 m0, %0\n\ts_nop 0\n\t");
#pragma unroll 1
    while (todo) STEP("s_mov_b32 m0, %0\n\ts_waitcnt vmcnt(%15)\n\ts_barrier\n\t");
    static_assert(AHEAD >= 2 && AHEAD <= 5 && AHEAD < (int)NS, "drain");
    if (AHEAD > 4) { wait_vm<CNT * 4>(); wg_barrier(); }
    if (AHEAD > 3) { wait_vm<CNT * 3>(); wg_barrier(); }
    if (AHEAD > 2) { wait_vm<CNT * 2>(); wg_barrier(); }
    wait_vm<CNT>(); wg_barrier();
    wait_vm<0>(); wg_barrier();
}
template <bool SYS>
__device__ __forceinline__ void load_tile_h(int h, int w, int lane, unsigned woff, unsigned xoff, unsigned first,
                                            u64 pw, u64 px, unsigned xbytes, u64* ts = nullptr) {
    const int nxw = (h - 8 * w + 31) >> 5;
    if (nxw == 2) load_tile<2, SYS>(w, lane, woff, xoff, first, pw, px, xbytes, ts);
    else if (nxw == 1) load_tile<1, SYS>(w, lane, woff, xoff, first, pw, px, xbytes, ts);
    else if (nxw == 3) load_tile<3, SYS>(w, lane, woff, xoff, first, pw, px, xbytes, ts);
    else load_tile<0, SYS>(w, lane, woff, xoff, first, pw, px, xbytes, ts);
}

template <int NU>
__device__ __forceinline__ void consume_tile(LAS uint8_t* smem, int c, int lane, v4f (&acc)[5][2]) {
    const int g = lane >> 4, q = (lane & 15) >> 1, p = lane & 1, m = lane & 15;
    const unsigned fw = ((q >> 1) & 3) | ((g & 1) << 2);
    const unsigned w0 = (16 * g + q) * 128 + (((2 * c) ^ fw) << 4) + p * 8;
    const unsigned x0 = 16384 + m * 128 + ((g ^ ((m >> 1) & 7)) << 4);
#pragma unroll
    for (int u = 0; u < 5; ++u) { acc[u][0] = v4f{0, 0, 0, 0}; acc[u][1] = v4f{0, 0, 0, 0}; }
    v8i wf[2][2], xf[2][NU];
    auto load = [&](int s, v8i (&wv)[2], v8i (&xv)[NU]) {
        LAS uint8_t* sl = smem + (s % NS) * SLOT;
#pragma unroll
        for (int cb = 0; cb < 2; ++cb) {
            LAS uint8_t* b = sl + (w0 ^ (cb << 4));
            v2i t0 = __builtin_amdgcn_ds_read_tr8_b64_v2i32((LAS v2i*)(b));
            v2i t1 = __builtin_amdgcn_ds_read_tr8_b64_v2i32((LAS v2i*)(b + 1024));
            v2i t2 = __builtin_amdgcn_ds_read_tr8_b64_v2i32((LAS v2i*)(b + 8192));
            v2i t3 = __builtin_amdgcn_ds_read_tr8_b64_v2i32((LAS v2i*)(b + 9216));
            wv[cb] = v8i{t0[0], t0[1], t1[0], t1[1], t2[0], t2[1], t3[0], t3[1]};
        }
#pragma unroll
        for (int u = 0; u < NU; ++u) {
            v4i a = *(LAS v4i*)(sl + x0 + u * 2048);
            v4i b = *(LAS v4i*)(sl + (x0 ^ 64) + u * 2048);
            xv[u] = v8i{a[0], a[1], a[2], a[3], b[0], b[1], b[2], b[3]};
        }
    };
    auto mma = [&](v8i (&wv)[2], v8i (&xv)[NU]) {
#pragma unroll
        for (int u = 0; u < NU; ++u)
#pragma unroll
            for (int cb = 0; cb < 2; ++cb)
                acc[u][cb] = __builtin_amdgcn_mfma_scale_f32_16x16x128_f8f6f4(wv[cb], xv[u], acc[u][cb], 1, 0, 0, 0, 0, 0);
    };
    auto step = [&](int s, v8i (&wn)[2], v8i (&xn)[NU], v8i (&wo)[2], v8i (&xo)[NU]) {
        wg_barrier();
        __builtin_amdgcn_sched_barrier(0);
        load(s, wn, xn);
        mma(wo, xo);
        constexpr int PER = (8 + 2 * NU + 2 * NU - 1) / (2 * NU);
#pragma unroll
        for (int i = 0; i < 2 * NU; ++i) {
            __builtin_amdgcn_sched_group_barrier(0x100, PER, 0);
            __builtin_amdgcn_sched_group_barrier(0x8, 1, 0);
        }
        __builtin_amdgcn_sched_barrier(0);
        wait_lds();
    };
    wg_barrier();
    load(0, wf[0], xf[0]);
    wait_lds();
    step(1, wf[1], xf[1], wf[0], xf[0]);
#pragma unroll 1
    for (int it = 0; it < (KSTEPS - 2) / (int)NS; ++it) {
#pragma unroll
        for (int s = 2; s < 2 + (int)NS; s += 2) {
            step(s, wf[0], xf[0], wf[1], xf[1]);
            step(s + 1, wf[1], xf[1], wf[0], xf[0]);
        }
    }
    static_assert((KSTEPS - 2) % NS == 0 && NS % 2 == 0, "trip count");
    mma(wf[1], xf[1]);
}

__global__ void __launch_bounds__(NTHREADS) gemm_a2a_kernel(Params P) {
    extern __shared__ __attribute__((aligned(16))) uint8_t smem_[];
    LAS uint8_t* smem = (LAS uint8_t*)smem_;
    const int grp = blockIdx.x & 7, idx = blockIdx.x >> 3, panel = idx & 3, rgp = ((idx >> 2) + 7) & 7;
    const CAS char* ka = (const CAS char*)__builtin_amdgcn_kernarg_segment_ptr();
#define KARG(off) (*(const volatile CAS u64*)(ka + (off)))
    const u64 px = KARG(offsetof(Params, x)), ctl = KARG(offsetof(Params, ctl));
    const u64 wcol = KARG(offsetof(Params, g) + grp * sizeof(Group)), parea = KARG(offsetof(Params, g) + grp * sizeof(Group) + 8);
    const u64 prows = KARG(offsetof(Params, g) + grp * sizeof(Group) + 16);
    const int lane = threadIdx.x & 63;
    const int wave = rfl(threadIdx.x >> 6);
    const bool local = grp == 7;
    const bool pusher = !local && rgp == 7;
#define CTR (ctl + C_CALL + blockIdx.x * 128)
#define PACE ((GAS u64*)(ctl + C_PACE + grp * 128))
#define SLOAD(a) (*(const volatile CAS unsigned*)(a))
#define TOUCH(a) do { unsigned t_ = SLOAD(a); asm volatile("" :: "s"(t_)); } while (0)
    unsigned call = 0;
    float scale = 0.f;
    const int g = lane >> 4, m = lane & 15;

    if (__builtin_expect(pusher, 1)) {
        constexpr int XI = RR / 8;
        const unsigned xo = (panel * 8 + wave) * XI * 1024;
        const unsigned lb = wave * XI * 1024;
        const __amdgpu_buffer_rsrc_t rp = make_rsrc(px, RR * KD);
        dma16(lb, lane * 16, rp, xo);
        dma16(lb + 1024, lane * 16, rp, xo + 1024);
        wg_barrier();
        dma16(lb + 2048, lane * 16, rp, xo + 2048);
        dma16(lb + 3072, lane * 16, rp, xo + 3072);
        const u64 xd = parea + A_ROWS + xo + lane * 16;
        LAS uint8_t* ls = smem + lb + lane * 16;
        wait_vm<3>();
        v4f v = *(LAS v4f*)ls;
#pragma unroll
        for (int i = 0; i < XI; ++i) {
            if (i >= 2) {
                if (i + 3 < XI) wait_vm<3>(); else if (i + 2 < XI) wait_vm<2>(); else wait_vm<1>();
            }
            st16_sys(xd + i * 1024, v);
            LOGT(i);
            if (i + 4 < XI) dma16(lb + (i + 4) * 1024, lane * 16, rp, xo + (i + 4) * 1024);
            if (i == 1 && wave == 7 && lane == 0) {
                __hip_atomic_fetch_max(PACE, rtime() << 6, __ATOMIC_RELAXED, __HIP_MEMORY_SCOPE_WORKGROUP);
                __hip_atomic_fetch_add(PACE, (u64)(RR * XTAU), __ATOMIC_RELAXED, __HIP_MEMORY_SCOPE_WORKGROUP);
            }
            if (i == 0) wait_vm<4>();
            if (i == 1) wait_vm<5>();
            if (i + 1 < XI) v = *(LAS v4f*)(ls + (i + 1) * 1024);
        }
        static_assert(XI >= 6 && XI <= 16, "wait counts");
        scale = __uint_as_float(SLOAD((u64)P.sx)) * __uint_as_float(SLOAD((u64)P.sw));
        call = SLOAD(CTR) + 1;
        wait_vm<0>();
        if (lane == 0) flag_sys(parea + A_XFLAG + (panel * 8 + wave) * 4, call);
        STAMP(40);
        LOGOUT(0);
    } else {
        const int h0 = local ? 64 : (int)(pack(HL) >> (8 * rgp)) & 255;
        const int row0 = local ? rgp * 64 : RR + 2 * ((int)(rpack() >> (8 * rgp)) & 255);
        const bool pulled = !local && panel >= EP && rgp == PRG;
        if (!local && panel >= EP) {
            constexpr int NH = (4 - EP) * 7, NP = 16 * EP + (MS - RR) / 8;
            static_assert(NH * 8 - NP >= 20, "spare waves");
            const int id = wave * NH + (panel - EP) * 7 + rgp;
            const u64 t0 = pack(LL) ? rtime() : 0;
            if (id < NP) {
                const bool isw = id < 16 * EP;
                unsigned vo = ((isw ? id & 15 : id - 16 * EP + RR / 8) * 8 + (lane >> 3)) * KD + (isw ? (id >> 4) * 128 : 0) + (lane & 7) * 16;
                unsigned st = isw ? 128u * ND : 128u, sz = isw ? WRANGE : (unsigned)(MS * KD);
                asm volatile("" : "+v"(vo), "+s"(st), "+s"(sz));
                __builtin_amdgcn_sched_barrier(0);
                const __amdgpu_buffer_rsrc_t rs = make_rsrc(isw ? wcol : px, sz);
#pragma unroll 1
                for (unsigned sg = 0; sg < KSTEPS; ++sg) { dma16(JUNK_OFF, vo, rs, sg * st); wait_vm<PWIN>(); }
            } else if (id - NP < 15) {
                const int j = id - NP;
                const u64 a = j < 1 ? parea + 4
                            : j < 9 ? (u64)P.w + ((u64)j << 21) - 4
                            : j < 10 ? px + MS * KD - 4
                            : j < 12 ? prows + (j - 10) * (MS * NC * 4 - 4)
                            : j < 13 ? (u64)P.g[grp].marea
                            : (u64)P.mrows[grp] + (j & 1) * (RR * NC * 4);
                if (j != 0 && j != 9) delay(rtime(), 60 + j * 10);
                TOUCH(a);
            } else {
                u64 pc; unsigned len;
                asm volatile("s_getpc_b64 %0\n.Lcodepf_pc:\n\ts_mov_b32 %1, .Lcodepf_end-.Lcodepf_pc" : "=s"(pc), "=s"(len));
                if (len > 65536u) len = 0;
                const __amdgpu_buffer_rsrc_t rc = make_rsrc(pc & ~15ull, len & ~15u);
#pragma unroll 1
                for (unsigned o = (id - NP - 15) * 1024u; o < len; o += (NH * 8 - NP - 15) * 1024u) dma16(JUNK_OFF, lane * 16, rc, o);
            }
            if (pack(LL)) delay(t0, 8 * ((unsigned)(pack(LL) >> (8 * rgp)) & 255));
            wg_barrier();
        } else if (!local && ((pack(DL) >> (8 * rgp)) & 255)) {
            delay(rtime(), 4 * ((unsigned)(pack(DL) >> (8 * rgp)) & 255));
        }
        if (wave >= 4) {
            const int w = wave - 4;
            const int r7 = rgp + 3 * (panel & 1);
            const unsigned first = local ? comb(8) << ((rgp + 2 * panel) & 7) : panel < EP ? 0u : comb(7) << (r7 >= 7 ? r7 - 7 : r7);
            u64 ts[3];
            load_tile_h<false>(h0, w, lane, panel * 128, row0 * KD, first, wcol, px, (row0 + h0) * KD, ts);
            if (!local && !pulled && w == 0) {
                if (lane == 0) {
                    const u64 now = (rtime() + LEAD) << 6;
                    __hip_atomic_fetch_max(PACE, now, __ATOMIC_RELAXED, __HIP_MEMORY_SCOPE_WORKGROUP);
                    const u64 t = __hip_atomic_fetch_add(PACE, (u64)(h0 / 2) * TAU, __ATOMIC_RELAXED, __HIP_MEMORY_SCOPE_WORKGROUP);
                    *(volatile LAS u64*)(smem + CTL_OFF + 8) = t - now < (4000ull << 6) ? t : now;
                    __hip_atomic_fetch_add((GAS unsigned*)(PACE + 128), 1u, __ATOMIC_RELAXED, __HIP_MEMORY_SCOPE_AGENT);
                }
                wait_lds();
            }
            call = SLOAD(CTR) + 1;
            asm volatile("" : "+s"(call));
        } else {
            v4f acc[5][2];
            if (local && rgp >= 4) {
                const int j = (rgp - 4) * 16 + panel * 4 + wave;
                const u64 a = j < 8 ? (u64)P.w + ((u64)(j + 1) << 21) - 4 : j < 16 ? (u64)P.mrows[j - 8] : (u64)P.g[(j - 16) & 7].marea;
                if (j < 23) TOUCH(a);
            }
            float sa = __uint_as_float(SLOAD((u64)P.sx)), sb = __uint_as_float(SLOAD((u64)P.sw));
            call = SLOAD(CTR) + 1;
            asm volatile("" : "+s"(sa), "+s"(sb), "+s"(call));
            scale = sa * sb;
            if (h0 > 64) consume_tile<5>(smem, wave, lane, acc);
            else if (h0 > 48) consume_tile<4>(smem, wave, lane, acc);
            else if (h0 > 32) consume_tile<3>(smem, wave, lane, acc);
            else if (h0 > 16) consume_tile<2>(smem, wave, lane, acc);
            else consume_tile<1>(smem, wave, lane, acc);
            if (pulled) {
                const u64 o = (u64)P.g[grp].marea + A_PULL + (((call & 1) * 2 + panel - EP) * HMAX + m) * 512 + wave * 128 + g * 16;
#pragma unroll
                for (int u = 0; u < 5; ++u)
                    if (u * 16 < h0) {
                        st16_sys(o + u * 8192, acc[u][0] * scale);
                        st16_sys(o + u * 8192 + 64, acc[u][1] * scale);
                    }
                wait_vm<0>();
            } else if (!local) {
#pragma unroll
                for (int u = 0; u < 5; ++u)
                    if (u * 16 < h0) {
                        const int r = u * 16 + m;
                        LAS uint8_t* d = smem + STG_OFF + r * 512 + wave * 128;
                        *(LAS v4f*)(d + ((g ^ (r & 7)) << 4)) = acc[u][0] * scale;
                        *(LAS v4f*)(d + (((4 + g) ^ (r & 7)) << 4)) = acc[u][1] * scale;
                    }
                wait_lds();
            } else {
                const u64 o = prows + ((row0 + m) * NC + panel * 128 + wave * 32 + g * 4) * 4;
#pragma unroll
                for (int u = 0; u < 4; ++u) {
                    st16_sys(o + u * 16 * NC * 4, acc[u][0] * scale);
                    st16_sys(o + u * 16 * NC * 4 + 64, acc[u][1] * scale);
                }
            }
        }

        if (pulled) {
            wg_barrier();
            if (wave == 0) {
                const u64 t0 = rtime();
                while (__hip_atomic_load((GAS unsigned*)(PACE + 128), __ATOMIC_RELAXED, __HIP_MEMORY_SCOPE_AGENT) < call * 26u && rtime() - t0 < 3000)
                    __builtin_amdgcn_s_sleep(8);
                u64 te = __hip_atomic_fetch_add(PACE, 0ull, __ATOMIC_RELAXED, __HIP_MEMORY_SCOPE_WORKGROUP);
                te = ((((u64)rfl((unsigned)(te >> 32)) << 32) | rfl((unsigned)te)) >> 6) - GOLEAD + (panel - EP) * (HL[PRG] / 2 * TAU / 64);
                if (lane == 0) {
                    const u64 dt = te - rtime();
                    __hip_atomic_store((GAS u64*)(parea + A_GO) + (panel - EP), ((u64)call << 32) | (dt < 4000 ? dt : 0), __ATOMIC_RELAXED, __HIP_MEMORY_SCOPE_SYSTEM);
                    *(GAS unsigned*)CTR = call;
                }
                STAMP(53);
            }
            return;
        }
        if (!local) {
            const u64 o = prows + ((row0 + (lane >> 5)) * NC + panel * 128 + (lane & 31) * 4) * 4;
            wg_barrier();
            const u64 st = *(volatile LAS u64*)(smem + CTL_OFF + 8);
            u64 slot = (((u64)rfl((unsigned)(st >> 32)) << 32) | rfl((unsigned)st)) + wave * STAU;
#pragma unroll 1
            for (int it = wave; it < h0 / 2; it += 8) {
                const int r = it * 2 + (lane >> 5);
                const v4f v = *(LAS v4f*)(smem + STG_OFF + r * 512 + (((lane & 31) ^ (r & 7)) << 4));
                for (;;) {
                    const long long d = (long long)(slot >> 6) - (long long)rtime();
                    if (d <= 0) break;
                    if (d > 30) __builtin_amdgcn_s_sleep(8);
                }
                st16_sys(o + it * (2 * NC * 4), v);
                LOGT(it / 8);
                slot += 8 * STAU;
            }
            asm volatile(".Lcodepf_end:");
            wait_vm<0>();
            wg_barrier();
            if (wave == 0 && lane == 0) {
                flag_sys(parea + A_DONE + (rgp * 4 + panel) * 4, call);
                *(GAS unsigned*)CTR = call;
            }
            STAMP(52);
            LOGOUT(1);
            if (rgp == 0 && panel < EP) {
                const u64 c3 = (u64)P.g[7].wcol + 3 * 128;
                if (((c3 >> 7) & 7) == 3) {
                    const __amdgpu_buffer_rsrc_t rw = make_rsrc(c3, WRANGE - 3 * 128);
                    const unsigned vl = ((panel * 8 + wave) * 8 + (lane >> 3)) * ND + (lane & 7) * 16;
#pragma unroll 1
                    for (int i = 4; i < KSTEPS; ++i) dma16(JUNK_OFF, vl, rw, i * (128u * ND));
                }
            }
            if (rgp < 2 && panel < EP) {
                constexpr int NI = HL[PRG] / 2, PER = (NI + 15) / 16;
                if (wave == 0) {
                    const u64 go = (u64)P.g[grp].marea + A_GO + panel * 8;
                    const bool ok = wait_flags<2>(go + 4, 1, call, ctl + C_TMO, lane);
                    const u64 now = rtime();
                    const unsigned dt = __hip_atomic_load((GAS unsigned*)go, __ATOMIC_RELAXED, __HIP_MEMORY_SCOPE_SYSTEM);
                    u64 start = now + (dt < 4000 ? dt : 0);
                    if (dt > 150 && dt < 4000) delay(now, dt - 100);
                    const u64 own = (__hip_atomic_fetch_add(PACE, 0ull, __ATOMIC_RELAXED, __HIP_MEMORY_SCOPE_WORKGROUP) >> 6) - OWNLEAD + panel * (NI * TAU / 64);
                    if (own - start < 1000) start = own;
                    *(volatile LAS u64*)(smem + CTL_OFF + 24) = ok ? start : 0;
                    wait_lds();
                }
                wg_barrier();
                const u64 ts = *(volatile LAS u64*)(smem + CTL_OFF + 24);
                const bool ok = ts != 0;
                for (;;) {
                    const long long dd = (long long)(((u64)rfl((unsigned)(ts >> 32)) << 32) | rfl((unsigned)ts)) + (rgp * 8 + wave) * PSP - (long long)rtime();
                    if (dd <= 0) break;
                    if (dd > 30) __builtin_amdgcn_s_sleep(8);
                }
                const u64 a = parea + A_PULL + ((call & 1) * 2 + panel) * HMAX * 512 + lane * 16;
                const u64 d = (u64)P.mrows[grp] + ((RR + hsum(PRG) + (lane >> 5)) * NC + (EP + panel) * 128 + (lane & 31) * 4) * 4;
                static_assert(PER <= 3 && NI >= 16, "registers below");
                const int i0 = rgp * 8 + wave, i1 = i0 + 16 < NI ? i0 + 16 : i0, i2 = i0 + 32 < NI ? i0 + 32 : i0;
                v4f r0, r1, r2;
                if (i2 != i0) {
                    asm volatile("global_load_dwordx4 %0, %3, off sc0 sc1\n\tglobal_load_dwordx4 %1, %4, off sc0 sc1\n\t"
                                 "global_load_dwordx4 %2, %5, off sc0 sc1\n\ts_waitcnt vmcnt(0)"
                                 : "=&v"(r0), "=&v"(r1), "=&v"(r2) : "v"(a + i0 * 1024), "v"(a + i1 * 1024), "v"(a + i2 * 1024) : "memory");
                } else if (i1 != i0) {
                    asm volatile("global_load_dwordx4 %0, %2, off sc0 sc1\n\tglobal_load_dwordx4 %1, %3, off sc0 sc1\n\ts_waitcnt vmcnt(0)"
                                 : "=&v"(r0), "=&v"(r1) : "v"(a + i0 * 1024), "v"(a + i1 * 1024) : "memory");
                    r2 = r0;
                } else {
                    asm volatile("global_load_dwordx4 %0, %1, off sc0 sc1\n\ts_waitcnt vmcnt(0)" : "=&v"(r0) : "v"(a + i0 * 1024) : "memory");
                    r2 = r1 = r0;
                }
                if (!ok) r0 = r1 = r2 = v4f{__builtin_nanf(""), 0, 0, 0};
                st16_sys(d + i0 * (2 * NC * 4), r0);
                if (i1 != i0) st16_sys(d + i1 * (2 * NC * 4), r1);
                if (i2 != i0) st16_sys(d + i2 * (2 * NC * 4), r2);
            }
            return;
        }
        if (rgp == 7) {
            wait_vm<0>();
            wg_barrier();
            if (wave == 7 && lane == 0) *(GAS unsigned*)CTR = call;
            if (panel == 0 && wave < 7) {
                const bool ok = wait_flags((u64)P.g[wave].marea + A_DONE, 36, call, ctl + C_TMO, lane, 3ull << (PRG * 4 + EP));
                if (!ok && lane == 0)
                    flag_sys(prows + row0 * NC * 4, 0x7fc00000u);
                STAMP(45 + wave);
            }
            return;
        }
    }

    const int sg = local ? rgp : grp;
    const int r2 = local ? 0 : RA, h2 = local ? RA : RB;
    const u64 marea = (u64)P.g[sg].marea;
    if (wave == 4) {
        const bool ok = wait_flags(marea + A_XFLAG, 32, call, ctl + C_TMO, lane);
        *(volatile LAS unsigned*)(smem + CTL_OFF) = ok ? 0u : 1u;
        wait_lds();
        STAMP(44);
    }
    wg_barrier();
    if (wave >= 4) {
        const int w = wave - 4;
        load_tile_h<true>(h2, w, lane, panel * 128, r2 * KD, 0u, (u64)P.g[7].wcol, marea + A_ROWS, (r2 + h2) * KD);
        if (w == 0 && lane == 0) flag_sys((u64)P.g[sg].parea + A_DONE + ((local ? 32 : 28) + panel) * 4, call);
    } else {
        if (wave == 0 && lane == 0) *(GAS unsigned*)CTR = call;
        v4f acc[5][2];
        if (local) consume_tile<(RA + 15) / 16>(smem, wave, lane, acc);
        else consume_tile<(RB + 15) / 16>(smem, wave, lane, acc);
        STAMP(32 + wave);
        if (*(volatile LAS unsigned*)(smem + CTL_OFF)) scale = __builtin_nanf("");
        const u64 o = (u64)P.mrows[sg] + ((r2 + m) * NC + panel * 128 + wave * 32 + g * 4) * 4;
#pragma unroll
        for (int u = 0; u < (RM + 15) / 16; ++u)
            if (u * 16 + m < h2) {
                st16_sys(o + u * 16 * NC * 4, acc[u][0] * scale);
                st16_sys(o + u * 16 * NC * 4 + 64, acc[u][1] * scale);
            }
    }
}

}

extern "C" void kernel_launch(int rank, int nranks, ncclComm_t comm,
                              hipStream_t stream,
                              void* const* d_in, const int* in_sizes, int n_in,
                              void* d_out, int out_size,
                              const int* in_dtypes, int out_dtype,
                              void* const* d_out_all,
                              void* const* d_ws_all, size_t ws_bytes) {
    (void)comm; (void)in_dtypes; (void)out_dtype;
    if (nranks != NR || (unsigned)rank >= (unsigned)NR || n_in != 4 || in_sizes[0] != MS * KD || in_sizes[1] != KD * ND
        || out_size != NR * MS * NC || ws_bytes < WS_NEED) {
        fprintf(stderr, "kernel_launch: unexpected problem shape\n");
        return;
    }
    static bool attr_set = false;
    if (!attr_set) {
        HIP_CHECK(hipFuncSetAttribute((const void*)gemm_a2a_kernel, hipFuncAttributeMaxDynamicSharedMemorySize, LDS_BYTES));
        attr_set = true;
    }
    Params P;
    P.x = (const uint8_t*)d_in[0];
    P.w = (const uint8_t*)d_in[1];
    P.sx = (const float*)d_in[2];
    P.sw = (const float*)d_in[3];
    P.ws = (uint8_t*)d_ws_all[rank];
    P.ctl = P.ws + WS_SRC + (size_t)rank * SRC_BYTES;
    P.rank = (unsigned)rank;
    for (int g = 0; g < NR; ++g) {
        const int p = (rank + 1 + g) % NR;
        P.g[g].wcol = P.w + (size_t)p * NC;
        P.g[g].parea = (uint8_t*)d_ws_all[p] + WS_SRC + (size_t)rank * SRC_BYTES;
        P.g[g].prows = (float*)d_out_all[p] + (size_t)rank * MS * NC;
        P.g[g].marea = P.ws + WS_SRC + (size_t)p * SRC_BYTES;
        P.mrows[g] = (float*)d_out + (size_t)p * MS * NC;
    }
    hipLaunchKernelGGL(gemm_a2a_kernel, dim3(NWG), dim3(NTHREADS), LDS_BYTES, stream, P);
    HIP_CHECK(hipGetLastError());
}
```

```cpp
#include <hip/hip_runtime.h>
#include <rccl/rccl.h>
#include <cstdio>
#include <cstdint>
#include <cstddef>

#define HIP_CHECK(expr) do { hipError_t _e = (expr); if (_e != hipSuccess) { \
    fprintf(stderr, "HIP error %s at %s:%d\n", hipGetErrorString(_e), __FILE__, __LINE__); \
    return; } } while (0)

namespace {

typedef int v2i __attribute__((ext_vector_type(2)));
typedef int v4i __attribute__((ext_vector_type(4)));
typedef int v8i __attribute__((ext_vector_type(8)));
typedef float v4f __attribute__((ext_vector_type(4)));
typedef unsigned long long u64;
#define LAS __attribute__((address_space(3)))
#define GAS __attribute__((address_space(1)))
#define CAS __attribute__((address_space(4)))

constexpr int NR = 8;
constexpr int KD = 4096;
constexpr int ND = 4096;
constexpr int MS = 512;
constexpr int NC = 512;
#define RR 96
#define RA 40
constexpr int RB = RR - RA;
constexpr int RM = RA > RB ? RA : RB;
constexpr int HMAX = 80;
#define HLIST 32, 48, 64, 64, 64, 64, 80
#define DLIST 0, 0, 0, 10, 30, 45, 50
#define LLIST 0, 0, 0, 0, 0, 0, 0
constexpr int HL[7] = {HLIST}, DL[7] = {DLIST}, LL[7] = {LLIST};
constexpr int hsum(int n) { int t = 0; for (int i = 0; i < n; ++i) t += HL[i]; return t; }
constexpr u64 pack(const int (&a)[7]) { u64 t = 0; for (int i = 0; i < 7; ++i) t |= (u64)a[i] << (8 * i); return t; }
constexpr u64 rpack() { u64 t = 0; for (int i = 0; i < 7; ++i) t |= (u64)(hsum(i) / 2) << (8 * i); return t; }
constexpr bool hok() { for (int i = 0; i < 7; ++i) if (HL[i] < 2 || HL[i] > HMAX || HL[i] % 2) return false; return true; }
static_assert(RR % 8 == 0 && RA >= 8 && RB >= 8 && RM <= HMAX && hok() && hsum(7) == MS - RR, "shape");
#define EP 2
#define PWIN 6
#define AH1 3
#define TAU 110
#define XTAU 104
#define STAU 135
#define LEAD 25
constexpr int NTHREADS = 512;
constexpr int NWG = 256;
constexpr int KSTEPS = KD / 128;

constexpr unsigned SLOT = 16384 + HMAX * 128, NS = 6;
constexpr unsigned STG_OFF = 2 * SLOT;
static_assert((KSTEPS - 1) % NS == 1, "the last two stages use slots 0 and 1");
constexpr unsigned CTL_OFF = NS * SLOT;
constexpr unsigned JUNK_OFF = CTL_OFF + 64;
constexpr unsigned LDS_BYTES = JUNK_OFF + 1024;

constexpr size_t WS_STAMP = 131072;
constexpr size_t WS_SRC = 2u << 20, SRC_BYTES = 512u << 10;
constexpr size_t A_DONE = 0;
constexpr size_t A_XFLAG = 256;
constexpr size_t A_ROWS = 4096;
constexpr size_t C_TMO = 0;
constexpr size_t C_PACE = 1024;
constexpr size_t C_CALL = 4096;
static_assert(A_ROWS + RR * KD <= SRC_BYTES, "area");
constexpr size_t WS_NEED = WS_SRC + NR * SRC_BYTES;
constexpr u64 DEADLINE_TICKS = 40000000ull;
constexpr unsigned WRANGE = KD * ND - (ND - NC);

struct Group {
    const uint8_t* wcol;
    uint8_t* parea;
    float* prows;
    uint8_t* marea;
};
struct Params {
    const uint8_t* x;
    uint8_t* ctl;
    const float* sx;
    const float* sw;
    Group g[NR];
    float* mrows[NR];
    const uint8_t* w;
    uint8_t* ws;
    unsigned rank;
};

constexpr unsigned comb(int n) { unsigned m = 0; for (int i = 0; i < 32; i += n) m |= 1u << i; return m; }

__device__ __forceinline__ unsigned rfl(unsigned v) { return __builtin_amdgcn_readfirstlane(v); }
__device__ __forceinline__ u64 rtime() { return __builtin_amdgcn_s_memrealtime(); }
__device__ __forceinline__ void delay(u64 t0, unsigned ticks) { while (rtime() - t0 < ticks) __builtin_amdgcn_s_sleep(4); }

__device__ __forceinline__ __amdgpu_buffer_rsrc_t make_rsrc(u64 p, unsigned bytes) {
    return __builtin_amdgcn_make_buffer_rsrc((void*)p, (short)0, (int)bytes, 0x00020000);
}

template <bool SYS = false>
__device__ __forceinline__ void dma16(unsigned lds, unsigned voff, __amdgpu_buffer_rsrc_t rs, unsigned soff) {
    if (SYS) asm volatile("s_mov_b32 m0, %0\n\ts_nop 0\n\tbuffer_load_dwordx4 %1, %2, %3 offen sc0 sc1 lds"
                          :: "s"(lds), "v"(voff), "s"(rs), "s"(soff) : "memory");
    else     asm volatile("s_mov_b32 m0, %0\n\ts_nop 0\n\tbuffer_load_dwordx4 %1, %2, %3 offen lds"
                          :: "s"(lds), "v"(voff), "s"(rs), "s"(soff) : "memory");
}
template <int N> __device__ __forceinline__ void wait_vm() { asm volatile("s_waitcnt vmcnt(%0)" :: "i"(N) : "memory"); }
__device__ __forceinline__ void wg_barrier() { asm volatile("s_barrier" ::: "memory"); }
__device__ __forceinline__ void wait_lds() { asm volatile("s_waitcnt lgkmcnt(0)" ::: "memory"); }
__device__ __forceinline__ void st16_sys(u64 p, v4f v) {
    asm volatile("global_store_dwordx4 %0, %1, off sc0 sc1\n\ts_nop 1" :: "v"(p), "v"(v) : "memory");
}
__device__ __forceinline__ void flag_sys(u64 p, unsigned v) {
    __hip_atomic_store((GAS unsigned*)p, v, __ATOMIC_RELAXED, __HIP_MEMORY_SCOPE_SYSTEM);
}

#define STAMP(i) do {} while (0)
#define LOGT(k) do {} while (0)
#define LOGOUT(half) do {} while (0)

__device__ __forceinline__ bool wait_flags(u64 flags, int n, unsigned call, u64 tmo, int lane) {
    const GAS unsigned* p = (const GAS unsigned*)(lane < n ? flags + 4 * lane : tmo);
    const u64 t0 = rtime();
    for (;;) {
        const u64 now = rtime();
        const unsigned v = __hip_atomic_load(p, __ATOMIC_RELAXED, __HIP_MEMORY_SCOPE_SYSTEM);
        const u64 ge = __ballot(v >= call), eq = __ballot(v == call);
        if ((eq >> 63) || now - t0 > DEADLINE_TICKS) break;
        if ((ge | (~0ull << n)) == ~0ull) return true;
    }
    if (lane == 0) __hip_atomic_store((GAS unsigned*)tmo, call, __ATOMIC_RELAXED, __HIP_MEMORY_SCOPE_AGENT);
    return false;
}

template <int NXW, bool SYS>
__device__ __forceinline__ void load_tile(int w, int lane, unsigned woff, unsigned xoff, unsigned first,
                                          u64 pw, u64 px, unsigned xbytes, u64* ts = nullptr) {
    constexpr int CNT = 4 + NXW, AHEAD = AH1;
    const unsigned v0 = (lane >> 3) * KD + (((lane & 7) ^ ((lane >> 4) & 3)) << 4);
    unsigned vw[4];
#pragma unroll
    for (int j = 0; j < 4; ++j) vw[j] = (v0 ^ (((j >> 1) & 1) << 6)) + (4 * w + j) * (8u * ND) - j * 1024u + woff;
    unsigned vx0 = (v0 ^ ((w & 1) << 6)) + w * (8u * KD) + xoff, vx1 = vx0 + 32u * KD, vx2 = vx0 + 64u * KD;
    unsigned ioff = w * 1024;
    u64 todo = ((u64)~first << 32) | first;
    asm volatile("" : "+v"(vw[0]), "+v"(vw[1]), "+v"(vw[2]), "+v"(vw[3]), "+v"(vx0), "+v"(vx1), "+v"(vx2), "+s"(ioff), "+s"(todo));
    __builtin_amdgcn_sched_barrier(0);
    asm volatile("" : "+s"(pw), "+s"(px));
    const __amdgpu_buffer_rsrc_t rw = make_rsrc(pw, WRANGE), rx = make_rsrc(px, xbytes);
#define STEP_W "buffer_load_dwordx4 %4, %11, %13 offen lds\n\t" \
               "buffer_load_dwordx4 %5, %11, %13 offen offset:1024 lds\n\t" \
               "buffer_load_dwordx4 %6, %11, %13 offen offset:2048 lds\n\t" \
               "buffer_load_dwordx4 %7, %11, %13 offen offset:3072 lds"
#define STEP_X(m, v, pol) "\n\ts_mov_b32 m0, %" m "\n\ts_nop 0\n\tbuffer_load_dwordx4 %" v ", %12, %14 offen" pol " lds"
#define STEP_OPS :: "s"(ioff + w * 3072), "s"(ioff + 16384), "s"(ioff + 20480), "s"(ioff + 24576), \
                    "v"(vw[0]), "v"(vw[1]), "v"(vw[2]), "v"(vw[3]), "v"(vx0), "v"(vx1), "v"(vx2), \
                    "s"(rw), "s"(rx), "s"(ks * (128u * ND)), "s"(ks * 128u), "i"(CNT * (AHEAD - 1)) : "memory"
#define STEP_P(head, pol) do { \
        if (NXW == 0) asm volatile(head STEP_W STEP_OPS); \
        if (NXW == 1) asm volatile(head STEP_W STEP_X("1", "8", pol) STEP_OPS); \
        if (NXW == 2) asm volatile(head STEP_W STEP_X("1", "8", pol) STEP_X("2", "9", pol) STEP_OPS); \
        if (NXW == 3) asm volatile(head STEP_W STEP_X("1", "8", pol) STEP_X("2", "9", pol) STEP_X("3", "10", pol) STEP_OPS); \
    } while (0)
#define STEP(head) do { \
        const unsigned ks = __builtin_ctzll(todo) & 31; \
        todo &= todo - 1; \
        if (SYS) STEP_P(head, " sc0 sc1"); else STEP_P(head, ""); \
        ioff += SLOT; \
        if (ioff >= SLOT * NS) ioff -= SLOT * NS; \
    } while (0)
#pragma unroll 1
    for (int i = 0; i < AHEAD; ++i) STEP("s_mov_b32 m0, %0\n\ts_nop 0\n\t");
#pragma unroll 1
    while (todo) STEP("s_mov_b32 m0, %0\n\ts_waitcnt vmcnt(%15)\n\ts_barrier\n\t");
    static_assert(AHEAD >= 2 && AHEAD <= 5 && AHEAD < (int)NS, "drain");
    if (AHEAD > 4) { wait_vm<CNT * 4>(); wg_barrier(); }
    if (AHEAD > 3) { wait_vm<CNT * 3>(); wg_barrier(); }
    if (AHEAD > 2) { wait_vm<CNT * 2>(); wg_barrier(); }
    wait_vm<CNT>(); wg_barrier();
    wait_vm<0>(); wg_barrier();
}
template <bool SYS>
__device__ __forceinline__ void load_tile_h(int h, int w, int lane, unsigned woff, unsigned xoff, unsigned first,
                                            u64 pw, u64 px, unsigned xbytes, u64* ts = nullptr) {
    const int nxw = (h - 8 * w + 31) >> 5;
    if (nxw == 2) load_tile<2, SYS>(w, lane, woff, xoff, first, pw, px, xbytes, ts);
    else if (nxw == 1) load_tile<1, SYS>(w, lane, woff, xoff, first, pw, px, xbytes, ts);
    else if (nxw == 3) load_tile<3, SYS>(w, lane, woff, xoff, first, pw, px, xbytes, ts);
    else load_tile<0, SYS>(w, lane, woff, xoff, first, pw, px, xbytes, ts);
}

template <int NU>
__device__ __forceinline__ void consume_tile(LAS uint8_t* smem, int c, int lane, v4f (&acc)[5][2]) {
    const int g = lane >> 4, q = (lane & 15) >> 1, p = lane & 1, m = lane & 15;
    const unsigned fw = ((q >> 1) & 3) | ((g & 1) << 2);
    const unsigned w0 = (16 * g + q) * 128 + (((2 * c) ^ fw) << 4) + p * 8;
    const unsigned x0 = 16384 + m * 128 + ((g ^ ((m >> 1) & 7)) << 4);
#pragma unroll
    for (int u = 0; u < 5; ++u) { acc[u][0] = v4f{0, 0, 0, 0}; acc[u][1] = v4f{0, 0, 0, 0}; }
    v8i wf[2][2], xf[2][NU];
    auto load = [&](int s, v8i (&wv)[2], v8i (&xv)[NU]) {
        LAS uint8_t* sl = smem + (s % NS) * SLOT;
#pragma unroll
        for (int cb = 0; cb < 2; ++cb) {
            LAS uint8_t* b = sl + (w0 ^ (cb << 4));
            v2i t0 = __builtin_amdgcn_ds_read_tr8_b64_v2i32((LAS v2i*)(b));
            v2i t1 = __builtin_amdgcn_ds_read_tr8_b64_v2i32((LAS v2i*)(b + 1024));
            v2i t2 = __builtin_amdgcn_ds_read_tr8_b64_v2i32((LAS v2i*)(b + 8192));
            v2i t3 = __builtin_amdgcn_ds_read_tr8_b64_v2i32((LAS v2i*)(b + 9216));
            wv[cb] = v8i{t0[0], t0[1], t1[0], t1[1], t2[0], t2[1], t3[0], t3[1]};
        }
#pragma unroll
        for (int u = 0; u < NU; ++u) {
            v4i a = *(LAS v4i*)(sl + x0 + u * 2048);
            v4i b = *(LAS v4i*)(sl + (x0 ^ 64) + u * 2048);
            xv[u] = v8i{a[0], a[1], a[2], a[3], b[0], b[1], b[2], b[3]};
        }
    };
    auto mma = [&](v8i (&wv)[2], v8i (&xv)[NU]) {
#pragma unroll
        for (int u = 0; u < NU; ++u)
#pragma unroll
            for (int cb = 0; cb < 2; ++cb)
                acc[u][cb] = __builtin_amdgcn_mfma_scale_f32_16x16x128_f8f6f4(wv[cb], xv[u], acc[u][cb], 1, 0, 0, 0, 0, 0);
    };
    auto step = [&](int s, v8i (&wn)[2], v8i (&xn)[NU], v8i (&wo)[2], v8i (&xo)[NU]) {
        wg_barrier();
        __builtin_amdgcn_sched_barrier(0);
        load(s, wn, xn);
        mma(wo, xo);
        constexpr int PER = (8 + 2 * NU + 2 * NU - 1) / (2 * NU);
#pragma unroll
        for (int i = 0; i < 2 * NU; ++i) {
            __builtin_amdgcn_sched_group_barrier(0x100, PER, 0);
            __builtin_amdgcn_sched_group_barrier(0x8, 1, 0);
        }
        __builtin_amdgcn_sched_barrier(0);
        wait_lds();
    };
    wg_barrier();
    load(0, wf[0], xf[0]);
    wait_lds();
    step(1, wf[1], xf[1], wf[0], xf[0]);
#pragma unroll 1
    for (int it = 0; it < (KSTEPS - 2) / (int)NS; ++it) {
#pragma unroll
        for (int s = 2; s < 2 + (int)NS; s += 2) {
            step(s, wf[0], xf[0], wf[1], xf[1]);
            step(s + 1, wf[1], xf[1], wf[0], xf[0]);
        }
    }
    static_assert((KSTEPS - 2) % NS == 0 && NS % 2 == 0, "trip count");
    mma(wf[1], xf[1]);
}

__global__ void __launch_bounds__(NTHREADS) gemm_a2a_kernel(Params P) {
    extern __shared__ __attribute__((aligned(16))) uint8_t smem_[];
    LAS uint8_t* smem = (LAS uint8_t*)smem_;
    const int grp = blockIdx.x & 7, idx = blockIdx.x >> 3, panel = idx & 3, rgp = ((idx >> 2) + 7) & 7;
    const CAS char* ka = (const CAS char*)__builtin_amdgcn_kernarg_segment_ptr();
#define KARG(off) (*(const volatile CAS u64*)(ka + (off)))
    const u64 px = KARG(offsetof(Params, x)), ctl = KARG(offsetof(Params, ctl));
    const u64 wcol = KARG(offsetof(Params, g) + grp * sizeof(Group)), parea = KARG(offsetof(Params, g) + grp * sizeof(Group) + 8);
    const u64 prows = KARG(offsetof(Params, g) + grp * sizeof(Group) + 16);
    const int lane = threadIdx.x & 63;
    const int wave = rfl(threadIdx.x >> 6);
    const bool local = grp == 7;
    const bool pusher = !local && rgp == 7;
#define CTR (ctl + C_CALL + blockIdx.x * 128)
#define PACE ((GAS u64*)(ctl + C_PACE + grp * 128))
#define SLOAD(a) (*(const volatile CAS unsigned*)(a))
#define TOUCH(a) do { unsigned t_ = SLOAD(a); asm volatile("" :: "s"(t_)); } while (0)
    unsigned call = 0;
    float scale = 0.f;
    const int g = lane >> 4, m = lane & 15;

    if (__builtin_expect(pusher, 1)) {
        constexpr int XI = RR / 8;
        const unsigned xo = (panel * 8 + wave) * XI * 1024;
        const unsigned lb = wave * XI * 1024;
        const __amdgpu_buffer_rsrc_t rp = make_rsrc(px, RR * KD);
        dma16(lb, lane * 16, rp, xo);
        dma16(lb + 1024, lane * 16, rp, xo + 1024);
        wg_barrier();
        dma16(lb + 2048, lane * 16, rp, xo + 2048);
        dma16(lb + 3072, lane * 16, rp, xo + 3072);
        const u64 xd = parea + A_ROWS + xo + lane * 16;
        LAS uint8_t* ls = smem + lb + lane * 16;
        wait_vm<3>();
        v4f v = *(LAS v4f*)ls;
#pragma unroll
        for (int i = 0; i < XI; ++i) {
            if (i >= 2) {
                if (i + 3 < XI) wait_vm<3>(); else if (i + 2 < XI) wait_vm<2>(); else wait_vm<1>();
            }
            st16_sys(xd + i * 1024, v);
            LOGT(i);
            if (i + 4 < XI) dma16(lb + (i + 4) * 1024, lane * 16, rp, xo + (i + 4) * 1024);
            if (i == 1 && wave == 7 && lane == 0) {
                __hip_atomic_fetch_max(PACE, rtime() << 6, __ATOMIC_RELAXED, __HIP_MEMORY_SCOPE_WORKGROUP);
                __hip_atomic_fetch_add(PACE, (u64)(RR * XTAU), __ATOMIC_RELAXED, __HIP_MEMORY_SCOPE_WORKGROUP);
            }
            if (i == 0) wait_vm<4>();
            if (i == 1) wait_vm<5>();
            if (i + 1 < XI) v = *(LAS v4f*)(ls + (i + 1) * 1024);
        }
        static_assert(XI >= 6 && XI <= 16, "wait counts");
        scale = __uint_as_float(SLOAD((u64)P.sx)) * __uint_as_float(SLOAD((u64)P.sw));
        call = SLOAD(CTR) + 1;
        wait_vm<0>();
        if (lane == 0) flag_sys(parea + A_XFLAG + (panel * 8 + wave) * 4, call);
        STAMP(40);
        LOGOUT(0);
    } else {
        const int h0 = local ? 64 : (int)(pack(HL) >> (8 * rgp)) & 255;
        const int row0 = local ? rgp * 64 : RR + 2 * ((int)(rpack() >> (8 * rgp)) & 255);
        if (!local && panel >= EP) {
            constexpr int NH = (4 - EP) * 7, NP = 16 * EP + (MS - RR) / 8;
            static_assert(NH * 8 - NP >= 20, "spare waves");
            const int id = wave * NH + (panel - EP) * 7 + rgp;
            const u64 t0 = pack(LL) ? rtime() : 0;
            if (id < NP) {
                const bool isw = id < 16 * EP;
                unsigned vo = ((isw ? id & 15 : id - 16 * EP + RR / 8) * 8 + (lane >> 3)) * KD + (isw ? (id >> 4) * 128 : 0) + (lane & 7) * 16;
                unsigned st = isw ? 128u * ND : 128u, sz = isw ? WRANGE : (unsigned)(MS * KD);
                asm volatile("" : "+v"(vo), "+s"(st), "+s"(sz));
                __builtin_amdgcn_sched_barrier(0);
                const __amdgpu_buffer_rsrc_t rs = make_rsrc(isw ? wcol : px, sz);
#pragma unroll 1
                for (unsigned sg = 0; sg < KSTEPS; ++sg) { dma16(JUNK_OFF, vo, rs, sg * st); wait_vm<PWIN>(); }
            } else if (id - NP < 15) {
                const int j = id - NP;
                const u64 a = j < 1 ? parea + 4
                            : j < 9 ? (u64)P.w + ((u64)j << 21) - 4
                            : j < 10 ? px + MS * KD - 4
                            : j < 12 ? prows + (j - 10) * (MS * NC * 4 - 4)
                            : j < 13 ? (u64)P.g[grp].marea
                            : (u64)P.mrows[grp] + (j & 1) * (RR * NC * 4);
                if (j != 0 && j != 9) delay(rtime(), 60 + j * 10);
                TOUCH(a);
            } else {
                u64 pc; unsigned len;
                asm volatile("s_getpc_b64 %0\n.Lcodepf_pc:\n\ts_mov_b32 %1, .Lcodepf_end-.Lcodepf_pc" : "=s"(pc), "=s"(len));
                if (len > 65536u) len = 0;
                const __amdgpu_buffer_rsrc_t rc = make_rsrc(pc & ~15ull, len & ~15u);
#pragma unroll 1
                for (unsigned o = (id - NP - 15) * 1024u; o < len; o += (NH * 8 - NP - 15) * 1024u) dma16(JUNK_OFF, lane * 16, rc, o);
            }
            if (pack(LL)) delay(t0, 8 * ((unsigned)(pack(LL) >> (8 * rgp)) & 255));
            wg_barrier();
        } else if (!local && ((pack(DL) >> (8 * rgp)) & 255)) {
            delay(rtime(), 4 * ((unsigned)(pack(DL) >> (8 * rgp)) & 255));
        }
        if (wave >= 4) {
            const int w = wave - 4;
            const int r7 = rgp + 3 * (panel & 1);
            const unsigned first = local ? comb(8) << ((rgp + 2 * panel) & 7) : panel < EP ? 0u : comb(7) << (r7 >= 7 ? r7 - 7 : r7);
            u64 ts[3];
            load_tile_h<false>(h0, w, lane, panel * 128, row0 * KD, first, wcol, px, (row0 + h0) * KD, ts);
            if (!local && w == 0) {
                if (lane == 0) {
                    const u64 now = (rtime() + LEAD) << 6;
                    __hip_atomic_fetch_max(PACE, now, __ATOMIC_RELAXED, __HIP_MEMORY_SCOPE_WORKGROUP);
                    const u64 t = __hip_atomic_fetch_add(PACE, (u64)(h0 / 2) * TAU, __ATOMIC_RELAXED, __HIP_MEMORY_SCOPE_WORKGROUP);
                    *(volatile LAS u64*)(smem + CTL_OFF + 8) = t - now < (4000ull << 6) ? t : now;
                }
                wait_lds();
            }
            call = SLOAD(CTR) + 1;
            asm volatile("" : "+s"(call));
        } else {
            v4f acc[5][2];
            if (local && rgp >= 4) {
                const int j = (rgp - 4) * 16 + panel * 4 + wave;
                const u64 a = j < 8 ? (u64)P.w + ((u64)(j + 1) << 21) - 4 : j < 16 ? (u64)P.mrows[j - 8] : (u64)P.g[(j - 16) & 7].marea;
                if (j < 23) TOUCH(a);
            }
            float sa = __uint_as_float(SLOAD((u64)P.sx)), sb = __uint_as_float(SLOAD((u64)P.sw));
            call = SLOAD(CTR) + 1;
            asm volatile("" : "+s"(sa), "+s"(sb), "+s"(call));
            scale = sa * sb;
            if (h0 > 64) consume_tile<5>(smem, wave, lane, acc);
            else if (h0 > 48) consume_tile<4>(smem, wave, lane, acc);
            else if (h0 > 32) consume_tile<3>(smem, wave, lane, acc);
            else if (h0 > 16) consume_tile<2>(smem, wave, lane, acc);
            else consume_tile<1>(smem, wave, lane, acc);
            if (!local) {
#pragma unroll
                for (int u = 0; u < 5; ++u)
                    if (u * 16 < h0) {
                        const int r = u * 16 + m;
                        LAS uint8_t* d = smem + STG_OFF + r * 512 + wave * 128;
                        *(LAS v4f*)(d + ((g ^ (r & 7)) << 4)) = acc[u][0] * scale;
                        *(LAS v4f*)(d + (((4 + g) ^ (r & 7)) << 4)) = acc[u][1] * scale;
                    }
                wait_lds();
            } else {
                const u64 o = prows + ((row0 + m) * NC + panel * 128 + wave * 32 + g * 4) * 4;
#pragma unroll
                for (int u = 0; u < 4; ++u) {
                    st16_sys(o + u * 16 * NC * 4, acc[u][0] * scale);
                    st16_sys(o + u * 16 * NC * 4 + 64, acc[u][1] * scale);
                }
            }
        }

        if (!local) {
            const u64 o = prows + ((row0 + (lane >> 5)) * NC + panel * 128 + (lane & 31) * 4) * 4;
            wg_barrier();
            const u64 st = *(volatile LAS u64*)(smem + CTL_OFF + 8);
            u64 slot = (((u64)rfl((unsigned)(st >> 32)) << 32) | rfl((unsigned)st)) + wave * STAU;
#pragma unroll 1
            for (int it = wave; it < h0 / 2; it += 8) {
                const int r = it * 2 + (lane >> 5);
                const v4f v = *(LAS v4f*)(smem + STG_OFF + r * 512 + (((lane & 31) ^ (r & 7)) << 4));
                for (;;) {
                    const long long d = (long long)(slot >> 6) - (long long)rtime();
                    if (d <= 0) break;
                    if (d > 30) __builtin_amdgcn_s_sleep(8);
                }
                st16_sys(o + it * (2 * NC * 4), v);
                LOGT(it / 8);
                slot += 8 * STAU;
            }
            asm volatile(".Lcodepf_end:");
            wait_vm<0>();
            wg_barrier();
            if (wave == 0 && lane == 0) {
                flag_sys(parea + A_DONE + (rgp * 4 + panel) * 4, call);
                *(GAS unsigned*)CTR = call;
            }
            STAMP(52);
            LOGOUT(1);
            if (rgp == 0 && panel < EP) {
                const u64 c3 = (u64)P.g[7].wcol + 3 * 128;
                if (((c3 >> 7) & 7) == 3) {
                    const __amdgpu_buffer_rsrc_t rw = make_rsrc(c3, WRANGE - 3 * 128);
                    const unsigned vl = ((panel * 8 + wave) * 8 + (lane >> 3)) * ND + (lane & 7) * 16;
#pragma unroll 1
                    for (int i = 4; i < KSTEPS; ++i) dma16(JUNK_OFF, vl, rw, i * (128u * ND));
                }
            }
            return;
        }
        if (rgp == 7) {
            wait_vm<0>();
            wg_barrier();
            if (wave == 7 && lane == 0) *(GAS unsigned*)CTR = call;
            if (panel == 0 && wave < 7) {
                const bool ok = wait_flags((u64)P.g[wave].marea + A_DONE, 36, call, ctl + C_TMO, lane);
                if (!ok && lane == 0)
                    flag_sys(prows + row0 * NC * 4, 0x7fc00000u);
                STAMP(45 + wave);
            }
            return;
        }
    }

    const int sg = local ? rgp : grp;
    const int r2 = local ? 0 : RA, h2 = local ? RA : RB;
    const u64 marea = (u64)P.g[sg].marea;
    if (wave == 4) {
        const bool ok = wait_flags(marea + A_XFLAG, 32, call, ctl + C_TMO, lane);
        *(volatile LAS unsigned*)(smem + CTL_OFF) = ok ? 0u : 1u;
        wait_lds();
        STAMP(44);
    }
    wg_barrier();
    if (wave >= 4) {
        const int w = wave - 4;
        load_tile_h<true>(h2, w, lane, panel * 128, r2 * KD, 0u, (u64)P.g[7].wcol, marea + A_ROWS, (r2 + h2) * KD);
        if (w == 0 && lane == 0) flag_sys((u64)P.g[sg].parea + A_DONE + ((local ? 32 : 28) + panel) * 4, call);
    } else {
        if (wave == 0 && lane == 0) *(GAS unsigned*)CTR = call;
        v4f acc[5][2];
        if (local) consume_tile<(RA + 15) / 16>(smem, wave, lane, acc);
        else consume_tile<(RB + 15) / 16>(smem, wave, lane, acc);
        STAMP(32 + wave);
        if (*(volatile LAS unsigned*)(smem + CTL_OFF)) scale = __builtin_nanf("");
        const u64 o = (u64)P.mrows[sg] + ((r2 + m) * NC + panel * 128 + wave * 32 + g * 4) * 4;
#pragma unroll
        for (int u = 0; u < (RM + 15) / 16; ++u)
            if (u * 16 + m < h2) {
                st16_sys(o + u * 16 * NC * 4, acc[u][0] * scale);
                st16_sys(o + u * 16 * NC * 4 + 64, acc[u][1] * scale);
            }
    }
}

}

extern "C" void kernel_launch(int rank, int nranks, ncclComm_t comm,
                              hipStream_t stream,
                              void* const* d_in, const int* in_sizes, int n_in,
                              void* d_out, int out_size,
                              const int* in_dtypes, int out_dtype,
                              void* const* d_out_all,
                              void* const* d_ws_all, size_t ws_bytes) {
    (void)comm; (void)in_dtypes; (void)out_dtype;
    if (nranks != NR || (unsigned)rank >= (unsigned)NR || n_in != 4 || in_sizes[0] != MS * KD || in_sizes[1] != KD * ND
        || out_size != NR * MS * NC || ws_bytes < WS_NEED) {
        fprintf(stderr, "kernel_launch: unexpected problem shape\n");
        return;
    }
    static bool attr_set = false;
    if (!attr_set) {
        HIP_CHECK(hipFuncSetAttribute((const void*)gemm_a2a_kernel, hipFuncAttributeMaxDynamicSharedMemorySize, LDS_BYTES));
        attr_set = true;
    }
    Params P;
    P.x = (const uint8_t*)d_in[0];
    P.w = (const uint8_t*)d_in[1];
    P.sx = (const float*)d_in[2];
    P.sw = (const float*)d_in[3];
    P.ws = (uint8_t*)d_ws_all[rank];
    P.ctl = P.ws + WS_SRC + (size_t)rank * SRC_BYTES;
    P.rank = (unsigned)rank;
    for (int g = 0; g < NR; ++g) {
        const int p = (rank + 1 + g) % NR;
        P.g[g].wcol = P.w + (size_t)p * NC;
        P.g[g].parea = (uint8_t*)d_ws_all[p] + WS_SRC + (size_t)rank * SRC_BYTES;
        P.g[g].prows = (float*)d_out_all[p] + (size_t)rank * MS * NC;
        P.g[g].marea = P.ws + WS_SRC + (size_t)p * SRC_BYTES;
        P.mrows[g] = (float*)d_out + (size_t)p * MS * NC;
    }
    hipLaunchKernelGGL(gemm_a2a_kernel, dim3(NWG), dim3(NTHREADS), LDS_BYTES, stream, P);
    HIP_CHECK(hipGetLastError());
}
```

```cpp
#include <hip/hip_runtime.h>
#include <rccl/rccl.h>
#include <cstdio>
#include <cstdint>
#include <cstddef>

#define HIP_CHECK(expr) do { hipError_t _e = (expr); if (_e != hipSuccess) { \
    fprintf(stderr, "HIP error %s at %s:%d\n", hipGetErrorString(_e), __FILE__, __LINE__); \
    return; } } while (0)

namespace {

typedef int v2i __attribute__((ext_vector_type(2)));
typedef int v4i __attribute__((ext_vector_type(4)));
typedef int v8i __attribute__((ext_vector_type(8)));
typedef float v4f __attribute__((ext_vector_type(4)));
typedef unsigned long long u64;
#define LAS __attribute__((address_space(3)))
#define GAS __attribute__((address_space(1)))
#define CAS __attribute__((address_space(4)))

constexpr int NR = 8;
constexpr int KD = 4096;
constexpr int ND = 4096;
constexpr int MS = 512;
constexpr int NC = 512;
constexpr int RR = 104;
constexpr int RA = 40;
constexpr int RB = RR - RA;
constexpr int RM = RA > RB ? RA : RB;
constexpr int HMAX = 80;
constexpr int HL[7] = {32, 48, 64, 64, 64, 64, 72};
constexpr int DL[7] = {0, 0, 0, 10, 30, 45, 50};
constexpr int hsum(int n) { int t = 0; for (int i = 0; i < n; ++i) t += HL[i]; return t; }
constexpr u64 pack(const int (&a)[7]) { u64 t = 0; for (int i = 0; i < 7; ++i) t |= (u64)a[i] << (8 * i); return t; }
constexpr u64 rpack() { u64 t = 0; for (int i = 0; i < 7; ++i) t |= (u64)(hsum(i) / 2) << (8 * i); return t; }
constexpr bool hok() { for (int i = 0; i < 7; ++i) if (HL[i] < 2 || HL[i] > HMAX || HL[i] % 2) return false; return true; }
static_assert(RR % 8 == 0 && RA >= 8 && RB >= 8 && RM <= HMAX && hok() && hsum(7) == MS - RR, "shape");
constexpr int EP = 2;
constexpr int PWIN = 6;
constexpr int AH1 = 3;
constexpr int TAU = 110;
constexpr int XTAU = 104;
constexpr int STAU = 135;
constexpr int LEAD = 25;
constexpr int NTHREADS = 512;
constexpr int NWG = 256;
constexpr int KSTEPS = KD / 128;

constexpr unsigned SLOT = 16384 + HMAX * 128, NS = 6;
constexpr unsigned STG_OFF = 2 * SLOT;
static_assert((KSTEPS - 1) % NS == 1, "the last two stages use slots 0 and 1");
constexpr unsigned CTL_OFF = NS * SLOT;
constexpr unsigned JUNK_OFF = CTL_OFF + 64;
constexpr unsigned LDS_BYTES = JUNK_OFF + 1024;

constexpr size_t WS_SRC = 2u << 20, SRC_BYTES = 512u << 10;
constexpr size_t A_DONE = 0;
constexpr size_t A_XFLAG = 256;
constexpr size_t A_ROWS = 4096;
constexpr size_t C_TMO = 0;
constexpr size_t C_PACE = 1024;
constexpr size_t C_CALL = 4096;
static_assert(A_ROWS + RR * KD <= SRC_BYTES, "area");
constexpr size_t WS_NEED = WS_SRC + NR * SRC_BYTES;
constexpr u64 DEADLINE_TICKS = 40000000ull;
constexpr unsigned WRANGE = KD * ND - (ND - NC);

struct Group {
    const uint8_t* wcol;
    uint8_t* parea;
    float* prows;
    uint8_t* marea;
};
struct Params {
    const uint8_t* x;
    uint8_t* ctl;
    const float* sx;
    const float* sw;
    Group g[NR];
    float* mrows[NR];
    const uint8_t* w;
};

constexpr unsigned comb(int n) { unsigned m = 0; for (int i = 0; i < 32; i += n) m |= 1u << i; return m; }

__device__ __forceinline__ unsigned rfl(unsigned v) { return __builtin_amdgcn_readfirstlane(v); }
__device__ __forceinline__ u64 rtime() { return __builtin_amdgcn_s_memrealtime(); }
__device__ __forceinline__ void delay(u64 t0, unsigned ticks) { while (rtime() - t0 < ticks) __builtin_amdgcn_s_sleep(4); }

__device__ __forceinline__ __amdgpu_buffer_rsrc_t make_rsrc(u64 p, unsigned bytes) {
    return __builtin_amdgcn_make_buffer_rsrc((void*)p, (short)0, (int)bytes, 0x00020000);
}

template <bool SYS = false>
__device__ __forceinline__ void dma16(unsigned lds, unsigned voff, __amdgpu_buffer_rsrc_t rs, unsigned soff) {
    if (SYS) asm volatile("s_mov_b32 m0, %0\n\ts_nop 0\n\tbuffer_load_dwordx4 %1, %2, %3 offen sc0 sc1 lds"
                          :: "s"(lds), "v"(voff), "s"(rs), "s"(soff) : "memory");
    else     asm volatile("s_mov_b32 m0, %0\n\ts_nop 0\n\tbuffer_load_dwordx4 %1, %2, %3 offen lds"
                          :: "s"(lds), "v"(voff), "s"(rs), "s"(soff) : "memory");
}
template <int N> __device__ __forceinline__ void wait_vm() { asm volatile("s_waitcnt vmcnt(%0)" :: "i"(N) : "memory"); }
__device__ __forceinline__ void wg_barrier() { asm volatile("s_barrier" ::: "memory"); }
__device__ __forceinline__ void wait_lds() { asm volatile("s_waitcnt lgkmcnt(0)" ::: "memory"); }
__device__ __forceinline__ void st16_sys(u64 p, v4f v) {
    asm volatile("global_store_dwordx4 %0, %1, off sc0 sc1\n\ts_nop 1" :: "v"(p), "v"(v) : "memory");
}
__device__ __forceinline__ void flag_sys(u64 p, unsigned v) {
    __hip_atomic_store((GAS unsigned*)p, v, __ATOMIC_RELAXED, __HIP_MEMORY_SCOPE_SYSTEM);
}

__device__ __forceinline__ bool wait_flags(u64 flags, int n, unsigned call, u64 tmo, int lane) {
    const GAS unsigned* p = (const GAS unsigned*)(lane < n ? flags + 4 * lane : tmo);
    const u64 t0 = rtime();
    for (;;) {
        const u64 now = rtime();
        const unsigned v = __hip_atomic_load(p, __ATOMIC_RELAXED, __HIP_MEMORY_SCOPE_SYSTEM);
        const u64 ge = __ballot(v >= call), eq = __ballot(v == call);
        if ((eq >> 63) || now - t0 > DEADLINE_TICKS) break;
        if ((ge | (~0ull << n)) == ~0ull) return true;
    }
    if (lane == 0) __hip_atomic_store((GAS unsigned*)tmo, call, __ATOMIC_RELAXED, __HIP_MEMORY_SCOPE_AGENT);
    return false;
}

template <int NXW, bool SYS>
__device__ __forceinline__ void load_tile(int w, int lane, unsigned woff, unsigned xoff, unsigned first,
                                          u64 pw, u64 px, unsigned xbytes) {
    constexpr int CNT = 4 + NXW, AHEAD = AH1;
    const unsigned v0 = (lane >> 3) * KD + (((lane & 7) ^ ((lane >> 4) & 3)) << 4);
    unsigned vw[4];
#pragma unroll
    for (int j = 0; j < 4; ++j) vw[j] = (v0 ^ (((j >> 1) & 1) << 6)) + (4 * w + j) * (8u * ND) - j * 1024u + woff;
    unsigned vx0 = (v0 ^ ((w & 1) << 6)) + w * (8u * KD) + xoff, vx1 = vx0 + 32u * KD, vx2 = vx0 + 64u * KD;
    unsigned ioff = w * 1024;
    u64 todo = ((u64)~first << 32) | first;
    asm volatile("" : "+v"(vw[0]), "+v"(vw[1]), "+v"(vw[2]), "+v"(vw[3]), "+v"(vx0), "+v"(vx1), "+v"(vx2), "+s"(ioff), "+s"(todo));
    __builtin_amdgcn_sched_barrier(0);
    asm volatile("" : "+s"(pw), "+s"(px));
    const __amdgpu_buffer_rsrc_t rw = make_rsrc(pw, WRANGE), rx = make_rsrc(px, xbytes);
#define STEP_W "buffer_load_dwordx4 %4, %11, %13 offen lds\n\t" \
               "buffer_load_dwordx4 %5, %11, %13 offen offset:1024 lds\n\t" \
               "buffer_load_dwordx4 %6, %11, %13 offen offset:2048 lds\n\t" \
               "buffer_load_dwordx4 %7, %11, %13 offen offset:3072 lds"
#define STEP_X(m, v, pol) "\n\ts_mov_b32 m0, %" m "\n\ts_nop 0\n\tbuffer_load_dwordx4 %" v ", %12, %14 offen" pol " lds"
#define STEP_OPS :: "s"(ioff + w * 3072), "s"(ioff + 16384), "s"(ioff + 20480), "s"(ioff + 24576), \
                    "v"(vw[0]), "v"(vw[1]), "v"(vw[2]), "v"(vw[3]), "v"(vx0), "v"(vx1), "v"(vx2), \
                    "s"(rw), "s"(rx), "s"(ks * (128u * ND)), "s"(ks * 128u), "i"(CNT * (AHEAD - 1)) : "memory"
#define STEP_P(head, pol) do { \
        if (NXW == 0) asm volatile(head STEP_W STEP_OPS); \
        if (NXW == 1) asm volatile(head STEP_W STEP_X("1", "8", pol) STEP_OPS); \
        if (NXW == 2) asm volatile(head STEP_W STEP_X("1", "8", pol) STEP_X("2", "9", pol) STEP_OPS); \
        if (NXW == 3) asm volatile(head STEP_W STEP_X("1", "8", pol) STEP_X("2", "9", pol) STEP_X("3", "10", pol) STEP_OPS); \
    } while (0)
#define STEP(head) do { \
        const unsigned ks = __builtin_ctzll(todo) & 31; \
        todo &= todo - 1; \
        if (SYS) STEP_P(head, " sc0 sc1"); else STEP_P(head, ""); \
        ioff += SLOT; \
        if (ioff >= SLOT * NS) ioff -= SLOT * NS; \
    } while (0)
#pragma unroll 1
    for (int i = 0; i < AHEAD; ++i) STEP("s_mov_b32 m0, %0\n\ts_nop 0\n\t");
#pragma unroll 1
    while (todo) STEP("s_mov_b32 m0, %0\n\ts_waitcnt vmcnt(%15)\n\ts_barrier\n\t");
    static_assert(AHEAD >= 2 && AHEAD <= 5 && AHEAD < (int)NS, "drain");
    if (AHEAD > 4) { wait_vm<CNT * 4>(); wg_barrier(); }
    if (AHEAD > 3) { wait_vm<CNT * 3>(); wg_barrier(); }
    if (AHEAD > 2) { wait_vm<CNT * 2>(); wg_barrier(); }
    wait_vm<CNT>(); wg_barrier();
    wait_vm<0>(); wg_barrier();
}
template <bool SYS>
__device__ __forceinline__ void load_tile_h(int h, int w, int lane, unsigned woff, unsigned xoff, unsigned first,
                                            u64 pw, u64 px, unsigned xbytes) {
    const int nxw = (h - 8 * w + 31) >> 5;
    if (nxw == 2) load_tile<2, SYS>(w, lane, woff, xoff, first, pw, px, xbytes);
    else if (nxw == 1) load_tile<1, SYS>(w, lane, woff, xoff, first, pw, px, xbytes);
    else if (nxw == 3) load_tile<3, SYS>(w, lane, woff, xoff, first, pw, px, xbytes);
    else load_tile<0, SYS>(w, lane, woff, xoff, first, pw, px, xbytes);
}

template <int NU>
__device__ __forceinline__ void consume_tile(LAS uint8_t* smem, int c, int lane, v4f (&acc)[5][2]) {
    const int g = lane >> 4, q = (lane & 15) >> 1, p = lane & 1, m = lane & 15;
    const unsigned fw = ((q >> 1) & 3) | ((g & 1) << 2);
    const unsigned w0 = (16 * g + q) * 128 + (((2 * c) ^ fw) << 4) + p * 8;
    const unsigned x0 = 16384 + m * 128 + ((g ^ ((m >> 1) & 7)) << 4);
#pragma unroll
    for (int u = 0; u < 5; ++u) { acc[u][0] = v4f{0, 0, 0, 0}; acc[u][1] = v4f{0, 0, 0, 0}; }
    v8i wf[2][2], xf[2][NU];
    auto load = [&](int s, v8i (&wv)[2], v8i (&xv)[NU]) {
        LAS uint8_t* sl = smem + (s % NS) * SLOT;
#pragma unroll
        for (int cb = 0; cb < 2; ++cb) {
            LAS uint8_t* b = sl + (w0 ^ (cb << 4));
            v2i t0 = __builtin_amdgcn_ds_read_tr8_b64_v2i32((LAS v2i*)(b));
            v2i t1 = __builtin_amdgcn_ds_read_tr8_b64_v2i32((LAS v2i*)(b + 1024));
            v2i t2 = __builtin_amdgcn_ds_read_tr8_b64_v2i32((LAS v2i*)(b + 8192));
            v2i t3 = __builtin_amdgcn_ds_read_tr8_b64_v2i32((LAS v2i*)(b + 9216));
            wv[cb] = v8i{t0[0], t0[1], t1[0], t1[1], t2[0], t2[1], t3[0], t3[1]};
        }
#pragma unroll
        for (int u = 0; u < NU; ++u) {
            v4i a = *(LAS v4i*)(sl + x0 + u * 2048);
            v4i b = *(LAS v4i*)(sl + (x0 ^ 64) + u * 2048);
            xv[u] = v8i{a[0], a[1], a[2], a[3], b[0], b[1], b[2], b[3]};
        }
    };
    auto mma = [&](v8i (&wv)[2], v8i (&xv)[NU]) {
#pragma unroll
        for (int u = 0; u < NU; ++u)
#pragma unroll
            for (int cb = 0; cb < 2; ++cb)
                acc[u][cb] = __builtin_amdgcn_mfma_scale_f32_16x16x128_f8f6f4(wv[cb], xv[u], acc[u][cb], 1, 0, 0, 0, 0, 0);
    };
    auto step = [&](int s, v8i (&wn)[2], v8i (&xn)[NU], v8i (&wo)[2], v8i (&xo)[NU]) {
        wg_barrier();
        __builtin_amdgcn_sched_barrier(0);
        load(s, wn, xn);
        mma(wo, xo);
        constexpr int PER = (8 + 2 * NU + 2 * NU - 1) / (2 * NU);
#pragma unroll
        for (int i = 0; i < 2 * NU; ++i) {
            __builtin_amdgcn_sched_group_barrier(0x100, PER, 0);
            __builtin_amdgcn_sched_group_barrier(0x8, 1, 0);
        }
        __builtin_amdgcn_sched_barrier(0);
        wait_lds();
    };
    wg_barrier();
    load(0, wf[0], xf[0]);
    wait_lds();
    step(1, wf[1], xf[1], wf[0], xf[0]);
#pragma unroll 1
    for (int it = 0; it < (KSTEPS - 2) / (int)NS; ++it) {
#pragma unroll
        for (int s = 2; s < 2 + (int)NS; s += 2) {
            step(s, wf[0], xf[0], wf[1], xf[1]);
            step(s + 1, wf[1], xf[1], wf[0], xf[0]);
        }
    }
    static_assert((KSTEPS - 2) % NS == 0 && NS % 2 == 0, "trip count");
    mma(wf[1], xf[1]);
}

__global__ void __launch_bounds__(NTHREADS) gemm_a2a_kernel(Params P) {
    extern __shared__ __attribute__((aligned(16))) uint8_t smem_[];
    LAS uint8_t* smem = (LAS uint8_t*)smem_;
    const int grp = blockIdx.x & 7, idx = blockIdx.x >> 3, panel = idx & 3, rgp = ((idx >> 2) + 7) & 7;
    const CAS char* ka = (const CAS char*)__builtin_amdgcn_kernarg_segment_ptr();
#define KARG(off) (*(const volatile CAS u64*)(ka + (off)))
    const u64 px = KARG(offsetof(Params, x)), ctl = KARG(offsetof(Params, ctl));
    const u64 wcol = KARG(offsetof(Params, g) + grp * sizeof(Group)), parea = KARG(offsetof(Params, g) + grp * sizeof(Group) + 8);
    const u64 prows = KARG(offsetof(Params, g) + grp * sizeof(Group) + 16);
    const int lane = threadIdx.x & 63;
    const int wave = rfl(threadIdx.x >> 6);
    const bool local = grp == 7;
    const bool pusher = !local && rgp == 7;
#define CTR (ctl + C_CALL + blockIdx.x * 128)
#define PACE ((GAS u64*)(ctl + C_PACE + grp * 128))
#define SLOAD(a) (*(const volatile CAS unsigned*)(a))
#define TOUCH(a) do { unsigned t_ = SLOAD(a); asm volatile("" :: "s"(t_)); } while (0)
    unsigned call = 0;
    float scale = 0.f;
    const int g = lane >> 4, m = lane & 15;

    if (__builtin_expect(pusher, 1)) {
        constexpr int XI = RR / 8;
        const unsigned xo = (panel * 8 + wave) * XI * 1024;
        const unsigned lb = wave * XI * 1024;
        const __amdgpu_buffer_rsrc_t rp = make_rsrc(px, RR * KD);
        dma16(lb, lane * 16, rp, xo);
        dma16(lb + 1024, lane * 16, rp, xo + 1024);
        wg_barrier();
        dma16(lb + 2048, lane * 16, rp, xo + 2048);
        dma16(lb + 3072, lane * 16, rp, xo + 3072);
        const u64 xd = parea + A_ROWS + xo + lane * 16;
        LAS uint8_t* ls = smem + lb + lane * 16;
        wait_vm<3>();
        v4f v = *(LAS v4f*)ls;
#pragma unroll
        for (int i = 0; i < XI; ++i) {
            if (i >= 2) {
                if (i + 3 < XI) wait_vm<3>(); else if (i + 2 < XI) wait_vm<2>(); else wait_vm<1>();
            }
            st16_sys(xd + i * 1024, v);
            if (i + 4 < XI) dma16(lb + (i + 4) * 1024, lane * 16, rp, xo + (i + 4) * 1024);
            if (i == 1 && wave == 7 && lane == 0) {
                __hip_atomic_fetch_max(PACE, rtime() << 6, __ATOMIC_RELAXED, __HIP_MEMORY_SCOPE_WORKGROUP);
                __hip_atomic_fetch_add(PACE, (u64)(RR * XTAU), __ATOMIC_RELAXED, __HIP_MEMORY_SCOPE_WORKGROUP);
            }
            if (i == 0) wait_vm<4>();
            if (i == 1) wait_vm<5>();
            if (i + 1 < XI) v = *(LAS v4f*)(ls + (i + 1) * 1024);
        }
        static_assert(XI >= 6 && XI <= 16, "wait counts");
        scale = __uint_as_float(SLOAD((u64)P.sx)) * __uint_as_float(SLOAD((u64)P.sw));
        call = SLOAD(CTR) + 1;
        wait_vm<0>();
        if (lane == 0) flag_sys(parea + A_XFLAG + (panel * 8 + wave) * 4, call);
    } else {
        const int h0 = local ? 64 : (int)(pack(HL) >> (8 * rgp)) & 255;
        const int row0 = local ? rgp * 64 : RR + 2 * ((int)(rpack() >> (8 * rgp)) & 255);
        if (!local && panel >= EP) {
            constexpr int NH = (4 - EP) * 7, NP = 16 * EP + (MS - RR) / 8;
            static_assert(NH * 8 - NP >= 20, "spare waves");
            const int id = wave * NH + (panel - EP) * 7 + rgp;
            if (id < NP) {
                const bool isw = id < 16 * EP;
                unsigned vo = ((isw ? id & 15 : id - 16 * EP + RR / 8) * 8 + (lane >> 3)) * KD + (isw ? (id >> 4) * 128 : 0) + (lane & 7) * 16;
                unsigned st = isw ? 128u * ND : 128u, sz = isw ? WRANGE : (unsigned)(MS * KD);
                asm volatile("" : "+v"(vo), "+s"(st), "+s"(sz));
                __builtin_amdgcn_sched_barrier(0);
                const __amdgpu_buffer_rsrc_t rs = make_rsrc(isw ? wcol : px, sz);
#pragma unroll 1
                for (unsigned sg = 0; sg < KSTEPS; ++sg) { dma16(JUNK_OFF, vo, rs, sg * st); wait_vm<PWIN>(); }
            } else if (id - NP < 15) {
                const int j = id - NP;
                const u64 a = j < 1 ? parea + 4
                            : j < 9 ? (u64)P.w + ((u64)j << 21) - 4
                            : j < 10 ? px + MS * KD - 4
                            : j < 12 ? prows + (j - 10) * (MS * NC * 4 - 4)
                            : j < 13 ? (u64)P.g[grp].marea
                            : (u64)P.mrows[grp] + (j & 1) * (RR * NC * 4);
                if (j != 0 && j != 9) delay(rtime(), 60 + j * 10);
                TOUCH(a);
            } else {
                u64 pc; unsigned len;
                asm volatile("s_getpc_b64 %0\n.Lcodepf_pc:\n\ts_mov_b32 %1, .Lcodepf_end-.Lcodepf_pc" : "=s"(pc), "=s"(len));
                if (len > 65536u) len = 0;
                const __amdgpu_buffer_rsrc_t rc = make_rsrc(pc & ~15ull, len & ~15u);
#pragma unroll 1
                for (unsigned o = (id - NP - 15) * 1024u; o < len; o += (NH * 8 - NP - 15) * 1024u) dma16(JUNK_OFF, lane * 16, rc, o);
            }
            wg_barrier();
        } else if (!local && ((pack(DL) >> (8 * rgp)) & 255)) {
            delay(rtime(), 4 * ((unsigned)(pack(DL) >> (8 * rgp)) & 255));
        }
        if (wave >= 4) {
            const int w = wave - 4;
            const int r7 = rgp + 3 * (panel & 1);
            const unsigned first = local ? comb(8) << ((rgp + 2 * panel) & 7) : panel < EP ? 0u : comb(7) << (r7 >= 7 ? r7 - 7 : r7);
            load_tile_h<false>(h0, w, lane, panel * 128, row0 * KD, first, wcol, px, (row0 + h0) * KD);
            if (!local && w == 0) {
                if (lane == 0) {
                    const u64 now = (rtime() + LEAD) << 6;
                    __hip_atomic_fetch_max(PACE, now, __ATOMIC_RELAXED, __HIP_MEMORY_SCOPE_WORKGROUP);
                    const u64 t = __hip_atomic_fetch_add(PACE, (u64)(h0 / 2) * TAU, __ATOMIC_RELAXED, __HIP_MEMORY_SCOPE_WORKGROUP);
                    *(volatile LAS u64*)(smem + CTL_OFF + 8) = t - now < (4000ull << 6) ? t : now;
                }
                wait_lds();
            }
            call = SLOAD(CTR) + 1;
            asm volatile("" : "+s"(call));
        } else {
            v4f acc[5][2];
            if (local && rgp >= 4) {
                const int j = (rgp - 4) * 16 + panel * 4 + wave;
                const u64 a = j < 8 ? (u64)P.w + ((u64)(j + 1) << 21) - 4 : j < 16 ? (u64)P.mrows[(j - 8) & 7] : (u64)P.g[(j - 16) & 7].marea;
                if (j < 23) TOUCH(a);
            }
            float sa = __uint_as_float(SLOAD((u64)P.sx)), sb = __uint_as_float(SLOAD((u64)P.sw));
            call = SLOAD(CTR) + 1;
            asm volatile("" : "+s"(sa), "+s"(sb), "+s"(call));
            scale = sa * sb;
            if (h0 > 64) consume_tile<5>(smem, wave, lane, acc);
            else if (h0 > 48) consume_tile<4>(smem, wave, lane, acc);
            else if (h0 > 32) consume_tile<3>(smem, wave, lane, acc);
            else consume_tile<2>(smem, wave, lane, acc);
            if (!local) {
#pragma unroll
                for (int u = 0; u < 5; ++u)
                    if (u * 16 < h0) {
                        const int r = u * 16 + m;
                        LAS uint8_t* d = smem + STG_OFF + r * 512 + wave * 128;
                        *(LAS v4f*)(d + ((g ^ (r & 7)) << 4)) = acc[u][0] * scale;
                        *(LAS v4f*)(d + (((4 + g) ^ (r & 7)) << 4)) = acc[u][1] * scale;
                    }
                wait_lds();
            } else {
                const u64 o = prows + ((row0 + m) * NC + panel * 128 + wave * 32 + g * 4) * 4;
#pragma unroll
                for (int u = 0; u < 4; ++u) {
                    st16_sys(o + u * 16 * NC * 4, acc[u][0] * scale);
                    st16_sys(o + u * 16 * NC * 4 + 64, acc[u][1] * scale);
                }
            }
        }

        if (!local) {
            const u64 o = prows + ((row0 + (lane >> 5)) * NC + panel * 128 + (lane & 31) * 4) * 4;
            wg_barrier();
            const u64 st = *(volatile LAS u64*)(smem + CTL_OFF + 8);
            u64 slot = (((u64)rfl((unsigned)(st >> 32)) << 32) | rfl((unsigned)st)) + wave * STAU;
#pragma unroll 1
            for (int it = wave; it < h0 / 2; it += 8) {
                const int r = it * 2 + (lane >> 5);
                const v4f v = *(LAS v4f*)(smem + STG_OFF + r * 512 + (((lane & 31) ^ (r & 7)) << 4));
                for (;;) {
                    const long long d = (long long)(slot >> 6) - (long long)rtime();
                    if (d <= 0) break;
                    if (d > 30) __builtin_amdgcn_s_sleep(8);
                }
                st16_sys(o + it * (2 * NC * 4), v);
                slot += 8 * STAU;
            }
            asm volatile(".Lcodepf_end:");
            wait_vm<0>();
            wg_barrier();
            if (wave == 0 && lane == 0) {
                flag_sys(parea + A_DONE + (rgp * 4 + panel) * 4, call);
                *(GAS unsigned*)CTR = call;
            }
            if (rgp == 0 && panel < EP) {
                const u64 c3 = (u64)P.g[7].wcol + 3 * 128;
                if (((c3 >> 7) & 7) == 3) {
                    const __amdgpu_buffer_rsrc_t rw = make_rsrc(c3, WRANGE - 3 * 128);
                    const unsigned vl = ((panel * 8 + wave) * 8 + (lane >> 3)) * ND + (lane & 7) * 16;
#pragma unroll 1
                    for (int i = 4; i < KSTEPS; ++i) dma16(JUNK_OFF, vl, rw, i * (128u * ND));
                }
            }
            return;
        }
        if (rgp == 7) {
            wait_vm<0>();
            wg_barrier();
            if (wave == 7 && lane == 0) *(GAS unsigned*)CTR = call;
            if (panel == 0 && wave < 7) {
                const bool ok = wait_flags((u64)P.g[wave].marea + A_DONE, 36, call, ctl + C_TMO, lane);
                if (!ok && lane == 0)
                    flag_sys(prows + row0 * NC * 4, 0x7fc00000u);
            }
            return;
        }
    }

    const int sg = local ? rgp : grp;
    const int r2 = local ? 0 : RA, h2 = local ? RA : RB;
    const u64 marea = (u64)P.g[sg].marea;
    if (wave == 4) {
        const bool ok = wait_flags(marea + A_XFLAG, 32, call, ctl + C_TMO, lane);
        *(volatile LAS unsigned*)(smem + CTL_OFF) = ok ? 0u : 1u;
        wait_lds();
    }
    wg_barrier();
    if (wave >= 4) {
        const int w = wave - 4;
        load_tile_h<true>(h2, w, lane, panel * 128, r2 * KD, 0u, (u64)P.g[7].wcol, marea + A_ROWS, (r2 + h2) * KD);
        if (w == 0 && lane == 0) flag_sys((u64)P.g[sg].parea + A_DONE + ((local ? 32 : 28) + panel) * 4, call);
    } else {
        if (wave == 0 && lane == 0) *(GAS unsigned*)CTR = call;
        v4f acc[5][2];
        if (local) consume_tile<(RA + 15) / 16>(smem, wave, lane, acc);
        else consume_tile<(RB + 15) / 16>(smem, wave, lane, acc);
        if (*(volatile LAS unsigned*)(smem + CTL_OFF)) scale = __builtin_nanf("");
        const u64 o = (u64)P.mrows[sg] + ((r2 + m) * NC + panel * 128 + wave * 32 + g * 4) * 4;
#pragma unroll
        for (int u = 0; u < (RM + 15) / 16; ++u)
            if (u * 16 + m < h2) {
                st16_sys(o + u * 16 * NC * 4, acc[u][0] * scale);
                st16_sys(o + u * 16 * NC * 4 + 64, acc[u][1] * scale);
            }
    }
}

}

extern "C" void kernel_launch(int rank, int nranks, ncclComm_t comm,
                              hipStream_t stream,
                              void* const* d_in, const int* in_sizes, int n_in,
                              void* d_out, int out_size,
                              const int* in_dtypes, int out_dtype,
                              void* const* d_out_all,
                              void* const* d_ws_all, size_t ws_bytes) {
    (void)comm; (void)in_dtypes; (void)out_dtype;
    if (nranks != NR || (unsigned)rank >= (unsigned)NR || n_in != 4 || in_sizes[0] != MS * KD || in_sizes[1] != KD * ND
        || out_size != NR * MS * NC || ws_bytes < WS_NEED) {
        fprintf(stderr, "kernel_launch: unexpected problem shape\n");
        return;
    }
    static bool attr_set = false;
    if (!attr_set) {
        HIP_CHECK(hipFuncSetAttribute((const void*)gemm_a2a_kernel, hipFuncAttributeMaxDynamicSharedMemorySize, LDS_BYTES));
        attr_set = true;
    }
    Params P;
    P.x = (const uint8_t*)d_in[0];
    P.w = (const uint8_t*)d_in[1];
    P.sx = (const float*)d_in[2];
    P.sw = (const float*)d_in[3];
    uint8_t* ws = (uint8_t*)d_ws_all[rank];
    P.ctl = ws + WS_SRC + (size_t)rank * SRC_BYTES;
    for (int g = 0; g < NR; ++g) {
        const int p = (rank + 1 + g) % NR;
        P.g[g].wcol = P.w + (size_t)p * NC;
        P.g[g].parea = (uint8_t*)d_ws_all[p] + WS_SRC + (size_t)rank * SRC_BYTES;
        P.g[g].prows = (float*)d_out_all[p] + (size_t)rank * MS * NC;
        P.g[g].marea = ws + WS_SRC + (size_t)p * SRC_BYTES;
        P.mrows[g] = (float*)d_out + (size_t)p * MS * NC;
    }
    hipLaunchKernelGGL(gemm_a2a_kernel, dim3(NWG), dim3(NTHREADS), LDS_BYTES, stream, P);
    HIP_CHECK(hipGetLastError());
}
```

```cpp
#include <hip/hip_runtime.h>
#include <rccl/rccl.h>
#include <cstdio>
#include <cstdint>
#include <cstddef>

#define HIP_CHECK(expr) do { hipError_t _e = (expr); if (_e != hipSuccess) { \
    fprintf(stderr, "HIP error %s at %s:%d\n", hipGetErrorString(_e), __FILE__, __LINE__); \
    return; } } while (0)

namespace {

typedef int v2i __attribute__((ext_vector_type(2)));
typedef int v4i __attribute__((ext_vector_type(4)));
typedef int v8i __attribute__((ext_vector_type(8)));
typedef float v4f __attribute__((ext_vector_type(4)));
typedef unsigned long long u64;
#define LAS __attribute__((address_space(3)))
#define GAS __attribute__((address_space(1)))
#define CAS __attribute__((address_space(4)))

constexpr int NR = 8;
constexpr int KD = 4096;
constexpr int ND = 4096;
constexpr int MS = 512;
constexpr int NC = 512;
constexpr int RR = 96;
constexpr int RA = 40;
constexpr int RB = RR - RA;
constexpr int RM = RA > RB ? RA : RB;
constexpr int HMAX = 80;
constexpr int HL[7] = {32, 48, 64, 64, 64, 64, 80};
constexpr int DL[7] = {0, 0, 0, 10, 30, 45, 50};
constexpr int hsum(int n) { int t = 0; for (int i = 0; i < n; ++i) t += HL[i]; return t; }
constexpr u64 pack(const int (&a)[7]) { u64 t = 0; for (int i = 0; i < 7; ++i) t |= (u64)a[i] << (8 * i); return t; }
constexpr u64 rpack() { u64 t = 0; for (int i = 0; i < 7; ++i) t |= (u64)(hsum(i) / 2) << (8 * i); return t; }
constexpr bool hok() { for (int i = 0; i < 7; ++i) if (HL[i] < 2 || HL[i] > HMAX || HL[i] % 2) return false; return true; }
static_assert(RR % 8 == 0 && RA >= 8 && RB >= 8 && RM <= HMAX && hok() && hsum(7) == MS - RR, "shape");
constexpr int EP = 2;
constexpr int PWIN = 6;
constexpr int AH1 = 3;
constexpr int TAU = 110;
constexpr int XTAU = 104;
constexpr int STAU = 135;
constexpr int XOFF = 5;
constexpr int LEAD = 50;
constexpr int NTHREADS = 512;
constexpr int NWG = 256;
constexpr int KSTEPS = KD / 128;

constexpr unsigned SLOT = 16384 + HMAX * 128, NS = 6;
constexpr unsigned STG_OFF = 2 * SLOT;
static_assert((KSTEPS - 1) % NS == 1, "the last two stages use slots 0 and 1");
constexpr unsigned CTL_OFF = NS * SLOT;
constexpr unsigned JUNK_OFF = CTL_OFF + 64;
constexpr unsigned LDS_BYTES = JUNK_OFF + 1024;

constexpr size_t WS_SRC = 2u << 20, SRC_BYTES = 512u << 10;
constexpr size_t A_DONE = 0;
constexpr size_t A_XFLAG = 256;
constexpr size_t A_ROWS = 4096;
constexpr size_t C_TMO = 0;
constexpr size_t C_PACE = 1024;
constexpr size_t C_CALL = 4096;
static_assert(A_ROWS + RR * KD <= SRC_BYTES, "area");
constexpr size_t WS_NEED = WS_SRC + NR * SRC_BYTES;
constexpr u64 DEADLINE_TICKS = 40000000ull;
constexpr unsigned WRANGE = KD * ND - (ND - NC);

struct Group {
    const uint8_t* wcol;
    uint8_t* parea;
    float* prows;
    uint8_t* marea;
};
struct Params {
    const uint8_t* x;
    uint8_t* ctl;
    const float* sx;
    const float* sw;
    Group g[NR];
    float* mrows[NR];
    const uint8_t* w;
};

constexpr unsigned comb(int n) { unsigned m = 0; for (int i = 0; i < 32; i += n) m |= 1u << i; return m; }

__device__ __forceinline__ unsigned rfl(unsigned v) { return __builtin_amdgcn_readfirstlane(v); }
__device__ __forceinline__ u64 rtime() { return __builtin_amdgcn_s_memrealtime(); }
__device__ __forceinline__ void delay(u64 t0, unsigned ticks) { while (rtime() - t0 < ticks) __builtin_amdgcn_s_sleep(4); }

__device__ __forceinline__ __amdgpu_buffer_rsrc_t make_rsrc(u64 p, unsigned bytes) {
    return __builtin_amdgcn_make_buffer_rsrc((void*)p, (short)0, (int)bytes, 0x00020000);
}

template <bool SYS = false>
__device__ __forceinline__ void dma16(unsigned lds, unsigned voff, __amdgpu_buffer_rsrc_t rs, unsigned soff) {
    if (SYS) asm volatile("s_mov_b32 m0, %0\n\ts_nop 0\n\tbuffer_load_dwordx4 %1, %2, %3 offen sc0 sc1 lds"
                          :: "s"(lds), "v"(voff), "s"(rs), "s"(soff) : "memory");
    else     asm volatile("s_mov_b32 m0, %0\n\ts_nop 0\n\tbuffer_load_dwordx4 %1, %2, %3 offen lds"
                          :: "s"(lds), "v"(voff), "s"(rs), "s"(soff) : "memory");
}
template <int N> __device__ __forceinline__ void wait_vm() { asm volatile("s_waitcnt vmcnt(%0)" :: "i"(N) : "memory"); }
__device__ __forceinline__ void wg_barrier() { asm volatile("s_barrier" ::: "memory"); }
__device__ __forceinline__ void wait_lds() { asm volatile("s_waitcnt lgkmcnt(0)" ::: "memory"); }
__device__ __forceinline__ void st16_sys(u64 p, v4f v) {
    asm volatile("global_store_dwordx4 %0, %1, off sc0 sc1\n\ts_nop 1" :: "v"(p), "v"(v) : "memory");
}
__device__ __forceinline__ void flag_sys(u64 p, unsigned v) {
    __hip_atomic_store((GAS unsigned*)p, v, __ATOMIC_RELAXED, __HIP_MEMORY_SCOPE_SYSTEM);
}

__device__ __forceinline__ bool wait_flags(u64 flags, int n, unsigned call, u64 tmo, int lane) {
    const GAS unsigned* p = (const GAS unsigned*)(lane < n ? flags + 4 * lane : tmo);
    const u64 t0 = rtime();
    for (;;) {
        const u64 now = rtime();
        const unsigned v = __hip_atomic_load(p, __ATOMIC_RELAXED, __HIP_MEMORY_SCOPE_SYSTEM);
        const u64 ge = __ballot(v >= call), eq = __ballot(v == call);
        if ((eq >> 63) || now - t0 > DEADLINE_TICKS) break;
        if ((ge | (~0ull << n)) == ~0ull) return true;
    }
    if (lane == 0) __hip_atomic_store((GAS unsigned*)tmo, call, __ATOMIC_RELAXED, __HIP_MEMORY_SCOPE_AGENT);
    return false;
}

struct Loader {
    unsigned vw[4], vx[3], ioff;
    u64 todo;
    __amdgpu_buffer_rsrc_t rw, rx;
    int nxw;
};
#define STEP_W "buffer_load_dwordx4 %4, %11, %13 offen lds\n\t" \
               "buffer_load_dwordx4 %5, %11, %13 offen offset:1024 lds\n\t" \
               "buffer_load_dwordx4 %6, %11, %13 offen offset:2048 lds\n\t" \
               "buffer_load_dwordx4 %7, %11, %13 offen offset:3072 lds"
#define STEP_X(m, v, pol) "\n\ts_mov_b32 m0, %" m "\n\ts_nop 0\n\tbuffer_load_dwordx4 %" v ", %12, %14 offen" pol " lds"
#define STEP_OPS(n) :: "s"(L.ioff + w * 3072), "s"(L.ioff + 16384), "s"(L.ioff + 20480), "s"(L.ioff + 24576), \
                       "v"(L.vw[0]), "v"(L.vw[1]), "v"(L.vw[2]), "v"(L.vw[3]), "v"(L.vx[0]), "v"(L.vx[1]), "v"(L.vx[2]), \
                       "s"(L.rw), "s"(L.rx), "s"(ks * (128u * ND)), "s"(ks * 128u), "i"(n) : "memory"
template <bool SYS>
__device__ __forceinline__ void load_begin(Loader& L, int h, int w, int lane, unsigned woff, unsigned xoff, unsigned first,
                                           u64 pw, u64 px, unsigned xbytes) {
    const unsigned v0 = (lane >> 3) * KD + (((lane & 7) ^ ((lane >> 4) & 3)) << 4);
#pragma unroll
    for (int j = 0; j < 4; ++j)
        L.vw[j] = (v0 ^ (((j >> 1) & 1) << 6)) + (4 * w + j) * (8u * ND) - j * 1024u + woff;
#pragma unroll
    for (int j = 0; j < 3; ++j) L.vx[j] = (v0 ^ ((w & 1) << 6)) + (w + 4 * j) * (8u * KD) + xoff;
    L.ioff = w * 1024;
    L.todo = ((u64)~first << 32) | first;
    L.nxw = (h - 8 * w + 31) >> 5;
    asm volatile("" : "+v"(L.vw[0]), "+v"(L.vw[1]), "+v"(L.vw[2]), "+v"(L.vw[3]), "+v"(L.vx[0]), "+v"(L.vx[1]), "+v"(L.vx[2]),
                      "+s"(L.ioff), "+s"(L.todo), "+s"(L.nxw), "+s"(xbytes));
    __builtin_amdgcn_sched_barrier(0);
    asm volatile("" : "+s"(pw), "+s"(px));
    L.rw = make_rsrc(pw, WRANGE);
    L.rx = make_rsrc(px, xbytes);
#pragma unroll 1
    for (int i = 0; i < AH1; ++i) {
        const unsigned ks = __builtin_ctzll(L.todo) & 31;
        L.todo &= L.todo - 1;
        asm volatile("s_mov_b32 m0, %0\n\ts_nop 0\n\t" STEP_W STEP_OPS(0));
        if (L.nxw > 0) dma16<SYS>(L.ioff + 16384, L.vx[0], L.rx, ks * 128u);
        if (L.nxw > 1) dma16<SYS>(L.ioff + 20480, L.vx[1], L.rx, ks * 128u);
        if (L.nxw > 2) dma16<SYS>(L.ioff + 24576, L.vx[2], L.rx, ks * 128u);
        L.ioff += SLOT;
    }
}
template <int NXW, bool SYS>
__device__ __forceinline__ void load_run_n(Loader& L, int w, u64 pace, unsigned kib) {
    constexpr int CNT = 4 + NXW, N = CNT * (AH1 - 1);
#define STEP_H "s_mov_b32 m0, %0\n\ts_waitcnt vmcnt(%15)\n\ts_barrier\n\t"
#define STEP_P(pol) do { \
        if (NXW == 0) asm volatile(STEP_H STEP_W STEP_OPS(N)); \
        if (NXW == 1) asm volatile(STEP_H STEP_W STEP_X("1", "8", pol) STEP_OPS(N)); \
        if (NXW == 2) asm volatile(STEP_H STEP_W STEP_X("1", "8", pol) STEP_X("2", "9", pol) STEP_OPS(N)); \
        if (NXW == 3) asm volatile(STEP_H STEP_W STEP_X("1", "8", pol) STEP_X("2", "9", pol) STEP_X("3", "10", pol) STEP_OPS(N)); \
    } while (0)
#pragma unroll 1
    while (L.todo) {
        const unsigned ks = __builtin_ctzll(L.todo) & 31;
        L.todo &= L.todo - 1;
        if (SYS) STEP_P(" sc0 sc1"); else STEP_P("");
        L.ioff += SLOT;
        if (L.ioff >= SLOT * NS) L.ioff -= SLOT * NS;
    }
    static_assert(AH1 == 3 && AH1 < (int)NS, "drain");
    if (!SYS && pace) {
        const u64 now = (rtime() + LEAD) << 6, len = (u64)kib * TAU;
        u64 t;
        asm volatile("s_mov_b64 exec, 1\n\tglobal_atomic_umax_x2 %1, %2, %4\n\tglobal_atomic_add_x2 %0, %1, %3, %4 sc0\n\ts_mov_b64 exec, -1\n\t"
                     "s_waitcnt vmcnt(%5)\n\ts_barrier\n\ts_waitcnt vmcnt(%6)\n\ts_barrier\n\ts_waitcnt vmcnt(0)\n\ts_barrier"
                     : "=&v"(t) : "v"(0u), "v"(now), "v"(len), "s"(pace), "i"(CNT * 2), "i"(CNT) : "memory");
        if ((threadIdx.x & 63) == 0) *(volatile LAS u64*)(uintptr_t)(CTL_OFF + 8) = t - now < (4000ull << 6) ? t : now;
        wait_lds();
    } else {
        wait_vm<CNT * 2>(); wg_barrier();
        wait_vm<CNT>(); wg_barrier();
        wait_vm<0>(); wg_barrier();
    }
}
template <bool SYS>
__device__ __forceinline__ void load_run(Loader& L, int w, u64 pace = 0, unsigned kib = 0) {
    if (L.nxw == 2) load_run_n<2, SYS>(L, w, pace, kib);
    else if (L.nxw == 1) load_run_n<1, SYS>(L, w, pace, kib);
    else if (L.nxw == 3) load_run_n<3, SYS>(L, w, pace, kib);
    else load_run_n<0, SYS>(L, w, pace, kib);
}

template <int NU>
__device__ __forceinline__ void consume_tile(LAS uint8_t* smem, int c, int lane, v4f (&acc)[5][2]) {
    const int g = lane >> 4, q = (lane & 15) >> 1, p = lane & 1, m = lane & 15;
    const unsigned fw = ((q >> 1) & 3) | ((g & 1) << 2);
    const unsigned w0 = (16 * g + q) * 128 + (((2 * c) ^ fw) << 4) + p * 8;
    const unsigned x0 = 16384 + m * 128 + ((g ^ ((m >> 1) & 7)) << 4);
#pragma unroll
    for (int u = 0; u < 5; ++u) { acc[u][0] = v4f{0, 0, 0, 0}; acc[u][1] = v4f{0, 0, 0, 0}; }
    v8i wf[2][2], xf[2][NU];
    unsigned aw0 = w0, aw1 = w0 ^ 16, ax0 = x0, ax1 = x0 ^ 64, base = 0;
    auto load = [&](int s, v8i (&wv)[2], v8i (&xv)[NU]) {
        LAS uint8_t* sl = (LAS uint8_t*)(uintptr_t)(s * SLOT);
#pragma unroll
        for (int cb = 0; cb < 2; ++cb) {
            LAS uint8_t* b = sl + (cb ? aw1 : aw0);
            v2i t0 = __builtin_amdgcn_ds_read_tr8_b64_v2i32((LAS v2i*)(b));
            v2i t1 = __builtin_amdgcn_ds_read_tr8_b64_v2i32((LAS v2i*)(b + 1024));
            v2i t2 = __builtin_amdgcn_ds_read_tr8_b64_v2i32((LAS v2i*)(b + 8192));
            v2i t3 = __builtin_amdgcn_ds_read_tr8_b64_v2i32((LAS v2i*)(b + 9216));
            wv[cb] = v8i{t0[0], t0[1], t1[0], t1[1], t2[0], t2[1], t3[0], t3[1]};
        }
#pragma unroll
        for (int u = 0; u < NU; ++u) {
            v4i a = *(LAS v4i*)(sl + ax0 + u * 2048);
            v4i b = *(LAS v4i*)(sl + ax1 + u * 2048);
            xv[u] = v8i{a[0], a[1], a[2], a[3], b[0], b[1], b[2], b[3]};
        }
    };
    auto mma = [&](v8i (&wv)[2], v8i (&xv)[NU]) {
#pragma unroll
        for (int u = 0; u < NU; ++u)
#pragma unroll
            for (int cb = 0; cb < 2; ++cb)
                acc[u][cb] = __builtin_amdgcn_mfma_scale_f32_16x16x128_f8f6f4(wv[cb], xv[u], acc[u][cb], 1, 0, 0, 0, 0, 0);
    };
    auto step = [&](int s, v8i (&wn)[2], v8i (&xn)[NU], v8i (&wo)[2], v8i (&xo)[NU]) {
        wg_barrier();
        __builtin_amdgcn_sched_barrier(0);
        load(s, wn, xn);
        mma(wo, xo);
        constexpr int PER = (8 + 2 * NU + 2 * NU - 1) / (2 * NU);
#pragma unroll
        for (int i = 0; i < 2 * NU; ++i) {
            __builtin_amdgcn_sched_group_barrier(0x100, PER, 0);
            __builtin_amdgcn_sched_group_barrier(0x8, 1, 0);
        }
        __builtin_amdgcn_sched_barrier(0);
        if (s == 1) {
            base += 2 * SLOT;
            unsigned d = 2 * SLOT;
            if (base == NS * SLOT) { base = 0; d -= NS * SLOT; }
            aw0 += d; aw1 += d; ax0 += d; ax1 += d;
            asm volatile("" : "+v"(aw0), "+v"(aw1), "+v"(ax0), "+v"(ax1));
            __builtin_amdgcn_sched_barrier(0);
        }
        __builtin_amdgcn_s_waitcnt(0xC07F);
        __builtin_amdgcn_sched_barrier(0);
    };
    wf[1][0] = wf[1][1] = v8i{0, 0, 0, 0, 0, 0, 0, 0};
#pragma unroll
    for (int u = 0; u < NU; ++u) xf[1][u] = v8i{0, 0, 0, 0, 0, 0, 0, 0};
    __builtin_amdgcn_s_waitcnt(0xC07F);
#pragma unroll 1
    for (int it = 0; it < KSTEPS / 2; ++it) {
        step(0, wf[0], xf[0], wf[1], xf[1]);
        step(1, wf[1], xf[1], wf[0], xf[0]);
    }
    static_assert(NS % 2 == 0, "slot pairs");
    mma(wf[1], xf[1]);
}

__global__ void __launch_bounds__(NTHREADS) gemm_a2a_kernel(Params P) {
    extern __shared__ __attribute__((aligned(16))) uint8_t smem_[];
    LAS uint8_t* smem = (LAS uint8_t*)smem_;
    const int grp = ~(blockIdx.x + XOFF) & 7;
    const int idx = blockIdx.x >> 3, panel = idx & 3, rgp = ((idx >> 2) + 7) & 7;
    const CAS char* ka = (const CAS char*)__builtin_amdgcn_kernarg_segment_ptr();
#define KARG(off) (*(const volatile CAS u64*)(ka + (off)))
    const u64 px = KARG(offsetof(Params, x)), ctl = KARG(offsetof(Params, ctl));
    const u64 wcol = KARG(offsetof(Params, g) + grp * sizeof(Group)), parea = KARG(offsetof(Params, g) + grp * sizeof(Group) + 8);
    const u64 prows = KARG(offsetof(Params, g) + grp * sizeof(Group) + 16);
    const int lane = threadIdx.x & 63;
    const int wave = rfl(threadIdx.x >> 6);
    const bool local = grp == 7;
    const bool pusher = !local && rgp == 7;
#define CTR (ctl + C_CALL + blockIdx.x * 128)
#define PACE ((GAS u64*)(ctl + C_PACE + grp * 128))
#define SLOAD(a) (*(const volatile CAS unsigned*)(a))
#define TOUCH(a) do { unsigned t_ = SLOAD(a); asm volatile("" :: "s"(t_)); } while (0)
    unsigned call = 0;
    float scale = 0.f;
    const int g = lane >> 4, m = lane & 15;

    constexpr int XI = RR / 8;
    const unsigned xo = (panel * 8 + wave) * XI * 1024;
    const unsigned lb = wave * XI * 1024;
    if (pusher) {
        const __amdgpu_buffer_rsrc_t rp = make_rsrc(px, RR * KD);
        dma16(lb, lane * 16, rp, xo);
        dma16(lb + 1024, lane * 16, rp, xo + 1024);
    }
    const bool helper = !local && !pusher && panel >= EP;
    const int h0 = local ? 64 : (int)(pack(HL) >> (8 * rgp)) & 255;
    const int row0 = local ? rgp * 64 : RR + 2 * ((int)(rpack() >> (8 * rgp)) & 255);
    const unsigned dly = local ? 0u : 4 * ((unsigned)(pack(DL) >> (8 * rgp)) & 255);
    const bool atonce = !local && !pusher && !helper && dly == 0;
    Loader L;
    if (atonce && wave >= 4) {
        load_begin<false>(L, h0, wave - 4, lane, panel * 128, row0 * KD, 0u, wcol, px, (row0 + h0) * KD);
    }
    unsigned sa = 0, sb = 0;
    if (!pusher && !helper && wave < 4) {
        u64 pc; unsigned len;
        asm volatile("s_getpc_b64 %0\n.Lcodepf_pc:\n\ts_mov_b32 %1, .Lcodepf_end-.Lcodepf_pc" : "=s"(pc), "=s"(len));
        dma16(JUNK_OFF, lane * 16, make_rsrc(pc & ~15ull, len < 65536u ? len & ~15u : 0u), (((rgp * 2 + panel) * 4 + wave) & 31) * 1024u);
        sa = SLOAD((u64)P.sx); sb = SLOAD((u64)P.sw); call = SLOAD(CTR);
    }
    constexpr int NH = (4 - EP) * 7, NP = 16 * EP + (MS - RR) / 8;
    const int id = wave * NH + (panel - EP) * 7 + rgp;
    if (helper && id < NP) {
        const bool isw = id < 16 * EP;
        unsigned vo = ((isw ? id & 15 : id - 16 * EP + RR / 8) * 8 + (lane >> 3)) * KD + (isw ? (id >> 4) * 128 : 0) + (lane & 7) * 16;
        unsigned st = isw ? 128u * ND : 128u, sz = isw ? WRANGE : (unsigned)(MS * KD);
        asm volatile("" : "+v"(vo), "+s"(st), "+s"(sz));
        __builtin_amdgcn_sched_barrier(0);
        const __amdgpu_buffer_rsrc_t rs = make_rsrc(isw ? wcol : px, sz);
#pragma unroll 1
        for (unsigned sg = 0; sg < KSTEPS; ++sg) { dma16(JUNK_OFF, vo, rs, sg * st); wait_vm<PWIN>(); }
    }

    if (__builtin_expect(pusher, 1)) {
        const __amdgpu_buffer_rsrc_t rp = make_rsrc(px, RR * KD);
        wg_barrier();
        dma16(lb + 2048, lane * 16, rp, xo + 2048);
        dma16(lb + 3072, lane * 16, rp, xo + 3072);
        const u64 xd = parea + A_ROWS + xo + lane * 16;
        LAS uint8_t* ls = smem + lb + lane * 16;
        wait_vm<3>();
        v4f v = *(LAS v4f*)ls;
#pragma unroll
        for (int i = 0; i < XI; ++i) {
            if (i >= 2) {
                if (i + 3 < XI) wait_vm<3>(); else if (i + 2 < XI) wait_vm<2>(); else wait_vm<1>();
            }
            st16_sys(xd + i * 1024, v);
            if (i + 4 < XI) dma16(lb + (i + 4) * 1024, lane * 16, rp, xo + (i + 4) * 1024);
            if (i == 1 && wave == 7 && lane == 0) {
                __hip_atomic_fetch_max(PACE, rtime() << 6, __ATOMIC_RELAXED, __HIP_MEMORY_SCOPE_WORKGROUP);
                __hip_atomic_fetch_add(PACE, (u64)(RR * XTAU), __ATOMIC_RELAXED, __HIP_MEMORY_SCOPE_WORKGROUP);
            }
            if (i == 0) wait_vm<4>();
            if (i == 1) wait_vm<5>();
            if (i + 1 < XI) v = *(LAS v4f*)(ls + (i + 1) * 1024);
        }
        static_assert(XI >= 6 && XI <= 16, "wait counts");
        scale = __uint_as_float(SLOAD((u64)P.sx)) * __uint_as_float(SLOAD((u64)P.sw));
        call = SLOAD(CTR) + 1;
        wait_vm<0>();
        if (lane == 0) flag_sys(parea + A_XFLAG + (panel * 8 + wave) * 4, call);
    } else {
        if (helper && (unsigned)(id - NP) < 15u) {
            const int j = id - NP;
            const u64 a = j < 1 ? parea + 4
                        : j < 2 ? px + MS * KD - 4
                        : j < 10 ? (u64)P.w + ((u64)(j - 1) << 21) - 4
                        : j < 12 ? prows + (j - 10) * (MS * NC * 4 - 4)
                        : j < 13 ? (u64)P.g[grp].marea
                        : (u64)P.mrows[grp] + (j & 1) * (RR * NC * 4);
            if (j > 1) delay(rtime(), 50 + j * 10);
            TOUCH(a);
        }
        if (helper) wg_barrier();
        if (!pusher && !helper && dly) delay(rtime(), dly);
        if (!pusher && !atonce && wave >= 4) {
            const int r7 = rgp + 3 * (panel & 1);
            const unsigned first = local ? comb(8) << ((rgp + 2 * panel) & 7) : panel < EP ? 0u : comb(7) << (r7 >= 7 ? r7 - 7 : r7);
            load_begin<false>(L, h0, wave - 4, lane, panel * 128, row0 * KD, first, wcol, px, (row0 + h0) * KD);
        }
        if (wave >= 4) {
            const int w = wave - 4;
            load_run<false>(L, w, !local && w == 0 ? (u64)PACE : 0, h0 / 2);
            call = SLOAD(CTR) + 1;
            asm volatile("" : "+s"(call));
        } else {
            v4f acc[5][2];
            if (local && rgp >= 4) {
                const int j = (rgp - 4) * 16 + panel * 4 + wave;
                const u64 a = j < 8 ? (u64)P.w + ((u64)(j + 1) << 21) - 4 : j < 16 ? (u64)P.mrows[(j - 8) & 7] : (u64)P.g[(j - 16) & 7].marea;
                if (j < 23) TOUCH(a);
            }
            if (helper) { sa = SLOAD((u64)P.sx); sb = SLOAD((u64)P.sw); call = SLOAD(CTR); }
            asm volatile("" : "+s"(sa), "+s"(sb), "+s"(call));
            call += 1;
            scale = __uint_as_float(sa) * __uint_as_float(sb);
            if (h0 <= 32) consume_tile<2>(smem, wave, lane, acc);
            else if (h0 <= 48) consume_tile<3>(smem, wave, lane, acc);
            else if (h0 <= 64) consume_tile<4>(smem, wave, lane, acc);
            else consume_tile<5>(smem, wave, lane, acc);
            if (!local) {
#pragma unroll
                for (int u = 0; u < 5; ++u)
                    if (u * 16 < h0) {
                        const int r = u * 16 + m;
                        LAS uint8_t* d = smem + STG_OFF + r * 512 + wave * 128;
                        *(LAS v4f*)(d + ((g ^ (r & 7)) << 4)) = acc[u][0] * scale;
                        *(LAS v4f*)(d + (((4 + g) ^ (r & 7)) << 4)) = acc[u][1] * scale;
                    }
                wait_lds();
            } else {
                const u64 o = prows + ((row0 + m) * NC + panel * 128 + wave * 32 + g * 4) * 4;
#pragma unroll
                for (int u = 0; u < 4; ++u) {
                    st16_sys(o + u * 16 * NC * 4, acc[u][0] * scale);
                    st16_sys(o + u * 16 * NC * 4 + 64, acc[u][1] * scale);
                }
            }
        }

        if (!local) {
            const u64 o = prows + ((row0 + (lane >> 5)) * NC + panel * 128 + (lane & 31) * 4) * 4;
            wg_barrier();
            const u64 st = *(volatile LAS u64*)(smem + CTL_OFF + 8);
            u64 slot = (((u64)rfl((unsigned)(st >> 32)) << 32) | rfl((unsigned)st)) + wave * STAU;
#pragma unroll 1
            for (int it = wave; it < h0 / 2; it += 8) {
                const int r = it * 2 + (lane >> 5);
                const v4f v = *(LAS v4f*)(smem + STG_OFF + r * 512 + (((lane & 31) ^ (r & 7)) << 4));
                for (;;) {
                    const long long d = (long long)(slot >> 6) - (long long)rtime();
                    if (d <= 0) break;
                    if (d > 30) __builtin_amdgcn_s_sleep(8);
                }
                st16_sys(o + it * (2 * NC * 4), v);
                slot += 8 * STAU;
            }
            asm volatile(".Lcodepf_end:");
            wait_vm<0>();
            wg_barrier();
            if (wave == 0 && lane == 0) {
                flag_sys(parea + A_DONE + (rgp * 4 + panel) * 4, call);
                *(GAS unsigned*)CTR = call;
            }
            if (rgp == 0 && panel < EP) {
                const u64 c3 = (u64)P.g[7].wcol + 3 * 128;
                if (((c3 >> 7) & 7) == 3) {
                    const __amdgpu_buffer_rsrc_t rw = make_rsrc(c3, WRANGE - 3 * 128);
                    const unsigned vl = ((panel * 8 + wave) * 8 + (lane >> 3)) * ND + (lane & 7) * 16;
#pragma unroll 1
                    for (int i = 4; i < KSTEPS; ++i) dma16(JUNK_OFF, vl, rw, i * (128u * ND));
                }
            }
            return;
        }
        if (rgp == 7) {
            wait_vm<0>();
            wg_barrier();
            if (wave == 7 && lane == 0) *(GAS unsigned*)CTR = call;
            if (panel == 0 && wave < 7) {
                const bool ok = wait_flags((u64)P.g[wave].marea + A_DONE, 36, call, ctl + C_TMO, lane);
                if (!ok && lane == 0)
                    flag_sys(prows + row0 * NC * 4, 0x7fc00000u);
            }
            return;
        }
    }

    const int sg = local ? rgp : grp;
    const int r2 = local ? 0 : RA, h2 = local ? RA : RB;
    const u64 marea = (u64)P.g[sg].marea;
    if (wave == 4) {
        const bool ok = wait_flags(marea + A_XFLAG, 32, call, ctl + C_TMO, lane);
        *(volatile LAS unsigned*)(smem + CTL_OFF) = ok ? 0u : 1u;
        wait_lds();
    }
    wg_barrier();
    if (wave >= 4) {
        const int w = wave - 4;
        load_begin<true>(L, h2, w, lane, panel * 128, r2 * KD, 0u, (u64)P.g[7].wcol, marea + A_ROWS, (r2 + h2) * KD);
        load_run<true>(L, w);
        if (w == 0 && lane == 0) flag_sys((u64)P.g[sg].parea + A_DONE + ((local ? 32 : 28) + panel) * 4, call);
    } else {
        if (wave == 0 && lane == 0) *(GAS unsigned*)CTR = call;
        v4f acc[5][2];
        if (local) consume_tile<(RA + 15) / 16>(smem, wave, lane, acc);
        else consume_tile<(RB + 15) / 16>(smem, wave, lane, acc);
        if (*(volatile LAS unsigned*)(smem + CTL_OFF)) scale = __builtin_nanf("");
        const u64 o = (u64)P.mrows[sg] + ((r2 + m) * NC + panel * 128 + wave * 32 + g * 4) * 4;
#pragma unroll
        for (int u = 0; u < (RM + 15) / 16; ++u)
            if (u * 16 + m < h2) {
                st16_sys(o + u * 16 * NC * 4, acc[u][0] * scale);
                st16_sys(o + u * 16 * NC * 4 + 64, acc[u][1] * scale);
            }
    }
}

}

extern "C" void kernel_launch(int rank, int nranks, ncclComm_t comm,
                              hipStream_t stream,
                              void* const* d_in, const int* in_sizes, int n_in,
                              void* d_out, int out_size,
                              const int* in_dtypes, int out_dtype,
                              void* const* d_out_all,
                              void* const* d_ws_all, size_t ws_bytes) {
    (void)comm; (void)in_dtypes; (void)out_dtype;
    if (nranks != NR || (unsigned)rank >= (unsigned)NR || n_in != 4 || in_sizes[0] != MS * KD || in_sizes[1] != KD * ND
        || out_size != NR * MS * NC || ws_bytes < WS_NEED) {
        fprintf(stderr, "kernel_launch: unexpected problem shape\n");
        return;
    }
    static bool attr_set = false;
    if (!attr_set) {
        HIP_CHECK(hipFuncSetAttribute((const void*)gemm_a2a_kernel, hipFuncAttributeMaxDynamicSharedMemorySize, LDS_BYTES));
        attr_set = true;
    }
    Params P;
    P.x = (const uint8_t*)d_in[0];
    P.w = (const uint8_t*)d_in[1];
    P.sx = (const float*)d_in[2];
    P.sw = (const float*)d_in[3];
    uint8_t* ws = (uint8_t*)d_ws_all[rank];
    P.ctl = ws + WS_SRC + (size_t)rank * SRC_BYTES;
    int order[NR], n = 0;
    const unsigned set = (0x4Du >> rank) & 1 ? 0x4Du : 0xB2u;
    for (int k = 1; k < NR; ++k) if (!((set >> ((rank + k) % NR)) & 1)) order[n++] = (rank + k) % NR;
    for (int k = 1; k <= NR; ++k) if ((set >> ((rank + k) % NR)) & 1) order[n++] = (rank + k) % NR;
    for (int g = 0; g < NR; ++g) {
        const int p = order[g];
        P.g[g].wcol = P.w + (size_t)p * NC;
        P.g[g].parea = (uint8_t*)d_ws_all[p] + WS_SRC + (size_t)rank * SRC_BYTES;
        P.g[g].prows = (float*)d_out_all[p] + (size_t)rank * MS * NC;
        P.g[g].marea = ws + WS_SRC + (size_t)p * SRC_BYTES;
        P.mrows[g] = (float*)d_out + (size_t)p * MS * NC;
    }
    hipLaunchKernelGGL(gemm_a2a_kernel, dim3(NWG), dim3(NTHREADS), LDS_BYTES, stream, P);
    HIP_CHECK(hipGetLastError());
}
```

```cpp
#include <hip/hip_runtime.h>
#include <rccl/rccl.h>
#include <cstdio>
#include <cstdint>
#include <cstddef>

#define HIP_CHECK(expr) do { hipError_t _e = (expr); if (_e != hipSuccess) { \
    fprintf(stderr, "HIP error %s at %s:%d\n", hipGetErrorString(_e), __FILE__, __LINE__); \
    return; } } while (0)

namespace {

typedef int v2i __attribute__((ext_vector_type(2)));
typedef int v4i __attribute__((ext_vector_type(4)));
typedef int v8i __attribute__((ext_vector_type(8)));
typedef float v4f __attribute__((ext_vector_type(4)));
typedef unsigned long long u64;
#define LAS __attribute__((address_space(3)))
#define GAS __attribute__((address_space(1)))
#define CAS __attribute__((address_space(4)))

constexpr int NR = 8;
constexpr int KD = 4096;
constexpr int ND = 4096;
constexpr int MS = 512;
constexpr int NC = 512;
constexpr int RR = 96;
constexpr int RA = 32;
constexpr int RB = RR - RA;
constexpr int RM = RA > RB ? RA : RB;
constexpr int HMAX = 80;
constexpr int HL[7] = {32, 48, 64, 64, 64, 64, 80};
constexpr int DL[7] = {0, 0, 0, 10, 30, 45, 50};
constexpr int hsum(int n) { int t = 0; for (int i = 0; i < n; ++i) t += HL[i]; return t; }
constexpr u64 pack(const int (&a)[7]) { u64 t = 0; for (int i = 0; i < 7; ++i) t |= (u64)a[i] << (8 * i); return t; }
constexpr u64 rpack() { u64 t = 0; for (int i = 0; i < 7; ++i) t |= (u64)(hsum(i) / 2) << (8 * i); return t; }
constexpr bool hok() { for (int i = 0; i < 7; ++i) if (HL[i] < 2 || HL[i] > HMAX || HL[i] % 2) return false; return true; }
static_assert(RR % 8 == 0 && RA >= 8 && RB >= 8 && RM <= HMAX && hok() && hsum(7) == MS - RR, "shape");
constexpr int EP = 2;
constexpr int PWIN = 6;
constexpr int AH1 = 3;
constexpr int TAU = 110;
constexpr int XTAU = 104;
constexpr int STAU = 135;
constexpr int XOFF = 5;
constexpr int LEAD = 50;
constexpr int NTHREADS = 512;
constexpr int NWG = 256;
constexpr int KSTEPS = KD / 128;

constexpr unsigned SLOT = 16384 + HMAX * 128, NS = 6;
constexpr unsigned STG_OFF = 2 * SLOT;
static_assert((KSTEPS - 1) % NS == 1, "the last two stages use slots 0 and 1");
constexpr unsigned CTL_OFF = NS * SLOT;
constexpr unsigned JUNK_OFF = CTL_OFF + 64;
constexpr unsigned LDS_BYTES = JUNK_OFF + 1024;

constexpr size_t WS_SRC = 2u << 20, SRC_BYTES = 512u << 10;
constexpr size_t A_DONE = 0;
constexpr size_t A_XFLAG = 256;
constexpr size_t A_ROWS = 4096;
constexpr size_t C_TMO = 0;
constexpr size_t C_PACE = 1024;
constexpr size_t C_CALL = 4096;
static_assert(A_ROWS + RR * KD <= SRC_BYTES, "area");
constexpr size_t WS_NEED = WS_SRC + NR * SRC_BYTES;
constexpr u64 DEADLINE_TICKS = 40000000ull;
constexpr unsigned WRANGE = KD * ND - (ND - NC);

struct Group {
    const uint8_t* wcol;
    uint8_t* parea;
    float* prows;
    uint8_t* marea;
};
struct Params {
    const uint8_t* x;
    uint8_t* ctl;
    const float* sx;
    const float* sw;
    Group g[NR];
    float* mrows[NR];
    const uint8_t* w;
};

constexpr unsigned comb(int n) { unsigned m = 0; for (int i = 0; i < 32; i += n) m |= 1u << i; return m; }

__device__ __forceinline__ unsigned rfl(unsigned v) { return __builtin_amdgcn_readfirstlane(v); }
__device__ __forceinline__ u64 rtime() { return __builtin_amdgcn_s_memrealtime(); }
__device__ __forceinline__ void delay(u64 t0, unsigned ticks) { while (rtime() - t0 < ticks) __builtin_amdgcn_s_sleep(4); }

__device__ __forceinline__ __amdgpu_buffer_rsrc_t make_rsrc(u64 p, unsigned bytes) {
    return __builtin_amdgcn_make_buffer_rsrc((void*)p, (short)0, (int)bytes, 0x00020000);
}

template <bool SYS = false>
__device__ __forceinline__ void dma16(unsigned lds, unsigned voff, __amdgpu_buffer_rsrc_t rs, unsigned soff) {
    if (SYS) asm volatile("s_mov_b32 m0, %0\n\ts_nop 0\n\tbuffer_load_dwordx4 %1, %2, %3 offen sc0 sc1 lds"
                          :: "s"(lds), "v"(voff), "s"(rs), "s"(soff) : "memory");
    else     asm volatile("s_mov_b32 m0, %0\n\ts_nop 0\n\tbuffer_load_dwordx4 %1, %2, %3 offen lds"
                          :: "s"(lds), "v"(voff), "s"(rs), "s"(soff) : "memory");
}
template <int N> __device__ __forceinline__ void wait_vm() { asm volatile("s_waitcnt vmcnt(%0)" :: "i"(N) : "memory"); }
__device__ __forceinline__ void wg_barrier() { asm volatile("s_barrier" ::: "memory"); }
__device__ __forceinline__ void wait_lds() { asm volatile("s_waitcnt lgkmcnt(0)" ::: "memory"); }
__device__ __forceinline__ void st16_sys(u64 p, v4f v) {
    asm volatile("global_store_dwordx4 %0, %1, off sc0 sc1\n\ts_nop 1" :: "v"(p), "v"(v) : "memory");
}
__device__ __forceinline__ void flag_sys(u64 p, unsigned v) {
    __hip_atomic_store((GAS unsigned*)p, v, __ATOMIC_RELAXED, __HIP_MEMORY_SCOPE_SYSTEM);
}

template <bool EXIT = false>
__device__ __forceinline__ bool wait_flags(u64 flags, int n, unsigned call, u64 tmo, int lane) {
    const GAS unsigned* p = (const GAS unsigned*)(lane < n ? flags + 4 * lane : tmo);
    const u64 t0 = rtime();
    for (;;) {
        const u64 now = rtime();
        const unsigned v = __hip_atomic_load(p, __ATOMIC_RELAXED, __HIP_MEMORY_SCOPE_SYSTEM);
        if (EXIT) asm volatile("v_cmp_gt_u32 vcc, %0, %1\n\ts_nop 4\n\ts_cbranch_vccnz 1f\n\ts_endpgm\n1:" :: "v"(lane < n ? call : 0u), "v"(v) : "vcc", "memory");
        const u64 ge = __ballot(v >= call), eq = __ballot(v == call);
        if ((eq >> 63) || now - t0 > DEADLINE_TICKS) break;
        if ((ge | (~0ull << n)) == ~0ull) return true;
    }
    if (lane == 0) __hip_atomic_store((GAS unsigned*)tmo, call, __ATOMIC_RELAXED, __HIP_MEMORY_SCOPE_AGENT);
    return false;
}

struct Loader {
    unsigned vw[4], vx[3], ioff;
    u64 todo;
    __amdgpu_buffer_rsrc_t rw, rx;
    int nxw;
};
#define STEP_W "buffer_load_dwordx4 %4, %11, %13 offen lds\n\t" \
               "buffer_load_dwordx4 %5, %11, %13 offen offset:1024 lds\n\t" \
               "buffer_load_dwordx4 %6, %11, %13 offen offset:2048 lds\n\t" \
               "buffer_load_dwordx4 %7, %11, %13 offen offset:3072 lds"
#define STEP_X(m, v, pol) "\n\ts_mov_b32 m0, %" m "\n\ts_nop 0\n\tbuffer_load_dwordx4 %" v ", %12, %14 offen" pol " lds"
#define STEP_OPS(n) :: "s"(L.ioff + w * 3072), "s"(L.ioff + 16384), "s"(L.ioff + 20480), "s"(L.ioff + 24576), \
                       "v"(L.vw[0]), "v"(L.vw[1]), "v"(L.vw[2]), "v"(L.vw[3]), "v"(L.vx[0]), "v"(L.vx[1]), "v"(L.vx[2]), \
                       "s"(L.rw), "s"(L.rx), "s"(ks * (128u * ND)), "s"(ks * 128u), "i"(n) : "memory"
template <bool SYS>
__device__ __forceinline__ void load_begin(Loader& L, int h, int w, int lane, unsigned woff, unsigned xoff, unsigned first,
                                           u64 pw, u64 px, unsigned xbytes) {
    const unsigned v0 = (lane >> 3) * KD + (((lane & 7) ^ ((lane >> 4) & 3)) << 4);
#pragma unroll
    for (int j = 0; j < 4; ++j)
        L.vw[j] = (v0 ^ (((j >> 1) & 1) << 6)) + (4 * w + j) * (8u * ND) - j * 1024u + woff;
#pragma unroll
    for (int j = 0; j < 3; ++j) L.vx[j] = (v0 ^ ((w & 1) << 6)) + (w + 4 * j) * (8u * KD) + xoff;
    L.ioff = w * 1024;
    L.todo = ((u64)~first << 32) | first;
    L.nxw = (h - 8 * w + 31) >> 5;
    asm volatile("" : "+v"(L.vw[0]), "+v"(L.vw[1]), "+v"(L.vw[2]), "+v"(L.vw[3]), "+v"(L.vx[0]), "+v"(L.vx[1]), "+v"(L.vx[2]),
                      "+s"(L.ioff), "+s"(L.todo), "+s"(L.nxw), "+s"(xbytes));
    __builtin_amdgcn_sched_barrier(0);
    asm volatile("" : "+s"(pw), "+s"(px));
    L.rw = make_rsrc(pw, WRANGE);
    L.rx = make_rsrc(px, xbytes);
#pragma unroll 1
    for (int i = 0; i < AH1; ++i) {
        const unsigned ks = __builtin_ctzll(L.todo) & 31;
        L.todo &= L.todo - 1;
        asm volatile("s_mov_b32 m0, %0\n\ts_nop 0\n\t" STEP_W STEP_OPS(0));
        if (L.nxw > 0) dma16<SYS>(L.ioff + 16384, L.vx[0], L.rx, ks * 128u);
        if (L.nxw > 1) dma16<SYS>(L.ioff + 20480, L.vx[1], L.rx, ks * 128u);
        if (L.nxw > 2) dma16<SYS>(L.ioff + 24576, L.vx[2], L.rx, ks * 128u);
        L.ioff += SLOT;
    }
}
template <int NXW, bool SYS>
__device__ __forceinline__ void load_run_n(Loader& L, int w, u64 pace, unsigned kib) {
    constexpr int CNT = 4 + NXW, N = CNT * (AH1 - 1);
#define STEP_H "s_mov_b32 m0, %0\n\ts_waitcnt vmcnt(%15)\n\ts_barrier\n\t"
#define STEP_P(pol) do { \
        if (NXW == 0) asm volatile(STEP_H STEP_W STEP_OPS(N)); \
        if (NXW == 1) asm volatile(STEP_H STEP_W STEP_X("1", "8", pol) STEP_OPS(N)); \
        if (NXW == 2) asm volatile(STEP_H STEP_W STEP_X("1", "8", pol) STEP_X("2", "9", pol) STEP_OPS(N)); \
        if (NXW == 3) asm volatile(STEP_H STEP_W STEP_X("1", "8", pol) STEP_X("2", "9", pol) STEP_X("3", "10", pol) STEP_OPS(N)); \
    } while (0)
#pragma unroll 1
    while (L.todo) {
        const unsigned ks = __builtin_ctzll(L.todo) & 31;
        L.todo &= L.todo - 1;
        if (SYS) STEP_P(" sc0 sc1"); else STEP_P("");
        L.ioff += SLOT;
        if (L.ioff >= SLOT * NS) L.ioff -= SLOT * NS;
    }
    static_assert(AH1 == 3 && AH1 < (int)NS, "drain");
    if (!SYS && pace) {
        const u64 now = (rtime() + LEAD) << 6, len = (u64)kib * TAU;
        u64 t;
        asm volatile("s_mov_b64 exec, 1\n\tglobal_atomic_umax_x2 %1, %2, %4\n\tglobal_atomic_add_x2 %0, %1, %3, %4 sc0\n\ts_mov_b64 exec, -1\n\t"
                     "s_waitcnt vmcnt(%5)\n\ts_barrier\n\ts_waitcnt vmcnt(%6)\n\ts_barrier\n\ts_waitcnt vmcnt(0)\n\ts_barrier"
                     : "=&v"(t) : "v"(0u), "v"(now), "v"(len), "s"(pace), "i"(CNT * 2), "i"(CNT) : "memory");
        if ((threadIdx.x & 63) == 0) *(volatile LAS u64*)(uintptr_t)(CTL_OFF + 8) = t - now < (4000ull << 6) ? t : now;
        wait_lds();
    } else {
        wait_vm<CNT * 2>(); wg_barrier();
        wait_vm<CNT>(); wg_barrier();
        wait_vm<0>(); wg_barrier();
    }
}
template <bool SYS>
__device__ __forceinline__ void load_run(Loader& L, int w, u64 pace = 0, unsigned kib = 0) {
    if (L.nxw == 2) load_run_n<2, SYS>(L, w, pace, kib);
    else if (L.nxw == 1) load_run_n<1, SYS>(L, w, pace, kib);
    else if (L.nxw == 3) load_run_n<3, SYS>(L, w, pace, kib);
    else load_run_n<0, SYS>(L, w, pace, kib);
}

template <int NU>
__device__ __forceinline__ void consume_tile(LAS uint8_t* smem, int c, int lane, v4f (&acc)[5][2]) {
    const int g = lane >> 4, q = (lane & 15) >> 1, p = lane & 1, m = lane & 15;
    const unsigned fw = ((q >> 1) & 3) | ((g & 1) << 2);
    const unsigned w0 = (16 * g + q) * 128 + (((2 * c) ^ fw) << 4) + p * 8;
    const unsigned x0 = 16384 + m * 128 + ((g ^ ((m >> 1) & 7)) << 4);
#pragma unroll
    for (int u = 0; u < 5; ++u) { acc[u][0] = v4f{0, 0, 0, 0}; acc[u][1] = v4f{0, 0, 0, 0}; }
    v8i wf[2][2], xf[2][NU];
    unsigned aw0 = w0, aw1 = w0 ^ 16, ax0 = x0, ax1 = x0 ^ 64, base = 0;
    auto load = [&](int s, v8i (&wv)[2], v8i (&xv)[NU]) {
        LAS uint8_t* sl = (LAS uint8_t*)(uintptr_t)(s * SLOT);
#pragma unroll
        for (int cb = 0; cb < 2; ++cb) {
            LAS uint8_t* b = sl + (cb ? aw1 : aw0);
            v2i t0 = __builtin_amdgcn_ds_read_tr8_b64_v2i32((LAS v2i*)(b));
            v2i t1 = __builtin_amdgcn_ds_read_tr8_b64_v2i32((LAS v2i*)(b + 1024));
            v2i t2 = __builtin_amdgcn_ds_read_tr8_b64_v2i32((LAS v2i*)(b + 8192));
            v2i t3 = __builtin_amdgcn_ds_read_tr8_b64_v2i32((LAS v2i*)(b + 9216));
            wv[cb] = v8i{t0[0], t0[1], t1[0], t1[1], t2[0], t2[1], t3[0], t3[1]};
        }
#pragma unroll
        for (int u = 0; u < NU; ++u) {
            v4i a = *(LAS v4i*)(sl + ax0 + u * 2048);
            v4i b = *(LAS v4i*)(sl + ax1 + u * 2048);
            xv[u] = v8i{a[0], a[1], a[2], a[3], b[0], b[1], b[2], b[3]};
        }
    };
    auto mma = [&](v8i (&wv)[2], v8i (&xv)[NU]) {
#pragma unroll
        for (int u = 0; u < NU; ++u)
#pragma unroll
            for (int cb = 0; cb < 2; ++cb)
                acc[u][cb] = __builtin_amdgcn_mfma_scale_f32_16x16x128_f8f6f4(wv[cb], xv[u], acc[u][cb], 1, 0, 0, 0, 0, 0);
    };
    auto step = [&](int s, v8i (&wn)[2], v8i (&xn)[NU], v8i (&wo)[2], v8i (&xo)[NU]) {
        wg_barrier();
        __builtin_amdgcn_sched_barrier(0);
        load(s, wn, xn);
        mma(wo, xo);
        constexpr int PER = (8 + 2 * NU + 2 * NU - 1) / (2 * NU);
#pragma unroll
        for (int i = 0; i < 2 * NU; ++i) {
            __builtin_amdgcn_sched_group_barrier(0x100, PER, 0);
            __builtin_amdgcn_sched_group_barrier(0x8, 1, 0);
        }
        __builtin_amdgcn_sched_barrier(0);
        if (s == 1) {
            base += 2 * SLOT;
            unsigned d = 2 * SLOT;
            if (base == NS * SLOT) { base = 0; d -= NS * SLOT; }
            aw0 += d; aw1 += d; ax0 += d; ax1 += d;
            asm volatile("" : "+v"(aw0), "+v"(aw1), "+v"(ax0), "+v"(ax1));
            __builtin_amdgcn_sched_barrier(0);
        }
        __builtin_amdgcn_s_waitcnt(0xC07F);
        __builtin_amdgcn_sched_barrier(0);
    };
    wf[1][0] = wf[1][1] = v8i{0, 0, 0, 0, 0, 0, 0, 0};
#pragma unroll
    for (int u = 0; u < NU; ++u) xf[1][u] = v8i{0, 0, 0, 0, 0, 0, 0, 0};
    __builtin_amdgcn_s_waitcnt(0xC07F);
#pragma unroll 1
    for (int it = 0; it < KSTEPS / 2; ++it) {
        step(0, wf[0], xf[0], wf[1], xf[1]);
        step(1, wf[1], xf[1], wf[0], xf[0]);
    }
    static_assert(NS % 2 == 0, "slot pairs");
    mma(wf[1], xf[1]);
}

__global__ void __launch_bounds__(NTHREADS) gemm_a2a_kernel(Params P) {
    extern __shared__ __attribute__((aligned(16))) uint8_t smem_[];
    LAS uint8_t* smem = (LAS uint8_t*)smem_;
    const int grp = ~(blockIdx.x + XOFF) & 7;
    const int idx = blockIdx.x >> 3, panel = idx & 3, rgp = ((idx >> 2) + 7) & 7;
    const CAS char* ka = (const CAS char*)__builtin_amdgcn_kernarg_segment_ptr();
#define KARG(off) (*(const volatile CAS u64*)(ka + (off)))
    const u64 px = KARG(offsetof(Params, x)), ctl = KARG(offsetof(Params, ctl));
    const u64 wcol = KARG(offsetof(Params, g) + grp * sizeof(Group)), parea = KARG(offsetof(Params, g) + grp * sizeof(Group) + 8);
    const u64 prows = KARG(offsetof(Params, g) + grp * sizeof(Group) + 16);
    const int lane = threadIdx.x & 63;
    const int wave = rfl(threadIdx.x >> 6);
    const bool local = grp == 7;
    const bool pusher = !local && rgp == 7;
#define CTR (ctl + C_CALL + blockIdx.x * 128)
#define PACE ((GAS u64*)(ctl + C_PACE + grp * 128))
#define SLOAD(a) (*(const volatile CAS unsigned*)(a))
#define TOUCH(a) do { unsigned t_ = SLOAD(a); asm volatile("" :: "s"(t_)); } while (0)
    unsigned call = 0;
    float scale = 0.f;
    const int g = lane >> 4, m = lane & 15;

    constexpr int XI = RR / 8;
    const unsigned xo = (panel * 8 + wave) * XI * 1024;
    const unsigned lb = wave * XI * 1024;
    if (pusher) {
        const __amdgpu_buffer_rsrc_t rp = make_rsrc(px, RR * KD);
        dma16(lb, lane * 16, rp, xo);
        dma16(lb + 1024, lane * 16, rp, xo + 1024);
    }
    const bool helper = !local && !pusher && panel >= EP;
    const int h0 = local ? 64 : (int)(pack(HL) >> (8 * rgp)) & 255;
    const int row0 = local ? rgp * 64 : RR + 2 * ((int)(rpack() >> (8 * rgp)) & 255);
    const unsigned dly = local ? 0u : 4 * ((unsigned)(pack(DL) >> (8 * rgp)) & 255);
    const bool atonce = !local && !pusher && !helper && dly == 0;
    Loader L;
    if (atonce && wave >= 4) {
        load_begin<false>(L, h0, wave - 4, lane, panel * 128, row0 * KD, 0u, wcol, px, (row0 + h0) * KD);
    }
    unsigned sa = 0, sb = 0;
    if (!pusher && !helper && wave < 4) {
        u64 pc; unsigned len;
        asm volatile("s_getpc_b64 %0\n.Lcodepf_pc:\n\ts_mov_b32 %1, .Lcodepf_end-.Lcodepf_pc" : "=s"(pc), "=s"(len));
        dma16(JUNK_OFF, lane * 16, make_rsrc(pc & ~15ull, len < 65536u ? len & ~15u : 0u), (((rgp * 2 + panel) * 4 + wave) & 31) * 1024u);
        sa = SLOAD((u64)P.sx); sb = SLOAD((u64)P.sw); call = SLOAD(CTR);
    }
    constexpr int NH = (4 - EP) * 7, NP = 16 * EP + (MS - RR) / 8;
    const int id = wave * NH + (panel - EP) * 7 + rgp;
    if (helper && id < NP) {
        const bool isw = id < 16 * EP;
        unsigned vo = ((isw ? id & 15 : id - 16 * EP + RR / 8) * 8 + (lane >> 3)) * KD + (isw ? (id >> 4) * 128 : 0) + (lane & 7) * 16;
        unsigned st = isw ? 128u * ND : 128u, sz = isw ? WRANGE : (unsigned)(MS * KD);
        asm volatile("" : "+v"(vo), "+s"(st), "+s"(sz));
        __builtin_amdgcn_sched_barrier(0);
        const __amdgpu_buffer_rsrc_t rs = make_rsrc(isw ? wcol : px, sz);
#pragma unroll 1
        for (unsigned sg = 0; sg < KSTEPS; ++sg) { dma16(JUNK_OFF, vo, rs, sg * st); wait_vm<PWIN>(); }
    }

    if (__builtin_expect(pusher, 1)) {
        const __amdgpu_buffer_rsrc_t rp = make_rsrc(px, RR * KD);
        wg_barrier();
        dma16(lb + 2048, lane * 16, rp, xo + 2048);
        dma16(lb + 3072, lane * 16, rp, xo + 3072);
        const u64 xd = parea + A_ROWS + xo + lane * 16;
        LAS uint8_t* ls = smem + lb + lane * 16;
        wait_vm<3>();
        v4f v = *(LAS v4f*)ls;
#pragma unroll
        for (int i = 0; i < XI; ++i) {
            if (i >= 2) {
                if (i + 3 < XI) wait_vm<3>(); else if (i + 2 < XI) wait_vm<2>(); else wait_vm<1>();
            }
            st16_sys(xd + i * 1024, v);
            if (i + 4 < XI) dma16(lb + (i + 4) * 1024, lane * 16, rp, xo + (i + 4) * 1024);
            if (i == 1 && wave == 7 && lane == 0) {
                __hip_atomic_fetch_max(PACE, rtime() << 6, __ATOMIC_RELAXED, __HIP_MEMORY_SCOPE_WORKGROUP);
                __hip_atomic_fetch_add(PACE, (u64)(RR * XTAU), __ATOMIC_RELAXED, __HIP_MEMORY_SCOPE_WORKGROUP);
            }
            if (i == 0) wait_vm<4>();
            if (i == 1) wait_vm<5>();
            if (i + 1 < XI) v = *(LAS v4f*)(ls + (i + 1) * 1024);
        }
        static_assert(XI >= 6 && XI <= 16, "wait counts");
        scale = __uint_as_float(SLOAD((u64)P.sx)) * __uint_as_float(SLOAD((u64)P.sw));
        call = SLOAD(CTR) + 1;
        wait_vm<0>();
        if (lane == 0) flag_sys(parea + A_XFLAG + (panel * 8 + wave) * 4, call);
    } else {
        if (helper && (unsigned)(id - NP) < 15u) {
            const int j = id - NP;
            const u64 a = j < 1 ? parea + 4
                        : j < 2 ? px + MS * KD - 4
                        : j < 10 ? (u64)P.w + ((u64)(j - 1) << 21) - 4
                        : j < 12 ? prows + (j - 10) * (MS * NC * 4 - 4)
                        : j < 13 ? (u64)P.g[grp].marea
                        : (u64)P.mrows[grp] + (j & 1) * (RR * NC * 4);
            if (j > 1) delay(rtime(), 50 + j * 10);
            TOUCH(a);
        }
        if (helper) wg_barrier();
        if (!pusher && !helper && dly) delay(rtime(), dly);
        if (!pusher && !atonce && wave >= 4) {
            const int r7 = rgp + 3 * (panel & 1);
            const unsigned first = local ? comb(8) << ((rgp + 2 * panel) & 7) : panel < EP ? 0u : comb(7) << (r7 >= 7 ? r7 - 7 : r7);
            load_begin<false>(L, h0, wave - 4, lane, panel * 128, row0 * KD, first, wcol, px, (row0 + h0) * KD);
        }
        if (wave >= 4) {
            const int w = wave - 4;
            load_run<false>(L, w, !local && w == 0 ? (u64)PACE : 0, h0 / 2);
            if (local) call = SLOAD(CTR) + 1;
            asm volatile("" : "+s"(call));
        } else {
            v4f acc[5][2];
            if (local && rgp >= 4) {
                const int j = (rgp - 4) * 16 + panel * 4 + wave;
                const u64 a = j < 8 ? (u64)P.w + ((u64)(j + 1) << 21) - 4 : j < 16 ? (u64)P.mrows[(j - 8) & 7] : (u64)P.g[(j - 16) & 7].marea;
                if (j < 23) TOUCH(a);
            }
            if (helper) { sa = SLOAD((u64)P.sx); sb = SLOAD((u64)P.sw); call = SLOAD(CTR); }
            asm volatile("" : "+s"(sa), "+s"(sb), "+s"(call));
            call += 1;
            scale = __uint_as_float(sa) * __uint_as_float(sb);
            if (h0 <= 32) consume_tile<2>(smem, wave, lane, acc);
            else if (h0 <= 48) consume_tile<3>(smem, wave, lane, acc);
            else if (h0 <= 64) consume_tile<4>(smem, wave, lane, acc);
            else consume_tile<5>(smem, wave, lane, acc);
            if (!local) {
#pragma unroll
                for (int u = 0; u < 5; ++u)
                    if (u * 16 < h0) {
                        const int r = u * 16 + m;
                        LAS uint8_t* d = smem + STG_OFF + r * 512 + wave * 128;
                        *(LAS v4f*)(d + ((g ^ (r & 7)) << 4)) = acc[u][0] * scale;
                        *(LAS v4f*)(d + (((4 + g) ^ (r & 7)) << 4)) = acc[u][1] * scale;
                    }
                wait_lds();
            } else {
                const u64 o = prows + ((row0 + m) * NC + panel * 128 + wave * 32 + g * 4) * 4;
#pragma unroll
                for (int u = 0; u < 4; ++u) {
                    st16_sys(o + u * 16 * NC * 4, acc[u][0] * scale);
                    st16_sys(o + u * 16 * NC * 4 + 64, acc[u][1] * scale);
                }
            }
        }

        if (!local) {
            const u64 o = prows + ((row0 + (lane >> 5)) * NC + panel * 128 + (lane & 31) * 4) * 4;
            wg_barrier();
            const u64 st = *(volatile LAS u64*)(smem + CTL_OFF + 8);
            u64 slot = (((u64)rfl((unsigned)(st >> 32)) << 32) | rfl((unsigned)st)) + wave * STAU;
#pragma unroll 1
            for (int it = wave; it < h0 / 2; it += 8) {
                const int r = it * 2 + (lane >> 5);
                const v4f v = *(LAS v4f*)(smem + STG_OFF + r * 512 + (((lane & 31) ^ (r & 7)) << 4));
                for (;;) {
                    const long long d = (long long)(slot >> 6) - (long long)rtime();
                    if (d <= 0) break;
                    if (d > 30) __builtin_amdgcn_s_sleep(8);
                }
                st16_sys(o + it * (2 * NC * 4), v);
                slot += 8 * STAU;
            }
            asm volatile(".Lcodepf_end:");
            asm volatile("s_cmp_lg_u32 %3, 0\n\ts_waitcnt vmcnt(0)\n\ts_barrier\n\ts_cbranch_scc1 1f\n\t"
                         "s_mov_b64 exec, 1\n\tglobal_store_dword %0, %1, %2 sc0 sc1\n\ts_mov_b64 exec, -1\n1:"
                         :: "v"((unsigned)A_DONE + (rgp * 4 + panel) * 4), "v"(call), "s"(parea), "s"(wave) : "scc", "memory");
            if (wave == 0 && lane == 0) *(GAS unsigned*)CTR = call;
            if (rgp == 0 && panel < EP) {
                const u64 c3 = (u64)P.g[7].wcol + 3 * 128;
                if (((c3 >> 7) & 7) == 3) {
                    const __amdgpu_buffer_rsrc_t rw = make_rsrc(c3, WRANGE - 3 * 128);
                    const unsigned vl = ((panel * 8 + wave) * 8 + (lane >> 3)) * ND + (lane & 7) * 16;
#pragma unroll 1
                    for (int i = 4; i < KSTEPS; ++i) dma16(JUNK_OFF, vl, rw, i * (128u * ND));
                }
            }
            return;
        }
        if (rgp == 7) {
            wait_vm<0>();
            wg_barrier();
            if (wave == 7 && lane == 0) *(GAS unsigned*)CTR = call;
            if (panel == 0 && wave < 7) {
                const bool ok = wait_flags<true>((u64)P.g[wave].marea + A_DONE, 36, call, ctl + C_TMO, lane);
                if (!ok && lane == 0)
                    flag_sys(prows + row0 * NC * 4, 0x7fc00000u);
            }
            return;
        }
    }

    const int sg = local ? rgp : grp;
    const int r2 = local ? 0 : RA, h2 = local ? RA : RB;
    const u64 marea = (u64)P.g[sg].marea;
    if (wave == 4) {
        const bool ok = wait_flags(marea + A_XFLAG, 32, call, ctl + C_TMO, lane);
        *(volatile LAS unsigned*)(smem + CTL_OFF) = ok ? 0u : 1u;
        wait_lds();
    }
    wg_barrier();
    if (wave >= 4) {
        const int w = wave - 4;
        load_begin<true>(L, h2, w, lane, panel * 128, r2 * KD, local ? comb(7) << rgp : 0u, (u64)P.g[7].wcol, marea + A_ROWS, (r2 + h2) * KD);
        load_run<true>(L, w);
        if (w == 0 && lane == 0) flag_sys((u64)P.g[sg].parea + A_DONE + ((local ? 32 : 28) + panel) * 4, call);
    } else {
        if (wave == 0 && lane == 0) *(GAS unsigned*)CTR = call;
        v4f acc[5][2];
        if (local) consume_tile<(RA + 15) / 16>(smem, wave, lane, acc);
        else consume_tile<(RB + 15) / 16>(smem, wave, lane, acc);
        if (*(volatile LAS unsigned*)(smem + CTL_OFF)) scale = __builtin_nanf("");
        const u64 o = (u64)P.mrows[sg] + ((r2 + m) * NC + panel * 128 + wave * 32 + g * 4) * 4;
#pragma unroll
        for (int u = 0; u < (RM + 15) / 16; ++u)
            if (u * 16 + m < h2) {
                st16_sys(o + u * 16 * NC * 4, acc[u][0] * scale);
                st16_sys(o + u * 16 * NC * 4 + 64, acc[u][1] * scale);
            }
    }
}

}

extern "C" void kernel_launch(int rank, int nranks, ncclComm_t comm,
                              hipStream_t stream,
                              void* const* d_in, const int* in_sizes, int n_in,
                              void* d_out, int out_size,
                              const int* in_dtypes, int out_dtype,
                              void* const* d_out_all,
                              void* const* d_ws_all, size_t ws_bytes) {
    (void)comm; (void)in_dtypes; (void)out_dtype;
    if (nranks != NR || (unsigned)rank >= (unsigned)NR || n_in != 4 || in_sizes[0] != MS * KD || in_sizes[1] != KD * ND
        || out_size != NR * MS * NC || ws_bytes < WS_NEED) {
        fprintf(stderr, "kernel_launch: unexpected problem shape\n");
        return;
    }
    static bool attr_set = false;
    if (!attr_set) {
        HIP_CHECK(hipFuncSetAttribute((const void*)gemm_a2a_kernel, hipFuncAttributeMaxDynamicSharedMemorySize, LDS_BYTES));
        attr_set = true;
    }
    Params P;
    P.x = (const uint8_t*)d_in[0];
    P.w = (const uint8_t*)d_in[1];
    P.sx = (const float*)d_in[2];
    P.sw = (const float*)d_in[3];
    uint8_t* ws = (uint8_t*)d_ws_all[rank];
    P.ctl = ws + WS_SRC + (size_t)rank * SRC_BYTES;
    int order[NR], n = 0;
    const unsigned set = (0x4Du >> rank) & 1 ? 0x4Du : 0xB2u;
    for (int k = 1; k < NR; ++k) if (!((set >> ((rank + k) % NR)) & 1)) order[n++] = (rank + k) % NR;
    for (int k = 1; k <= NR; ++k) if ((set >> ((rank + k) % NR)) & 1) order[n++] = (rank + k) % NR;
    for (int g = 0; g < NR; ++g) {
        const int p = order[g];
        P.g[g].wcol = P.w + (size_t)p * NC;
        P.g[g].parea = (uint8_t*)d_ws_all[p] + WS_SRC + (size_t)rank * SRC_BYTES;
        P.g[g].prows = (float*)d_out_all[p] + (size_t)rank * MS * NC;
        P.g[g].marea = ws + WS_SRC + (size_t)p * SRC_BYTES;
        P.mrows[g] = (float*)d_out + (size_t)p * MS * NC;
    }
    hipLaunchKernelGGL(gemm_a2a_kernel, dim3(NWG), dim3(NTHREADS), LDS_BYTES, stream, P);
    HIP_CHECK(hipGetLastError());
}
```
